```python
import numpy as np

kernel_name = "gpu_a2a_gemm_m4096_k4096_n4096_w8a8_dequant"


ABSMAX_RTOL = 0.001

COMPUTE_DTYPE = "int8"
EPILOGUE = "dequant"
IN_QUANT = "int8"
OUT_QUANT = None
IN_DTYPES = {"x": "int8", "w_mat": "int8",
             "scale_x": "float32", "scale_w": "float32"}
WORLD_SIZE = 8

def setup_inputs():

    x = np.random.randint(-127, 128, size=(4096, 4096)).astype(np.int8)
    w_mat = np.random.randint(-127, 128, size=(4096, 4096)).astype(np.int8)
    scale_x = (0.02 * (1.0 + np.random.rand(1))).astype(np.float32)
    scale_w = (0.02 * (1.0 + np.random.rand(1))).astype(np.float32)
    return {"x": x, "w_mat": w_mat, "scale_x": scale_x, "scale_w": scale_w}

def reference(x, w_mat, scale_x, scale_w):
    acc = x.astype(np.float64) @ w_mat.astype(np.float64)
    y = (acc.astype(np.float32) * (scale_x * scale_w)).astype(np.float32)
    return y
```

```cpp
#include <hip/hip_runtime.h>
#include <rccl/rccl.h>
#include <cstdint>
#include <cstdio>

#define HIP_CHECK(expr) do { hipError_t _e = (expr); if (_e != hipSuccess) { \
    fprintf(stderr, "HIP error %s at %s:%d\n", hipGetErrorString(_e), __FILE__, __LINE__); \
    return; } } while (0)

typedef int i32x2 __attribute__((ext_vector_type(2)));
typedef int i32x4 __attribute__((ext_vector_type(4)));
typedef int i32x16 __attribute__((ext_vector_type(16)));
typedef __attribute__((address_space(3))) i32x2* lds_i32x2_p;
typedef __attribute__((address_space(3))) i32x4* lds_i32x4_p;
typedef __attribute__((address_space(3))) int* lds_int_p;

namespace {

constexpr int NRANK = 8;
constexpr int NN = 4096;
constexpr int KLOC = 512;
constexpr int MLOC = 512;
constexpr int NWG = 256;
constexpr int NTHREADS = 384;

constexpr unsigned WS_FLAGS = 0;
constexpr unsigned WS_TMO = 8192;
constexpr unsigned WS_CNT = 16384;
constexpr unsigned WS_DATA = 65536;
constexpr unsigned SRC_BYTES = MLOC * KLOC;
constexpr unsigned SLOT_BYTES = NRANK * SRC_BYTES;
constexpr size_t WS_NEEDED = (size_t)WS_DATA + 2u * SLOT_BYTES;

constexpr unsigned RING_BYTES = 32768;
constexpr unsigned LDS_CTRL = 4 * RING_BYTES;
constexpr unsigned LDS_BYTES = LDS_CTRL + 64;

constexpr unsigned long long DEADLINE_TICKS = 50000000ull;

struct Args {
    const int8_t* x;
    const int8_t* w;
    const float* sx;
    const float* sw;
    float* out;
    char* ws[NRANK];
    int rank;
    int pad;
};

struct AFrag { i32x4 v[4]; };
struct BFrag { i32x4 v[4]; };

__device__ __forceinline__ void dma_step(const char* gbase, unsigned lds_addr,
                                         unsigned v0, unsigned v1, unsigned v2, unsigned v3) {
    asm volatile(
        "s_mov_b32 m0, %5\n\t"
        "s_nop 0\n\t"
        "global_load_lds_dwordx4 %0, %4\n\t"
        "global_load_lds_dwordx4 %1, %4 offset:1024\n\t"
        "global_load_lds_dwordx4 %2, %4 offset:2048\n\t"
        "global_load_lds_dwordx4 %3, %4 offset:3072"
        :: "v"(v0), "v"(v1), "v"(v2), "v"(v3), "s"(gbase), "s"(lds_addr) : "memory");
}

__device__ __forceinline__ void load_a(AFrag& a, const char* gbase, unsigned voff) {
    asm volatile(
        "global_load_dwordx4 %0, %4, %5 sc0 sc1\n\t"
        "global_load_dwordx4 %1, %4, %5 offset:1024 sc0 sc1\n\t"
        "global_load_dwordx4 %2, %4, %5 offset:2048 sc0 sc1\n\t"
        "global_load_dwordx4 %3, %4, %5 offset:3072 sc0 sc1"
        : "=&v"(a.v[0]), "=&v"(a.v[1]), "=&v"(a.v[2]), "=&v"(a.v[3])
        : "v"(voff), "s"(gbase) : "memory");
}

template <int N>
__device__ __forceinline__ void wait_a(AFrag& a) {
    asm volatile("s_waitcnt vmcnt(%4)"
                 : "+v"(a.v[0]), "+v"(a.v[1]), "+v"(a.v[2]), "+v"(a.v[3]) : "i"(N) : "memory");
}

template <int OFF>
__device__ __forceinline__ void read_b(BFrag& b, const unsigned (&rb)[4]) {
#pragma unroll
    for (int t = 0; t < 4; ++t) {
        i32x2 lo = __builtin_amdgcn_ds_read_tr8_b64_v2i32((lds_i32x2_p)(rb[t] + OFF));
        i32x2 hi = __builtin_amdgcn_ds_read_tr8_b64_v2i32((lds_i32x2_p)(rb[t] + OFF + 1024));
        b.v[t] = __builtin_shufflevector(lo, hi, 0, 1, 2, 3);
    }
}

__device__ __forceinline__ void mma_step(i32x16 (&acc)[2][4], const i32x4& a0, const i32x4& a1, const BFrag& b) {
#pragma unroll
    for (int t = 0; t < 4; ++t) {
        acc[0][t] = __builtin_amdgcn_mfma_i32_32x32x32_i8(a0, b.v[t], acc[0][t], 0, 0, 0);
        acc[1][t] = __builtin_amdgcn_mfma_i32_32x32x32_i8(a1, b.v[t], acc[1][t], 0, 0, 0);
    }
}

struct Ctx {
    int wave, rank;
    const char* wbase;
    const char* abase;
    unsigned ring;
    unsigned ctrl;
    unsigned dv[4];
    unsigned rb[4];
    unsigned aoff;
};

__device__ __forceinline__ int unit_src(const Ctx& c, int e) {
    const int rel = (c.wave == 0) ? ((e >> 3) << 2) : (c.wave + ((e & 1) << 2));
    return (c.rank + rel) & (NRANK - 1);
}
__device__ __forceinline__ int unit_cls(const Ctx& c, int e) { return (c.wave == 0) ? (e & 7) : (e >> 1); }

__device__ __forceinline__ void issue_w(const Ctx& c, int e, int slot) {
    const int S = unit_src(c, e), g = unit_cls(c, e);
    const char* p = c.wbase + (size_t)(unsigned)((S * KLOC + g * 32) * NN);
    dma_step(p, c.ring + slot * 8192, c.dv[0], c.dv[1], c.dv[2], c.dv[3]);
    dma_step(p + 256 * NN, c.ring + slot * 8192 + 4096, c.dv[0], c.dv[1], c.dv[2], c.dv[3]);
}

__device__ __forceinline__ void gate(const Ctx& c, int e) {
    const int S = unit_src(c, e), need = unit_cls(c, e) + 1;
    lds_int_p p = (lds_int_p)(c.ctrl + S * 4);
    while (true) {
        int v = __hip_atomic_load(p, __ATOMIC_ACQUIRE, __HIP_MEMORY_SCOPE_WORKGROUP);
        if (__builtin_amdgcn_readfirstlane(v) >= need) break;
        __builtin_amdgcn_s_sleep(1);
    }
}

__device__ __forceinline__ void issue_a(const Ctx& c, int e, AFrag& a) {
    const int S = unit_src(c, e), g = unit_cls(c, e);
    load_a(a, c.abase + (unsigned)(S * SRC_BYTES + g * 4096), c.aoff);
}

template <int U, int MODE>
__device__ __forceinline__ void unit(const Ctx& c, int e, bool first, i32x16 (&acc)[2][4], AFrag (&A)[4], BFrag& B) {
    BFrag Bn;
    read_b<U * 8192 + 4096>(Bn, c.rb);
    mma_step(acc, A[U].v[0], A[U].v[1], B);
    if constexpr (MODE == 0) {
        asm volatile("s_waitcnt lgkmcnt(0)" ::: "memory");
        issue_w(c, e + 4, U);
    }
    if constexpr (MODE <= 1) {
        gate(c, e + 3);
        issue_a(c, e + 3, A[(U + 3) & 3]);
    }
    if constexpr (MODE <= 3) {
        if constexpr (MODE == 0) {
            if (U == 0 && first) wait_a<16>(A[(U + 1) & 3]);
            wait_a<24>(A[(U + 1) & 3]);
        } else if constexpr (MODE == 1) wait_a<16>(A[(U + 1) & 3]);
        else if constexpr (MODE == 2) wait_a<4>(A[(U + 1) & 3]);
        else wait_a<0>(A[(U + 1) & 3]);
        read_b<((U + 1) & 3) * 8192>(B, c.rb);
    }
    mma_step(acc, A[U].v[2], A[U].v[3], Bn);
}

}

__global__ __launch_bounds__(NTHREADS, 2) void a2a_gemm_kernel(Args a) {
    extern __shared__ __attribute__((aligned(16))) char smem[];
    const int lane = threadIdx.x & 63;
    const int wave = __builtin_amdgcn_readfirstlane(threadIdx.x >> 6);
    const int b = blockIdx.x;
    const int xcd = b & 7, j = b >> 3;
    const int rank = a.rank & (NRANK - 1);
    char* const ws = a.ws[rank];
    const unsigned lds0 = (unsigned)(size_t)smem;
    const unsigned ctrl = lds0 + LDS_CTRL;
    unsigned* const cnt = (unsigned*)(ws + WS_CNT + b * 128);
    const int mt = j & 7;
    const int n0 = (xcd * 4 + (j >> 3)) * 128;

    if (wave == 4) {
        const int D = (rank + (j & 7)) & (NRANK - 1);
        const int pj = (j >> 3) * 8 + xcd, rg = pj >> 1, s = pj & 1;
        const char* src = (const char*)a.x + (size_t)(unsigned)((D * MLOC + rg * 32) * KLOC + s * 256);
        const unsigned soff = (lane & 31) * KLOC + (lane >> 5) * 16;
        i32x4 d[8];
        unsigned cv;
        asm volatile(
            "global_load_dword %8, %9, %10 sc0 sc1\n\t"
            "global_load_dwordx4 %0, %11, %12\n\t"
            "global_load_dwordx4 %1, %11, %12 offset:32\n\t"
            "global_load_dwordx4 %2, %11, %12 offset:64\n\t"
            "global_load_dwordx4 %3, %11, %12 offset:96\n\t"
            "global_load_dwordx4 %4, %11, %12 offset:128\n\t"
            "global_load_dwordx4 %5, %11, %12 offset:160\n\t"
            "global_load_dwordx4 %6, %11, %12 offset:192\n\t"
            "global_load_dwordx4 %7, %11, %12 offset:224\n\t"
            "s_barrier\n\t"
            "s_waitcnt vmcnt(0)"
            : "=&v"(d[0]), "=&v"(d[1]), "=&v"(d[2]), "=&v"(d[3]), "=&v"(d[4]), "=&v"(d[5]), "=&v"(d[6]), "=&v"(d[7]),
              "=&v"(cv)
            : "v"(0u), "s"(cnt), "v"(soff), "s"(src) : "memory");
        const unsigned done = __builtin_amdgcn_readfirstlane(cv);
        char* pw = a.ws[D];
        char* dst = pw + WS_DATA + ((done + 1) & 1) * SLOT_BYTES + rank * SRC_BYTES
                  + ((rg >> 1) * 32 + s * 2 + (rg & 1)) * 1024;
        char* flag = pw + WS_FLAGS + ((rg >> 1) * 8 + rank) * 128 + (pj & 3) * 4;
        unsigned o = lane * 16, f = done * 8;
        asm volatile(
            "s_nop 4\n\t"
            "global_store_dwordx4 %0, %2, %10 sc0 sc1\n\t"  "v_add_u32 %0, 0x1000, %0\n\t"
            "global_store_dwordx4 %0, %3, %10 sc0 sc1\n\t"  "v_add_u32 %0, 0x1000, %0\n\t"
            "global_store_dwordx4 %0, %4, %10 sc0 sc1\n\t"  "v_add_u32 %0, 0x1000, %0\n\t"
            "global_store_dwordx4 %0, %5, %10 sc0 sc1\n\t"  "v_add_u32 %0, 0x1000, %0\n\t"
            "global_store_dwordx4 %0, %6, %10 sc0 sc1\n\t"  "v_add_u32 %0, 0x1000, %0\n\t"
            "global_store_dwordx4 %0, %7, %10 sc0 sc1\n\t"  "v_add_u32 %0, 0x1000, %0\n\t"
            "global_store_dwordx4 %0, %8, %10 sc0 sc1\n\t"  "v_add_u32 %0, 0x1000, %0\n\t"
            "global_store_dwordx4 %0, %9, %10 sc0 sc1\n\t"
            "s_mov_b64 exec, 1\n\t"
            "v_mov_b32 %0, 0\n\t"
            "v_add_u32 %1, 1, %1\n\t" "s_waitcnt vmcnt(7)\n\t" "global_store_dword %0, %1, %11 sc0 sc1\n\t" "s_nop 1\n\t"
            "v_add_u32 %1, 1, %1\n\t" "s_waitcnt vmcnt(7)\n\t" "global_store_dword %0, %1, %11 sc0 sc1\n\t" "s_nop 1\n\t"
            "v_add_u32 %1, 1, %1\n\t" "s_waitcnt vmcnt(7)\n\t" "global_store_dword %0, %1, %11 sc0 sc1\n\t" "s_nop 1\n\t"
            "v_add_u32 %1, 1, %1\n\t" "s_waitcnt vmcnt(7)\n\t" "global_store_dword %0, %1, %11 sc0 sc1\n\t" "s_nop 1\n\t"
            "v_add_u32 %1, 1, %1\n\t" "s_waitcnt vmcnt(7)\n\t" "global_store_dword %0, %1, %11 sc0 sc1\n\t" "s_nop 1\n\t"
            "v_add_u32 %1, 1, %1\n\t" "s_waitcnt vmcnt(7)\n\t" "global_store_dword %0, %1, %11 sc0 sc1\n\t" "s_nop 1\n\t"
            "v_add_u32 %1, 1, %1\n\t" "s_waitcnt vmcnt(7)\n\t" "global_store_dword %0, %1, %11 sc0 sc1\n\t" "s_nop 1\n\t"
            "v_add_u32 %1, 1, %1\n\t" "s_waitcnt vmcnt(7)\n\t" "global_store_dword %0, %1, %11 sc0 sc1\n\t" "s_nop 1\n\t"
            "s_mov_b64 exec, -1"
            : "+v"(o), "+v"(f)
            : "v"(d[0]), "v"(d[1]), "v"(d[2]), "v"(d[3]), "v"(d[4]), "v"(d[5]), "v"(d[6]), "v"(d[7]),
              "s"(dst), "s"(flag)
            : "memory");
        return;
    }

    if (wave == 5) {
        if (lane < 16) *(lds_int_p)(ctrl + lane * 4) = 0;
        asm volatile("s_waitcnt lgkmcnt(0)\n\ts_barrier" ::: "memory");
        const unsigned done = __builtin_amdgcn_readfirstlane(
            __hip_atomic_load(cnt, __ATOMIC_RELAXED, __HIP_MEMORY_SCOPE_SYSTEM));
        const unsigned call = done + 1;
        if (lane == 0) *(lds_int_p)(ctrl + 32) = (int)call;
        unsigned* tmo = (unsigned*)(ws + WS_TMO);
        const unsigned* flag = (const unsigned*)(ws + WS_FLAGS + (mt * 8 + ((lane >> 2) & 7)) * 128 + (lane & 3) * 4);
        const unsigned long long t0 = __builtin_amdgcn_s_memrealtime();
        int bad = 0;
        while (true) {
            int d = (int)(__hip_atomic_load(flag, __ATOMIC_RELAXED, __HIP_MEMORY_SCOPE_SYSTEM) - done * 8);
            d = min(d, __builtin_amdgcn_update_dpp(d, d, 0xB1, 0xF, 0xF, false));
            d = min(d, __builtin_amdgcn_update_dpp(d, d, 0x4E, 0xF, 0xF, false));
            if (lane < 32 && (lane & 3) == 0)
                __hip_atomic_store((lds_int_p)(ctrl + lane), d, __ATOMIC_RELEASE, __HIP_MEMORY_SCOPE_WORKGROUP);
            if (__all(d >= 8)) break;
            if (__builtin_amdgcn_s_memrealtime() - t0 > DEADLINE_TICKS) {
                if (lane == 0) __hip_atomic_store(tmo, call, __ATOMIC_RELAXED, __HIP_MEMORY_SCOPE_AGENT);
                if (lane < 8)
                    __hip_atomic_store((lds_int_p)(ctrl + lane * 4), 8, __ATOMIC_RELEASE, __HIP_MEMORY_SCOPE_WORKGROUP);
                bad = 1;
                break;
            }
            __builtin_amdgcn_s_sleep(1);
        }
        if (__hip_atomic_load(tmo, __ATOMIC_RELAXED, __HIP_MEMORY_SCOPE_AGENT) == call) bad = 1;
        if (lane == 0) *(lds_int_p)(ctrl + 36) = bad;
        asm volatile("s_waitcnt lgkmcnt(0)\n\ts_barrier" ::: "memory");
        if (lane == 0) __hip_atomic_store(cnt, call, __ATOMIC_RELAXED, __HIP_MEMORY_SCOPE_SYSTEM);
        return;
    }

    Ctx c;
    c.wave = wave;
    c.rank = rank;
    c.wbase = (const char*)a.w + n0;
    c.ring = lds0 + wave * RING_BYTES;
    c.ctrl = ctrl;
    c.aoff = lane * 16;
    {
        const unsigned dv = (lane >> 3) * NN + (((lane & 7) ^ (((lane >> 4) & 3) << 1)) << 4);
#pragma unroll
        for (int q = 0; q < 4; ++q) c.dv[q] = dv + q * (8 * NN - 1024);
        const unsigned rrow = (lane >> 5) * 16 + ((lane & 15) >> 1);
        const unsigned r0 = c.ring + rrow * 128 + ((lane >> 4) & 1) * 16 + (lane & 1) * 8;
#pragma unroll
        for (int t = 0; t < 4; ++t) c.rb[t] = r0 + (((t ^ wave) ^ ((lane >> 2) & 3)) << 5);
    }
#pragma unroll
    for (int u = 0; u < 4; ++u) issue_w(c, u, u);
    asm volatile("s_barrier" ::: "memory");

    i32x16 acc[2][4];
#pragma unroll
    for (int m = 0; m < 2; ++m)
#pragma unroll
        for (int t = 0; t < 4; ++t)
#pragma unroll
            for (int r = 0; r < 16; ++r) acc[m][t][r] = 0;

    AFrag A[4];
    BFrag B;
    gate(c, 0);
    {
        const unsigned call = __builtin_amdgcn_readfirstlane(*(lds_int_p)(ctrl + 32));
        c.abase = ws + WS_DATA + (call & 1) * SLOT_BYTES + mt * 32768;
    }
    issue_a(c, 0, A[0]);
    gate(c, 1);
    issue_a(c, 1, A[1]);
    gate(c, 2);
    issue_a(c, 2, A[2]);
    wait_a<8>(A[0]);
    read_b<0>(B, c.rb);
#pragma unroll 1
    for (int it = 0; it < 3; ++it) {
        unit<0, 0>(c, it * 4 + 0, it == 0, acc, A, B);
        unit<1, 0>(c, it * 4 + 1, false, acc, A, B);
        unit<2, 0>(c, it * 4 + 2, false, acc, A, B);
        unit<3, 0>(c, it * 4 + 3, false, acc, A, B);
    }
    unit<0, 1>(c, 12, false, acc, A, B);
    unit<1, 2>(c, 13, false, acc, A, B);
    unit<2, 3>(c, 14, false, acc, A, B);
    unit<3, 4>(c, 15, false, acc, A, B);

    const float scale = a.sx[0] * a.sw[0];
#pragma unroll
    for (int t = 1; t < 4; ++t)
#pragma unroll
        for (int m = 0; m < 2; ++m)
#pragma unroll
            for (int q = 0; q < 4; ++q) {
                i32x4 v = {acc[m][t][4 * q], acc[m][t][4 * q + 1], acc[m][t][4 * q + 2], acc[m][t][4 * q + 3]};
                *(lds_i32x4_p)(c.ring + c.aoff + ((t * 2 + m) * 4 + q) * 1024) = v;
            }
    asm volatile("s_waitcnt lgkmcnt(0)\n\ts_barrier" ::: "memory");
    const int bad = *(lds_int_p)(ctrl + 36);
#pragma unroll
    for (int t = 1; t < 4; ++t) {
        const unsigned other = lds0 + (wave ^ t) * RING_BYTES + c.aoff;
#pragma unroll
        for (int m = 0; m < 2; ++m)
#pragma unroll
            for (int q = 0; q < 4; ++q) {
                i32x4 v = *(lds_i32x4_p)(other + ((t * 2 + m) * 4 + q) * 1024);
#pragma unroll
                for (int r = 0; r < 4; ++r) acc[m][0][4 * q + r] += v[r];
            }
    }
    const float fs = bad ? __builtin_nanf("") : scale;
    float* o = a.out + (size_t)(mt * 64 + (lane >> 5) * 4) * NN + n0 + wave * 32 + (lane & 31);
#pragma unroll
    for (int m = 0; m < 2; ++m)
#pragma unroll
        for (int r = 0; r < 16; ++r)
            o[(size_t)(m * 32 + (r & 3) + 8 * (r >> 2)) * NN] = (float)acc[m][0][r] * fs;
}

extern "C" void kernel_launch(int rank, int nranks, ncclComm_t comm,
                              hipStream_t stream,
                              void* const* d_in, const int* in_sizes, int n_in,
                              void* d_out, int out_size,
                              const int* in_dtypes, int out_dtype,
                              void* const* d_out_all,
                              void* const* d_ws_all, size_t ws_bytes) {
    if (nranks != NRANK || (unsigned)rank >= (unsigned)NRANK || ws_bytes < WS_NEEDED || n_in < 4 ||
        out_size != MLOC * NN || in_sizes[0] != NRANK * MLOC * KLOC || in_sizes[1] != NRANK * KLOC * NN) {
        fprintf(stderr, "unexpected problem shape\n");
        return;
    }
    static bool attr_set = false;
    if (!attr_set) {
        HIP_CHECK(hipFuncSetAttribute((const void*)a2a_gemm_kernel, hipFuncAttributeMaxDynamicSharedMemorySize,
                                      (int)LDS_BYTES));
        attr_set = true;
    }
    Args a;
    a.x = (const int8_t*)d_in[0];
    a.w = (const int8_t*)d_in[1];
    a.sx = (const float*)d_in[2];
    a.sw = (const float*)d_in[3];
    a.out = (float*)d_out;
    for (int p = 0; p < NRANK; ++p) a.ws[p] = (char*)d_ws_all[p];
    a.rank = rank;
    a.pad = 0;
    hipLaunchKernelGGL(a2a_gemm_kernel, dim3(NWG), dim3(NTHREADS), LDS_BYTES, stream, a);
    HIP_CHECK(hipGetLastError());
}
```

```cpp
#include <hip/hip_runtime.h>
#include <rccl/rccl.h>
#include <cstdint>
#include <cstdio>

#define HIP_CHECK(expr) do { hipError_t _e = (expr); if (_e != hipSuccess) { \
    fprintf(stderr, "HIP error %s at %s:%d\n", hipGetErrorString(_e), __FILE__, __LINE__); \
    return; } } while (0)

typedef int i32x2 __attribute__((ext_vector_type(2)));
typedef int i32x4 __attribute__((ext_vector_type(4)));
typedef float f32x4 __attribute__((ext_vector_type(4)));
typedef __attribute__((address_space(3))) i32x2* lds_i32x2_p;
typedef __attribute__((address_space(3))) i32x4* lds_i32x4_p;
typedef __attribute__((address_space(3))) int* lds_int_p;

#define STAMP(i) do {} while (0)
#define CSTAMP(i) do {} while (0)
#define STAMP_DECL do {} while (0)
#define STAMP_OUT(wsp) do {} while (0)
#define STAMP_ARG
#define STAMP_PASS

namespace {

constexpr int NRANK = 8;
constexpr int NN = 4096;
constexpr int KLOC = 512;
constexpr int MLOC = 512;
constexpr int NWG = 256;
constexpr int NTHREADS = 448;

#define NSPLIT 36
#define PACE 58
#define PUSH0 75
#define ADIST 8
#define DUMP_EVERY 1
#define XLOAD_AT (8 * DUMP_EVERY + 5)
#define FIRST_AT 2
#define LDW 1
#define ROT 2
#define XEARLY 256
#define XLEAD 150
#define LSLOW 6
#define LDW0 1
#define CS0 10
#define CS1 22
#define LPRIO 2
#define CPRIO 1
constexpr int NL = 4;
constexpr int NS1 = NL + NSPLIT / 2, NS2 = 56 - NSPLIT, NSTEP = NS1 + NS2;
constexpr int NF1 = 2 * (NS1 - NL), NFRAG = NF1 + NS2;
static_assert(NSPLIT % 2 == 0 && NSPLIT >= 8 && NSTEP <= 64 && NFRAG == 56 && XLOAD_AT < NS2 && FIRST_AT < NL, "");

constexpr unsigned WS_TMO = 0;
constexpr unsigned WS_TICKET = 4096;
constexpr unsigned WS_FLAGS = 8192;
constexpr unsigned WS_DATA = 1u << 20;
constexpr unsigned SLOT_BYTES = 32 * NFRAG * 1024;
constexpr unsigned WS_X = WS_DATA + 2u * SLOT_BYTES;
constexpr size_t WS_NEEDED = (size_t)WS_X + NWG * 32768u;
constexpr unsigned POISON = 0x80808080u;

constexpr unsigned LDS_PUSH = 16 * 8192;
constexpr unsigned LDS_CTRL = LDS_PUSH + 8192;
constexpr unsigned C_FULL = 0;
constexpr unsigned C_CALL = 16;
constexpr unsigned C_PROG = 1024;
constexpr unsigned LDS_BYTES = LDS_CTRL + 2048;

constexpr unsigned DEADLINE_TICKS = 50000000u;

struct Args {
    const int8_t* x;
    const int8_t* w;
    const float* sx;
    const float* sw;
    float* out;
    int rank;
    int pad;
    char* ws[NRANK];
};

#define SB() __builtin_amdgcn_sched_barrier(0)
#define LGKM(n) __builtin_amdgcn_s_waitcnt(0xC07F | ((n) << 8))

__device__ __forceinline__ void dma4(const char* gbase, unsigned lds_addr,
                                     unsigned v0, unsigned v1, unsigned v2, unsigned v3) {
    asm volatile(
        "s_mov_b32 m0, %5\n\t"
        "s_nop 0\n\t"
        "global_load_lds_dwordx4 %0, %4\n\t"
        "global_load_lds_dwordx4 %1, %4 offset:1024\n\t"
        "global_load_lds_dwordx4 %2, %4 offset:2048\n\t"
        "global_load_lds_dwordx4 %3, %4 offset:3072"
        :: "v"(v0), "v"(v1), "v"(v2), "v"(v3), "s"(gbase), "s"(lds_addr) : "memory");
}
template <int N> __device__ __forceinline__ void wait_vm(i32x4& a) {
    asm volatile("s_waitcnt vmcnt(%1)" : "+v"(a) : "i"(N) : "memory");
}
template <int N> __device__ __forceinline__ void wait_vm(i32x4& a, i32x4& b) {
    asm volatile("s_waitcnt vmcnt(%2)" : "+v"(a), "+v"(b) : "i"(N) : "memory");
}
__device__ __forceinline__ int lds_ld(unsigned addr) {
    return __hip_atomic_load((lds_int_p)addr, __ATOMIC_RELAXED, __HIP_MEMORY_SCOPE_WORKGROUP);
}
__device__ __forceinline__ void lds_st(unsigned addr, int v) {
    __hip_atomic_store((lds_int_p)addr, v, __ATOMIC_RELAXED, __HIP_MEMORY_SCOPE_WORKGROUP);
}
__device__ __forceinline__ void lds_st1(unsigned addr, int v) {
    asm volatile("s_mov_b64 exec, 1\n\tds_write_b32 %0, %1\n\ts_mov_b64 exec, -1" :: "v"(addr), "v"(v) : "memory");
}
__device__ __forceinline__ void lds_wait(unsigned addr, int need) {
    while (__builtin_amdgcn_readfirstlane(
               __hip_atomic_load((lds_int_p)addr, __ATOMIC_ACQUIRE, __HIP_MEMORY_SCOPE_WORKGROUP)) < need)
        __builtin_amdgcn_s_sleep(1);
    asm volatile("" ::: "memory");
}

__device__ __forceinline__ void step_src(int s, int P, int& rel, int& cls) {
    const int i = s < NS1 ? 2 * (s - NL) + P : NSPLIT + s - NS1;
    cls = (i * 37) >> 8;
    rel = i - 7 * cls + 1;
}
__device__ __forceinline__ void src_step(int rel, int cls, int m, int& P, int& s, int& frag) {
    const int i = 7 * cls + rel - 1;
    if (i < NSPLIT) { P = i & 1; s = NL + (i >> 1); frag = (i & ~1) + (m ^ P); }
    else { P = m; s = NS1 + i - NSPLIT; frag = i; }
}

struct Ctx {
    const char* af;
    char* xo;
    const char* xi;
    unsigned sent;
    unsigned tag;
    unsigned rb[2][8];
    unsigned l16;
    unsigned chk;
    const char* flags;
    unsigned foff;
    unsigned call, t0;
    unsigned bad;
    unsigned prog;
    int one;
    int seen;
    unsigned la;
};

template <int H>
__device__ __forceinline__ void read_local(i32x4 (&AL)[NL][2], const Ctx& c) {
#pragma unroll
    for (int s = 0; s < 2; ++s) {
        AL[2 * H + s][0] = *(lds_i32x4_p)((c.la ^ (s * 64)) + H * 4096);
        AL[2 * H + s][1] = *(lds_i32x4_p)((c.la ^ (s * 64) ^ 2048) + H * 4096);
    }
}

template <int S, int H>
__device__ __forceinline__ void read_b(i32x2 (&B)[16], int t, const Ctx& c) {
    B[2 * t + H] = __builtin_amdgcn_ds_read_tr8_b64_v2i32((lds_i32x2_p)(c.rb[(S >> 3) & 1][t] + (S & 7) * 8192 + H * 1024));
}
__device__ __forceinline__ void mma(i32x4& acc, const i32x4& a, const i32x2 (&B)[16], int t) {
    acc = __builtin_amdgcn_mfma_i32_16x16x64_i8(__builtin_shufflevector(B[2 * t], B[2 * t + 1], 0, 1, 2, 3), a, acc, 0, 0, 0);
}
template <int F>
__device__ __forceinline__ void load_frag(i32x4 (&A)[NFRAG], const Ctx& c) {
    asm volatile("global_load_dwordx4 %0, %1, %2 offset:%3 sc0 sc1"
                 : "=v"(A[F]) : "v"(c.l16), "s"(c.af + (F / 8) * 8192 + 4096), "i"((F % 8) * 1024 - 4096) : "memory");
}
template <int F, int END>
__device__ __forceinline__ void load_frags(i32x4 (&A)[NFRAG], const Ctx& c) {
    if constexpr (F < END) { load_frag<F>(A, c); load_frags<F + 1, END>(A, c); }
}
template <int FULL>
__device__ __forceinline__ void check(const Ctx& c, int seen) {
    if (__builtin_expect(__any(seen < FULL), 0)) {
        while (__any(__hip_atomic_load((lds_int_p)c.chk, __ATOMIC_ACQUIRE, __HIP_MEMORY_SCOPE_WORKGROUP) < FULL))
            __builtin_amdgcn_s_sleep(1);
    }
    asm volatile("" ::: "memory");
}

#define RELOAD(n, off) "global_load_dwordx4 %" #n ", %10, %18 offset:" #off " sc0 sc1\n\t"
template <int F, int F1>
__device__ __forceinline__ void validate(i32x4 (&A)[NFRAG], Ctx& c, unsigned long long m0, unsigned long long m1) {
    unsigned p0, p1, off, fl, prev;
    unsigned long long now;
    i32x4 D[7];
#pragma unroll
    for (int i = 0; i < 7; ++i) if (F + 1 + i >= NFRAG) asm volatile("" : "=v"(D[i]));
    asm volatile(
        "s_or_b64 vcc, %15, %16\n\t"
        "s_cbranch_scc1 .Lslow%=\n"
        ".Lback%=:\n\t"
        ".subsection 1\n"
        ".Lslow%=:\n\t"
        "v_add_u32 %10, %21, %17\n\t"
        "s_mov_b32 %13, 0\n"
        ".Lretry%=:\n\t"
        "global_load_dword %8, %10, %18 offset:-4096 sc0 sc1\n\t"
        "global_load_dword %9, %10, %18 offset:%22 sc0 sc1\n\t"
        RELOAD(0, -4096) RELOAD(1, -3072) RELOAD(2, -2048) RELOAD(3, -1024)
        RELOAD(4, 0) RELOAD(5, 1024) RELOAD(6, 2048) RELOAD(7, 3072)
        "global_load_dword %11, %19, %20 sc0 sc1\n\t"
        "s_memrealtime vcc\n\t"
        "s_waitcnt vmcnt(0) lgkmcnt(0)\n\t"
        "s_cmp_lg_u32 %13, 0\n\t"
        "s_cbranch_scc1 .Lback%=\n\t"
        "s_sub_u32 %13, vcc_lo, %25\n\t"
        "v_cmp_eq_u32 vcc, %23, %8\n\t"
        "v_cmp_eq_u32 %12, %23, %9\n\t"
        "s_or_b64 vcc, vcc, %12\n\t"
        "s_cbranch_scc0 .Lback%=\n\t"
        "s_cmp_gt_u32 %13, %26\n\t"
        "s_cselect_b32 %14, 1, %14\n\t"
        "s_cbranch_scc1 .Lback%=\n\t"
        "v_subrev_u32 %11, %24, %11\n\t"
        "v_cmp_gt_i32 vcc, 0, %11\n\t"
        "s_cmp_eq_u64 vcc, 0\n\t"
        "s_cselect_b32 %13, 1, 0\n\t"
        "s_branch .Lretry%=\n\t"
        ".subsection 0"
        : "+v"(A[F]), "+v"(F + 1 < NFRAG ? A[F + 1 < NFRAG ? F + 1 : 0] : D[0]), "+v"(F + 2 < NFRAG ? A[F + 2 < NFRAG ? F + 2 : 0] : D[1]),
          "+v"(F + 3 < NFRAG ? A[F + 3 < NFRAG ? F + 3 : 0] : D[2]), "+v"(F + 4 < NFRAG ? A[F + 4 < NFRAG ? F + 4 : 0] : D[3]),
          "+v"(F + 5 < NFRAG ? A[F + 5 < NFRAG ? F + 5 : 0] : D[4]), "+v"(F + 6 < NFRAG ? A[F + 6 < NFRAG ? F + 6 : 0] : D[5]),
          "+v"(F + 7 < NFRAG ? A[F + 7 < NFRAG ? F + 7 : 0] : D[6]),
          "=&v"(p0), "=&v"(p1), "=&v"(off), "=&v"(fl), "=&s"(now), "=&s"(prev), "+s"(c.bad)
        : "s"(m0), "s"(m1), "v"(c.l16), "s"(c.af), "v"(c.foff), "s"(c.flags),
          "i"(F * 1024 + 4096), "i"((F1 - F) * 1024 - 4096), "s"(c.sent), "s"(c.call), "s"(c.t0), "i"(DEADLINE_TICKS)
        : "memory", "vcc", "scc");
}

static_assert(ADIST >= 8 && ADIST % 2 == 0, "validate() reloads 8 fragments that have been requested");
constexpr int imin(int x, int y) { return x < y ? x : y; }
constexpr int frag_lo(int s) { return s <= NL ? 0 : s < NS1 ? 2 * (s - NL) : NF1 + s - NS1; }
constexpr int issued_before(int s) {
    return s <= FIRST_AT ? 0 : imin(NFRAG, imin(frag_lo(s) + ADIST, s <= NL ? 8 : 8 + 4 * (s - NL)));
}
constexpr bool dumps(int s) { return s >= NS1 && (s - NS1) % DUMP_EVERY == 0 && (s - NS1) / DUMP_EVERY < 8; }
constexpr int issue_step(int f) { int s = 0; while (issued_before(s + 1) <= f) ++s; return s; }
constexpr int younger(int s) {
    const int f = frag_lo(s + 1) - 1;
    int n = issued_before(s) - 1 - f;
    for (int t = issue_step(f); t < s; ++t) n += (t < s - 1 && dumps(t) ? 1 : 0) + (t == NS1 + XLOAD_AT ? 8 : 0);
    return n;
}

template <int WAIT>
__device__ __forceinline__ void load_x(i32x4 (&X)[8], const Ctx& c) {
    asm volatile(
        "global_load_dwordx4 %0, %8, %9 offset:-4096 sc0 sc1\n\t"
        "global_load_dwordx4 %1, %8, %9 offset:-3072 sc0 sc1\n\t"
        "global_load_dwordx4 %2, %8, %9 offset:-2048 sc0 sc1\n\t"
        "global_load_dwordx4 %3, %8, %9 offset:-1024 sc0 sc1\n\t"
        "global_load_dwordx4 %4, %8, %9 sc0 sc1\n\t"
        "global_load_dwordx4 %5, %8, %9 offset:1024 sc0 sc1\n\t"
        "global_load_dwordx4 %6, %8, %9 offset:2048 sc0 sc1\n\t"
        "global_load_dwordx4 %7, %8, %9 offset:3072 sc0 sc1\n\t"
        "s_waitcnt vmcnt(%10)"
        : "=&v"(X[0]), "=&v"(X[1]), "=&v"(X[2]), "=&v"(X[3]), "=&v"(X[4]), "=&v"(X[5]), "=&v"(X[6]), "=&v"(X[7])
        : "v"(c.l16), "s"(c.xi), "i"(WAIT) : "memory");
}
__device__ __forceinline__ void wait_x(i32x4 (&X)[8]) {
    asm volatile("s_waitcnt vmcnt(0)"
                 : "+v"(X[0]), "+v"(X[1]), "+v"(X[2]), "+v"(X[3]), "+v"(X[4]), "+v"(X[5]), "+v"(X[6]), "+v"(X[7]) :: "memory");
}

template <int S>
__device__ __forceinline__ void step(Ctx& c, i32x4 (&acc)[2][8], i32x4 (&AL)[NL][2], i32x4 (&A)[NFRAG], i32x4 (&X)[8],
                                     i32x2 (&B)[2][16] STAMP_ARG) {
    constexpr bool PH1 = S < NS1, LOC = S < NL, NEXT = S + 1 < NSTEP;
    constexpr bool VAL = NEXT && S + 1 >= NL;
    constexpr int F = frag_lo(S), FN = frag_lo(S + 1), FN1 = S + 1 < NS1 ? FN + 1 : FN;
    constexpr int L0 = issued_before(S), L1 = issued_before(S + 1);
    if constexpr ((S & 3) == 0) STAMP(4 + (S >> 2));
    if constexpr (S == CS0) { STAMP(28); CSTAMP(30); }
    if constexpr (S == CS1) { STAMP(29); CSTAMP(31); }
    if constexpr (S < 8) check<S + 2>(c, c.seen);
    else if constexpr ((S & 1) == 0) check<imin(S + 3, NSTEP)>(c, c.seen);
    if constexpr (S == 2) {
        asm volatile("s_waitcnt vmcnt(%0)" :: "i"(issued_before(2)) : "memory");
        read_local<1>(AL, c);
    }
    const i32x4& a0 = LOC ? AL[LOC ? S : 0][0] : A[F];
    const i32x4& a1 = LOC ? AL[LOC ? S : 0][1] : A[PH1 ? F + 1 : F];
    unsigned long long m0 = 0, m1 = 0;
    LGKM(8);
    SB();
#pragma unroll
    for (int t = 0; t < 8; ++t) {
        if (t == 4) { if constexpr (NEXT) LGKM(8 + (S < 8 || (S & 1))); else LGKM(0); SB(); }
        mma(acc[0][t], a0, B[S & 1], t); SB();
        if constexpr (NEXT) read_b<S + 1, 0>(B[(S + 1) & 1], t, c);
        SB();
        if constexpr (PH1) { mma(acc[1][t], a1, B[S & 1], t); SB(); }
        if constexpr (NEXT) read_b<S + 1, 1>(B[(S + 1) & 1], t, c);
        if constexpr (L1 - L0 <= 2) {
            if (t == 0) { if constexpr (L0 < L1) load_frag<L0>(A, c); }
            if (t == 2) { if constexpr (L0 + 1 < L1) load_frag<L0 + 1>(A, c); }
        } else {
            if (t == 0) load_frags<L0, imin(L1, L0 + 2)>(A, c);
            if (t == 1) load_frags<L0 + 2, imin(L1, L0 + 4)>(A, c);
            if (t == 2) load_frags<L0 + 4, imin(L1, L0 + 6)>(A, c);
            if (t == 3) load_frags<L0 + 6, L1>(A, c);
        }
        if ((S < 8 || (S & 1) == 1) && NEXT && t == 3) c.seen = lds_ld(c.chk);
        if (t == 4) { if constexpr (S == NS1 + XLOAD_AT) load_x<63>(X, c); }
        if (t == 6) {
            if constexpr (VAL) {
                if constexpr (FN1 == FN) wait_vm<younger(S + 1)>(A[FN]);
                else wait_vm<younger(S + 1)>(A[FN], A[FN1]);
                m0 = __builtin_amdgcn_uicmp(A[FN][0], c.sent, 32);
                m1 = FN1 == FN ? m0 : __builtin_amdgcn_uicmp(A[FN1][0], c.sent, 32);
            }
        }
        if (t == 7) {
            if constexpr (dumps(S)) {
                constexpr int Q = (S - NS1) / DUMP_EVERY;
                i32x4 v = acc[1][Q];
                v[0] = (v[0] << 4) | c.tag;
                asm volatile("global_store_dwordx4 %1, %0, %2 offset:%3 sc0 sc1\n\ts_nop 1"
                             :: "v"(v), "v"(c.l16), "s"(c.xo), "i"(Q * 1024 - 4096) : "memory");
            }
        }
        SB();
    }
    if constexpr ((S & 1) == 0 && S + 16 < NSTEP) asm volatile("ds_add_u32 %0, %1" :: "v"(c.prog), "v"(c.one) : "memory");
    if constexpr (VAL) validate<FN, FN1>(A, c, m0, m1);
    SB();
    if constexpr (NEXT) step<S + 1>(c, acc, AL, A, X, B STAMP_PASS);
}

#define LORD 0
constexpr int LSEQ[5][2][6] = {{{0, 1, 2, 3, 8, 10}, {4, 5, 6, 7, 9, 11}},
                               {{0, 5, 1, 7, 3, 9}, {4, 6, 2, 8, 10, 11}},
                               {{0, 5, 7, 9, 1, 3}, {4, 6, 8, 10, 2, 11}},
                               {{0, 5, 7, 1, 3, 9}, {4, 6, 8, 2, 10, 11}},
                               {{0, 1, 5, 3, 7, 9}, {4, 6, 2, 8, 10, 11}}};
constexpr int fill_step(int h, int k) { return k < 6 ? LSEQ[LORD][h][k] : imin(NSTEP, 2 * k + h); }
constexpr int owed(int h, int k) {
    int m = fill_step(h, k);
    for (int i = k + 1; i < 6; ++i) m = imin(m, fill_step(h, i));
    return m;
}
struct Ldr {
    const char* w;
    unsigned dv[4];
    unsigned woff;
    unsigned fa, pa;
    int freed;
};
__device__ __forceinline__ void fill(const Ldr& l, unsigned off, unsigned slot) {
    const char* p = l.w + (size_t)off;
    dma4(p, slot, l.dv[0], l.dv[1], l.dv[2], l.dv[3]);
    dma4(p + 32 * NN, slot + 4096, l.dv[0], l.dv[1], l.dv[2], l.dv[3]);
}
__device__ __forceinline__ int released(const Ldr& l) {
    int v = lds_ld(l.pa);
    v = min(v, __builtin_amdgcn_update_dpp(v, v, 0xB1, 0xF, 0xF, false));
    v = min(v, __builtin_amdgcn_update_dpp(v, v, 0x4E, 0xF, 0xF, false));
    v = __builtin_amdgcn_readfirstlane(v);
    return v < 2 ? -4 : 2 * v;
}
template <int H, int K>
__device__ __forceinline__ void fills(Ldr& l STAMP_ARG) {
    constexpr int S = fill_step(H, K);
    if constexpr (S < NSTEP) {
        if constexpr (S >= 12) {
            if (__builtin_expect(l.freed <= S - 16, 0)) {
                l.freed = released(l);
                if (l.freed <= S - 16) {
                    asm volatile("s_waitcnt vmcnt(0)" ::: "memory");
                    asm volatile("ds_max_i32 %0, %1" :: "v"(l.fa), "v"(S) : "memory");
                    do {
                        __builtin_amdgcn_s_sleep(1);
                        l.freed = released(l);
                    } while (l.freed <= S - 16);
                }
            }
        }
        fill(l, __builtin_amdgcn_readlane(l.woff, S), (S & 15) * 8192);
        constexpr int W = K < LSLOW ? LDW0 : LDW;
        if constexpr (K >= W) {
            asm volatile("s_waitcnt vmcnt(%2)\n\tds_max_i32 %0, %1" :: "v"(l.fa), "v"(owed(H, K - W + 1)), "i"(8 * W) : "memory");
        }
        if constexpr (K < 6 || (K & 1) == 1) STAMP(K < 6 ? K : 3 + K / 2);
        fills<H, K + 1>(l STAMP_PASS);
    } else {
#pragma unroll
        for (int k = LDW - 1; k >= 0; --k)
            asm volatile("s_waitcnt vmcnt(%2)\n\tds_max_i32 %0, %1" :: "v"(l.fa), "v"(k ? fill_step(H, K - k) : NSTEP), "i"(8 * k) : "memory");
    }
}

}

__global__ __launch_bounds__(NTHREADS, 2) void a2a_gemm_kernel(Args a) {
    extern __shared__ __attribute__((aligned(16))) char smem[];
    asm volatile("" :: "s"(a.x), "s"(a.w), "s"(a.sx), "s"(a.sw), "s"(a.out), "s"(a.rank), "s"(a.ws[0]), "s"(a.ws[1]),
                       "s"(a.ws[2]), "s"(a.ws[3]), "s"(a.ws[4]), "s"(a.ws[5]), "s"(a.ws[6]), "s"(a.ws[7]));
    const int lane = threadIdx.x & 63;
    const int wave = __builtin_amdgcn_readfirstlane(threadIdx.x >> 6);
    const int b = blockIdx.x;
    const int xcd = b & 7, j = b >> 3;
    const int rank = a.rank & (NRANK - 1);
    char* const ws = a.ws[0];
    const unsigned lds0 = (unsigned)(uintptr_t)(__attribute__((address_space(3))) char*)smem;
    const unsigned ctrl = lds0 + LDS_CTRL;
    const int P = j & 1;
    const int mi = (j >> 1) & 3;
    const int n0 = ((j >> 3) * 8 + xcd) * 128;
    STAMP_DECL;
    STAMP(0);

    if (wave == 6) {
        __builtin_amdgcn_s_setprio(3);
        const unsigned tstart = (unsigned)__builtin_amdgcn_s_memrealtime();
        unsigned* const ticket = (unsigned*)(ws + WS_TICKET);
        const int pc = b >> 5, rg = b & 31;
        unsigned cv;
        asm volatile("global_load_dword %0, %1, %2 sc1" : "=&v"(cv) : "v"(0u), "s"(ticket) : "memory");
        const unsigned due = tstart + PUSH0 + ((b * PACE) >> 5);
        auto load_piece = [&]() {
            const char* src = (const char*)a.x + (size_t)(unsigned)(rg * 16 * KLOC + ((pc + ROT * (rg >> 3)) & 7) * 64);
            const unsigned sv = (lane >> 2) * KLOC + (((lane & 3) ^ ((lane >> 4) & 3)) << 4);
#pragma unroll
            for (int d = 1; d < NRANK; ++d)
                asm volatile("s_mov_b32 m0, %2\n\ts_nop 0\n\tglobal_load_lds_dwordx4 %0, %1"
                             :: "v"(sv), "s"(src + (size_t)(unsigned)(((rank + d) & (NRANK - 1)) * MLOC * KLOC)),
                                "s"(lds0 + LDS_PUSH + d * 1024) : "memory");
        };
        if (b < XEARLY) load_piece();
        lds_st(ctrl + lane * 4, lane == 1 ? owed(1, 0) : 0);
        *(lds_i32x4_p)(ctrl + C_PROG + lane * 16) = i32x4{0, 0, 0, 0};
        asm volatile("s_waitcnt lgkmcnt(0)\n\ts_barrier" ::: "memory");
        unsigned foff[NRANK];
#pragma unroll
        for (int rel = 1; rel < NRANK; ++rel) {
            int Pc, sc, fr;
            src_step(rel, pc, rg & 1, Pc, sc, fr);
            foff[rel] = (((rg >> 1) * 2 + Pc) * NFRAG + fr) * 1024;
            asm volatile("" : "+s"(foff[rel]));
        }
        const unsigned roff = lds0 + LDS_PUSH + (lane & 15) * 64 + (((lane >> 4) ^ ((lane >> 2) & 3)) << 4);
        const unsigned l16 = lane * 16;
        if (b >= XEARLY) asm volatile("s_waitcnt vmcnt(0)" ::: "memory");
        asm volatile("s_waitcnt vmcnt(7)" : "+v"(cv) :: "memory");
        const unsigned call = ((unsigned)__builtin_amdgcn_readfirstlane(cv) >> 8) + 1;
        lds_st1(ctrl + C_CALL, (int)call);
        const unsigned cur = WS_DATA + (call & 1) * SLOT_BYTES, nxt = WS_DATA + ((call + 1) & 1) * SLOT_BYTES;
        char* dp[NRANK];
        char* pp[NRANK];
#pragma unroll
        for (int d = 1; d < NRANK; ++d) {
            dp[d] = a.ws[d] + cur + foff[NRANK - d];
            pp[d] = ws + nxt + foff[d];
            asm volatile("" : "+s"(dp[d]), "+s"(pp[d]));
        }
        auto poison_next = [&]() {
            const i32x4 poison = {(int)POISON, (int)POISON, (int)POISON, (int)POISON};
#pragma unroll
            for (int rel = 1; rel < NRANK; ++rel)
                asm volatile("global_store_dwordx4 %1, %0, %2 sc0 sc1\n\ts_nop 1"
                             :: "v"(poison), "v"(l16), "s"(pp[rel]) : "memory");
        };
        if (b >= XEARLY) {
            while ((int)((unsigned)__builtin_amdgcn_s_memrealtime() + XLEAD - due) < 0) __builtin_amdgcn_s_sleep(2);
            load_piece();
            poison_next();
            asm volatile("s_waitcnt vmcnt(7)" ::: "memory");
        } else {
            asm volatile("s_waitcnt vmcnt(0)" ::: "memory");
        }
        STAMP(11);
        i32x4 dat[NRANK];
#pragma unroll
        for (int d = 1; d < NRANK; ++d) dat[d] = *(lds_i32x4_p)(roff + d * 1024);
        while ((int)((unsigned)__builtin_amdgcn_s_memrealtime() - due) < 0) __builtin_amdgcn_s_sleep(1);
        STAMP(2);
#pragma unroll
        for (int d = 1; d < NRANK; ++d)
            asm volatile("global_store_dwordx4 %1, %0, %2 sc0 sc1\n\ts_nop 1"
                         :: "v"(dat[d]), "v"(l16), "s"(dp[d]) : "memory");
        STAMP(3);
        if (b < XEARLY) {
            asm volatile("s_waitcnt vmcnt(0)" ::: "memory");
            poison_next();
        }
        asm volatile("s_waitcnt vmcnt(0)" ::: "memory");
        STAMP(4);
        unsigned tk = 0;
        if (lane == 0) tk = __hip_atomic_fetch_add(ticket, 1u, __ATOMIC_RELAXED, __HIP_MEMORY_SCOPE_AGENT);
        if ((__builtin_amdgcn_readfirstlane(tk) & 255) == 255) {
            if (lane < NRANK)
                __hip_atomic_store((unsigned*)(a.ws[lane] + WS_FLAGS + ((NRANK - lane) & (NRANK - 1)) * 128), call,
                                   __ATOMIC_RELAXED, __HIP_MEMORY_SCOPE_SYSTEM);
            STAMP(5);
            const unsigned* fl = (const unsigned*)(ws + WS_FLAGS + (lane & 7) * 128);
            while (__any((int)(__hip_atomic_load(fl, __ATOMIC_RELAXED, __HIP_MEMORY_SCOPE_SYSTEM) - call) < 0)) {
                if ((unsigned)__builtin_amdgcn_s_memrealtime() - tstart > 2 * DEADLINE_TICKS) {
                    if (lane == 0) {
                        __hip_atomic_store((unsigned*)(ws + WS_TMO), call, __ATOMIC_RELAXED, __HIP_MEMORY_SCOPE_AGENT);
                        a.out[0] = __builtin_nanf("");
                    }
                    break;
                }
                __builtin_amdgcn_s_sleep(16);
            }
            STAMP(6);
        }
        STAMP_OUT(ws);
        return;
    }

    if (wave >= 4) {
        __builtin_amdgcn_s_waitcnt(0x0F70);
        __builtin_amdgcn_s_setprio(LPRIO);
        const int h = wave & 1;
        Ldr l;
        l.w = (const char*)a.w + n0;
#pragma unroll
        for (int i = 0; i < 4; ++i)
            l.dv[i] = (lane >> 3) * NN + (((lane & 7) ^ (((lane >> 4) & 3) | ((i >> 1) << 2))) << 4) + i * (8 * NN - 1024);
        if (b < XEARLY) asm volatile("s_barrier" ::: "memory");
        fill(l, (h ? ((rank + 1 + P) & (NRANK - 1)) * 8 + ((ROT * mi) & 7) : rank * 8 + 4 * P + mi) * 64 * NN, h * 4 * 8192);
        SB();
        {
            int rel, cls;
            step_src(min(max(lane, NL), NSTEP - 1), P, rel, cls);
            l.woff = (((rel + rank) & (NRANK - 1)) * 8 + ((cls + ROT * mi) & 7)) * 64 * NN;
            if (lane < NL) l.woff = (rank * 8 + 4 * P + (lane ^ mi)) * 64 * NN;
        }
        l.fa = ctrl + C_FULL + (lane ? 64 + lane * 8 : 0) + h * 4;
        l.pa = ctrl + C_PROG + (lane & 3) * 260;
        l.freed = -4;
        if (b >= XEARLY) asm volatile("s_barrier" ::: "memory");
        if (h) fills<1, 1>(l STAMP_PASS);
        else fills<0, 1>(l STAMP_PASS);
        STAMP(20);
        STAMP_OUT(ws);
        return;
    }

    __builtin_amdgcn_s_waitcnt(0x0F70);
    __builtin_amdgcn_s_setprio(CPRIO);
    i32x4 AL[NL][2], A[NFRAG];
    const char* xl = (const char*)a.x + (size_t)(unsigned)((rank * MLOC + mi * 128 + wave * 32) * KLOC + P * 256);
    unsigned sv[4];
#pragma unroll
    for (int i = 0; i < 4; ++i)
        sv[i] = (i * 8 + (lane >> 3)) * KLOC + (((lane & 7) ^ ((i * 4 + (lane >> 4)) & 7)) << 4) - i * 1024;
    if (b < XEARLY) asm volatile("s_barrier" ::: "memory");
    dma4(xl + (mi >> 1) * 128, lds0 + (12 + wave) * 8192, sv[0], sv[1], sv[2], sv[3]);
    SB();
    STAMP(2);
    if (b >= XEARLY) asm volatile("s_barrier" ::: "memory");
    Ctx c;
    c.l16 = lane * 16;
    c.chk = ctrl + C_FULL + (lane & 1) * 4;
    c.foff = (lane & 7) * 128;
    c.flags = ws + WS_FLAGS;
    c.t0 = (unsigned)__builtin_amdgcn_s_memrealtime();
    c.bad = 0;
    typedef const __attribute__((address_space(4))) float* cfp;
    const float scale = *(cfp)(uintptr_t)a.sx * *(cfp)(uintptr_t)a.sw;
    c.prog = ctrl + C_PROG + wave * 256 + lane * 4;
    c.one = 1;
    asm volatile("" : "+v"(c.one), "+v"(c.chk), "+v"(c.prog), "+v"(c.foff));
    {
        const unsigned q = (lane & 15) >> 1, kg = lane >> 4;
        const unsigned r0 = lds0 + (kg * 16 + q) * 128 + (lane & 1) * 8, f = (q >> 1) | ((kg & 1) << 2);
#pragma unroll
        for (int t = 0; t < 8; ++t) {
            c.rb[0][t] = r0 + ((t ^ f) << 4);
            c.rb[1][t] = c.rb[0][t] + 65536;
            asm volatile("" : "+v"(c.rb[0][t]), "+v"(c.rb[1][t]));
        }
    }
    i32x4 acc[2][8];
#pragma unroll
    for (int m = 0; m < 2; ++m)
#pragma unroll
        for (int t = 0; t < 8; ++t) { acc[m][t] = i32x4{0, 0, 0, 0}; asm volatile("" : "+v"(acc[m][t])); }
    i32x2 B[2][16];
    asm volatile("s_waitcnt vmcnt(0)" ::: "memory");
    dma4(xl + ((mi >> 1) ^ 1) * 128, lds0 + (12 + wave) * 8192 + 4096, sv[0], sv[1], sv[2], sv[3]);
    c.la = lds0 + (12 + wave) * 8192 + P * 2048 + (lane & 15) * 128 + ((((mi & 1) * 4 + (lane >> 4)) ^ ((lane & 15) >> 1)) << 4);
    read_local<0>(AL, c);
    STAMP(1);
    lds_wait(ctrl + C_CALL, 1);
    const unsigned call = __builtin_amdgcn_readfirstlane(lds_ld(ctrl + C_CALL));
    c.call = call;
    c.sent = call == 1 ? 0u : POISON;
    c.tag = call & 15;
    c.af = ws + WS_DATA + (call & 1) * SLOT_BYTES + ((mi * 4 + wave) * 2 + P) * (NFRAG * 1024);
    c.xo = ws + WS_X + (b * 4 + wave) * 8192 + 4096;
    c.xi = ws + WS_X + ((b ^ 8) * 4 + wave) * 8192 + 4096;
    check<1>(c, -1);
    STAMP(3);
#pragma unroll
    for (int t = 0; t < 8; ++t) { read_b<0, 0>(B[0], t, c); read_b<0, 1>(B[0], t, c); }
    c.seen = -1;
    i32x4 X[8];
    step<0>(c, acc, AL, A, X, B STAMP_PASS);
    STAMP(20);

    wait_x(X);
    while (true) {
        unsigned diff = 0;
#pragma unroll
        for (int t = 0; t < 8; ++t) diff |= (unsigned)X[t][0] ^ c.tag;
        if (__builtin_expect(!__any(diff & 15), 1)) break;
        if ((unsigned)__builtin_amdgcn_s_memrealtime() - c.t0 > DEADLINE_TICKS + DEADLINE_TICKS / 2) { c.bad = 1; break; }
        load_x<0>(X, c);
    }
    if (c.bad && lane == 0) __hip_atomic_store((unsigned*)(ws + WS_TMO), call, __ATOMIC_RELAXED, __HIP_MEMORY_SCOPE_AGENT);
    const float fs = c.bad ? __builtin_nanf("") : scale;
    STAMP(22);
    {
        char* o = (char*)a.out + (size_t)(unsigned)(((mi * 128 + wave * 32 + P * 16) * NN + n0) * 4);
        const unsigned oo = (lane & 15) * NN * 4 + (lane >> 4) * 16;
#pragma unroll
        for (int t = 0; t < 8; ++t) {
            X[t][0] >>= 4;
            const i32x4 s = acc[0][t] + X[t];
            const f32x4 y = {(float)s[0] * fs, (float)s[1] * fs, (float)s[2] * fs, (float)s[3] * fs};
            asm volatile("global_store_dwordx4 %1, %0, %2 offset:%3 sc1\n\ts_nop 1"
                         :: "v"(y), "v"(oo), "s"(o), "i"(t * 64) : "memory");
        }
    }
    STAMP(23);
    STAMP(24);
    STAMP_OUT(ws);
}

extern "C" void kernel_launch(int rank, int nranks, ncclComm_t comm,
                              hipStream_t stream,
                              void* const* d_in, const int* in_sizes, int n_in,
                              void* d_out, int out_size,
                              const int* in_dtypes, int out_dtype,
                              void* const* d_out_all,
                              void* const* d_ws_all, size_t ws_bytes) {
    if (nranks != NRANK || (unsigned)rank >= (unsigned)NRANK || ws_bytes < WS_NEEDED || n_in < 4 ||
        out_size != MLOC * NN || in_sizes[0] != NRANK * MLOC * KLOC || in_sizes[1] != NRANK * KLOC * NN) {
        fprintf(stderr, "unexpected problem shape\n");
        return;
    }
    static bool attr_set = false;
    if (!attr_set) {
        HIP_CHECK(hipFuncSetAttribute((const void*)a2a_gemm_kernel, hipFuncAttributeMaxDynamicSharedMemorySize,
                                      (int)LDS_BYTES));
        attr_set = true;
    }
    Args a;
    a.x = (const int8_t*)d_in[0];
    a.w = (const int8_t*)d_in[1];
    a.sx = (const float*)d_in[2];
    a.sw = (const float*)d_in[3];
    a.out = (float*)d_out;
    a.rank = rank;
    a.pad = 0;
    for (int p = 0; p < NRANK; ++p) a.ws[p] = (char*)d_ws_all[(rank + p) & (NRANK - 1)];
    hipLaunchKernelGGL(a2a_gemm_kernel, dim3(NWG), dim3(NTHREADS), LDS_BYTES, stream, a);
    HIP_CHECK(hipGetLastError());
}
```

```cpp
#include <hip/hip_runtime.h>
#include <rccl/rccl.h>
#include <cstdint>
#include <cstdio>

#define HIP_CHECK(expr) do { hipError_t _e = (expr); if (_e != hipSuccess) { \
    fprintf(stderr, "HIP error %s at %s:%d\n", hipGetErrorString(_e), __FILE__, __LINE__); \
    return; } } while (0)

typedef int i32x2 __attribute__((ext_vector_type(2)));
typedef int i32x4 __attribute__((ext_vector_type(4)));
typedef float f32x4 __attribute__((ext_vector_type(4)));
typedef __attribute__((address_space(3))) i32x2* lds_i32x2_p;
typedef __attribute__((address_space(3))) i32x4* lds_i32x4_p;
typedef __attribute__((address_space(3))) int* lds_int_p;

#define STAMP(i) do {} while (0)
#define CSTAMP(i) do {} while (0)
#define STAMP_DECL do {} while (0)
#define STAMP_OUT(wsp) do {} while (0)
#define STAMP_ARG
#define STAMP_PASS

namespace {

constexpr int NRANK = 8;
constexpr int NN = 4096;
constexpr int KLOC = 512;
constexpr int MLOC = 512;
constexpr int NWG = 256;
constexpr int NTHREADS = 448;

#define NSPLIT 36
#define LP0 0
#define LP1 2
#define PACE 66
#define PUSH0 95
#define TFIRST 250
#define PACE0 57
#define NFAST 2
#define POISON_BITS ""
#define OUT_BITS "sc1"
#define XSTRIDE 9216
#define QROT 1
#define ADIST 8
#define DUMP_EVERY 1
#define XLOAD_AT (8 * DUMP_EVERY + 5)
#define FIRST_AT 3
#define LDW 1
#define ROT 4
#define LPRIO 0
#define CPRIO 1
#define CS0 10
#define CS1 20
constexpr int NS1 = 4 + NSPLIT / 2, NS2 = 56 - NSPLIT, NSTEP = NS1 + NS2;
constexpr int NFRAG = 56;
static_assert(NSPLIT % 2 == 0 && NSPLIT >= 8 && NSTEP <= 64 && XLOAD_AT < NS2 && LP0 + 2 <= LP1 && LP1 + 2 <= NS1, "");

constexpr unsigned WS_TMO = 0;
constexpr unsigned WS_TICKET = 4096;
constexpr unsigned WS_FLAGS = 8192;
constexpr unsigned WS_DATA = 1u << 20;
constexpr unsigned SLOT_BYTES = 32 * NFRAG * 1024;
constexpr unsigned WS_X = WS_DATA + 3u * SLOT_BYTES;
constexpr size_t WS_NEEDED = (size_t)WS_X + NWG * 4u * XSTRIDE;
constexpr unsigned POISON = 0x80808080u;

constexpr unsigned LDS_PUSH = 16 * 8192;
constexpr unsigned LDS_LOC = LDS_PUSH + 8192;
constexpr unsigned LDS_CTRL = LDS_LOC + 16384;
constexpr unsigned C_FULL = 0;
constexpr unsigned C_CALL = 16;
constexpr unsigned C_TICKET = 768;
constexpr unsigned C_PROG = 1024;
constexpr unsigned LDS_BYTES = LDS_CTRL + 2048;

constexpr unsigned DEADLINE_TICKS = 50000000u;

struct Args {
    const int8_t* x;
    const int8_t* w;
    const float* sx;
    const float* sw;
    float* out;
    int rank;
    int pad;
    char* ws[NRANK];
};

#define SB() __builtin_amdgcn_sched_barrier(0)
#define LGKM(n) __builtin_amdgcn_s_waitcnt(0xC07F | ((n) << 8))

__device__ __forceinline__ void dma4(const char* gbase, unsigned lds_addr,
                                     unsigned v0, unsigned v1, unsigned v2, unsigned v3) {
    asm volatile(
        "s_mov_b32 m0, %5\n\t"
        "s_nop 0\n\t"
        "global_load_lds_dwordx4 %0, %4\n\t"
        "global_load_lds_dwordx4 %1, %4 offset:1024\n\t"
        "global_load_lds_dwordx4 %2, %4 offset:2048\n\t"
        "global_load_lds_dwordx4 %3, %4 offset:3072"
        :: "v"(v0), "v"(v1), "v"(v2), "v"(v3), "s"(gbase), "s"(lds_addr) : "memory");
}
template <int N> __device__ __forceinline__ void wait_vm(i32x4& a) {
    asm volatile("s_waitcnt vmcnt(%1)" : "+v"(a) : "i"(N) : "memory");
}
template <int N> __device__ __forceinline__ void wait_vm(i32x4& a, i32x4& b) {
    asm volatile("s_waitcnt vmcnt(%2)" : "+v"(a), "+v"(b) : "i"(N) : "memory");
}
__device__ __forceinline__ int lds_ld(unsigned addr) {
    return __hip_atomic_load((lds_int_p)addr, __ATOMIC_RELAXED, __HIP_MEMORY_SCOPE_WORKGROUP);
}
__device__ __forceinline__ void lds_st(unsigned addr, int v) {
    __hip_atomic_store((lds_int_p)addr, v, __ATOMIC_RELAXED, __HIP_MEMORY_SCOPE_WORKGROUP);
}
__device__ __forceinline__ void lds_st1(unsigned addr, int v) {
    asm volatile("s_mov_b64 exec, 1\n\tds_write_b32 %0, %1\n\ts_mov_b64 exec, -1" :: "v"(addr), "v"(v) : "memory");
}
__device__ __forceinline__ void lds_wait(unsigned addr, int need) {
    while (__builtin_amdgcn_readfirstlane(
               __hip_atomic_load((lds_int_p)addr, __ATOMIC_ACQUIRE, __HIP_MEMORY_SCOPE_WORKGROUP)) < need)
        __builtin_amdgcn_s_sleep(1);
    asm volatile("" ::: "memory");
}

constexpr int imin(int x, int y) { return x < y ? x : y; }
constexpr int imax(int x, int y) { return x > y ? x : y; }
constexpr int local_of(int s) { return s >= LP0 && s < LP0 + 2 ? s - LP0 : s >= LP1 && s < LP1 + 2 ? 2 + s - LP1 : -1; }
constexpr int frag_lo(int s) {
    return s >= NS1 ? NSPLIT + s - NS1 : 2 * (s - imin(2, imax(0, s - LP0)) - imin(2, imax(0, s - LP1)));
}
__device__ __forceinline__ int pos_of(int rel, int mi) {
    if (!QROT) return rel - 1;
    const int v = (rel + 2 * mi) & 7;
    return v - (v > 2 * mi ? 1 : 0);
}
__device__ __forceinline__ int rel_of(int pos, int mi) {
    if (!QROT) return pos + 1;
    return (pos + (pos >= 2 * mi ? 1 : 0) - 2 * mi) & 7;
}
__device__ __forceinline__ unsigned frag_off(int rel, int cls, int rg) {
    const int i = 7 * cls + pos_of(rel, rg >> 3), m = rg & 1;
    const int P = i < NSPLIT ? i & 1 : m, frag = i < NSPLIT ? (i & ~1) + (m ^ P) : i;
    return (((rg >> 1) * 2 + P) * NFRAG + frag) * 1024;
}

struct Ctx {
    const char* af;
    char* xo;
    const char* xi;
    const char* xl;
    const char* xl2;
    unsigned sv[4];
    unsigned ll;
    unsigned la;
    unsigned sent;
    unsigned tag;
    unsigned rb[2][8];
    unsigned l16;
    unsigned chk;
    const char* flags;
    unsigned foff;
    unsigned call, t0;
    unsigned bad;
    unsigned prog;
    int one;
    int seen;
};

__device__ __forceinline__ void read_local(i32x4 (&AL)[2][2], const Ctx& c) {
#pragma unroll
    for (int s = 0; s < 2; ++s) {
        AL[s][0] = *(lds_i32x4_p)(c.la ^ (s * 64));
        AL[s][1] = *(lds_i32x4_p)(c.la ^ (s * 64) ^ 2048);
    }
}
template <int S, int H>
__device__ __forceinline__ void read_b(i32x2 (&B)[16], int t, const Ctx& c) {
    B[2 * t + H] = __builtin_amdgcn_ds_read_tr8_b64_v2i32((lds_i32x2_p)(c.rb[(S >> 3) & 1][t] + (S & 7) * 8192 + H * 1024));
}
__device__ __forceinline__ void mma(i32x4& acc, const i32x4& a, const i32x2 (&B)[16], int t) {
    acc = __builtin_amdgcn_mfma_i32_16x16x64_i8(__builtin_shufflevector(B[2 * t], B[2 * t + 1], 0, 1, 2, 3), a, acc, 0, 0, 0);
}
template <int F>
__device__ __forceinline__ void load_frag(i32x4 (&A)[NFRAG], const Ctx& c) {
    asm volatile("global_load_dwordx4 %0, %1, %2 offset:%3 sc0 sc1"
                 : "=v"(A[F]) : "v"(c.l16), "s"(c.af + (F / 8) * 8192 + 4096), "i"((F % 8) * 1024 - 4096) : "memory");
}
template <int F, int END>
__device__ __forceinline__ void load_frags(i32x4 (&A)[NFRAG], const Ctx& c) {
    if constexpr (F < END) { load_frag<F>(A, c); load_frags<F + 1, END>(A, c); }
}
template <int FULL>
__device__ __forceinline__ void check(const Ctx& c, int seen) {
    if (__builtin_expect(__any(seen < FULL), 0)) {
        while (__any(__hip_atomic_load((lds_int_p)c.chk, __ATOMIC_ACQUIRE, __HIP_MEMORY_SCOPE_WORKGROUP) < FULL))
            __builtin_amdgcn_s_sleep(1);
    }
    asm volatile("" ::: "memory");
}

#define RELOAD(n, off) "global_load_dwordx4 %" #n ", %10, %18 offset:" #off " sc0 sc1\n\t"
template <int F, int F1, bool ASYNC>
__device__ __forceinline__ void validate(i32x4 (&A)[NFRAG], Ctx& c, unsigned long long m0, unsigned long long m1) {
    unsigned p0, p1, off, fl, prev;
    unsigned long long now;
    i32x4 D[7];
#pragma unroll
    for (int i = 0; i < 7; ++i) if (F + 1 + i >= NFRAG) asm volatile("" : "=v"(D[i]));
    asm volatile(
        "s_or_b64 vcc, %15, %16\n\t"
        "s_cbranch_scc1 .Lslow%=\n"
        ".Lback%=:\n\t"
        ".subsection 1\n"
        ".Lslow%=:\n\t"
        "v_add_u32 %10, %21, %17\n\t"
        "s_mov_b32 %13, 0\n"
        ".Lretry%=:\n\t"
        "global_load_dword %8, %10, %18 offset:-4096 sc0 sc1\n\t"
        "global_load_dword %9, %10, %18 offset:%22 sc0 sc1\n\t"
        RELOAD(0, -4096) RELOAD(1, -3072)
        "global_load_dword %11, %19, %20 sc0 sc1\n\t"
        "s_memrealtime vcc\n\t"
        "s_waitcnt vmcnt(0) lgkmcnt(0)\n\t"
        "s_cmp_lg_u32 %13, 0\n\t"
        "s_cbranch_scc1 .Lgo%=\n\t"
        "s_sub_u32 %13, vcc_lo, %25\n\t"
        "v_cmp_eq_u32 vcc, %23, %8\n\t"
        "v_cmp_eq_u32 %12, %23, %9\n\t"
        "s_or_b64 vcc, vcc, %12\n\t"
        "s_cbranch_scc0 .Lgo%=\n\t"
        "s_cmp_gt_u32 %13, %26\n\t"
        "s_cselect_b32 %14, 1, %14\n\t"
        "s_cbranch_scc1 .Lgo%=\n\t"
        "v_subrev_u32 %11, %24, %11\n\t"
        "v_cmp_gt_i32 vcc, 0, %11\n\t"
        "s_cmp_eq_u64 vcc, 0\n\t"
        "s_cselect_b32 %13, 1, 0\n\t"
        "s_branch .Lretry%=\n"
        ".Lgo%=:\n\t"
        RELOAD(2, -2048) RELOAD(3, -1024) RELOAD(4, 0) RELOAD(5, 1024) RELOAD(6, 2048) RELOAD(7, 3072)
        "s_waitcnt vmcnt(%27)\n\t"
        "s_branch .Lback%=\n\t"
        ".subsection 0"
        : "+v"(A[F]), "+v"(F + 1 < NFRAG ? A[F + 1 < NFRAG ? F + 1 : 0] : D[0]), "+v"(F + 2 < NFRAG ? A[F + 2 < NFRAG ? F + 2 : 0] : D[1]),
          "+v"(F + 3 < NFRAG ? A[F + 3 < NFRAG ? F + 3 : 0] : D[2]), "+v"(F + 4 < NFRAG ? A[F + 4 < NFRAG ? F + 4 : 0] : D[3]),
          "+v"(F + 5 < NFRAG ? A[F + 5 < NFRAG ? F + 5 : 0] : D[4]), "+v"(F + 6 < NFRAG ? A[F + 6 < NFRAG ? F + 6 : 0] : D[5]),
          "+v"(F + 7 < NFRAG ? A[F + 7 < NFRAG ? F + 7 : 0] : D[6]),
          "=&v"(p0), "=&v"(p1), "=&v"(off), "=&v"(fl), "=&s"(now), "=&s"(prev), "+s"(c.bad)
        : "s"(m0), "s"(m1), "v"(c.l16), "s"(c.af), "v"(c.foff), "s"(c.flags),
          "i"(F * 1024 + 4096), "i"((F1 - F) * 1024 - 4096), "s"(c.sent), "s"(c.call), "s"(c.t0), "i"(DEADLINE_TICKS),
          "i"(ASYNC ? 63 : 0)
        : "memory", "vcc", "scc");
}

static_assert(ADIST >= 8 && ADIST % 2 == 0, "validate() reloads 8 fragments that have been requested");
constexpr int FS = LP0 > 0 ? 0 : LP1 > 2 ? 2 : 4;
static_assert(FIRST_AT <= FS, "");
constexpr int issued_before(int s) {
    return s < FIRST_AT ? 0 : imin(NFRAG, imin(frag_lo(s) + ADIST, s <= FS ? 8 : 8 + 4 * (s - FS)));
}
constexpr int LD0 = LP0 == 0 ? -1 : imax(0, LP0 - 5), LD1 = LP1 == 2 ? -1 : imax(LP0, LP1 - 5);
constexpr bool dumps(int s) { return s >= NS1 && (s - NS1) % DUMP_EVERY == 0 && (s - NS1) / DUMP_EVERY < 8; }
constexpr int extra_pre(int s) { return (s == LD0 || s == LD1 ? 4 : 0) + (s == NS1 + XLOAD_AT ? 8 : 0); }
constexpr int ops_before_wait(int s) {
    int n = issued_before(s + 1 == FIRST_AT ? s : s + 1);
    for (int t = 0; t <= s; ++t) n += extra_pre(t) + (t < s && dumps(t) ? 1 : 0);
    return n;
}
constexpr int ops_through_frag(int f) {
    int s = FIRST_AT - 1;
    while (issued_before(s + 1) <= f) ++s;
    int n = f + 1;
    for (int t = 0; t < (f < issued_before(FIRST_AT) ? FIRST_AT : s); ++t) n += extra_pre(t) + (dumps(t) ? 1 : 0);
    return n;
}
constexpr int ops_through_local(int ld) { return ld < 0 ? 0 : ops_before_wait(ld); }

template <int WAIT>
__device__ __forceinline__ void load_x(i32x4 (&X)[8], const Ctx& c) {
    asm volatile(
        "global_load_dwordx4 %0, %8, %9 offset:-4096 sc0 sc1\n\t"
        "global_load_dwordx4 %1, %8, %9 offset:-3072 sc0 sc1\n\t"
        "global_load_dwordx4 %2, %8, %9 offset:-2048 sc0 sc1\n\t"
        "global_load_dwordx4 %3, %8, %9 offset:-1024 sc0 sc1\n\t"
        "global_load_dwordx4 %4, %8, %9 sc0 sc1\n\t"
        "global_load_dwordx4 %5, %8, %9 offset:1024 sc0 sc1\n\t"
        "global_load_dwordx4 %6, %8, %9 offset:2048 sc0 sc1\n\t"
        "global_load_dwordx4 %7, %8, %9 offset:3072 sc0 sc1\n\t"
        "s_waitcnt vmcnt(%10)"
        : "=&v"(X[0]), "=&v"(X[1]), "=&v"(X[2]), "=&v"(X[3]), "=&v"(X[4]), "=&v"(X[5]), "=&v"(X[6]), "=&v"(X[7])
        : "v"(c.l16), "s"(c.xi), "i"(WAIT) : "memory");
}
__device__ __forceinline__ void wait_x(i32x4 (&X)[8]) {
    asm volatile("s_waitcnt vmcnt(0)"
                 : "+v"(X[0]), "+v"(X[1]), "+v"(X[2]), "+v"(X[3]), "+v"(X[4]), "+v"(X[5]), "+v"(X[6]), "+v"(X[7]) :: "memory");
}

template <int FN, int FN1, int WAIT>
__device__ __forceinline__ void probe(i32x4 (&A)[NFRAG], const Ctx& c, unsigned long long& m0, unsigned long long& m1) {
    if constexpr (FN1 == FN) wait_vm<WAIT>(A[FN]);
    else wait_vm<WAIT>(A[FN], A[FN1]);
    m0 = __builtin_amdgcn_uicmp(A[FN][0], c.sent, 32);
    m1 = FN1 == FN ? m0 : __builtin_amdgcn_uicmp(A[FN1][0], c.sent, 32);
}

template <int S>
__device__ __forceinline__ void step(Ctx& c, i32x4 (&acc)[2][8], i32x4 (&AL)[2][2][2], i32x4 (&A)[NFRAG], i32x4 (&X)[8],
                                     i32x2 (&B)[2][16] STAMP_ARG) {
    constexpr int LI = local_of(S);
    constexpr bool PH1 = S < NS1, LOC = LI >= 0, NEXT = S + 1 < NSTEP, FINE = S < 8;
    constexpr bool VAL = NEXT && local_of(S + 1) < 0 && S + 1 != FIRST_AT;
    constexpr int F = frag_lo(S), FN = frag_lo(S + 1), FN1 = S + 1 < NS1 ? FN + 1 : FN;
    constexpr int L0 = issued_before(S), L1 = S + 1 == FIRST_AT ? L0 : issued_before(S + 1);
    unsigned long long m0 = 0, m1 = 0;
    if constexpr ((S & 3) == 0) STAMP(4 + (S >> 2));
    if constexpr (S == 2) STAMP(15);
    if constexpr (S == CS0) { STAMP(28); CSTAMP(30); }
    if constexpr (S == CS1) { STAMP(29); CSTAMP(31); }
    if constexpr (S == FIRST_AT) {
        while ((int)((unsigned)__builtin_amdgcn_s_memrealtime() - c.t0 - TFIRST) < 0) __builtin_amdgcn_s_sleep(1);
        load_frags<0, 8>(A, c);
        if constexpr (S == FS) {
            probe<0, 1, 6>(A, c, m0, m1);
            validate<0, 1, true>(A, c, m0, m1);
        }
    }
    if constexpr (FINE) check<S + 2>(c, c.seen);
    else if constexpr ((S & 1) == 0) check<imin(S + 3, NSTEP)>(c, c.seen);
    const i32x4& a0 = LOC ? AL[LOC ? LI / 2 : 0][LI & 1][0] : A[F];
    const i32x4& a1 = LOC ? AL[LOC ? LI / 2 : 0][LI & 1][1] : A[PH1 ? F + 1 : F];
    LGKM(8);
    SB();
#pragma unroll
    for (int t = 0; t < 8; ++t) {
        if (t == 4) { if constexpr (NEXT) LGKM(8 + (FINE || (S & 1))); else LGKM(0); SB(); }
        mma(acc[0][t], a0, B[S & 1], t); SB();
        if constexpr (NEXT) read_b<S + 1, 0>(B[(S + 1) & 1], t, c);
        SB();
        if constexpr (PH1) { mma(acc[1][t], a1, B[S & 1], t); SB(); }
        if constexpr (NEXT) read_b<S + 1, 1>(B[(S + 1) & 1], t, c);
        if constexpr (L1 - L0 <= 2) {
            if (t == 0) { if constexpr (L0 < L1) load_frag<L0>(A, c); }
            if (t == 2) { if constexpr (L0 + 1 < L1) load_frag<L0 + 1>(A, c); }
        } else {
            if (t == 0) load_frags<L0, L0 + 2>(A, c);
            if (t == 2) load_frags<L0 + 2, L1>(A, c);
        }
        if ((FINE || (S & 1) == 1) && NEXT && t == 3) c.seen = lds_ld(c.chk);
        if (t == 4) {
            if constexpr (S == LD0) dma4(c.xl, c.ll, c.sv[0], c.sv[1], c.sv[2], c.sv[3]);
            if constexpr (S == LD1) dma4(c.xl2, c.ll, c.sv[0], c.sv[1], c.sv[2], c.sv[3]);
            if constexpr (S == NS1 + XLOAD_AT) load_x<63>(X, c);
        }
        if (t == 6) {
            if constexpr (VAL) probe<FN, FN1, ops_before_wait(S) - ops_through_frag(FN1)>(A, c, m0, m1);
            if constexpr (S + 1 == LP0 || S + 1 == LP1) {
                asm volatile("s_waitcnt vmcnt(%0)" :: "i"(ops_before_wait(S) - ops_through_local(S + 1 == LP0 ? LD0 : LD1)) : "memory");
                read_local(AL[S + 1 == LP1], c);
            }
        }
        if (t == 7) {
            if constexpr (dumps(S)) {
                constexpr int Q = (S - NS1) / DUMP_EVERY;
                i32x4 v = acc[1][Q];
                v[0] = (v[0] << 4) | c.tag;
                asm volatile("global_store_dwordx4 %1, %0, %2 offset:%3 sc0 sc1\n\ts_nop 1"
                             :: "v"(v), "v"(c.l16), "s"(c.xo), "i"(Q * 1024 - 4096) : "memory");
            }
        }
        SB();
    }
    if constexpr ((S & 1) == 0 && S + 16 < NSTEP) asm volatile("ds_add_u32 %0, %1" :: "v"(c.prog), "v"(c.one) : "memory");
    if constexpr (VAL)
        validate<FN, FN1, (FN1 > FN && L1 == FN + 8 && ops_before_wait(S) - ops_through_frag(FN1) == 6)>(A, c, m0, m1);
    SB();
    if constexpr (NEXT) step<S + 1>(c, acc, AL, A, X, B STAMP_PASS);
}

#define LORD 0
constexpr int LSEQ[3][2][3] = {{{0, 2, 4}, {1, 3, 5}}, {{0, 3, 1}, {2, 4, 5}}, {{0, 1, 4}, {2, 3, 5}}};
constexpr int fill_step(int h, int k) { return k < 3 ? LSEQ[LORD][h][k] : imin(NSTEP, 2 * k + h); }
constexpr int owed(int h, int k) {
    int m = fill_step(h, k);
    for (int i = k + 1; i < 3; ++i) m = imin(m, fill_step(h, i));
    return m;
}
__device__ __forceinline__ unsigned w_off(int li, int fl, int s, int P, int mi, int rank) {
    const int i = s < NS1 ? fl + P : fl, cls = (i * 37) >> 8;
    const int rem = (((rel_of(i - 7 * cls, mi) + rank) & (NRANK - 1)) * 8 + ((cls + ROT * mi) & 7));
    return (li >= 0 ? rank * 8 + 4 * P + (li ^ mi) : rem) * 64 * NN;
}
struct Ldr {
    const char* w;
    unsigned dv[4];
    unsigned woff;
    unsigned fa, pa;
    int freed;
};
__device__ __forceinline__ void fill(const Ldr& l, unsigned off, unsigned slot) {
    const char* p = l.w + (size_t)off;
    dma4(p, slot, l.dv[0], l.dv[1], l.dv[2], l.dv[3]);
    dma4(p + 32 * NN, slot + 4096, l.dv[0], l.dv[1], l.dv[2], l.dv[3]);
}
__device__ __forceinline__ int released(const Ldr& l) {
    int v = lds_ld(l.pa);
    v = min(v, __builtin_amdgcn_update_dpp(v, v, 0xB1, 0xF, 0xF, false));
    v = min(v, __builtin_amdgcn_update_dpp(v, v, 0x4E, 0xF, 0xF, false));
    return 2 * __builtin_amdgcn_readfirstlane(v);
}
template <int H, int K>
__device__ __forceinline__ void fills(Ldr& l STAMP_ARG) {
    constexpr int S = fill_step(H, K);
    if constexpr (S < NSTEP) {
        if constexpr (S >= 16) {
            if (__builtin_expect(l.freed <= S - 16, 0)) {
                l.freed = released(l);
                if (l.freed <= S - 16) {
                    asm volatile("s_waitcnt vmcnt(0)" ::: "memory");
                    asm volatile("ds_max_i32 %0, %1" :: "v"(l.fa), "v"(S) : "memory");
                    do {
                        __builtin_amdgcn_s_sleep(1);
                        l.freed = released(l);
                    } while (l.freed <= S - 16);
                }
            }
        }
        fill(l, __builtin_amdgcn_readlane(l.woff, S), (S & 15) * 8192);
        if constexpr (K >= LDW)
            asm volatile("s_waitcnt vmcnt(%2)\n\tds_max_i32 %0, %1" :: "v"(l.fa), "v"(owed(H, K - LDW + 1)), "i"(8 * LDW) : "memory");
        if constexpr (K < 6 || (K & 1) == 1) STAMP(K < 6 ? K : 3 + K / 2);
        fills<H, K + 1>(l STAMP_PASS);
    } else {
#pragma unroll
        for (int k = LDW - 1; k >= 0; --k)
            asm volatile("s_waitcnt vmcnt(%2)\n\tds_max_i32 %0, %1" :: "v"(l.fa), "v"(k ? fill_step(H, K - k) : NSTEP), "i"(8 * k) : "memory");
    }
}

}

__global__ __launch_bounds__(NTHREADS, 2) void a2a_gemm_kernel(Args a) {
    extern __shared__ __attribute__((aligned(16))) char smem[];
    const unsigned tstart = (unsigned)__builtin_amdgcn_s_memrealtime();
    asm volatile("" :: "s"(a.x), "s"(a.w), "s"(a.sx), "s"(a.sw), "s"(a.out), "s"(a.rank), "s"(a.ws[0]), "s"(a.ws[1]),
                       "s"(a.ws[2]), "s"(a.ws[3]), "s"(a.ws[4]), "s"(a.ws[5]), "s"(a.ws[6]), "s"(a.ws[7]));
    const int lane = threadIdx.x & 63;
    int wave = __builtin_amdgcn_readfirstlane(threadIdx.x >> 6);
    const int b = blockIdx.x;
    const int xcd = b & 7, j = b >> 3;
    const int rank = a.rank & (NRANK - 1);
    char* const ws = a.ws[0];
    const unsigned lds0 = (unsigned)(uintptr_t)(__attribute__((address_space(3))) char*)smem;
    const unsigned ctrl = lds0 + LDS_CTRL;
    const int P = j & 1;
    const int mi = (j >> 1) & 3;
    const int n0 = ((j >> 3) * 8 + xcd) * 128;
    const int xs = (xcd + (int)((uintptr_t)a.w >> 7) + 4) & 7;
    const int pd = xs == 7 ? 0 : (xs + j) % 7 + 1, pc = j >> 2, pm = (j - (QROT ? (NRANK - pd) >> 1 : 0)) & 3;
    STAMP_DECL;
    STAMP(0);

    unsigned v4[4];
    const char* base;
    if (wave == 6) {
        __builtin_amdgcn_s_setprio(3);
        base = (const char*)a.x + (size_t)(unsigned)((((rank + pd) & (NRANK - 1)) * MLOC + pm * 128) * KLOC + ((pc + ROT * pm) & 7) * 64);
        const unsigned sv = (lane >> 2) * KLOC + (((lane & 3) ^ ((lane >> 4) & 3)) << 4);
        constexpr unsigned G = 16 * KLOC - 1024;
        if (pd) {
            dma4(base, lds0 + LDS_PUSH, sv, sv + G, sv + 2 * G, sv + 3 * G);
            dma4(base + 64 * KLOC, lds0 + LDS_PUSH + 4096, sv, sv + G, sv + 2 * G, sv + 3 * G);
        }
        asm volatile("s_mov_b32 m0, %2\n\ts_mov_b64 exec, 1\n\tglobal_load_lds_dword %0, %1 sc1\n\ts_mov_b64 exec, -1"
                     :: "v"(WS_TICKET), "s"(ws), "i"(LDS_CTRL + C_TICKET) : "memory");
        lds_st(ctrl + lane * 4, lane == 1 ? owed(1, 0) : 0);
        *(lds_i32x4_p)(ctrl + C_PROG + lane * 16) = i32x4{0, 0, 0, 0};
        v4[0] = v4[1] = v4[2] = v4[3] = 0;
        asm volatile("s_waitcnt lgkmcnt(0)\n\ts_barrier" ::: "memory");
    } else if (wave >= 4) {
        __builtin_amdgcn_s_setprio(LPRIO);
        base = (const char*)a.w + n0;
#pragma unroll
        for (int i = 0; i < 4; ++i)
            v4[i] = (lane >> 3) * NN + (((lane & 7) ^ (((lane >> 4) & 3) | ((i >> 1) << 2))) << 4) + i * (8 * NN - 1024);
        constexpr int S0 = fill_step(0, 0), S1 = fill_step(1, 0);
        const bool h = wave & 1;
        const char* p = base + (size_t)(h ? w_off(local_of(S1), frag_lo(S1), S1, P, mi, rank) : w_off(local_of(S0), frag_lo(S0), S0, P, mi, rank));
        const unsigned slot = lds0 + (h ? S1 : S0) * 8192;
        asm volatile("s_barrier" :: "s"(p), "s"(slot), "v"(v4[0]), "v"(v4[1]), "v"(v4[2]), "v"(v4[3]) : "memory");
        dma4(p, slot, v4[0], v4[1], v4[2], v4[3]);
        dma4(p + 32 * NN, slot + 4096, v4[0], v4[1], v4[2], v4[3]);
    } else {
        __builtin_amdgcn_s_setprio(CPRIO);
        base = (const char*)a.x + (size_t)(unsigned)((rank * MLOC + mi * 128 + wave * 32) * KLOC + P * 256 + (mi >> 1) * 128);
#pragma unroll
        for (int i = 0; i < 4; ++i)
            v4[i] = (i * 8 + (lane >> 3)) * KLOC + (((lane & 7) ^ ((i * 4 + (lane >> 4)) & 7)) << 4) - i * 1024;
        asm volatile("s_barrier" :: "s"(base), "v"(v4[0]), "v"(v4[1]), "v"(v4[2]), "v"(v4[3]) : "memory");
        if (LP0 == 0) dma4(base, lds0 + LDS_LOC + wave * 4096, v4[0], v4[1], v4[2], v4[3]);
    }
    asm volatile("" : "+s"(wave) :: "memory");
    STAMP(19);

    if (wave == 6) {
        unsigned* const ticket = (unsigned*)(ws + WS_TICKET);
        const unsigned due = tstart + PUSH0 + ((j < 4 * NFAST ? j * PACE0 : j * PACE - 4 * NFAST * (PACE - PACE0)) >> 2);
        char* const pw = a.ws[pd];
        unsigned doff[2], poff[2];
#pragma unroll
        for (int m = 0; m < 2; ++m) {
            doff[m] = lane * 16 + WS_DATA + frag_off(NRANK - pd, pc, pm * 8 + m);
            poff[m] = lane * 16 + WS_DATA + frag_off(pd, pc, pm * 8 + m);
            asm volatile("" : "+v"(doff[m]), "+v"(poff[m]));
        }
        const unsigned roff = lds0 + LDS_PUSH + (lane & 15) * 64 + (((lane >> 4) ^ ((lane >> 2) & 3)) << 4);
        asm volatile("s_waitcnt vmcnt(0)" :: "s"(pw), "s"(due), "v"(roff) : "memory");
        STAMP(11);
        const unsigned cv = lds_ld(ctrl + C_TICKET);
        i32x4 dat[8];
#pragma unroll
        for (int g = 0; g < 8; ++g) dat[g] = *(lds_i32x4_p)(roff + g * 1024);
        const unsigned call = ((unsigned)__builtin_amdgcn_readfirstlane(cv) >> 8) + 1;
        lds_st1(ctrl + C_CALL, (int)call);
        const unsigned cur = (call % 3) * SLOT_BYTES, old = ((call + 2) % 3) * SLOT_BYTES;
        if (pd) {
            char* dp[4];
#pragma unroll
            for (int w = 0; w < 4; ++w) { dp[w] = pw + cur + w * (2 * NFRAG * 1024); asm volatile("" : "+s"(dp[w])); }
            while ((int)((unsigned)__builtin_amdgcn_s_memrealtime() - due) < 0) __builtin_amdgcn_s_sleep(1);
            STAMP(2);
#pragma unroll
            for (int g = 0; g < 8; ++g)
                asm volatile("global_store_dwordx4 %1, %0, %2 sc0 sc1\n\ts_nop 1" :: "v"(dat[g]), "v"(doff[g & 1]), "s"(dp[g >> 1]) : "memory");
            STAMP(3);
            asm volatile("s_waitcnt vmcnt(0)" ::: "memory");
            STAMP(7);
            const i32x4 poison = {(int)POISON, (int)POISON, (int)POISON, (int)POISON};
#pragma unroll
            for (int g = 0; g < 8; ++g)
                asm volatile("global_store_dwordx4 %1, %0, %2 " POISON_BITS "\n\ts_nop 1"
                             :: "v"(poison), "v"(poff[g & 1]), "s"(ws + old + (g >> 1) * (2 * NFRAG * 1024)) : "memory");
        }
        while (__any(lds_ld(ctrl + C_FULL + (lane & 1) * 4) < NSTEP) && (unsigned)__builtin_amdgcn_s_memrealtime() - tstart < 2500)
            __builtin_amdgcn_s_sleep(8);
        STAMP(4);
        unsigned tk = 0;
        if (lane == 0) tk = __hip_atomic_fetch_add(ticket, 1u, __ATOMIC_RELAXED, __HIP_MEMORY_SCOPE_AGENT);
        if ((__builtin_amdgcn_readfirstlane(tk) & 255) == 255) {
            if (lane < NRANK)
                __hip_atomic_store((unsigned*)(a.ws[lane] + WS_FLAGS + ((NRANK - lane) & (NRANK - 1)) * 128), call,
                                   __ATOMIC_RELAXED, __HIP_MEMORY_SCOPE_SYSTEM);
        }
        STAMP(5);
        STAMP_OUT(ws);
        return;
    }

    if (wave >= 4) {
        __builtin_amdgcn_s_waitcnt(0xCF7F);
        Ldr l;
        l.w = base;
#pragma unroll
        for (int i = 0; i < 4; ++i) l.dv[i] = v4[i];
        {
            const int s = min(lane, NSTEP - 1);
            const int li = (unsigned)(s - LP0) < 2 ? s - LP0 : (unsigned)(s - LP1) < 2 ? 2 + s - LP1 : -1;
            const int fl = s >= NS1 ? NSPLIT + s - NS1 : 2 * (s - (s > LP0 ? 2 : 0) - (s > LP1 ? 2 : 0));
            l.woff = w_off(li, max(fl, 0), s, P, mi, rank);
        }
        l.fa = ctrl + C_FULL + (lane ? 64 + lane * 8 : 0) + (wave & 1) * 4;
        l.pa = ctrl + C_PROG + (lane & 3) * 260;
        l.freed = 0;
        if (wave & 1) fills<1, 1>(l STAMP_PASS);
        else fills<0, 1>(l STAMP_PASS);
        STAMP(20);
        STAMP_OUT(ws);
        return;
    }

    Ctx c;
    c.xl = base;
    c.xl2 = base + (128 - (mi >> 1) * 256);
#pragma unroll
    for (int i = 0; i < 4; ++i) c.sv[i] = v4[i];
    c.ll = lds0 + LDS_LOC + wave * 4096;
    c.la = c.ll + P * 2048 + (lane & 15) * 128 + ((((mi & 1) * 4 + (lane >> 4)) ^ ((lane & 15) >> 1)) << 4);
    c.l16 = lane * 16;
    c.chk = ctrl + C_FULL + (lane & 1) * 4;
    c.foff = (lane & 7) * 128;
    c.flags = ws + WS_FLAGS;
    c.t0 = tstart;
    c.bad = 0;
    typedef const __attribute__((address_space(4))) float* cfp;
    const float scale = *(cfp)(uintptr_t)a.sx * *(cfp)(uintptr_t)a.sw;
    c.prog = ctrl + C_PROG + wave * 256 + lane * 4;
    c.one = 1;
    asm volatile("" : "+v"(c.one), "+v"(c.chk), "+v"(c.prog), "+v"(c.foff));
    {
        const unsigned q = (lane & 15) >> 1, kg = lane >> 4;
        const unsigned r0 = lds0 + (kg * 16 + q) * 128 + (lane & 1) * 8, f = (q >> 1) | ((kg & 1) << 2);
#pragma unroll
        for (int t = 0; t < 8; ++t) {
            c.rb[0][t] = r0 + ((t ^ f) << 4);
            c.rb[1][t] = c.rb[0][t] + 65536;
            asm volatile("" : "+v"(c.rb[0][t]), "+v"(c.rb[1][t]));
        }
    }
    i32x4 acc[2][8];
#pragma unroll
    for (int m = 0; m < 2; ++m)
#pragma unroll
        for (int t = 0; t < 8; ++t) { acc[m][t] = i32x4{0, 0, 0, 0}; asm volatile("" : "+v"(acc[m][t])); }
    i32x2 B[2][16];
    i32x4 AL[2][2][2], A[NFRAG], X[8];
    if constexpr (LP0 == 0) {
        asm volatile("s_waitcnt vmcnt(0)" ::: "memory");
        read_local(AL[0], c);
        if constexpr (LD1 < 0) {
            LGKM(0);
            dma4(c.xl2, c.ll, c.sv[0], c.sv[1], c.sv[2], c.sv[3]);
        }
    }
    STAMP(1);
    unsigned call;
    while ((call = __builtin_amdgcn_readfirstlane(lds_ld(ctrl + C_CALL))) == 0) __builtin_amdgcn_s_sleep(1);
    c.call = call;
    c.sent = call <= 2 ? 0u : POISON;
    c.tag = call & 15;
    c.af = ws + WS_DATA + (call % 3) * SLOT_BYTES + ((mi * 4 + wave) * 2 + P) * (NFRAG * 1024);
    c.xo = ws + WS_X + (b * 4 + wave) * XSTRIDE + 4096;
    c.xi = ws + WS_X + ((b ^ 8) * 4 + wave) * XSTRIDE + 4096;
    check<1>(c, -1);
    STAMP(3);
#pragma unroll
    for (int t = 0; t < 8; ++t) { read_b<0, 0>(B[0], t, c); read_b<0, 1>(B[0], t, c); }
    c.seen = -1;
    step<0>(c, acc, AL, A, X, B STAMP_PASS);
    STAMP(20);

    wait_x(X);
    while (true) {
        unsigned diff = 0;
#pragma unroll
        for (int t = 0; t < 8; ++t) diff |= (unsigned)X[t][0] ^ c.tag;
        if (__builtin_expect(!__any(diff & 15), 1)) break;
        if ((unsigned)__builtin_amdgcn_s_memrealtime() - c.t0 > DEADLINE_TICKS + DEADLINE_TICKS / 2) { c.bad = 1; break; }
        load_x<0>(X, c);
    }
    if (c.bad && lane == 0) __hip_atomic_store((unsigned*)(ws + WS_TMO), call, __ATOMIC_RELAXED, __HIP_MEMORY_SCOPE_AGENT);
    const float fs = c.bad ? __builtin_nanf("") : scale;
    STAMP(22);
    {
        char* o = (char*)a.out + (size_t)(unsigned)(((mi * 128 + wave * 32 + P * 16) * NN + n0) * 4);
        const unsigned oo = (lane & 15) * NN * 4 + (lane >> 4) * 16;
#pragma unroll
        for (int t = 0; t < 8; ++t) {
            X[t][0] >>= 4;
            const i32x4 s = acc[0][t] + X[t];
            const f32x4 y = {(float)s[0] * fs, (float)s[1] * fs, (float)s[2] * fs, (float)s[3] * fs};
            asm volatile("global_store_dwordx4 %1, %0, %2 offset:%3 " OUT_BITS "\n\ts_nop 1"
                         :: "v"(y), "v"(oo), "s"(o), "i"(t * 64) : "memory");
        }
    }
    STAMP(23);
    STAMP(24);
    STAMP_OUT(ws);
}

extern "C" void kernel_launch(int rank, int nranks, ncclComm_t comm,
                              hipStream_t stream,
                              void* const* d_in, const int* in_sizes, int n_in,
                              void* d_out, int out_size,
                              const int* in_dtypes, int out_dtype,
                              void* const* d_out_all,
                              void* const* d_ws_all, size_t ws_bytes) {
    if (nranks != NRANK || (unsigned)rank >= (unsigned)NRANK || ws_bytes < WS_NEEDED || n_in < 4 ||
        out_size != MLOC * NN || in_sizes[0] != NRANK * MLOC * KLOC || in_sizes[1] != NRANK * KLOC * NN) {
        fprintf(stderr, "unexpected problem shape\n");
        return;
    }
    static bool attr_set = false;
    if (!attr_set) {
        HIP_CHECK(hipFuncSetAttribute((const void*)a2a_gemm_kernel, hipFuncAttributeMaxDynamicSharedMemorySize,
                                      (int)LDS_BYTES));
        attr_set = true;
    }
    Args a;
    a.x = (const int8_t*)d_in[0];
    a.w = (const int8_t*)d_in[1];
    a.sx = (const float*)d_in[2];
    a.sw = (const float*)d_in[3];
    a.out = (float*)d_out;
    a.rank = rank;
    a.pad = 0;
    for (int p = 0; p < NRANK; ++p) a.ws[p] = (char*)d_ws_all[(rank + p) & (NRANK - 1)];
    hipLaunchKernelGGL(a2a_gemm_kernel, dim3(NWG), dim3(NTHREADS), LDS_BYTES, stream, a);
    HIP_CHECK(hipGetLastError());
}
```

```cpp
#include <hip/hip_runtime.h>
#include <rccl/rccl.h>
#include <cstdint>
#include <cstdio>

#define HIP_CHECK(expr) do { hipError_t _e = (expr); if (_e != hipSuccess) { \
    fprintf(stderr, "HIP error %s at %s:%d\n", hipGetErrorString(_e), __FILE__, __LINE__); \
    return; } } while (0)

typedef int i32x2 __attribute__((ext_vector_type(2)));
typedef int i32x4 __attribute__((ext_vector_type(4)));
typedef float f32x4 __attribute__((ext_vector_type(4)));
typedef __attribute__((address_space(3))) i32x2* lds_i32x2_p;
typedef __attribute__((address_space(3))) i32x4* lds_i32x4_p;
typedef __attribute__((address_space(3))) int* lds_int_p;

#define STAMP(i) do {} while (0)
#define CSTAMP(i) do {} while (0)
#define STAMP_DECL do {} while (0)
#define STAMP_OUT(wsp) do {} while (0)
#define STAMP_ARG
#define STAMP_PASS

namespace {

constexpr int NRANK = 8;
constexpr int NN = 4096;
constexpr int KLOC = 512;
constexpr int MLOC = 512;
constexpr int NWG = 256;
constexpr int NTHREADS = 448;

#define NSPLIT 36
#define LP0 0
#define LP1 2
#define PACE 66
#define PUSH0 95
#define TFIRST 250
#define PACE0 57
#define NFAST 2
#define POISON_BITS ""
#define OUT_BITS "sc1"
#define XST_BITS "sc0 sc1"
#define XLD_BITS "sc0 sc1"
#define XRE_BITS "sc0 sc1"
#define XPF_BITS "sc1"
#define XSTRIDE 9216
#define QROT 1
#define ADIST 8
#define DUMP_EVERY 1
#define XLOAD_AT (8 * DUMP_EVERY + 8)
#define XGATE 3
#define FIRST_AT 3
#define LDW 1
#define ROT 4
#define LPRIO 0
#define CPRIO 1
#define CS0 10
#define CS1 20
constexpr int NS1 = 4 + NSPLIT / 2, NS2 = 56 - NSPLIT, NSTEP = NS1 + NS2;
constexpr int NFRAG = 56;
static_assert(NSPLIT % 2 == 0 && NSPLIT >= 8 && NSTEP <= 64 && XLOAD_AT < NS2 && LP0 + 2 <= LP1 && LP1 + 2 <= NS1, "");

constexpr unsigned WS_TMO = 0;
constexpr unsigned WS_TICKET = 4096;
constexpr unsigned WS_FLAGS = 8192;
constexpr unsigned WS_DATA = 1u << 20;
constexpr unsigned SLOT_BYTES = 32 * NFRAG * 1024;
constexpr unsigned WS_X = WS_DATA + 3u * SLOT_BYTES;
constexpr size_t WS_NEEDED = (size_t)WS_X + NWG * 4u * XSTRIDE;
constexpr unsigned POISON = 0x80808080u;

constexpr unsigned LDS_PUSH = 16 * 8192;
constexpr unsigned LDS_LOC = LDS_PUSH + 8192;
constexpr unsigned LDS_CTRL = LDS_LOC + 16384;
constexpr unsigned C_FULL = 0;
constexpr unsigned C_CALL = 16;
constexpr unsigned C_TICKET = 768;
constexpr unsigned C_PROG = 1024;
constexpr unsigned LDS_BYTES = LDS_CTRL + 2048;

#define DEADLINE 50000000u
constexpr unsigned DEADLINE_TICKS = DEADLINE;

struct Args {
    const int8_t* x;
    const int8_t* w;
    const float* sx;
    const float* sw;
    float* out;
    int rank;
    int pad;
    char* ws[NRANK];
};

#define SB() __builtin_amdgcn_sched_barrier(0)
#define LGKM(n) __builtin_amdgcn_s_waitcnt(0xC07F | ((n) << 8))

__device__ __forceinline__ void dma4(const char* gbase, unsigned lds_addr,
                                     unsigned v0, unsigned v1, unsigned v2, unsigned v3) {
    asm volatile(
        "s_mov_b32 m0, %5\n\t"
        "s_nop 0\n\t"
        "global_load_lds_dwordx4 %0, %4\n\t"
        "global_load_lds_dwordx4 %1, %4 offset:1024\n\t"
        "global_load_lds_dwordx4 %2, %4 offset:2048\n\t"
        "global_load_lds_dwordx4 %3, %4 offset:3072"
        :: "v"(v0), "v"(v1), "v"(v2), "v"(v3), "s"(gbase), "s"(lds_addr) : "memory");
}
template <int N> __device__ __forceinline__ void wait_vm(i32x4& a) {
    asm volatile("s_waitcnt vmcnt(%1)" : "+v"(a) : "i"(N) : "memory");
}
template <int N> __device__ __forceinline__ void wait_vm(i32x4& a, i32x4& b) {
    asm volatile("s_waitcnt vmcnt(%2)" : "+v"(a), "+v"(b) : "i"(N) : "memory");
}
__device__ __forceinline__ int lds_ld(unsigned addr) {
    return __hip_atomic_load((lds_int_p)addr, __ATOMIC_RELAXED, __HIP_MEMORY_SCOPE_WORKGROUP);
}
__device__ __forceinline__ void lds_st(unsigned addr, int v) {
    __hip_atomic_store((lds_int_p)addr, v, __ATOMIC_RELAXED, __HIP_MEMORY_SCOPE_WORKGROUP);
}
__device__ __forceinline__ void lds_st1(unsigned addr, int v) {
    asm volatile("s_mov_b64 exec, 1\n\tds_write_b32 %0, %1\n\ts_mov_b64 exec, -1" :: "v"(addr), "v"(v) : "memory");
}
__device__ __forceinline__ void lds_wait(unsigned addr, int need) {
    while (__builtin_amdgcn_readfirstlane(
               __hip_atomic_load((lds_int_p)addr, __ATOMIC_ACQUIRE, __HIP_MEMORY_SCOPE_WORKGROUP)) < need)
        __builtin_amdgcn_s_sleep(1);
    asm volatile("" ::: "memory");
}

constexpr int imin(int x, int y) { return x < y ? x : y; }
constexpr int imax(int x, int y) { return x > y ? x : y; }
constexpr int local_of(int s) { return s >= LP0 && s < LP0 + 2 ? s - LP0 : s >= LP1 && s < LP1 + 2 ? 2 + s - LP1 : -1; }
constexpr int frag_lo(int s) {
    return s >= NS1 ? NSPLIT + s - NS1 : 2 * (s - imin(2, imax(0, s - LP0)) - imin(2, imax(0, s - LP1)));
}
__device__ __forceinline__ int pos_of(int rel, int mi) {
    if (!QROT) return rel - 1;
    const int v = (rel + 2 * mi) & 7;
    return v - (v > 2 * mi ? 1 : 0);
}
__device__ __forceinline__ int rel_of(int pos, int mi) {
    if (!QROT) return pos + 1;
    return (pos + (pos >= 2 * mi ? 1 : 0) - 2 * mi) & 7;
}
__device__ __forceinline__ unsigned frag_off(int rel, int cls, int rg) {
    const int i = 7 * cls + pos_of(rel, rg >> 3), m = rg & 1;
    const int P = i < NSPLIT ? i & 1 : m, frag = i < NSPLIT ? (i & ~1) + (m ^ P) : i;
    return (((rg >> 1) * 2 + P) * NFRAG + frag) * 1024;
}

struct Ctx {
    const char* af;
    char* xo;
    const char* xi;
    const char* xl;
    const char* xl2;
    unsigned sv[4];
    unsigned ll;
    unsigned la;
    unsigned sent;
    unsigned tag;
    unsigned xp, xok;
    unsigned rb[2][8];
    unsigned l16;
    unsigned chk;
    const char* flags;
    unsigned foff;
    unsigned call, t0;
    unsigned bad;
    unsigned prog;
    int one;
    int seen;
};

__device__ __forceinline__ void read_local(i32x4 (&AL)[2][2], const Ctx& c) {
#pragma unroll
    for (int s = 0; s < 2; ++s) {
        AL[s][0] = *(lds_i32x4_p)(c.la ^ (s * 64));
        AL[s][1] = *(lds_i32x4_p)(c.la ^ (s * 64) ^ 2048);
    }
}
template <int S, int H>
__device__ __forceinline__ void read_b(i32x2 (&B)[16], int t, const Ctx& c) {
    B[2 * t + H] = __builtin_amdgcn_ds_read_tr8_b64_v2i32((lds_i32x2_p)(c.rb[(S >> 3) & 1][t] + (S & 7) * 8192 + H * 1024));
}
__device__ __forceinline__ void mma(i32x4& acc, const i32x4& a, const i32x2 (&B)[16], int t) {
    acc = __builtin_amdgcn_mfma_i32_16x16x64_i8(__builtin_shufflevector(B[2 * t], B[2 * t + 1], 0, 1, 2, 3), a, acc, 0, 0, 0);
}
template <int F>
__device__ __forceinline__ void load_frag(i32x4 (&A)[NFRAG], const Ctx& c) {
    asm volatile("global_load_dwordx4 %0, %1, %2 offset:%3 sc0 sc1"
                 : "=v"(A[F]) : "v"(c.l16), "s"(c.af + (F / 8) * 8192 + 4096), "i"((F % 8) * 1024 - 4096) : "memory");
}
template <int F, int END>
__device__ __forceinline__ void load_frags(i32x4 (&A)[NFRAG], const Ctx& c) {
    if constexpr (F < END) { load_frag<F>(A, c); load_frags<F + 1, END>(A, c); }
}
template <int FULL>
__device__ __forceinline__ void check(const Ctx& c, int seen) {
    if (__builtin_expect(__any(seen < FULL), 0)) {
        while (__any(__hip_atomic_load((lds_int_p)c.chk, __ATOMIC_ACQUIRE, __HIP_MEMORY_SCOPE_WORKGROUP) < FULL))
            __builtin_amdgcn_s_sleep(1);
    }
    asm volatile("" ::: "memory");
}

#define RELOAD(n, off) "global_load_dwordx4 %" #n ", %10, %18 offset:" #off " sc0 sc1\n\t"
template <int F, int F1, bool ASYNC>
__device__ __forceinline__ void validate(i32x4 (&A)[NFRAG], Ctx& c, unsigned long long m0, unsigned long long m1) {
    unsigned p0, p1, off, fl, prev;
    unsigned long long now;
    i32x4 D[7];
#pragma unroll
    for (int i = 0; i < 7; ++i) if (F + 1 + i >= NFRAG) asm volatile("" : "=v"(D[i]));
    asm volatile(
        "s_or_b64 vcc, %15, %16\n\t"
        "s_cbranch_scc1 .Lslow%=\n"
        ".Lback%=:\n\t"
        ".subsection 1\n"
        ".Lslow%=:\n\t"
        "v_add_u32 %10, %21, %17\n\t"
        "s_mov_b32 %13, 0\n"
        ".Lretry%=:\n\t"
        "global_load_dword %8, %10, %18 offset:-4096 sc0 sc1\n\t"
        "global_load_dword %9, %10, %18 offset:%22 sc0 sc1\n\t"
        RELOAD(0, -4096) RELOAD(1, -3072)
        "global_load_dword %11, %19, %20 sc0 sc1\n\t"
        "s_memrealtime vcc\n\t"
        "s_waitcnt vmcnt(0) lgkmcnt(0)\n\t"
        "s_cmp_lg_u32 %13, 0\n\t"
        "s_cbranch_scc1 .Lgo%=\n\t"
        "s_sub_u32 %13, vcc_lo, %25\n\t"
        "v_cmp_eq_u32 vcc, %23, %8\n\t"
        "v_cmp_eq_u32 %12, %23, %9\n\t"
        "s_or_b64 vcc, vcc, %12\n\t"
        "s_cbranch_scc0 .Lgo%=\n\t"
        "s_cmp_gt_u32 %13, %26\n\t"
        "s_cselect_b32 %14, 1, %14\n\t"
        "s_cbranch_scc1 .Lgo%=\n\t"
        "v_subrev_u32 %11, %24, %11\n\t"
        "v_cmp_gt_i32 vcc, 0, %11\n\t"
        "s_cmp_eq_u64 vcc, 0\n\t"
        "s_cselect_b32 %13, 1, 0\n\t"
        "s_branch .Lretry%=\n"
        ".Lgo%=:\n\t"
        RELOAD(2, -2048) RELOAD(3, -1024) RELOAD(4, 0) RELOAD(5, 1024) RELOAD(6, 2048) RELOAD(7, 3072)
        "s_waitcnt vmcnt(%27)\n\t"
        "s_branch .Lback%=\n\t"
        ".subsection 0"
        : "+v"(A[F]), "+v"(F + 1 < NFRAG ? A[F + 1 < NFRAG ? F + 1 : 0] : D[0]), "+v"(F + 2 < NFRAG ? A[F + 2 < NFRAG ? F + 2 : 0] : D[1]),
          "+v"(F + 3 < NFRAG ? A[F + 3 < NFRAG ? F + 3 : 0] : D[2]), "+v"(F + 4 < NFRAG ? A[F + 4 < NFRAG ? F + 4 : 0] : D[3]),
          "+v"(F + 5 < NFRAG ? A[F + 5 < NFRAG ? F + 5 : 0] : D[4]), "+v"(F + 6 < NFRAG ? A[F + 6 < NFRAG ? F + 6 : 0] : D[5]),
          "+v"(F + 7 < NFRAG ? A[F + 7 < NFRAG ? F + 7 : 0] : D[6]),
          "=&v"(p0), "=&v"(p1), "=&v"(off), "=&v"(fl), "=&s"(now), "=&s"(prev), "+s"(c.bad)
        : "s"(m0), "s"(m1), "v"(c.l16), "s"(c.af), "v"(c.foff), "s"(c.flags),
          "i"(F * 1024 + 4096), "i"((F1 - F) * 1024 - 4096), "s"(c.sent), "s"(c.call), "s"(c.t0), "i"(DEADLINE_TICKS),
          "i"(ASYNC ? 63 : 0)
        : "memory", "vcc", "scc");
}

static_assert(ADIST >= 8 && ADIST % 2 == 0, "validate() reloads 8 fragments that have been requested");
constexpr int FS = LP0 > 0 ? 0 : LP1 > 2 ? 2 : 4;
static_assert(FIRST_AT <= FS, "");
constexpr int issued_before(int s) {
    return s < FIRST_AT ? 0 : imin(NFRAG, imin(frag_lo(s) + ADIST, s <= FS ? 8 : 8 + 4 * (s - FS)));
}
constexpr int LD0 = LP0 == 0 ? -1 : imax(0, LP0 - 5), LD1 = LP1 == 2 ? -1 : imax(LP0, LP1 - 5);
constexpr bool dumps(int s) { return s >= NS1 && (s - NS1) % DUMP_EVERY == 0 && (s - NS1) / DUMP_EVERY < 8; }
constexpr int extra_pre(int s) { return (s == LD0 || s == LD1 ? 4 : 0) + (s == NS1 + XLOAD_AT ? 8 : 0) + (XGATE && s == NS1 + XLOAD_AT - XGATE ? 1 : 0); }
constexpr int ops_before_wait(int s) {
    int n = issued_before(s + 1 == FIRST_AT ? s : s + 1);
    for (int t = 0; t <= s; ++t) n += extra_pre(t) + (t < s && dumps(t) ? 1 : 0);
    return n;
}
constexpr int ops_through_frag(int f) {
    int s = FIRST_AT - 1;
    while (issued_before(s + 1) <= f) ++s;
    int n = f + 1;
    for (int t = 0; t < (f < issued_before(FIRST_AT) ? FIRST_AT : s); ++t) n += extra_pre(t) + (dumps(t) ? 1 : 0);
    return n;
}
constexpr int ops_through_local(int ld) { return ld < 0 ? 0 : ops_before_wait(ld); }

#define LOAD_X8 \
        "global_load_dwordx4 %0, %10, %11 offset:-4096 sc0 sc1\n\t" \
        "global_load_dwordx4 %1, %10, %11 offset:-3072 sc0 sc1\n\t" \
        "global_load_dwordx4 %2, %10, %11 offset:-2048 sc0 sc1\n\t" \
        "global_load_dwordx4 %3, %10, %11 offset:-1024 sc0 sc1\n\t" \
        "global_load_dwordx4 %4, %10, %11 sc0 sc1\n\t" \
        "global_load_dwordx4 %5, %10, %11 offset:1024 sc0 sc1\n\t" \
        "global_load_dwordx4 %6, %10, %11 offset:2048 sc0 sc1\n\t" \
        "global_load_dwordx4 %7, %10, %11 offset:3072 sc0 sc1\n\t"
__device__ __forceinline__ void look_x(Ctx& c) {
    asm volatile("global_load_dword %0, %1, %2 offset:3072 sc0 sc1" : "=v"(c.xp) : "v"(c.foff), "s"(c.xi) : "memory");
}
template <int W0, int W1>
__device__ __forceinline__ void load_x(i32x4 (&X)[8], Ctx& c) {
    asm volatile(
        "s_waitcnt vmcnt(%13)\n\t"
        "v_and_b32 %8, 15, %8\n\t"
        "v_cmp_ne_u32 vcc, %12, %8\n\t"
        "s_cmp_eq_u64 vcc, 0\n\t"
        "s_cselect_b64 exec, -1, 0\n\t"
        "s_cselect_b32 %9, 1, 0\n\t"
        LOAD_X8
        "s_mov_b64 exec, -1\n\t"
        "s_waitcnt vmcnt(%14)"
        : "=&v"(X[0]), "=&v"(X[1]), "=&v"(X[2]), "=&v"(X[3]), "=&v"(X[4]), "=&v"(X[5]), "=&v"(X[6]), "=&v"(X[7]),
          "+v"(c.xp), "=&s"(c.xok)
        : "v"(c.l16), "s"(c.xi), "s"(c.tag), "i"(W0), "i"(W1) : "memory", "vcc", "scc");
}
__device__ __forceinline__ void wait_x(i32x4 (&X)[8]) {
    asm volatile("s_waitcnt vmcnt(0)"
                 : "+v"(X[0]), "+v"(X[1]), "+v"(X[2]), "+v"(X[3]), "+v"(X[4]), "+v"(X[5]), "+v"(X[6]), "+v"(X[7]) :: "memory");
}

template <int FN, int FN1, int WAIT>
__device__ __forceinline__ void probe(i32x4 (&A)[NFRAG], const Ctx& c, unsigned long long& m0, unsigned long long& m1) {
    if constexpr (FN1 == FN) wait_vm<WAIT>(A[FN]);
    else wait_vm<WAIT>(A[FN], A[FN1]);
    m0 = __builtin_amdgcn_uicmp(A[FN][0], c.sent, 32);
    m1 = FN1 == FN ? m0 : __builtin_amdgcn_uicmp(A[FN1][0], c.sent, 32);
}

template <int S>
__device__ __forceinline__ void step(Ctx& c, i32x4 (&acc)[2][8], i32x4 (&AL)[2][2][2], i32x4 (&A)[NFRAG], i32x4 (&X)[8],
                                     i32x2 (&B)[2][16] STAMP_ARG) {
    constexpr int LI = local_of(S);
    constexpr bool PH1 = S < NS1, LOC = LI >= 0, NEXT = S + 1 < NSTEP, FINE = S < 8;
    constexpr bool VAL = NEXT && local_of(S + 1) < 0 && S + 1 != FIRST_AT;
    constexpr int F = frag_lo(S), FN = frag_lo(S + 1), FN1 = S + 1 < NS1 ? FN + 1 : FN;
    constexpr int L0 = issued_before(S), L1 = S + 1 == FIRST_AT ? L0 : issued_before(S + 1);
    unsigned long long m0 = 0, m1 = 0;
    if constexpr ((S & 3) == 0) STAMP(4 + (S >> 2));
    if constexpr (S == 2) STAMP(15);
    if constexpr (S == CS0) { STAMP(28); CSTAMP(30); }
    if constexpr (S == CS1) { STAMP(29); CSTAMP(31); }
    if constexpr (S == FIRST_AT) {
        while ((int)((unsigned)__builtin_amdgcn_s_memrealtime() - c.t0 - TFIRST) < 0) __builtin_amdgcn_s_sleep(1);
        load_frags<0, 8>(A, c);
        if constexpr (S == FS) {
            probe<0, 1, 6>(A, c, m0, m1);
            validate<0, 1, true>(A, c, m0, m1);
        }
    }
    if constexpr (FINE) check<S + 2>(c, c.seen);
    else if constexpr ((S & 1) == 0) check<imin(S + 3, NSTEP)>(c, c.seen);
    const i32x4& a0 = LOC ? AL[LOC ? LI / 2 : 0][LI & 1][0] : A[F];
    const i32x4& a1 = LOC ? AL[LOC ? LI / 2 : 0][LI & 1][1] : A[PH1 ? F + 1 : F];
    LGKM(8);
    SB();
#pragma unroll
    for (int t = 0; t < 8; ++t) {
        if (t == 4) { if constexpr (NEXT) LGKM(8 + (FINE || (S & 1))); else LGKM(0); SB(); }
        mma(acc[0][t], a0, B[S & 1], t); SB();
        if constexpr (NEXT) read_b<S + 1, 0>(B[(S + 1) & 1], t, c);
        SB();
        if constexpr (PH1) { mma(acc[1][t], a1, B[S & 1], t); SB(); }
        if constexpr (NEXT) read_b<S + 1, 1>(B[(S + 1) & 1], t, c);
        if constexpr (L1 - L0 <= 2) {
            if (t == 0) { if constexpr (L0 < L1) load_frag<L0>(A, c); }
            if (t == 2) { if constexpr (L0 + 1 < L1) load_frag<L0 + 1>(A, c); }
        } else {
            if (t == 0) load_frags<L0, L0 + 2>(A, c);
            if (t == 2) load_frags<L0 + 2, L1>(A, c);
        }
        if ((FINE || (S & 1) == 1) && NEXT && t == 3) c.seen = lds_ld(c.chk);
        if (t == 4) {
            if constexpr (S == LD0) dma4(c.xl, c.ll, c.sv[0], c.sv[1], c.sv[2], c.sv[3]);
            if constexpr (S == LD1) dma4(c.xl2, c.ll, c.sv[0], c.sv[1], c.sv[2], c.sv[3]);
            if constexpr (XGATE && S == NS1 + XLOAD_AT - XGATE) look_x(c);
            if constexpr (S == NS1 + XLOAD_AT) load_x<XGATE ? ops_before_wait(S) - 8 - ops_before_wait(S - XGATE) : 63, 63>(X, c);
        }
        if (t == 6) {
            if constexpr (VAL) probe<FN, FN1, ops_before_wait(S) - ops_through_frag(FN1)>(A, c, m0, m1);
            if constexpr (S + 1 == LP0 || S + 1 == LP1) {
                asm volatile("s_waitcnt vmcnt(%0)" :: "i"(ops_before_wait(S) - ops_through_local(S + 1 == LP0 ? LD0 : LD1)) : "memory");
                read_local(AL[S + 1 == LP1], c);
            }
        }
        if (t == 7) {
            if constexpr (dumps(S)) {
                constexpr int Q = (S - NS1) / DUMP_EVERY;
                i32x4 v = acc[1][Q];
                v[0] = (v[0] << 4) | c.tag;
                asm volatile("global_store_dwordx4 %1, %0, %2 offset:%3 " XST_BITS "\n\ts_nop 1"
                             :: "v"(v), "v"(c.l16), "s"(c.xo), "i"(Q * 1024 - 4096) : "memory");
            }
        }
        SB();
    }
    if constexpr ((S & 1) == 0 && S + 16 < NSTEP) asm volatile("ds_add_u32 %0, %1" :: "v"(c.prog), "v"(c.one) : "memory");
    if constexpr (VAL)
        validate<FN, FN1, (FN1 > FN && L1 == FN + 8 && ops_before_wait(S) - ops_through_frag(FN1) == 6)>(A, c, m0, m1);
    SB();
    if constexpr (NEXT) step<S + 1>(c, acc, AL, A, X, B STAMP_PASS);
}

#define LORD 0
constexpr int LSEQ[3][2][3] = {{{0, 2, 4}, {1, 3, 5}}, {{0, 3, 1}, {2, 4, 5}}, {{0, 1, 4}, {2, 3, 5}}};
constexpr int fill_step(int h, int k) { return k < 3 ? LSEQ[LORD][h][k] : imin(NSTEP, 2 * k + h); }
constexpr int owed(int h, int k) {
    int m = fill_step(h, k);
    for (int i = k + 1; i < 3; ++i) m = imin(m, fill_step(h, i));
    return m;
}
__device__ __forceinline__ unsigned w_off(int li, int fl, int s, int P, int mi, int rank) {
    const int i = s < NS1 ? fl + P : fl, cls = (i * 37) >> 8;
    const int rem = (((rel_of(i - 7 * cls, mi) + rank) & (NRANK - 1)) * 8 + ((cls + ROT * mi) & 7));
    return (li >= 0 ? rank * 8 + 4 * P + (li ^ mi) : rem) * 64 * NN;
}
struct Ldr {
    const char* w;
    unsigned dv[4];
    unsigned woff;
    unsigned fa, pa;
    int freed;
};
__device__ __forceinline__ void fill(const Ldr& l, unsigned off, unsigned slot) {
    const char* p = l.w + (size_t)off;
    dma4(p, slot, l.dv[0], l.dv[1], l.dv[2], l.dv[3]);
    dma4(p + 32 * NN, slot + 4096, l.dv[0], l.dv[1], l.dv[2], l.dv[3]);
}
__device__ __forceinline__ int released(const Ldr& l) {
    int v = lds_ld(l.pa);
    v = min(v, __builtin_amdgcn_update_dpp(v, v, 0xB1, 0xF, 0xF, false));
    v = min(v, __builtin_amdgcn_update_dpp(v, v, 0x4E, 0xF, 0xF, false));
    return 2 * __builtin_amdgcn_readfirstlane(v);
}
template <int H, int K>
__device__ __forceinline__ void fills(Ldr& l STAMP_ARG) {
    constexpr int S = fill_step(H, K);
    if constexpr (S < NSTEP) {
        if constexpr (S >= 16) {
            if (__builtin_expect(l.freed <= S - 16, 0)) {
                l.freed = released(l);
                if (l.freed <= S - 16) {
                    asm volatile("s_waitcnt vmcnt(0)" ::: "memory");
                    asm volatile("ds_max_i32 %0, %1" :: "v"(l.fa), "v"(S) : "memory");
                    do {
                        __builtin_amdgcn_s_sleep(1);
                        l.freed = released(l);
                    } while (l.freed <= S - 16);
                }
            }
        }
        fill(l, __builtin_amdgcn_readlane(l.woff, S), (S & 15) * 8192);
        if constexpr (K >= LDW)
            asm volatile("s_waitcnt vmcnt(%2)\n\tds_max_i32 %0, %1" :: "v"(l.fa), "v"(owed(H, K - LDW + 1)), "i"(8 * LDW) : "memory");
        if constexpr (K < 6 || (K & 1) == 1) STAMP(K < 6 ? K : 3 + K / 2);
        fills<H, K + 1>(l STAMP_PASS);
    } else {
#pragma unroll
        for (int k = LDW - 1; k >= 0; --k)
            asm volatile("s_waitcnt vmcnt(%2)\n\tds_max_i32 %0, %1" :: "v"(l.fa), "v"(k ? fill_step(H, K - k) : NSTEP), "i"(8 * k) : "memory");
    }
}

}

__global__ __launch_bounds__(NTHREADS, 2) void a2a_gemm_kernel(Args a) {
    extern __shared__ __attribute__((aligned(16))) char smem[];
    const unsigned tstart = (unsigned)__builtin_amdgcn_s_memrealtime();
    asm volatile("" :: "s"(a.x), "s"(a.w), "s"(a.sx), "s"(a.sw), "s"(a.out), "s"(a.rank), "s"(a.ws[0]), "s"(a.ws[1]),
                       "s"(a.ws[2]), "s"(a.ws[3]), "s"(a.ws[4]), "s"(a.ws[5]), "s"(a.ws[6]), "s"(a.ws[7]));
    const int lane = threadIdx.x & 63;
    int wave = __builtin_amdgcn_readfirstlane(threadIdx.x >> 6);
    const int b = blockIdx.x;
    const int xcd = b & 7, j = b >> 3;
    const int rank = a.rank & (NRANK - 1);
    char* const ws = a.ws[0];
    const unsigned lds0 = (unsigned)(uintptr_t)(__attribute__((address_space(3))) char*)smem;
    const unsigned ctrl = lds0 + LDS_CTRL;
    const int P = j & 1;
    const int mi = (j >> 1) & 3;
    const int n0 = ((j >> 3) * 8 + xcd) * 128;
    const int xs = (xcd + (int)((uintptr_t)a.w >> 7) + 4) & 7;
    const int pd = xs == 7 ? 0 : (xs + j) % 7 + 1, pc = j >> 2, pm = (j - (QROT ? (NRANK - pd) >> 1 : 0)) & 3;
    STAMP_DECL;
    STAMP(0);

    unsigned v4[4];
    const char* base;
    if (wave == 6) {
        __builtin_amdgcn_s_setprio(3);
        base = (const char*)a.x + (size_t)(unsigned)((((rank + pd) & (NRANK - 1)) * MLOC + pm * 128) * KLOC + ((pc + ROT * pm) & 7) * 64);
        const unsigned sv = (lane >> 2) * KLOC + (((lane & 3) ^ ((lane >> 4) & 3)) << 4);
        constexpr unsigned G = 16 * KLOC - 1024;
        if (pd) {
            dma4(base, lds0 + LDS_PUSH, sv, sv + G, sv + 2 * G, sv + 3 * G);
            dma4(base + 64 * KLOC, lds0 + LDS_PUSH + 4096, sv, sv + G, sv + 2 * G, sv + 3 * G);
        }
        asm volatile("s_mov_b32 m0, %2\n\ts_mov_b64 exec, 1\n\tglobal_load_lds_dword %0, %1 sc1\n\ts_mov_b64 exec, -1"
                     :: "v"(WS_TICKET), "s"(ws), "i"(LDS_CTRL + C_TICKET) : "memory");
        lds_st(ctrl + lane * 4, lane == 1 ? owed(1, 0) : 0);
        *(lds_i32x4_p)(ctrl + C_PROG + lane * 16) = i32x4{0, 0, 0, 0};
        v4[0] = v4[1] = v4[2] = v4[3] = 0;
        asm volatile("s_waitcnt lgkmcnt(0)\n\ts_barrier" ::: "memory");
    } else if (wave >= 4) {
        __builtin_amdgcn_s_setprio(LPRIO);
        base = (const char*)a.w + n0;
#pragma unroll
        for (int i = 0; i < 4; ++i)
            v4[i] = (lane >> 3) * NN + (((lane & 7) ^ (((lane >> 4) & 3) | ((i >> 1) << 2))) << 4) + i * (8 * NN - 1024);
        constexpr int S0 = fill_step(0, 0), S1 = fill_step(1, 0);
        const bool h = wave & 1;
        const char* p = base + (size_t)(h ? w_off(local_of(S1), frag_lo(S1), S1, P, mi, rank) : w_off(local_of(S0), frag_lo(S0), S0, P, mi, rank));
        const unsigned slot = lds0 + (h ? S1 : S0) * 8192;
        asm volatile("s_barrier" :: "s"(p), "s"(slot), "v"(v4[0]), "v"(v4[1]), "v"(v4[2]), "v"(v4[3]) : "memory");
        dma4(p, slot, v4[0], v4[1], v4[2], v4[3]);
        dma4(p + 32 * NN, slot + 4096, v4[0], v4[1], v4[2], v4[3]);
    } else {
        __builtin_amdgcn_s_setprio(CPRIO);
        base = (const char*)a.x + (size_t)(unsigned)((rank * MLOC + mi * 128 + wave * 32) * KLOC + P * 256 + (mi >> 1) * 128);
#pragma unroll
        for (int i = 0; i < 4; ++i)
            v4[i] = (i * 8 + (lane >> 3)) * KLOC + (((lane & 7) ^ ((i * 4 + (lane >> 4)) & 7)) << 4) - i * 1024;
        asm volatile("s_barrier" :: "s"(base), "v"(v4[0]), "v"(v4[1]), "v"(v4[2]), "v"(v4[3]) : "memory");
        if (LP0 == 0) dma4(base, lds0 + LDS_LOC + wave * 4096, v4[0], v4[1], v4[2], v4[3]);
    }
    asm volatile("" : "+s"(wave) :: "memory");
    STAMP(19);

    if (wave == 6) {
        unsigned* const ticket = (unsigned*)(ws + WS_TICKET);
        const unsigned due = tstart + PUSH0 + ((j < 4 * NFAST ? j * PACE0 : j * PACE - 4 * NFAST * (PACE - PACE0)) >> 2);
        char* const pw = a.ws[pd];
        unsigned doff[2], poff[2];
#pragma unroll
        for (int m = 0; m < 2; ++m) {
            doff[m] = lane * 16 + WS_DATA + frag_off(NRANK - pd, pc, pm * 8 + m);
            poff[m] = lane * 16 + WS_DATA + frag_off(pd, pc, pm * 8 + m);
            asm volatile("" : "+v"(doff[m]), "+v"(poff[m]));
        }
        const unsigned roff = lds0 + LDS_PUSH + (lane & 15) * 64 + (((lane >> 4) ^ ((lane >> 2) & 3)) << 4);
        asm volatile("s_waitcnt vmcnt(0)" :: "s"(pw), "s"(due), "v"(roff) : "memory");
        STAMP(11);
        const unsigned cv = lds_ld(ctrl + C_TICKET);
        i32x4 dat[8];
#pragma unroll
        for (int g = 0; g < 8; ++g) dat[g] = *(lds_i32x4_p)(roff + g * 1024);
        const unsigned call = ((unsigned)__builtin_amdgcn_readfirstlane(cv) >> 8) + 1;
        lds_st1(ctrl + C_CALL, (int)call);
        const unsigned cur = (call % 3) * SLOT_BYTES, old = ((call + 2) % 3) * SLOT_BYTES;
        if (pd) {
            char* dp[4];
#pragma unroll
            for (int w = 0; w < 4; ++w) { dp[w] = pw + cur + w * (2 * NFRAG * 1024); asm volatile("" : "+s"(dp[w])); }
            while ((int)((unsigned)__builtin_amdgcn_s_memrealtime() - due) < 0) __builtin_amdgcn_s_sleep(1);
            STAMP(2);
#pragma unroll
            for (int g = 0; g < 8; ++g)
                asm volatile("global_store_dwordx4 %1, %0, %2 sc0 sc1\n\ts_nop 1" :: "v"(dat[g]), "v"(doff[g & 1]), "s"(dp[g >> 1]) : "memory");
            STAMP(3);
            asm volatile("s_waitcnt vmcnt(0)" ::: "memory");
            STAMP(7);
            const i32x4 poison = {(int)POISON, (int)POISON, (int)POISON, (int)POISON};
#pragma unroll
            for (int g = 0; g < 8; ++g)
                asm volatile("global_store_dwordx4 %1, %0, %2 " POISON_BITS "\n\ts_nop 1"
                             :: "v"(poison), "v"(poff[g & 1]), "s"(ws + old + (g >> 1) * (2 * NFRAG * 1024)) : "memory");
        }
        {
            unsigned d;
            const unsigned po = (lane & 7) * 128 + ((lane >> 3) & 3) * XSTRIDE + 7168;
            asm volatile("global_load_dword %0, %1, %2 sc1\n\ts_waitcnt vmcnt(0)" : "=&v"(d) : "v"(po), "s"(ws + WS_X + b * 4 * XSTRIDE) : "memory");
        }
        while (__any(lds_ld(ctrl + C_FULL + (lane & 1) * 4) < NSTEP) && (unsigned)__builtin_amdgcn_s_memrealtime() - tstart < 2500)
            __builtin_amdgcn_s_sleep(8);
        STAMP(4);
        unsigned tk = 0;
        if (lane == 0) tk = __hip_atomic_fetch_add(ticket, 1u, __ATOMIC_RELAXED, __HIP_MEMORY_SCOPE_AGENT);
        if ((__builtin_amdgcn_readfirstlane(tk) & 255) == 255) {
            if (lane < NRANK)
                __hip_atomic_store((unsigned*)(a.ws[lane] + WS_FLAGS + ((NRANK - lane) & (NRANK - 1)) * 128), call,
                                   __ATOMIC_RELAXED, __HIP_MEMORY_SCOPE_SYSTEM);
        }
        STAMP(5);
        STAMP_OUT(ws);
        return;
    }

    if (wave >= 4) {
        __builtin_amdgcn_s_waitcnt(0xCF7F);
        Ldr l;
        l.w = base;
#pragma unroll
        for (int i = 0; i < 4; ++i) l.dv[i] = v4[i];
        {
            const int s = min(lane, NSTEP - 1);
            const int li = (unsigned)(s - LP0) < 2 ? s - LP0 : (unsigned)(s - LP1) < 2 ? 2 + s - LP1 : -1;
            const int fl = s >= NS1 ? NSPLIT + s - NS1 : 2 * (s - (s > LP0 ? 2 : 0) - (s > LP1 ? 2 : 0));
            l.woff = w_off(li, max(fl, 0), s, P, mi, rank);
        }
        l.fa = ctrl + C_FULL + (lane ? 64 + lane * 8 : 0) + (wave & 1) * 4;
        l.pa = ctrl + C_PROG + (lane & 3) * 260;
        l.freed = 0;
        if (wave & 1) fills<1, 1>(l STAMP_PASS);
        else fills<0, 1>(l STAMP_PASS);
        STAMP(20);
        STAMP_OUT(ws);
        return;
    }

    Ctx c;
    c.xl = base;
    c.xl2 = base + (128 - (mi >> 1) * 256);
#pragma unroll
    for (int i = 0; i < 4; ++i) c.sv[i] = v4[i];
    c.ll = lds0 + LDS_LOC + wave * 4096;
    c.la = c.ll + P * 2048 + (lane & 15) * 128 + ((((mi & 1) * 4 + (lane >> 4)) ^ ((lane & 15) >> 1)) << 4);
    c.l16 = lane * 16;
    c.chk = ctrl + C_FULL + (lane & 1) * 4;
    c.foff = (lane & 7) * 128;
    c.flags = ws + WS_FLAGS;
    c.t0 = tstart;
    c.bad = 0;
    typedef const __attribute__((address_space(4))) float* cfp;
    const float scale = *(cfp)(uintptr_t)a.sx * *(cfp)(uintptr_t)a.sw;
    c.prog = ctrl + C_PROG + wave * 256 + lane * 4;
    c.one = 1;
    asm volatile("" : "+v"(c.one), "+v"(c.chk), "+v"(c.prog), "+v"(c.foff));
    {
        const unsigned q = (lane & 15) >> 1, kg = lane >> 4;
        const unsigned r0 = lds0 + (kg * 16 + q) * 128 + (lane & 1) * 8, f = (q >> 1) | ((kg & 1) << 2);
#pragma unroll
        for (int t = 0; t < 8; ++t) {
            c.rb[0][t] = r0 + ((t ^ f) << 4);
            c.rb[1][t] = c.rb[0][t] + 65536;
            asm volatile("" : "+v"(c.rb[0][t]), "+v"(c.rb[1][t]));
        }
    }
    i32x4 acc[2][8];
#pragma unroll
    for (int m = 0; m < 2; ++m)
#pragma unroll
        for (int t = 0; t < 8; ++t) { acc[m][t] = i32x4{0, 0, 0, 0}; asm volatile("" : "+v"(acc[m][t])); }
    i32x2 B[2][16];
    i32x4 AL[2][2][2], A[NFRAG], X[8];
    if constexpr (LP0 == 0) {
        asm volatile("s_waitcnt vmcnt(0)" ::: "memory");
        read_local(AL[0], c);
        if constexpr (LD1 < 0) {
            LGKM(0);
            dma4(c.xl2, c.ll, c.sv[0], c.sv[1], c.sv[2], c.sv[3]);
        }
    }
    STAMP(1);
    unsigned call;
    while ((call = __builtin_amdgcn_readfirstlane(lds_ld(ctrl + C_CALL))) == 0) __builtin_amdgcn_s_sleep(1);
    c.call = call;
    c.sent = call <= 2 ? 0u : POISON;
    c.tag = call & 15;
    c.xp = 0;
    c.af = ws + WS_DATA + (call % 3) * SLOT_BYTES + ((mi * 4 + wave) * 2 + P) * (NFRAG * 1024);
    c.xo = ws + WS_X + (b * 4 + wave) * XSTRIDE + 4096;
    c.xi = ws + WS_X + ((b ^ 8) * 4 + wave) * XSTRIDE + 4096;
    check<1>(c, -1);
    STAMP(3);
#pragma unroll
    for (int t = 0; t < 8; ++t) { read_b<0, 0>(B[0], t, c); read_b<0, 1>(B[0], t, c); }
    c.seen = -1;
    step<0>(c, acc, AL, A, X, B STAMP_PASS);
    STAMP(20);

    wait_x(X);
    while (true) {
        unsigned diff = 0;
#pragma unroll
        for (int t = 0; t < 8; ++t) diff |= (unsigned)X[t][0] ^ c.tag;
        if (__builtin_expect(c.xok && !__any(diff & 15), 1)) break;
        if ((unsigned)__builtin_amdgcn_s_memrealtime() - c.t0 > DEADLINE_TICKS + DEADLINE_TICKS / 2) { c.bad = 1; break; }
        look_x(c);
        load_x<0, 0>(X, c);
    }
    if (c.bad && lane == 0) __hip_atomic_store((unsigned*)(ws + WS_TMO), call, __ATOMIC_RELAXED, __HIP_MEMORY_SCOPE_AGENT);
    const float fs = c.bad ? __builtin_nanf("") : scale;
    STAMP(22);
    {
        char* o = (char*)a.out + (size_t)(unsigned)(((mi * 128 + wave * 32 + P * 16) * NN + n0) * 4);
        const unsigned oo = (lane & 15) * NN * 4 + (lane >> 4) * 16;
#pragma unroll
        for (int t = 0; t < 8; ++t) {
            X[t][0] >>= 4;
            const i32x4 s = acc[0][t] + X[t];
            const f32x4 y = {(float)s[0] * fs, (float)s[1] * fs, (float)s[2] * fs, (float)s[3] * fs};
            asm volatile("global_store_dwordx4 %1, %0, %2 offset:%3 " OUT_BITS "\n\ts_nop 1"
                         :: "v"(y), "v"(oo), "s"(o), "i"(t * 64) : "memory");
        }
    }
    STAMP(23);
    STAMP(24);
    STAMP_OUT(ws);
}

extern "C" void kernel_launch(int rank, int nranks, ncclComm_t comm,
                              hipStream_t stream,
                              void* const* d_in, const int* in_sizes, int n_in,
                              void* d_out, int out_size,
                              const int* in_dtypes, int out_dtype,
                              void* const* d_out_all,
                              void* const* d_ws_all, size_t ws_bytes) {
    if (nranks != NRANK || (unsigned)rank >= (unsigned)NRANK || ws_bytes < WS_NEEDED || n_in < 4 ||
        out_size != MLOC * NN || in_sizes[0] != NRANK * MLOC * KLOC || in_sizes[1] != NRANK * KLOC * NN) {
        fprintf(stderr, "unexpected problem shape\n");
        return;
    }
    static bool attr_set = false;
    if (!attr_set) {
        HIP_CHECK(hipFuncSetAttribute((const void*)a2a_gemm_kernel, hipFuncAttributeMaxDynamicSharedMemorySize,
                                      (int)LDS_BYTES));
        attr_set = true;
    }
    Args a;
    a.x = (const int8_t*)d_in[0];
    a.w = (const int8_t*)d_in[1];
    a.sx = (const float*)d_in[2];
    a.sw = (const float*)d_in[3];
    a.out = (float*)d_out;
    a.rank = rank;
    a.pad = 0;
    for (int p = 0; p < NRANK; ++p) a.ws[p] = (char*)d_ws_all[(rank + p) & (NRANK - 1)];
    hipLaunchKernelGGL(a2a_gemm_kernel, dim3(NWG), dim3(NTHREADS), LDS_BYTES, stream, a);
    HIP_CHECK(hipGetLastError());
}
```

```cpp
#include <hip/hip_runtime.h>
#include <rccl/rccl.h>
#include <cstdint>
#include <cstdio>

#define HIP_CHECK(expr) do { hipError_t _e = (expr); if (_e != hipSuccess) { \
    fprintf(stderr, "HIP error %s at %s:%d\n", hipGetErrorString(_e), __FILE__, __LINE__); \
    return; } } while (0)

typedef int i32x2 __attribute__((ext_vector_type(2)));
typedef int i32x4 __attribute__((ext_vector_type(4)));
typedef float f32x4 __attribute__((ext_vector_type(4)));
typedef __attribute__((address_space(3))) i32x2* lds_i32x2_p;
typedef __attribute__((address_space(3))) i32x4* lds_i32x4_p;
typedef __attribute__((address_space(3))) int* lds_int_p;

#define STAMP(i) do {} while (0)
#define CSTAMP(i) do {} while (0)
#define STAMP_DECL do {} while (0)
#define STAMP_OUT(wsp) do {} while (0)
#define STAMP_ARG
#define STAMP_PASS

namespace {

constexpr int NRANK = 8;
constexpr int NN = 4096;
constexpr int KLOC = 512;
constexpr int MLOC = 512;
constexpr int NWG = 256;
constexpr int NTHREADS = 512;

#define NSPLIT 36
#define LP0 0
#define LP1 2
#define PACE 60
#define PUSH0 95
#define TFIRST 250
#define PACE0 57
#define NFAST 2
#define POISON_BITS ""
#define OUT_BITS "sc1"
#define XST_BITS "sc0 sc1"
#define XLD_BITS "sc0 sc1"
#define XRE_BITS "sc0 sc1"
#define XPF_BITS "sc1"
#define XSTRIDE 9216
#define QROT 1
#define ADIST 8
#define DPS 1
#define TICKET_AT 0
#define LEAD 0
#define ADIST2 ADIST
#define AINIT ADIST
#define SPF 0
#define FIRST_AT 3
#define LDW 3
#define LDW_FROM 14
#define LSLEEP 2
#define XLEAD 1
#define ROT 4
#define LPRIO 0
#define CPRIO 1
#define CS0 10
#define CS1 20
constexpr int NS1 = 4 + NSPLIT / 2, NS2 = 56 - NSPLIT, NSTEP = NS1 + NS2;
constexpr int NFRAG = 56;
static_assert(NSPLIT % 2 == 0 && NSPLIT >= 8 && NSTEP <= 64 && NSTEP % 2 == 0 && LP0 == 0 && LP1 == 2, "");

constexpr unsigned WS_TMO = 0;
constexpr unsigned WS_TICKET = 4096;
constexpr unsigned WS_FLAGS = 8192;
constexpr unsigned WS_DATA = 1u << 20;
constexpr unsigned SLOT_BYTES = 32 * NFRAG * 1024;
constexpr unsigned WS_X = WS_DATA + 3u * SLOT_BYTES;
constexpr size_t WS_NEEDED = (size_t)WS_X + NWG * 4u * XSTRIDE;
constexpr unsigned POISON = 0x80808080u;

constexpr unsigned LDS_PUSH = 16 * 8192;
constexpr unsigned LDS_LOC = LDS_PUSH + 8192;
constexpr unsigned LDS_X3 = (NSTEP & 15) * 8192;
constexpr unsigned LDS_CTRL = LDS_LOC + 16384;
constexpr unsigned C_FULL = 0;
constexpr unsigned C_DUE = 8;
constexpr unsigned C_CNT = 12;
constexpr unsigned C_CALL = 16;
constexpr unsigned C_XSEEN = 24;
constexpr unsigned C_XRDY = 32;
constexpr unsigned C_TICKET = 768;
constexpr unsigned C_PROG = 1024;
constexpr unsigned LDS_BYTES = LDS_CTRL + 2048 + 256;

#define DEADLINE 50000000u
constexpr unsigned WATCH_TICKS = 3000;
constexpr unsigned DEADLINE_TICKS = DEADLINE;

struct Args {
    const int8_t* x;
    const int8_t* w;
    const float* sx;
    const float* sw;
    float* out;
    int rank;
    int pad;
    char* ws[NRANK];
};

#define SB() __builtin_amdgcn_sched_barrier(0)
#define LGKM(n) __builtin_amdgcn_s_waitcnt(0xC07F | ((n) << 8))

#define DMA4(BITS) asm volatile( \
        "s_mov_b32 m0, %5\n\t" \
        "s_nop 0\n\t" \
        "global_load_lds_dwordx4 %0, %4 " BITS "\n\t" \
        "global_load_lds_dwordx4 %1, %4 offset:1024 " BITS "\n\t" \
        "global_load_lds_dwordx4 %2, %4 offset:2048 " BITS "\n\t" \
        "global_load_lds_dwordx4 %3, %4 offset:3072 " BITS \
        :: "v"(v0), "v"(v1), "v"(v2), "v"(v3), "s"(gbase), "s"(lds_addr) : "memory")
template <bool SYS = false>
__device__ __forceinline__ void dma4(const char* gbase, unsigned lds_addr,
                                     unsigned v0, unsigned v1, unsigned v2, unsigned v3) {
    if constexpr (SYS) DMA4("sc0 sc1");
    else DMA4("");
}
template <int N> __device__ __forceinline__ void wait_vm(i32x4& a) {
    asm volatile("s_waitcnt vmcnt(%1)" : "+v"(a) : "i"(N) : "memory");
}
template <int N> __device__ __forceinline__ void wait_vm(i32x4& a, i32x4& b) {
    asm volatile("s_waitcnt vmcnt(%2)" : "+v"(a), "+v"(b) : "i"(N) : "memory");
}
__device__ __forceinline__ int lds_ld(unsigned addr) {
    return __hip_atomic_load((lds_int_p)addr, __ATOMIC_RELAXED, __HIP_MEMORY_SCOPE_WORKGROUP);
}
__device__ __forceinline__ void lds_st(unsigned addr, int v) {
    __hip_atomic_store((lds_int_p)addr, v, __ATOMIC_RELAXED, __HIP_MEMORY_SCOPE_WORKGROUP);
}
__device__ __forceinline__ void lds_st1(unsigned addr, int v) {
    asm volatile("s_mov_b64 exec, 1\n\tds_write_b32 %0, %1\n\ts_mov_b64 exec, -1" :: "v"(addr), "v"(v) : "memory");
}
__device__ __forceinline__ void lds_st2(unsigned addr, int v0, int v1) {
    asm volatile("s_mov_b64 exec, 1\n\tds_write_b64 %0, %1\n\ts_mov_b64 exec, -1" :: "v"(addr), "v"(i32x2{v0, v1}) : "memory");
}
__device__ __forceinline__ void lds_wait(unsigned addr, int need) {
    while (__builtin_amdgcn_readfirstlane(
               __hip_atomic_load((lds_int_p)addr, __ATOMIC_ACQUIRE, __HIP_MEMORY_SCOPE_WORKGROUP)) < need)
        __builtin_amdgcn_s_sleep(1);
    asm volatile("" ::: "memory");
}

constexpr int imin(int x, int y) { return x < y ? x : y; }
constexpr int imax(int x, int y) { return x > y ? x : y; }
constexpr int local_of(int s) { return s >= LP0 && s < LP0 + 2 ? s - LP0 : s >= LP1 && s < LP1 + 2 ? 2 + s - LP1 : -1; }
constexpr int frag_lo(int s) {
    return s >= NS1 ? NSPLIT + s - NS1 : 2 * (s - imin(2, imax(0, s - LP0)) - imin(2, imax(0, s - LP1)));
}
__device__ __forceinline__ int pos_of(int rel, int mi) {
    if (!QROT) return rel - 1;
    const int v = (rel + 2 * mi) & 7;
    return v - (v > 2 * mi ? 1 : 0);
}
__device__ __forceinline__ int rel_of(int pos, int mi) {
    if (!QROT) return pos + 1;
    return (pos + (pos >= 2 * mi ? 1 : 0) - 2 * mi) & 7;
}
__device__ __forceinline__ unsigned frag_off(int rel, int cls, int rg) {
    const int i = 7 * cls + pos_of(rel, rg >> 3), m = rg & 1;
    const int P = i < NSPLIT ? i & 1 : m, frag = i < NSPLIT ? (i & ~1) + (m ^ P) : i;
    return (((rg >> 1) * 2 + P) * NFRAG + frag) * 1024;
}

struct Ctx {
    const char* ab[7];
    char* xo;
    const char* xi;
    const char* xl;
    const char* xl2;
    unsigned sv[4];
    unsigned ll;
    unsigned la;
    unsigned sent;
    unsigned tag;
    unsigned rb[2][8];
    unsigned l16;
    unsigned chk;
    const char* flags;
    unsigned foff;
    unsigned call, t0;
    unsigned bad;
    unsigned prog;
    int one;
    int seen;
};

__device__ __forceinline__ void read_local(i32x4 (&AL)[2][2], const Ctx& c) {
#pragma unroll
    for (int s = 0; s < 2; ++s) {
        AL[s][0] = *(lds_i32x4_p)(c.la ^ (s * 64));
        AL[s][1] = *(lds_i32x4_p)(c.la ^ (s * 64) ^ 2048);
    }
}
template <int S, int H>
__device__ __forceinline__ void read_b(i32x2 (&B)[16], int t, const Ctx& c) {
    B[2 * t + H] = __builtin_amdgcn_ds_read_tr8_b64_v2i32((lds_i32x2_p)(c.rb[(S >> 3) & 1][t] + (S & 7) * 8192 + H * 1024));
}
__device__ __forceinline__ void mma(i32x4& acc, const i32x4& a, const i32x2 (&B)[16], int t) {
    asm volatile("v_mfma_i32_16x16x64_i8 %0, %1, %2, %0"
                 : "+v"(acc) : "v"(__builtin_shufflevector(B[2 * t], B[2 * t + 1], 0, 1, 2, 3)), "v"(a));
}
template <int F>
__device__ __forceinline__ void load_frag(i32x4 (&A)[NFRAG], const Ctx& c) {
    asm volatile("global_load_dwordx4 %0, %1, %2 offset:%3 sc0 sc1"
                 : "=v"(A[F]) : "v"(c.l16), "s"(c.ab[F / 8]), "i"((F % 8) * 1024 - 4096) : "memory");
}
template <int F, int END>
__device__ __forceinline__ void first_frags(i32x4 (&A)[NFRAG], const Ctx& c, int cnt) {
    if constexpr (F < END) {
        asm volatile(
            "v_cmp_gt_i32 vcc, %[need], %[cnt]\n\t"
            "s_cbranch_vccz .Lgo%=\n"
            ".Lagain%=:\n\t"
            "ds_read_b32 %[cnt], %[at]\n\t"
            "s_waitcnt lgkmcnt(0)\n\t"
            "v_cmp_gt_i32 vcc, %[need], %[cnt]\n\t"
            "s_cbranch_vccnz .Lagain%=\n"
            ".Lgo%=:"
            : [cnt] "+v"(cnt) : [at] "v"(LDS_CTRL + C_CNT), [need] "i"(F / 2 + 1) : "memory", "vcc");
        load_frag<F>(A, c);
        load_frag<F + 1>(A, c);
        first_frags<F + 2, END>(A, c, cnt);
    }
}
template <int FULL>
__device__ __forceinline__ void check(const Ctx& c) {
    while (__any(__hip_atomic_load((lds_int_p)c.chk, __ATOMIC_ACQUIRE, __HIP_MEMORY_SCOPE_WORKGROUP) < FULL))
        __builtin_amdgcn_s_sleep(1);
    asm volatile("" ::: "memory");
}

#define CP_WAIT \
        "s_or_b64 vcc, %[low], %[miss]\n\t" \
        "s_cbranch_scc1 .Lslow%=\n" \
        ".Lback%=:\n\t" \
        ".subsection 1\n" \
        ".Lagain%=:\n\t" \
        "s_sleep 1\n" \
        ".Lslow%=:\n\t" \
        "ds_read_b32 %[p0], %[chk]\n\t" \
        "s_waitcnt lgkmcnt(0)\n\t" \
        "v_cmp_gt_i32 vcc, %[full], %[p0]\n\t" \
        "s_cbranch_vccnz .Lagain%=\n\t"
template <int FULL>
__device__ __forceinline__ void checkpoint(Ctx& c, unsigned long long low) {
    unsigned p0;
    asm volatile(CP_WAIT "s_branch .Lback%=\n\t.subsection 0"
                 : [p0] "=&v"(p0) : [low] "s"(low), [miss] "i"(0), [chk] "v"(c.chk), [full] "i"(FULL) : "memory", "vcc", "scc");
}
template <int FULL, int F>
__device__ __forceinline__ void checkpoint(i32x4 (&A)[NFRAG], Ctx& c, unsigned long long low, unsigned long long miss) {
    unsigned p0, p1, fl, prev;
    asm volatile(
        CP_WAIT
        "s_cmp_eq_u64 %[miss], 0\n\t"
        "s_cbranch_scc1 .Lback%=\n\t"
        "s_mov_b32 %[prev], 0\n"
        ".Lretry%=:\n\t"
        "global_load_dword %[p0], %[l16], %[ab] offset:%[o0] sc0 sc1\n\t"
        "global_load_dword %[p1], %[l16], %[ab] offset:%[o1] sc0 sc1\n\t"
        "global_load_dwordx4 %[a0], %[l16], %[ab] offset:%[o0] sc0 sc1\n\t"
        "global_load_dwordx4 %[a1], %[l16], %[ab] offset:%[o1] sc0 sc1\n\t"
        "global_load_dword %[fl], %[foff], %[flags] sc0 sc1\n\t"
        "s_memrealtime vcc\n\t"
        "s_waitcnt vmcnt(0) lgkmcnt(0)\n\t"
        "s_cmp_lg_u32 %[prev], 0\n\t"
        "s_cbranch_scc1 .Lback%=\n\t"
        "s_sub_u32 %[prev], vcc_lo, %[t0]\n\t"
        "v_cmp_eq_u32 vcc, %[sent], %[p0]\n\t"
        "v_cmp_eq_u32 %[miss], %[sent], %[p1]\n\t"
        "s_or_b64 vcc, vcc, %[miss]\n\t"
        "s_cbranch_scc0 .Lback%=\n\t"
        "s_cmp_gt_u32 %[prev], %[dl]\n\t"
        "s_cselect_b32 %[bad], 1, %[bad]\n\t"
        "s_cbranch_scc1 .Lback%=\n\t"
        "v_subrev_u32 %[fl], %[call], %[fl]\n\t"
        "v_cmp_gt_i32 vcc, 0, %[fl]\n\t"
        "s_cmp_eq_u64 vcc, 0\n\t"
        "s_cselect_b32 %[prev], 1, 0\n\t"
        "s_branch .Lretry%=\n\t"
        ".subsection 0"
        : [a0] "+v"(A[F]), [a1] "+v"(A[F + 1]), [p0] "=&v"(p0), [p1] "=&v"(p1), [fl] "=&v"(fl), [prev] "=&s"(prev),
          [bad] "+s"(c.bad), [miss] "+s"(miss)
        : [low] "s"(low), [chk] "v"(c.chk), [full] "i"(FULL), [l16] "v"(c.l16), [ab] "s"(c.ab[F / 8]), [foff] "v"(c.foff), [flags] "s"(c.flags),
          [o0] "i"((F % 8) * 1024 - 4096), [o1] "i"((F % 8) * 1024 - 3072), [sent] "s"(c.sent), [call] "s"(c.call),
          [t0] "s"(c.t0), [dl] "i"(DEADLINE_TICKS)
        : "memory", "vcc", "scc");
}

static_assert(ADIST % 2 == 0 && AINIT % 2 == 0 && AINIT <= ADIST && FIRST_AT >= 2 && FIRST_AT <= 3 && NS2 % 2 == 0, "");
constexpr int issued_before(int s) {
    return s < FIRST_AT ? 0 : imin(NFRAG, imin(frag_lo(s) + (s > NS1 ? ADIST2 : ADIST), AINIT + 4 * (s - FIRST_AT)));
}
constexpr int span(int s) {
    return s + 1 >= NSTEP ? 0 : s + 1 < NS1 ? 1 : (s + 1 - NS1) % 2 == 0 ? 2 : 0;
}
constexpr int dumps(int s) { return s >= NS1 && (s - NS1) * DPS < 8 ? DPS : 0; }
constexpr int ops_before_wait(int s) {
    int n = issued_before(s + 1 == FIRST_AT ? s : s + 1);
    for (int t = 0; t <= s; ++t) n += t < s ? dumps(t) : 0;
    return n;
}
constexpr int ops_through_frag(int f) {
    int s = FIRST_AT - 1;
    while (issued_before(s + 1) <= f) ++s;
    int n = f + 1;
    for (int t = 0; t < s; ++t) n += dumps(t);
    return n;
}

__device__ __forceinline__ bool untagged(const i32x4& x0, const i32x4& x1, unsigned tag) {
    return __any((((unsigned)x0[0] ^ tag) | ((unsigned)x1[0] ^ tag)) & 15);
}
constexpr unsigned x_lds(int p) { return p == 3 ? LDS_X3 : LDS_PUSH + p * 8192; }
__device__ __forceinline__ void bring_x(const char* xp, int p, int lane) {
    const unsigned v0 = lane * 16, v2 = lane * 16 + XSTRIDE - 2048;
    dma4<true>(xp + p * 2048, x_lds(p), v0, v0, v2, v2);
    dma4<true>(xp + p * 2048 + 2 * XSTRIDE, x_lds(p) + 4096, v0, v0, v2, v2);
}
__device__ __forceinline__ unsigned long long or64(unsigned long long x, unsigned long long y) {
    unsigned long long r;
    asm volatile("s_or_b64 %0, %1, %2" : "=s"(r) : "s"(x), "s"(y) : "scc");
    return r;
}

template <int S>
__device__ __forceinline__ void step(Ctx& c, i32x4 (&acc)[2][8], i32x4 (&AL)[2][2][2], i32x4 (&A)[NFRAG],
                                     i32x2 (&B)[2][16] STAMP_ARG) {
    constexpr bool PH1 = S < NS1, LOC = S < 4, NEXT = S + 1 < NSTEP;
    constexpr int SPAN = span(S), FULL = imin(S + SPAN + 2, NSTEP);
    constexpr bool CP = SPAN > 0, VAL = CP && S + 1 >= 4;
    constexpr bool FREE = (S & 1) == 1 && S + 13 < NSTEP;
    constexpr int F = frag_lo(S), FN = frag_lo(S + 1);
    constexpr int L0 = issued_before(S), L1 = S + 1 == FIRST_AT ? L0 : issued_before(S + 1);
    unsigned long long m0 = 0, m1 = 0, m2 = 0;
    if constexpr ((S & 3) == 0) STAMP(4 + (S >> 2));
    if constexpr (S == 2) STAMP(15);
    if constexpr (S == CS0) { STAMP(28); CSTAMP(30); }
    if constexpr (S == CS1) { STAMP(29); CSTAMP(31); }
    if constexpr (S == FIRST_AT) first_frags<0, AINIT>(A, c, 0);
    const i32x4& a0 = LOC ? AL[LOC ? S / 2 : 0][S & 1][0] : A[F];
    const i32x4& a1 = LOC ? AL[LOC ? S / 2 : 0][S & 1][1] : A[PH1 ? F + 1 : F];
    LGKM(S == 2 ? 6 : 8);
    SB();
#pragma unroll
    for (int g = 0; g < 16; ++g) {
        const int t = g >> 1;
        if (g == 8) { if constexpr (NEXT) LGKM(8 + FREE); else LGKM(0); SB(); }
        if ((g & 1) == 0) { mma(acc[0][t], a0, B[S & 1], t); SB(); }
        else if constexpr (PH1) { mma(acc[1][t], a1, B[S & 1], t); SB(); }
        if (CP && g == 0) { c.seen = lds_ld(c.chk); SB(); }
        if constexpr (NEXT) {
            if (g & 1) read_b<S + 1, 1>(B[(S + 1) & 1], t, c);
            else if (g != 14) read_b<S + 1, 0>(B[(S + 1) & 1], t, c);
            if (g == 13) read_b<S + 1, 0>(B[(S + 1) & 1], 7, c);
        }
        if (g == 1) { if constexpr (L0 < L1) load_frag<L0>(A, c); }
        if (FREE && g == 2) __hip_atomic_fetch_add((lds_int_p)c.prog, c.one, __ATOMIC_RELAXED, __HIP_MEMORY_SCOPE_WORKGROUP);
        if (g == 3) { if constexpr (L0 + 2 < L1) load_frag<L0 + 1>(A, c); }
        if (g == 4) { if constexpr (L0 + 3 < L1) load_frag<L0 + 2>(A, c); }
        if (g == 5) { if constexpr (L0 + 1 < L1) load_frag<L1 - 1>(A, c); }
        if (CP && g == 8) m2 = __builtin_amdgcn_sicmp(c.seen, FULL, 40);
        if constexpr (VAL) {
            if (g == 9) wait_vm<ops_before_wait(S) - ops_through_frag(FN + 1)>(A[FN], A[FN + 1]);
            if (g == 10) m0 = __builtin_amdgcn_uicmp(A[FN][0], c.sent, 32);
            if (g == 11) m1 = __builtin_amdgcn_uicmp(A[FN + 1][0], c.sent, 32);
            if (g == 12) m0 = or64(m0, m1);
            if (g == 14) checkpoint<FULL, FN>(A, c, m2, m0);
        } else if constexpr (CP) {
            if (g == 14) checkpoint<FULL>(c, m2);
        }
        if (S == 1 && g == 9) {
            asm volatile("s_waitcnt vmcnt(0)" ::: "memory");
            read_local(AL[1], c);
        }
        if constexpr (dumps(S) > 0) {
            if (g == 15 || (DPS == 2 && g == 11)) {
                const int q = (S - NS1) * DPS + (g == 15 ? DPS - 1 : 0);
                acc[1][q][0] = (acc[1][q][0] << 4) | c.tag;
                asm volatile("global_store_dwordx4 %1, %0, %2 offset:%3 " XST_BITS "\n\ts_nop 1"
                             :: "v"(acc[1][q]), "v"(c.l16), "s"(c.xo), "i"(q * 1024 - 4096) : "memory");
            }
        }
        SB();
    }
    if constexpr (NEXT) step<S + 1>(c, acc, AL, A, B STAMP_PASS);
}

template <int S>
__device__ __forceinline__ void allowed(int& due, int n) {
    if constexpr (span(S) > 0) {
        constexpr int F = issued_before(S + span(S) + 1), NEED = F <= NSPLIT ? (F + 1) / 2 : F - NSPLIT / 2;
        if (n < NEED) due = imin(S + span(S) + 2, NSTEP) - 1;
    }
    if constexpr (S > FIRST_AT) allowed<S - 1>(due, n);
}
__device__ __forceinline__ void warm(const char* ws, int b, int lane) {
    asm volatile("s_mov_b32 m0, %2\n\ts_nop 0\n\tglobal_load_lds_dword %0, %1 sc1"
                 :: "v"(WS_DATA + (b >> 1) * (NFRAG * 1024) + (b & 1) * 8192 + lane * 128), "s"(ws), "i"(LDS_CTRL + 2048) : "memory");
}

#define LORD 0
constexpr int LSEQ[3][2][3] = {{{0, 2, 4}, {1, 3, 5}}, {{0, 3, 1}, {2, 4, 5}}, {{0, 1, 4}, {2, 3, 5}}};
constexpr int fill_step(int h, int k) { return k < 3 ? LSEQ[LORD][h][k] : imin(NSTEP, 2 * k + h); }
constexpr int owed(int h, int k) {
    int m = fill_step(h, k);
    for (int i = k + 1; i < 3; ++i) m = imin(m, fill_step(h, i));
    return m;
}
__device__ __forceinline__ unsigned w_off(int li, int fl, int s, int P, int mi, int rank) {
    const int i = s < NS1 ? fl + P : fl, cls = (i * 37) >> 8;
    const int rem = (((rel_of(i - 7 * cls, mi) + rank) & (NRANK - 1)) * 8 + ((cls + ROT * mi) & 7));
    return (li >= 0 ? rank * 8 + 4 * P + (li ^ mi) : rem) * 64 * NN;
}
struct Ldr {
    const char* w;
    unsigned dv[4];
    unsigned woff;
    unsigned fa, pa;
    int freed;
};
__device__ __forceinline__ void fill(const Ldr& l, unsigned off, unsigned slot) {
    const char* p = l.w + (size_t)off;
    dma4(p, slot, l.dv[0], l.dv[1], l.dv[2], l.dv[3]);
    dma4(p + 32 * NN, slot + 4096, l.dv[0], l.dv[1], l.dv[2], l.dv[3]);
}
__device__ __forceinline__ int released(unsigned pa) {
    int v = lds_ld(pa);
    v = min(v, __builtin_amdgcn_update_dpp(v, v, 0xB1, 0xF, 0xF, false));
    v = min(v, __builtin_amdgcn_update_dpp(v, v, 0x4E, 0xF, 0xF, false));
    return 2 * __builtin_amdgcn_readfirstlane(v);
}
template <int H, int K>
__device__ __forceinline__ void fills(Ldr& l STAMP_ARG) {
    constexpr int S = fill_step(H, K);
    if constexpr (S < NSTEP) {
        if constexpr (S >= 16) {
            if (__builtin_expect(l.freed <= S - 16, 0)) {
                while ((l.freed = released(l.pa)) <= S - 16) __builtin_amdgcn_s_sleep(LSLEEP);
            }
        }
        fill(l, __builtin_amdgcn_readlane(l.woff, S), (S & 15) * 8192);
        constexpr int W = imin(LDW, imax(1, K - LDW_FROM + 2));
        asm volatile("s_waitcnt vmcnt(%2)\n\tds_max_i32 %0, %1" :: "v"(l.fa), "v"(owed(H, K - W + 1)), "i"(8 * W) : "memory");
        if constexpr (K < 6 || (K & 1) == 1) STAMP(K < 6 ? K : 3 + K / 2);
        fills<H, K + 1>(l STAMP_PASS);
    } else {
#pragma unroll
        for (int k = LDW - 1; k >= 0; --k)
            asm volatile("s_waitcnt vmcnt(%2)\n\tds_max_i32 %0, %1" :: "v"(l.fa), "v"(k ? fill_step(H, K - k) : NSTEP), "i"(8 * k) : "memory");
    }
}

__device__ __forceinline__ void take_ticket(const Args& a, unsigned call, int lane) {
    unsigned tk = 0;
    if (lane == 0) tk = __hip_atomic_fetch_add((unsigned*)(a.ws[0] + WS_TICKET), 1u, __ATOMIC_RELAXED, __HIP_MEMORY_SCOPE_AGENT);
    if ((__builtin_amdgcn_readfirstlane(tk) & 255) == 255) {
        if (lane < NRANK)
            __hip_atomic_store((unsigned*)(a.ws[lane] + WS_FLAGS + ((NRANK - lane) & (NRANK - 1)) * 128), call,
                               __ATOMIC_RELAXED, __HIP_MEMORY_SCOPE_SYSTEM);
    }
}

}

__global__ __launch_bounds__(NTHREADS, 2) void a2a_gemm_kernel(Args a) {
    extern __shared__ __attribute__((aligned(16))) char smem[];
    const unsigned tstart = (unsigned)__builtin_amdgcn_s_memrealtime();
    asm volatile("" :: "s"(a.x), "s"(a.w), "s"(a.sx), "s"(a.sw), "s"(a.out), "s"(a.rank), "s"(a.ws[0]), "s"(a.ws[1]),
                       "s"(a.ws[2]), "s"(a.ws[3]), "s"(a.ws[4]), "s"(a.ws[5]), "s"(a.ws[6]), "s"(a.ws[7]));
    const int lane = threadIdx.x & 63;
    int wave = __builtin_amdgcn_readfirstlane(threadIdx.x >> 6);
    const int b = blockIdx.x;
    const int xcd = b & 7, j = b >> 3;
    const int rank = a.rank & (NRANK - 1);
    char* const ws = a.ws[0];
    const unsigned lds0 = (unsigned)(uintptr_t)(__attribute__((address_space(3))) char*)smem;
    const unsigned ctrl = lds0 + LDS_CTRL;
    const int P = j & 1;
    const int mi = (j >> 1) & 3;
    const int n0 = ((j >> 3) * 8 + xcd) * 128;
    const int xs = (xcd + (int)((uintptr_t)a.w >> 7) + 4) & 7;
    const int pd = xs == 7 ? 0 : (xs + j) % 7 + 1, pc = j >> 2, pm = (j - (QROT ? (NRANK - pd) >> 1 : 0)) & 3;
    STAMP_DECL;
    STAMP(0);

    unsigned v4[4];
    const char* base;
    if (wave == 7) {
        asm volatile("s_barrier" ::: "memory");
        v4[0] = v4[1] = v4[2] = v4[3] = 0;
        base = nullptr;
    } else if (wave == 6) {
        __builtin_amdgcn_s_setprio(3);
        base = (const char*)a.x + (size_t)(unsigned)((((rank + pd) & (NRANK - 1)) * MLOC + pm * 128) * KLOC + ((pc + ROT * pm) & 7) * 64);
        const unsigned sv = (lane >> 2) * KLOC + (((lane & 3) ^ ((lane >> 4) & 3)) << 4);
        constexpr unsigned G = 16 * KLOC - 1024;
        if (pd) {
            dma4(base, lds0 + LDS_PUSH, sv, sv + G, sv + 2 * G, sv + 3 * G);
            dma4(base + 64 * KLOC, lds0 + LDS_PUSH + 4096, sv, sv + G, sv + 2 * G, sv + 3 * G);
        }
        asm volatile("s_mov_b32 m0, %2\n\ts_mov_b64 exec, 1\n\tglobal_load_lds_dword %0, %1 sc1\n\ts_mov_b64 exec, -1"
                     :: "v"(WS_TICKET), "s"(ws), "i"(LDS_CTRL + C_TICKET) : "memory");
        if (SPF == 1 && b < 192) warm(ws, b, lane);
        lds_st(ctrl + lane * 4, lane == 1 ? owed(1, 0) : lane == C_DUE / 4 ? FIRST_AT + 2 : 0);
        *(lds_i32x4_p)(ctrl + C_PROG + lane * 16) = i32x4{0, 0, 0, 0};
        v4[0] = v4[1] = v4[2] = v4[3] = 0;
        asm volatile("s_waitcnt lgkmcnt(0)\n\ts_barrier" ::: "memory");
    } else if (wave >= 4) {
        __builtin_amdgcn_s_setprio(LPRIO);
        base = (const char*)a.w + n0;
#pragma unroll
        for (int i = 0; i < 4; ++i)
            v4[i] = (lane >> 3) * NN + (((lane & 7) ^ (((lane >> 4) & 3) | ((i >> 1) << 2))) << 4) + i * (8 * NN - 1024);
        constexpr int S0 = fill_step(0, 0), S1 = fill_step(1, 0);
        const bool h = wave & 1;
        const char* p = base + (size_t)(h ? w_off(local_of(S1), frag_lo(S1), S1, P, mi, rank) : w_off(local_of(S0), frag_lo(S0), S0, P, mi, rank));
        const unsigned slot = lds0 + (h ? S1 : S0) * 8192;
        asm volatile("s_barrier" :: "s"(p), "s"(slot), "v"(v4[0]), "v"(v4[1]), "v"(v4[2]), "v"(v4[3]) : "memory");
        dma4(p, slot, v4[0], v4[1], v4[2], v4[3]);
        dma4(p + 32 * NN, slot + 4096, v4[0], v4[1], v4[2], v4[3]);
        if (SPF == 2 && h && b < 192) warm(ws, b, lane);
    } else {
        __builtin_amdgcn_s_setprio(CPRIO);
        base = (const char*)a.x + (size_t)(unsigned)((rank * MLOC + mi * 128 + wave * 32) * KLOC + P * 256 + (mi >> 1) * 128);
#pragma unroll
        for (int i = 0; i < 4; ++i)
            v4[i] = (i * 8 + (lane >> 3)) * KLOC + (((lane & 7) ^ ((i * 4 + (lane >> 4)) & 7)) << 4) - i * 1024;
        asm volatile("s_barrier" :: "s"(base), "v"(v4[0]), "v"(v4[1]), "v"(v4[2]), "v"(v4[3]) : "memory");
        if (LP0 == 0) dma4(base, lds0 + LDS_LOC + wave * 4096, v4[0], v4[1], v4[2], v4[3]);
    }
    asm volatile("" : "+s"(wave) :: "memory");
    STAMP(19);

    if (wave == 7) {
        unsigned d[8];
        unsigned long long now;
        const char* const xi = ws + WS_X + (b ^ 8) * 4 * XSTRIDE + 2032 - XLEAD * 1024;
        const char *xp = xi, *xq = xi + 6144;
        {
            unsigned w;
#define TOUCH4(O) "s_load_dword %0, %1, " #O "\n\ts_load_dword %0, %1, %2+" #O "\n\ts_load_dword %0, %1, %3+" #O "\n\ts_load_dword %0, %1, %4+" #O "\n\t"
            asm volatile(TOUCH4(0) TOUCH4(2048) TOUCH4(4096) TOUCH4(6144) "s_waitcnt lgkmcnt(0)"
                         : "=&s"(w) : "s"(xi), "i"(XSTRIDE), "i"(2 * XSTRIDE), "i"(3 * XSTRIDE) : "memory");
        }
#define LOOK_X(BITS) asm volatile( \
            "s_load_dword %0, %9, 0 " BITS "\n\ts_load_dword %1, %9, %11 " BITS "\n\ts_load_dword %2, %9, %12 " BITS "\n\t" \
            "s_load_dword %3, %9, %13 " BITS "\n\ts_load_dword %4, %10, 0 " BITS "\n\ts_load_dword %5, %10, %11 " BITS "\n\t" \
            "s_load_dword %6, %10, %12 " BITS "\n\ts_load_dword %7, %10, %13 " BITS "\n\ts_memrealtime %8\n\ts_waitcnt lgkmcnt(0)" \
            : "=&s"(d[0]), "=&s"(d[1]), "=&s"(d[2]), "=&s"(d[3]), "=&s"(d[4]), "=&s"(d[5]), "=&s"(d[6]), "=&s"(d[7]), "=&s"(now) \
            : "s"(xp), "s"(xq), "i"(XSTRIDE), "i"(2 * XSTRIDE), "i"(3 * XSTRIDE) : "memory")
        int due = 1000;
        allowed<NSTEP - 1>(due, lane);
        constexpr int NI = NSPLIT / 2 + NS2;
        const int q = min(lane, NI - 1);
        const int offs = (q < NSPLIT / 2 ? 2 * q + (j >> 4) : q + NSPLIT / 2) * 1024;
        unsigned call;
        while ((call = __builtin_amdgcn_readfirstlane(lds_ld(ctrl + C_CALL))) == 0) __builtin_amdgcn_s_sleep(1);
        const unsigned sent = call <= 2 ? 0u : POISON;
        const char* af = ws + WS_DATA + (call % 3) * SLOT_BYTES + ((mi * 4 + 2 + ((j >> 3) & 1)) * 2 + P) * (NFRAG * 1024) + xcd * 128 + 112;
        int n = 0;
        STAMP(1);
        while (n < NSPLIT / 2) {
            unsigned o[4];
#pragma unroll
            for (int i = 0; i < 4; ++i) o[i] = __builtin_amdgcn_readlane(offs, n + i);
            do {
                asm volatile("s_load_dword %0, %5, %6 glc\n\ts_load_dword %1, %5, %7 glc\n\t"
                             "s_load_dword %2, %5, %8 glc\n\ts_load_dword %3, %5, %9 glc\n\t"
                             "s_memrealtime %4\n\ts_waitcnt lgkmcnt(0)"
                             : "=&s"(d[0]), "=&s"(d[1]), "=&s"(d[2]), "=&s"(d[3]), "=&s"(now)
                             : "s"(af), "s"(o[0]), "s"(o[1]), "s"(o[2]), "s"(o[3]) : "memory");
            } while (d[0] == sent && (unsigned)now - tstart < WATCH_TICKS);
            n += d[1] == sent ? 1 : d[2] == sent ? 2 : d[3] == sent ? 3 : 4;
            lds_st2(ctrl + C_DUE, __builtin_amdgcn_readlane(due, n + (n >= 4 ? LEAD : 0)), n);
        }
        while (n + 2 * LEAD < NI) {
            const char* p = af + (n + NSPLIT / 2) * 1024;
            do {
                asm volatile("s_load_dword %0, %9, 0 glc\n\ts_load_dword %1, %9, 0x400 glc\n\ts_load_dword %2, %9, 0x800 glc\n\t"
                             "s_load_dword %3, %9, 0xc00 glc\n\ts_load_dword %4, %9, 0x1000 glc\n\ts_load_dword %5, %9, 0x1400 glc\n\t"
                             "s_load_dword %6, %9, 0x1800 glc\n\ts_load_dword %7, %9, 0x1c00 glc\n\t"
                             "s_memrealtime %8\n\ts_waitcnt lgkmcnt(0)"
                             : "=&s"(d[0]), "=&s"(d[1]), "=&s"(d[2]), "=&s"(d[3]), "=&s"(d[4]), "=&s"(d[5]), "=&s"(d[6]), "=&s"(d[7]), "=&s"(now)
                             : "s"(p) : "memory");
            } while (d[0] == sent && (unsigned)now - tstart < WATCH_TICKS);
            n += d[1] == sent ? 1 : d[2] == sent ? 2 : d[3] == sent ? 3 : d[4] == sent ? 4 : d[5] == sent ? 5 : d[6] == sent ? 6 : d[7] == sent ? 7 : 8;
            lds_st1(ctrl + C_DUE, __builtin_amdgcn_readlane(due, n + 2 * LEAD));
        }
        lds_st2(ctrl + C_DUE, 1000, 1000);
        STAMP(7);
        const unsigned tag = call & 15, pa = ctrl + C_PROG + (lane & 3) * 260;
        bool late = false;
        while (released(pa) < imin(NS1, NSTEP - 14) && !(late = (unsigned)__builtin_amdgcn_s_memrealtime() - tstart > WATCH_TICKS))
            __builtin_amdgcn_s_sleep(6);
        STAMP(8);
        int p = 0;
        while (p < 4 && !late) {
            LOOK_X("glc");
            late = (unsigned)now - tstart > WATCH_TICKS;
            if (((d[0] ^ tag) | (d[1] ^ tag) | (d[2] ^ tag) | (d[3] ^ tag)) & 15) continue;
            p = ((d[4] ^ tag) | (d[5] ^ tag) | (d[6] ^ tag) | (d[7] ^ tag)) & 15 ? p + 1 : 4;
            xp += 2048;
            lds_st1(ctrl + C_XSEEN, p);
        }
        if (p < 4) lds_st1(ctrl + C_XSEEN, 5);
        STAMP(9);
        while (!late && released(pa) <= NSTEP - 16) late = (unsigned)__builtin_amdgcn_s_memrealtime() - tstart > WATCH_TICKS;
        if (!late) bring_x(xi - 2032 + XLEAD * 1024, 3, lane);
        asm volatile("s_waitcnt vmcnt(0)" ::: "memory");
        lds_st1(ctrl + C_XRDY + 12, late ? 2 : 1);
        STAMP(10);
        STAMP_OUT(ws);
        return;
    }
    if (wave == 6) {
        const unsigned due = tstart + PUSH0 + ((j < 4 * NFAST ? j * PACE0 : j * PACE - 4 * NFAST * (PACE - PACE0)) >> 2);
        char* const pw = a.ws[pd];
        unsigned doff[2], poff[2];
#pragma unroll
        for (int m = 0; m < 2; ++m) {
            doff[m] = lane * 16 + WS_DATA + frag_off(NRANK - pd, pc, pm * 8 + m);
            poff[m] = lane * 16 + WS_DATA + frag_off(pd, pc, pm * 8 + m);
            asm volatile("" : "+v"(doff[m]), "+v"(poff[m]));
        }
        const unsigned roff = lds0 + LDS_PUSH + (lane & 15) * 64 + (((lane >> 4) ^ ((lane >> 2) & 3)) << 4);
        asm volatile("s_waitcnt vmcnt(0)" :: "s"(pw), "s"(due), "v"(roff) : "memory");
        STAMP(11);
        const unsigned cv = lds_ld(ctrl + C_TICKET);
        i32x4 dat[8];
#pragma unroll
        for (int g = 0; g < 8; ++g) dat[g] = *(lds_i32x4_p)(roff + g * 1024);
        const unsigned call = ((unsigned)__builtin_amdgcn_readfirstlane(cv) >> 8) + 1;
        lds_st1(ctrl + C_CALL, (int)call);
        const unsigned cur = (call % 3) * SLOT_BYTES, old = ((call + 2) % 3) * SLOT_BYTES;
        if (pd) {
            char* dp[4];
#pragma unroll
            for (int w = 0; w < 4; ++w) { dp[w] = pw + cur + w * (2 * NFRAG * 1024); asm volatile("" : "+s"(dp[w])); }
            __builtin_amdgcn_s_setprio(0);
            for (int left; (left = (int)(due - (unsigned)__builtin_amdgcn_s_memrealtime())) > 0;) {
                if (left > 40) __builtin_amdgcn_s_sleep(8);
                else __builtin_amdgcn_s_sleep(1);
            }
            __builtin_amdgcn_s_setprio(3);
            STAMP(2);
#pragma unroll
            for (int g = 0; g < 8; ++g)
                asm volatile("global_store_dwordx4 %1, %0, %2 sc0 sc1\n\ts_nop 1" :: "v"(dat[g]), "v"(doff[g & 1]), "s"(dp[g >> 1]) : "memory");
            __builtin_amdgcn_s_setprio(0);
            STAMP(3);
            asm volatile("s_waitcnt vmcnt(0)" ::: "memory");
            STAMP(7);
            const i32x4 poison = {(int)POISON, (int)POISON, (int)POISON, (int)POISON};
#pragma unroll
            for (int g = 0; g < 8; ++g)
                asm volatile("global_store_dwordx4 %1, %0, %2 " POISON_BITS "\n\ts_nop 1"
                             :: "v"(poison), "v"(poff[g & 1]), "s"(ws + old + (g >> 1) * (2 * NFRAG * 1024)) : "memory");
        }
        take_ticket(a, call, lane);
        __builtin_amdgcn_s_setprio(0);
        for (int p = 0; p < 3;) {
            int seen;
            while ((seen = __builtin_amdgcn_readfirstlane(lds_ld(ctrl + C_XSEEN))) <= p) __builtin_amdgcn_s_sleep(3);
            const int p0 = p;
            if (seen > 4) p = 3;
            else for (; p < min(seen, 3); ++p) bring_x(ws + WS_X + (b ^ 8) * 4 * XSTRIDE, p, lane);
            asm volatile("s_waitcnt vmcnt(0)" ::: "memory");
            if (lane >= p0 && lane < p) lds_st(ctrl + C_XRDY + lane * 4, seen > 4 ? 2 : 1);
        }
        STAMP(4);
        STAMP(5);
        STAMP_OUT(ws);
        return;
    }

    if (wave >= 4) {
        __builtin_amdgcn_s_waitcnt(0xCF7F);
        Ldr l;
        l.w = base;
#pragma unroll
        for (int i = 0; i < 4; ++i) l.dv[i] = v4[i];
        {
            const int s = min(lane, NSTEP - 1);
            const int li = (unsigned)(s - LP0) < 2 ? s - LP0 : (unsigned)(s - LP1) < 2 ? 2 + s - LP1 : -1;
            const int fl = s >= NS1 ? NSPLIT + s - NS1 : 2 * (s - (s > LP0 ? 2 : 0) - (s > LP1 ? 2 : 0));
            l.woff = w_off(li, max(fl, 0), s, P, mi, rank);
        }
        l.fa = ctrl + C_FULL + (lane ? 64 + lane * 8 : 0) + (wave & 1) * 4;
        l.pa = ctrl + C_PROG + (lane & 3) * 260;
        l.freed = 0;
        if (wave & 1) fills<1, 1>(l STAMP_PASS);
        else fills<0, 1>(l STAMP_PASS);
        STAMP(20);
        STAMP_OUT(ws);
        return;
    }

    Ctx c;
    c.xl = base;
    c.xl2 = base + (128 - (mi >> 1) * 256);
#pragma unroll
    for (int i = 0; i < 4; ++i) c.sv[i] = v4[i];
    c.ll = lds0 + LDS_LOC + wave * 4096;
    c.la = c.ll + P * 2048 + (lane & 15) * 128 + ((((mi & 1) * 4 + (lane >> 4)) ^ ((lane & 15) >> 1)) << 4);
    c.l16 = lane * 16;
    c.chk = ctrl + C_FULL + min(lane & 3, 2) * 4;
    c.foff = (lane & 7) * 128;
    c.flags = ws + WS_FLAGS;
    c.t0 = tstart;
    c.bad = 0;
    typedef const __attribute__((address_space(4))) float* cfp;
    const float scale = *(cfp)(uintptr_t)a.sx * *(cfp)(uintptr_t)a.sw;
    c.prog = ctrl + C_PROG + wave * 256 + lane * 4;
    c.one = 1;
    asm volatile("" : "+v"(c.one), "+v"(c.chk), "+v"(c.prog), "+v"(c.foff));
    {
        const unsigned q = (lane & 15) >> 1, kg = lane >> 4;
        const unsigned r0 = lds0 + (kg * 16 + q) * 128 + (lane & 1) * 8, f = (q >> 1) | ((kg & 1) << 2);
#pragma unroll
        for (int t = 0; t < 8; ++t) {
            c.rb[0][t] = r0 + ((t ^ f) << 4);
            c.rb[1][t] = c.rb[0][t] + 65536;
            asm volatile("" : "+v"(c.rb[0][t]), "+v"(c.rb[1][t]));
        }
    }
    i32x4 acc[2][8];
#pragma unroll
    for (int m = 0; m < 2; ++m)
#pragma unroll
        for (int t = 0; t < 8; ++t) { acc[m][t] = i32x4{0, 0, 0, 0}; asm volatile("" : "+v"(acc[m][t])); }
    i32x2 B[2][16];
    i32x4 AL[2][2][2], A[NFRAG], X[8];
    if constexpr (LP0 == 0) {
        asm volatile("s_waitcnt vmcnt(0)" ::: "memory");
        read_local(AL[0], c);
        {
            LGKM(0);
            dma4(c.xl2, c.ll, c.sv[0], c.sv[1], c.sv[2], c.sv[3]);
        }
    }
    STAMP(1);
    unsigned call;
    while ((call = __builtin_amdgcn_readfirstlane(lds_ld(ctrl + C_CALL))) == 0) __builtin_amdgcn_s_sleep(1);
    c.call = call;
    c.sent = call <= 2 ? 0u : POISON;
    c.tag = call & 15;
#pragma unroll
    for (int i = 0; i < 7; ++i) {
        c.ab[i] = ws + WS_DATA + (call % 3) * SLOT_BYTES + ((mi * 4 + wave) * 2 + P) * (NFRAG * 1024) + i * 8192 + 4096;
        asm volatile("" : "+s"(c.ab[i]));
    }
    c.xo = ws + WS_X + (b * 4 + wave) * XSTRIDE + 4096;
    c.xi = ws + WS_X + ((b ^ 8) * 4 + wave) * XSTRIDE + 4096;
    asm volatile("" : "+s"(c.xo), "+s"(c.xi), "+s"(c.sent), "+s"(c.tag), "+s"(c.flags), "+s"(c.bad), "+v"(c.l16));
    check<1>(c);
    STAMP(3);
#pragma unroll
    for (int t = 0; t < 8; ++t) { read_b<0, 0>(B[0], t, c); read_b<0, 1>(B[0], t, c); }
    check<2>(c);
    step<0>(c, acc, AL, A, B STAMP_PASS);
    asm volatile("s_nop 7\n\ts_nop 1" ::: "memory");
    SB();
    STAMP(20);

    const unsigned xa = lds0 + wave * 2048 + c.l16;
    const i32x4 xr = *(volatile lds_i32x4_p)(ctrl + C_XRDY);
    asm volatile("" ::: "memory");
#pragma unroll
    for (int t = 0; t < 8; ++t) X[t] = *(lds_i32x4_p)(xa + x_lds(t >> 1) + (t & 1) * 1024);
    char* const o = (char*)a.out + (size_t)(unsigned)(((mi * 128 + wave * 32 + P * 16) * NN + n0) * 4);
    const unsigned oo = (lane & 15) * NN * 4 + (lane >> 4) * 16;
#pragma unroll
    for (int p = 0; p < 4; ++p) {
        i32x4 &x0 = X[2 * p], &x1 = X[2 * p + 1];
        int f = __builtin_amdgcn_readfirstlane(xr[p]);
        while (f == 0) {
            const int fv = lds_ld(ctrl + C_XRDY + p * 4);
            asm volatile("" ::: "memory");
            x0 = *(lds_i32x4_p)(xa + x_lds(p));
            x1 = *(lds_i32x4_p)(xa + x_lds(p) + 1024);
            f = __builtin_amdgcn_readfirstlane(fv);
        }
        if (__builtin_expect(untagged(x0, x1, c.tag) | (f != 1), 0)) {
            bool late;
            do {
                asm volatile("global_load_dwordx4 %0, %2, %3 offset:%4 sc0 sc1\n\tglobal_load_dwordx4 %1, %2, %3 offset:%5 sc0 sc1\n\t"
                             "s_waitcnt vmcnt(0)"
                             : "=&v"(x0), "=&v"(x1) : "v"(c.l16), "s"(c.xi), "i"(p * 2048 - 4096), "i"(p * 2048 - 3072) : "memory");
                late = (unsigned)__builtin_amdgcn_s_memrealtime() - c.t0 > DEADLINE_TICKS + DEADLINE_TICKS / 2;
            } while (untagged(x0, x1, c.tag) && !late);
            if (late) c.bad = 1;
        }
        const float fs = c.bad ? __builtin_nanf("") : scale;
        if (p == 0) STAMP(21);
        if (p == 3) STAMP(22);
#pragma unroll
        for (int t = 2 * p; t < 2 * p + 2; ++t) {
            X[t][0] >>= 4;
            const i32x4 s = acc[0][t] + X[t];
            const f32x4 y = {(float)s[0] * fs, (float)s[1] * fs, (float)s[2] * fs, (float)s[3] * fs};
            asm volatile("global_store_dwordx4 %1, %0, %2 offset:%3 " OUT_BITS "\n\ts_nop 1"
                         :: "v"(y), "v"(oo), "s"(o), "i"(t * 64) : "memory");
        }
    }
    if (c.bad && lane == 0) __hip_atomic_store((unsigned*)(ws + WS_TMO), call, __ATOMIC_RELAXED, __HIP_MEMORY_SCOPE_AGENT);
    STAMP(23);
    STAMP(24);
    STAMP_OUT(ws);
}

extern "C" void kernel_launch(int rank, int nranks, ncclComm_t comm,
                              hipStream_t stream,
                              void* const* d_in, const int* in_sizes, int n_in,
                              void* d_out, int out_size,
                              const int* in_dtypes, int out_dtype,
                              void* const* d_out_all,
                              void* const* d_ws_all, size_t ws_bytes) {
    if (nranks != NRANK || (unsigned)rank >= (unsigned)NRANK || ws_bytes < WS_NEEDED || n_in < 4 ||
        out_size != MLOC * NN || in_sizes[0] != NRANK * MLOC * KLOC || in_sizes[1] != NRANK * KLOC * NN) {
        fprintf(stderr, "unexpected problem shape\n");
        return;
    }
    static bool attr_set = false;
    if (!attr_set) {
        HIP_CHECK(hipFuncSetAttribute((const void*)a2a_gemm_kernel, hipFuncAttributeMaxDynamicSharedMemorySize,
                                      (int)LDS_BYTES));
        attr_set = true;
    }
    Args a;
    a.x = (const int8_t*)d_in[0];
    a.w = (const int8_t*)d_in[1];
    a.sx = (const float*)d_in[2];
    a.sw = (const float*)d_in[3];
    a.out = (float*)d_out;
    a.rank = rank;
    a.pad = 0;
    for (int p = 0; p < NRANK; ++p) a.ws[p] = (char*)d_ws_all[(rank + p) & (NRANK - 1)];
    hipLaunchKernelGGL(a2a_gemm_kernel, dim3(NWG), dim3(NTHREADS), LDS_BYTES, stream, a);
    HIP_CHECK(hipGetLastError());
}
```

```cpp
#include <hip/hip_runtime.h>
#include <rccl/rccl.h>
#include <cstdint>
#include <cstdio>

#define HIP_CHECK(expr) do { hipError_t _e = (expr); if (_e != hipSuccess) { \
    fprintf(stderr, "HIP error %s at %s:%d\n", hipGetErrorString(_e), __FILE__, __LINE__); \
    return; } } while (0)

typedef int i32x2 __attribute__((ext_vector_type(2)));
typedef int i32x4 __attribute__((ext_vector_type(4)));
typedef float f32x4 __attribute__((ext_vector_type(4)));
typedef __attribute__((address_space(3))) i32x2* lds_i32x2_p;
typedef __attribute__((address_space(3))) i32x4* lds_i32x4_p;
typedef __attribute__((address_space(3))) int* lds_int_p;
typedef volatile __attribute__((address_space(3))) int* lds_vint_p;
typedef volatile __attribute__((address_space(3))) i32x4* lds_vi32x4_p;

#define STAMP(i) do {} while (0)
#define CSTAMP(i) do {} while (0)
#define STAMP_DECL do {} while (0)
#define STAMP_OUT(wsp) do {} while (0)
#define STAMP_ARG
#define STAMP_PASS

namespace {

constexpr int NRANK = 8;
constexpr int NN = 4096;
constexpr int KLOC = 512;
constexpr int MLOC = 512;
constexpr int NWG = 256;
constexpr int NTHREADS = 512;

#define NSPLIT 40
#define LP0 0
#define LP1 2
#define PACE 60
#define PUSH0 70
#define TFIRST 250
#define PACE0 57
#define NFAST 2
#define POISON_BITS ""
#define OUT_BITS "sc1"
#define XST_BITS "sc0 sc1"
#define XLD_BITS "sc0 sc1"
#define XRE_BITS "sc0 sc1"
#define XPF_BITS "sc1"
#define XSTRIDE 9216
#define QROT 1
#define ADIST 6
#define DPS 2
#define TICKET_AT 0
#define LEAD 0
#define ADIST2 ADIST
#define AINIT ADIST
#define SPF 0
#define FIRST_AT 2
#define LDW 3
#define LDW_FROM 14
#define LSLEEP 2
#define XLEAD 1
#define XJIT 8
#define XAHEAD 150
#define HOLDW 8
#define HOLDX 2
#define CMAP 2
#define ROT 4
#define LPRIO 0
#define CPRIO 1
#define CS0 10
#define CS1 20
constexpr int NS1 = 4 + NSPLIT / 2, NS2 = 56 - NSPLIT, NSTEP = NS1 + NS2;
constexpr int NFRAG = 56;
static_assert(NSPLIT % 2 == 0 && NSPLIT >= 8 && NSTEP <= 64 && NSTEP % 2 == 0 && LP0 == 0 && LP1 == 2, "");

constexpr unsigned WS_TMO = 0;
constexpr unsigned WS_TICKET = 4096;
constexpr unsigned WS_FLAGS = 8192;
constexpr unsigned WS_DATA = 1u << 20;
constexpr unsigned SLOT_BYTES = 32 * NFRAG * 1024;
constexpr unsigned WS_X = WS_DATA + 3u * SLOT_BYTES;
constexpr size_t WS_NEEDED = (size_t)WS_X + NWG * 4u * XSTRIDE;
constexpr unsigned POISON = 0x80808080u;

constexpr unsigned LDS_PUSH = 16 * 8192;
constexpr unsigned LDS_LOC = LDS_PUSH + 8192;
constexpr unsigned LDS_X3 = (NSTEP & 15) * 8192;
constexpr unsigned LDS_CTRL = LDS_LOC + 16384;
constexpr unsigned C_FULL = 0;
constexpr unsigned C_DUE = 8;
constexpr unsigned C_CNT = 12;
constexpr unsigned C_CALL = 16;
constexpr unsigned C_XSEEN = 24;
constexpr unsigned C_XRDY = 32;
constexpr unsigned C_TICKET = 768;
constexpr unsigned C_PROG = 1024;
constexpr unsigned LDS_BYTES = LDS_CTRL + 2048 + 256;

#define DEADLINE 50000000u
constexpr unsigned WATCH_TICKS = 3000;
constexpr unsigned DEADLINE_TICKS = DEADLINE;

struct Args {
    struct { char* ws; const int8_t* x; } peer[NRANK];
    struct { const int8_t* w; float* out; } col[NRANK];
    const float* sx;
    const float* sw;
    int rank;
    int pad;
};
typedef unsigned long long u64;
typedef u64 u64x2 __attribute__((ext_vector_type(2)));
__device__ __forceinline__ u64x2 args_ld(unsigned off) {
    u64x2 r;
    asm volatile("s_load_dwordx4 %0, %1, %2" : "=s"(r) : "s"(__builtin_amdgcn_kernarg_segment_ptr()), "s"(off) : "memory");
    return r;
}

#define SB() __builtin_amdgcn_sched_barrier(0)
#define LGKM(n) __builtin_amdgcn_s_waitcnt(0xC07F | ((n) << 8))

#define DMA4(BITS) asm volatile( \
        "s_mov_b32 m0, %5\n\t" \
        "s_nop 0\n\t" \
        "global_load_lds_dwordx4 %0, %4 " BITS "\n\t" \
        "global_load_lds_dwordx4 %1, %4 offset:1024 " BITS "\n\t" \
        "global_load_lds_dwordx4 %2, %4 offset:2048 " BITS "\n\t" \
        "global_load_lds_dwordx4 %3, %4 offset:3072 " BITS \
        :: "v"(v0), "v"(v1), "v"(v2), "v"(v3), "s"(gbase), "s"(lds_addr) : "memory")
template <bool SYS = false>
__device__ __forceinline__ void dma4(const char* gbase, unsigned lds_addr,
                                     unsigned v0, unsigned v1, unsigned v2, unsigned v3) {
    if constexpr (SYS) DMA4("sc0 sc1");
    else DMA4("");
}
template <int N> __device__ __forceinline__ void wait_vm(i32x4& a) {
    asm volatile("s_waitcnt vmcnt(%1)" : "+v"(a) : "i"(N) : "memory");
}
template <int N> __device__ __forceinline__ void wait_vm(i32x4& a, i32x4& b) {
    asm volatile("s_waitcnt vmcnt(%2)" : "+v"(a), "+v"(b) : "i"(N) : "memory");
}
__device__ __forceinline__ int lds_ld(unsigned addr) {
    return __hip_atomic_load((lds_int_p)addr, __ATOMIC_RELAXED, __HIP_MEMORY_SCOPE_WORKGROUP);
}
__device__ __forceinline__ void lds_st(unsigned addr, int v) {
    __hip_atomic_store((lds_int_p)addr, v, __ATOMIC_RELAXED, __HIP_MEMORY_SCOPE_WORKGROUP);
}
__device__ __forceinline__ void lds_st1(unsigned addr, int v) {
    asm volatile("s_mov_b64 exec, 1\n\tds_write_b32 %0, %1\n\ts_mov_b64 exec, -1" :: "v"(addr), "v"(v) : "memory");
}
__device__ __forceinline__ void lds_st2(unsigned addr, int v0, int v1) {
    asm volatile("s_mov_b64 exec, 1\n\tds_write_b64 %0, %1\n\ts_mov_b64 exec, -1" :: "v"(addr), "v"(i32x2{v0, v1}) : "memory");
}
__device__ __forceinline__ void lds_wait(unsigned addr, int need) {
    while (__builtin_amdgcn_readfirstlane(
               __hip_atomic_load((lds_int_p)addr, __ATOMIC_ACQUIRE, __HIP_MEMORY_SCOPE_WORKGROUP)) < need)
        __builtin_amdgcn_s_sleep(1);
    asm volatile("" ::: "memory");
}

constexpr int imin(int x, int y) { return x < y ? x : y; }
constexpr int imax(int x, int y) { return x > y ? x : y; }
constexpr int local_of(int s) { return s >= LP0 && s < LP0 + 2 ? s - LP0 : s >= LP1 && s < LP1 + 2 ? 2 + s - LP1 : -1; }
constexpr int frag_lo(int s) {
    return s >= NS1 ? NSPLIT + s - NS1 : 2 * (s - imin(2, imax(0, s - LP0)) - imin(2, imax(0, s - LP1)));
}
__device__ __forceinline__ int pos_of(int rel, int mi) {
    if (!QROT) return rel - 1;
    const int v = (rel + 2 * mi) & 7;
    return v - (v > 2 * mi ? 1 : 0);
}
__device__ __forceinline__ int rel_of(int pos, int mi) {
    if (!QROT) return pos + 1;
    return (pos + (pos >= 2 * mi ? 1 : 0) - 2 * mi) & 7;
}
__device__ __forceinline__ unsigned frag_off(int rel, int cls, int rg) {
    const int i = 7 * cls + pos_of(rel, rg >> 3), m = rg & 1;
    const int P = i < NSPLIT ? i & 1 : m, frag = i < NSPLIT ? (i & ~1) + (m ^ P) : i;
    return (((rg >> 1) * 2 + P) * NFRAG + frag) * 1024;
}

struct Ctx {
    const char* ab[7];
    char* xo;
    const char* xi;
    const char* xl;
    const char* xl2;
    unsigned sv[4];
    unsigned ll;
    unsigned la;
    unsigned sent;
    unsigned tag;
    unsigned xf, xa, xa3;
    unsigned rb[2][8];
    unsigned l16;
    unsigned chk;
    const char* flags;
    unsigned foff;
    unsigned call, t0;
    unsigned bad;
    unsigned prog;
    int one;
    int seen;
};

__device__ __forceinline__ void read_local(i32x4 (&AL)[2][2], const Ctx& c) {
#pragma unroll
    for (int s = 0; s < 2; ++s) {
        AL[s][0] = *(lds_i32x4_p)(c.la ^ (s * 64));
        AL[s][1] = *(lds_i32x4_p)(c.la ^ (s * 64) ^ 2048);
    }
}
template <int S, int H>
__device__ __forceinline__ void read_b(i32x2 (&B)[16], int t, const Ctx& c) {
    B[2 * t + H] = __builtin_amdgcn_ds_read_tr8_b64_v2i32((lds_i32x2_p)(c.rb[(S >> 3) & 1][t] + (S & 7) * 8192 + H * 1024));
}
__device__ __forceinline__ void mma(i32x4& acc, const i32x4& a, const i32x2 (&B)[16], int t) {
    asm volatile("v_mfma_i32_16x16x64_i8 %0, %1, %2, %0"
                 : "+v"(acc) : "v"(__builtin_shufflevector(B[2 * t], B[2 * t + 1], 0, 1, 2, 3)), "v"(a));
}
template <int F>
__device__ __forceinline__ void load_frag(i32x4 (&A)[NFRAG], const Ctx& c) {
    asm volatile("global_load_dwordx4 %0, %1, %2 offset:%3 sc0 sc1"
                 : "=v"(A[F]) : "v"(c.l16), "s"(c.ab[F / 8]), "i"((F % 8) * 1024 - 4096) : "memory");
}
template <int F, int END>
__device__ __forceinline__ void first_frags(i32x4 (&A)[NFRAG], const Ctx& c, int cnt) {
    if constexpr (F < END) {
        asm volatile(
            "v_cmp_gt_i32 vcc, %[need], %[cnt]\n\t"
            "s_cbranch_vccz .Lgo%=\n"
            ".Lagain%=:\n\t"
            "ds_read_b32 %[cnt], %[at]\n\t"
            "s_waitcnt lgkmcnt(0)\n\t"
            "v_cmp_gt_i32 vcc, %[need], %[cnt]\n\t"
            "s_cbranch_vccnz .Lagain%=\n"
            ".Lgo%=:"
            : [cnt] "+v"(cnt) : [at] "v"(LDS_CTRL + C_CNT), [need] "i"(F / 2 + 1) : "memory", "vcc");
        load_frag<F>(A, c);
        load_frag<F + 1>(A, c);
        first_frags<F + 2, END>(A, c, cnt);
    }
}
template <int FULL>
__device__ __forceinline__ void check(const Ctx& c) {
    while (__any(__hip_atomic_load((lds_int_p)c.chk, __ATOMIC_ACQUIRE, __HIP_MEMORY_SCOPE_WORKGROUP) < FULL))
        __builtin_amdgcn_s_sleep(1);
    asm volatile("" ::: "memory");
}

#define CP_WAIT \
        "s_or_b64 vcc, %[low], %[miss]\n\t" \
        "s_cbranch_scc1 .Lslow%=\n" \
        ".Lback%=:\n\t" \
        ".subsection 1\n" \
        ".Lagain%=:\n\t" \
        "s_sleep 1\n" \
        ".Lslow%=:\n\t" \
        "ds_read_b32 %[p0], %[chk]\n\t" \
        "s_waitcnt lgkmcnt(0)\n\t" \
        "v_cmp_gt_i32 vcc, %[full], %[p0]\n\t" \
        "s_cbranch_vccnz .Lagain%=\n\t"
template <int FULL>
__device__ __forceinline__ void checkpoint(Ctx& c, unsigned long long low) {
    unsigned p0;
    asm volatile(CP_WAIT "s_branch .Lback%=\n\t.subsection 0"
                 : [p0] "=&v"(p0) : [low] "s"(low), [miss] "i"(0), [chk] "v"(c.chk), [full] "i"(FULL) : "memory", "vcc", "scc");
}
template <int FULL, int F>
__device__ __forceinline__ void checkpoint(i32x4 (&A)[NFRAG], Ctx& c, unsigned long long low, unsigned long long miss) {
    unsigned p0, p1, fl, prev;
    asm volatile(
        CP_WAIT
        "s_cmp_eq_u64 %[miss], 0\n\t"
        "s_cbranch_scc1 .Lback%=\n\t"
        "s_mov_b32 %[prev], 0\n"
        ".Lretry%=:\n\t"
        "global_load_dword %[p0], %[l16], %[ab] offset:%[o0] sc0 sc1\n\t"
        "global_load_dword %[p1], %[l16], %[ab] offset:%[o1] sc0 sc1\n\t"
        "global_load_dwordx4 %[a0], %[l16], %[ab] offset:%[o0] sc0 sc1\n\t"
        "global_load_dwordx4 %[a1], %[l16], %[ab] offset:%[o1] sc0 sc1\n\t"
        "global_load_dword %[fl], %[foff], %[flags] sc0 sc1\n\t"
        "s_memrealtime vcc\n\t"
        "s_waitcnt vmcnt(0) lgkmcnt(0)\n\t"
        "s_cmp_lg_u32 %[prev], 0\n\t"
        "s_cbranch_scc1 .Lback%=\n\t"
        "s_sub_u32 %[prev], vcc_lo, %[t0]\n\t"
        "v_cmp_eq_u32 vcc, %[sent], %[p0]\n\t"
        "v_cmp_eq_u32 %[miss], %[sent], %[p1]\n\t"
        "s_or_b64 vcc, vcc, %[miss]\n\t"
        "s_cbranch_scc0 .Lback%=\n\t"
        "s_cmp_gt_u32 %[prev], %[dl]\n\t"
        "s_cselect_b32 %[bad], 1, %[bad]\n\t"
        "s_cbranch_scc1 .Lback%=\n\t"
        "v_subrev_u32 %[fl], %[call], %[fl]\n\t"
        "v_cmp_gt_i32 vcc, 0, %[fl]\n\t"
        "s_cmp_eq_u64 vcc, 0\n\t"
        "s_cselect_b32 %[prev], 1, 0\n\t"
        "s_branch .Lretry%=\n\t"
        ".subsection 0"
        : [a0] "+v"(A[F]), [a1] "+v"(A[F + 1]), [p0] "=&v"(p0), [p1] "=&v"(p1), [fl] "=&v"(fl), [prev] "=&s"(prev),
          [bad] "+s"(c.bad), [miss] "+s"(miss)
        : [low] "s"(low), [chk] "v"(c.chk), [full] "i"(FULL), [l16] "v"(c.l16), [ab] "s"(c.ab[F / 8]), [foff] "v"(c.foff), [flags] "s"(c.flags),
          [o0] "i"((F % 8) * 1024 - 4096), [o1] "i"((F % 8) * 1024 - 3072), [sent] "s"(c.sent), [call] "s"(c.call),
          [t0] "s"(c.t0), [dl] "i"(DEADLINE_TICKS)
        : "memory", "vcc", "scc");
}

static_assert(ADIST2 >= ADIST && ADIST % 2 == 0 && AINIT % 2 == 0 && AINIT <= ADIST && FIRST_AT >= 2 && FIRST_AT <= 3 && NS2 % 2 == 0, "");
constexpr int issued_before(int s) {
    return s < FIRST_AT ? 0 : imin(NFRAG, imin(frag_lo(s) + (s > NS1 ? ADIST2 : ADIST), AINIT + 4 * (s - FIRST_AT)));
}
constexpr int span(int s) {
    return s + 1 >= NSTEP ? 0 : s + 1 < NS1 ? 1 : (s + 1 - NS1) % 2 == 0 ? 2 : 0;
}
constexpr int dumps(int s) { return s >= NS1 && (s - NS1) * DPS < 8 ? DPS : 0; }
constexpr int ops_before_wait(int s) {
    int n = issued_before(s + 1 == FIRST_AT ? s : s + 1);
    for (int t = 0; t <= s; ++t) n += t < s ? dumps(t) : 0;
    return n;
}
constexpr int ops_through_frag(int f) {
    int s = FIRST_AT - 1;
    while (issued_before(s + 1) <= f) ++s;
    int n = f + 1;
    for (int t = 0; t < s; ++t) n += dumps(t);
    return n;
}

__device__ __forceinline__ bool untagged(const i32x4& x0, const i32x4& x1, unsigned tag) {
    return __any((((unsigned)x0[0] ^ tag) | ((unsigned)x1[0] ^ tag)) & 15);
}
constexpr unsigned x_lds(int p) { return p == 3 ? LDS_X3 : LDS_PUSH + p * 8192; }
__device__ __forceinline__ void bring_x(const char* xp, int p, int lane) {
    const unsigned v0 = lane * 16, v2 = lane * 16 + XSTRIDE - 2048;
    dma4<true>(xp + p * 2048, x_lds(p), v0, v0, v2, v2);
    dma4<true>(xp + p * 2048 + 2 * XSTRIDE, x_lds(p) + 4096, v0, v0, v2, v2);
}
struct Part { int f; i32x4 x[2]; };
__device__ __forceinline__ Part fetch_x(const Ctx& c, int p) {
    Part r;
    r.f = *(lds_vint_p)(c.xf + p * 4);
    r.x[0] = *(lds_vi32x4_p)(p < 3 ? c.xa + p * 8192 : c.xa3);
    r.x[1] = *(lds_vi32x4_p)((p < 3 ? c.xa + p * 8192 : c.xa3) + 1024);
    return r;
}
__device__ __forceinline__ void touch_x(const char* xi) {
    unsigned w;
#define TOUCH4(O) "s_load_dword %0, %1, " #O "\n\ts_load_dword %0, %1, %2+" #O "\n\ts_load_dword %0, %1, %3+" #O "\n\ts_load_dword %0, %1, %4+" #O "\n\t"
    asm volatile(TOUCH4(0) TOUCH4(2048) TOUCH4(4096) TOUCH4(6144) "s_waitcnt lgkmcnt(0)"
                 : "=&s"(w) : "s"(xi + 2032 - XLEAD * 1024), "i"(XSTRIDE), "i"(2 * XSTRIDE), "i"(3 * XSTRIDE) : "memory");
}
__device__ __forceinline__ unsigned long long or64(unsigned long long x, unsigned long long y) {
    unsigned long long r;
    asm volatile("s_or_b64 %0, %1, %2" : "=s"(r) : "s"(x), "s"(y) : "scc");
    return r;
}

template <int S>
__device__ __forceinline__ void step(Ctx& c, i32x4 (&acc)[2][8], i32x4 (&AL)[2][2][2], i32x4 (&A)[NFRAG],
                                     i32x2 (&B)[2][16], Part& X STAMP_ARG) {
    constexpr bool PH1 = S < NS1, LOC = S < 4, NEXT = S + 1 < NSTEP;
    constexpr int SPAN = span(S), FULL = imin(S + SPAN + 2, NSTEP);
    constexpr bool CP = SPAN > 0, VAL = CP && S + 1 >= 4;
    constexpr bool FREE = (S & 1) == 1 && S + 13 < NSTEP;
    constexpr int F = frag_lo(S), FN = frag_lo(S + 1);
    constexpr int L0 = issued_before(S), L1 = S + 1 == FIRST_AT ? L0 : issued_before(S + 1);
    unsigned long long m0 = 0, m1 = 0, m2 = 0;
    if constexpr ((S & 3) == 0) STAMP(4 + (S >> 2));
    if constexpr (S == 2) STAMP(15);
    if constexpr (S == CS0) { STAMP(28); CSTAMP(30); }
    if constexpr (S == CS1) { STAMP(29); CSTAMP(31); }
    if constexpr (S == FIRST_AT) first_frags<0, AINIT>(A, c, 0);
    const i32x4& a0 = LOC ? AL[LOC ? S / 2 : 0][S & 1][0] : A[F];
    const i32x4& a1 = LOC ? AL[LOC ? S / 2 : 0][S & 1][1] : A[PH1 ? F + 1 : F];
    LGKM(S == 2 ? 6 : 8);
    SB();
#pragma unroll
    for (int g = 0; g < 16; ++g) {
        const int t = g >> 1;
        if (g == 8) { if constexpr (NEXT) LGKM(8 + FREE); else LGKM(0); SB(); }
        if ((g & 1) == 0) { mma(acc[0][t], a0, B[S & 1], t); SB(); }
        else if constexpr (PH1) { mma(acc[1][t], a1, B[S & 1], t); SB(); }
        if (CP && g == 0) { c.seen = lds_ld(c.chk); SB(); }
        if constexpr (NEXT) {
            if (g & 1) read_b<S + 1, 1>(B[(S + 1) & 1], t, c);
            else if (g != 14) read_b<S + 1, 0>(B[(S + 1) & 1], t, c);
            if (g == 13) read_b<S + 1, 0>(B[(S + 1) & 1], 7, c);
        }
        if (!NEXT && g == 8) X = fetch_x(c, 0);
        if (g == 1) { if constexpr (L0 < L1) load_frag<L0>(A, c); }
        if (FREE && g == 2) __hip_atomic_fetch_add((lds_int_p)c.prog, c.one, __ATOMIC_RELAXED, __HIP_MEMORY_SCOPE_WORKGROUP);
        if (g == 3) { if constexpr (L0 + 2 < L1) load_frag<L0 + 1>(A, c); }
        if (g == 4) { if constexpr (L0 + 3 < L1) load_frag<L0 + 2>(A, c); }
        if (g == 5) { if constexpr (L0 + 1 < L1) load_frag<L1 - 1>(A, c); }
        if (CP && g == 8) m2 = __builtin_amdgcn_sicmp(c.seen, FULL, 40);
        if constexpr (VAL) {
            if (g == 9) wait_vm<ops_before_wait(S) - ops_through_frag(FN + 1)>(A[FN], A[FN + 1]);
            if (g == 10) m0 = __builtin_amdgcn_uicmp(A[FN][0], c.sent, 32);
            if (g == 11) m1 = __builtin_amdgcn_uicmp(A[FN + 1][0], c.sent, 32);
            if (g == 12) m0 = or64(m0, m1);
            if (g == 14) checkpoint<FULL, FN>(A, c, m2, m0);
        } else if constexpr (CP) {
            if (g == 14) checkpoint<FULL>(c, m2);
        }
        if (S == 1 && g == 9) {
            asm volatile("s_waitcnt vmcnt(0)" ::: "memory");
            read_local(AL[1], c);
        }
        if constexpr (dumps(S) > 0) {
            if (g == 15 || (DPS == 2 && g == 11)) {
                const int q = (S - NS1) * DPS + (g == 15 ? DPS - 1 : 0);
                acc[1][q][0] = (acc[1][q][0] << 4) | c.tag;
                asm volatile("global_store_dwordx4 %1, %0, %2 offset:%3 " XST_BITS "\n\ts_nop 1"
                             :: "v"(acc[1][q]), "v"(c.l16), "s"(c.xo), "i"(q * 1024 - 4096) : "memory");
            }
        }
        SB();
    }
    if constexpr (NEXT) step<S + 1>(c, acc, AL, A, B, X STAMP_PASS);
}

template <int S>
__device__ __forceinline__ void allowed(int& due, int n) {
    if constexpr (span(S) > 0) {
        constexpr int F = issued_before(S + span(S) + 1), NEED = F <= NSPLIT ? (F + 1) / 2 : F - NSPLIT / 2;
        if (n < NEED) due = imin(S + span(S) + 2, NSTEP) - 1;
    }
    if constexpr (S > FIRST_AT) allowed<S - 1>(due, n);
}
#define LORD 0
constexpr int LSEQ[1][2][3] = {{{0, 2, 4}, {1, 3, 5}}};
constexpr int fill_step(int h, int k) { return k < 3 ? LSEQ[LORD][h][k] : imin(NSTEP, 2 * k + h); }
constexpr int owed(int h, int k) {
    int m = fill_step(h, k);
    for (int i = k + 1; i < 3; ++i) m = imin(m, fill_step(h, i));
    return m;
}
__device__ __forceinline__ int w_chunk(int s, int P, int mi) {
    const int i = max(s < NS1 ? 2 * s - 8 + P : NSPLIT + s - NS1, 0), cls = (i * 37) >> 8;
    const int rem = rel_of(i - 7 * cls, mi) * 8 + ((cls + ROT * mi) & 7);
    return s < 4 ? 4 * P + (s ^ mi) : rem;
}
struct Ldr {
    const char* w;
    unsigned dv[4];
    int woff;
    unsigned fa, pa;
    int freed;
};
__device__ __forceinline__ void fill(const Ldr& l, int off, unsigned slot) {
    const char* p = l.w + off;
    dma4(p, slot, l.dv[0], l.dv[1], l.dv[2], l.dv[3]);
    dma4(p + 32 * NN, slot + 4096, l.dv[0], l.dv[1], l.dv[2], l.dv[3]);
}
__device__ __forceinline__ int released(unsigned pa) {
    int v = lds_ld(pa);
    v = min(v, __builtin_amdgcn_update_dpp(v, v, 0xB1, 0xF, 0xF, false));
    v = min(v, __builtin_amdgcn_update_dpp(v, v, 0x4E, 0xF, 0xF, false));
    return 2 * __builtin_amdgcn_readfirstlane(v);
}
template <int H, int K>
__device__ __forceinline__ void fills(Ldr& l STAMP_ARG) {
    constexpr int S = fill_step(H, K);
    if constexpr (S < NSTEP) {
        if constexpr (S >= 16) {
            if (__builtin_expect(l.freed <= S - 16, 0)) {
                while ((l.freed = released(l.pa)) <= S - 16) __builtin_amdgcn_s_sleep(LSLEEP);
            }
        }
        fill(l, __builtin_amdgcn_readlane(l.woff, S), (S & 15) * 8192);
        constexpr int W = imin(LDW, imax(1, K - LDW_FROM + 2));
        asm volatile("s_waitcnt vmcnt(%2)\n\tds_max_i32 %0, %1" :: "v"(l.fa), "v"(owed(H, K - W + 1)), "i"(8 * W) : "memory");
        if constexpr (K < 6 || (K & 1) == 1) STAMP(K < 6 ? K : 3 + K / 2);
        fills<H, K + 1>(l STAMP_PASS);
    } else {
#pragma unroll
        for (int k = LDW - 1; k >= 0; --k)
            asm volatile("s_waitcnt vmcnt(%2)\n\tds_max_i32 %0, %1" :: "v"(l.fa), "v"(k ? fill_step(H, K - k) : NSTEP), "i"(8 * k) : "memory");
    }
}

__device__ __forceinline__ void take_ticket(const Args& a, unsigned call, int lane) {
    unsigned tk = 0;
    if (lane == 0) tk = __hip_atomic_fetch_add((unsigned*)(a.peer[0].ws + WS_TICKET), 1u, __ATOMIC_RELAXED, __HIP_MEMORY_SCOPE_AGENT);
    if ((__builtin_amdgcn_readfirstlane(tk) & 255) == 255) {
        if (lane < NRANK)
            __hip_atomic_store((unsigned*)(a.peer[lane].ws + WS_FLAGS + ((NRANK - lane) & (NRANK - 1)) * 128), call,
                               __ATOMIC_RELAXED, __HIP_MEMORY_SCOPE_SYSTEM);
    }
}

}

__global__ __launch_bounds__(NTHREADS, 2) void a2a_gemm_kernel(Args a) {
    extern __shared__ __attribute__((aligned(16))) char smem[];
    const int lane = threadIdx.x & 63;
    const int wave = __builtin_amdgcn_readfirstlane(threadIdx.x >> 6);
    const int b = blockIdx.x;
    const int xcd = b & 7, j = b >> 3;
    const unsigned lds0 = (unsigned)(uintptr_t)(__attribute__((address_space(3))) char*)smem;
    const unsigned ctrl = lds0 + LDS_CTRL;
    const int P = j & 1;
    const int mi = (j >> 1) & 3;
    const int ct = (xcd + CMAP * (j >> 3)) & 7;
    const int n0 = (j >> 3) * 1024;
    STAMP_DECL;
#define AFTER_X() do { if (j < XJIT) asm volatile("s_barrier" ::: "memory"); } while (0)
    u64 t64;

    if (wave == 7) {
        lds_st(ctrl + lane * 4, lane == 1 ? owed(1, 0) : lane == C_DUE / 4 ? FIRST_AT + 2 : 0);
        *(lds_i32x4_p)(ctrl + C_PROG + lane * 16) = i32x4{0, 0, 0, 0};
        asm volatile("s_waitcnt lgkmcnt(0)\n\ts_barrier" ::: "memory");
        AFTER_X();
        const unsigned tstart = (unsigned)__builtin_amdgcn_s_memrealtime();
        char* const ws = a.peer[0].ws;
        STAMP(0);
        unsigned d[8];
        unsigned long long now;
        const char* const xi = ws + WS_X + (b ^ 8) * 4 * XSTRIDE + 2032 - XLEAD * 1024;
        const char *xp = xi, *xq = xi + 6144;
#define LOOK_X(BITS) asm volatile( \
            "s_load_dword %0, %9, 0 " BITS "\n\ts_load_dword %1, %9, %11 " BITS "\n\ts_load_dword %2, %9, %12 " BITS "\n\t" \
            "s_load_dword %3, %9, %13 " BITS "\n\ts_load_dword %4, %10, 0 " BITS "\n\ts_load_dword %5, %10, %11 " BITS "\n\t" \
            "s_load_dword %6, %10, %12 " BITS "\n\ts_load_dword %7, %10, %13 " BITS "\n\ts_memrealtime %8\n\ts_waitcnt lgkmcnt(0)" \
            : "=&s"(d[0]), "=&s"(d[1]), "=&s"(d[2]), "=&s"(d[3]), "=&s"(d[4]), "=&s"(d[5]), "=&s"(d[6]), "=&s"(d[7]), "=&s"(now) \
            : "s"(xp), "s"(xq), "i"(XSTRIDE), "i"(2 * XSTRIDE), "i"(3 * XSTRIDE) : "memory")
        int due = 1000;
        allowed<NSTEP - 1>(due, lane);
        constexpr int NI = NSPLIT / 2 + NS2;
        const int q = min(lane, NI - 1);
        const int offs = (q < NSPLIT / 2 ? 2 * q + (j >> 4) : q + NSPLIT / 2) * 1024;
        unsigned call;
        while ((call = __builtin_amdgcn_readfirstlane(lds_ld(ctrl + C_CALL))) == 0) __builtin_amdgcn_s_sleep(1);
        const unsigned sent = call <= 2 ? 0u : POISON;
        const char* af = ws + WS_DATA + (call % 3) * SLOT_BYTES + ((mi * 4 + 2 + ((j >> 3) & 1)) * 2 + P) * (NFRAG * 1024) + xcd * 128 + 112;
        int n = 0;
        STAMP(1);
        while (n < NSPLIT / 2) {
            unsigned o[4];
#pragma unroll
            for (int i = 0; i < 4; ++i) o[i] = __builtin_amdgcn_readlane(offs, n + i);
            do {
                asm volatile("s_load_dword %0, %5, %6 glc\n\ts_load_dword %1, %5, %7 glc\n\t"
                             "s_load_dword %2, %5, %8 glc\n\ts_load_dword %3, %5, %9 glc\n\t"
                             "s_memrealtime %4\n\ts_waitcnt lgkmcnt(0)"
                             : "=&s"(d[0]), "=&s"(d[1]), "=&s"(d[2]), "=&s"(d[3]), "=&s"(now)
                             : "s"(af), "s"(o[0]), "s"(o[1]), "s"(o[2]), "s"(o[3]) : "memory");
            } while (d[0] == sent && (unsigned)now - tstart < WATCH_TICKS);
            n += d[1] == sent ? 1 : d[2] == sent ? 2 : d[3] == sent ? 3 : 4;
            lds_st2(ctrl + C_DUE, __builtin_amdgcn_readlane(due, n + (n >= 4 ? LEAD : 0)), n);
        }
        while (n + 2 * LEAD < NI) {
            const char* p = af + (n + NSPLIT / 2) * 1024;
            do {
                asm volatile("s_load_dword %0, %9, 0 glc\n\ts_load_dword %1, %9, 0x400 glc\n\ts_load_dword %2, %9, 0x800 glc\n\t"
                             "s_load_dword %3, %9, 0xc00 glc\n\ts_load_dword %4, %9, 0x1000 glc\n\ts_load_dword %5, %9, 0x1400 glc\n\t"
                             "s_load_dword %6, %9, 0x1800 glc\n\ts_load_dword %7, %9, 0x1c00 glc\n\t"
                             "s_memrealtime %8\n\ts_waitcnt lgkmcnt(0)"
                             : "=&s"(d[0]), "=&s"(d[1]), "=&s"(d[2]), "=&s"(d[3]), "=&s"(d[4]), "=&s"(d[5]), "=&s"(d[6]), "=&s"(d[7]), "=&s"(now)
                             : "s"(p) : "memory");
            } while (d[0] == sent && (unsigned)now - tstart < WATCH_TICKS);
            n += d[1] == sent ? 1 : d[2] == sent ? 2 : d[3] == sent ? 3 : d[4] == sent ? 4 : d[5] == sent ? 5 : d[6] == sent ? 6 : d[7] == sent ? 7 : 8;
            lds_st1(ctrl + C_DUE, __builtin_amdgcn_readlane(due, n + 2 * LEAD));
        }
        lds_st2(ctrl + C_DUE, 1000, 1000);
        STAMP(7);
        const unsigned tag = call & 15, pa = ctrl + C_PROG + (lane & 3) * 260;
        bool late = false;
        while (released(pa) < imin(NS1, NSTEP - 14) && !(late = (unsigned)__builtin_amdgcn_s_memrealtime() - tstart > WATCH_TICKS))
            __builtin_amdgcn_s_sleep(6);
        STAMP(8);
        int p = 0;
        while (!late) {
            LOOK_X("glc");
            if (!(((d[4] ^ tag) | (d[5] ^ tag) | (d[6] ^ tag) | (d[7] ^ tag)) & 15)) break;
            late = (unsigned)now - tstart > WATCH_TICKS;
            if ((((d[0] ^ tag) | (d[1] ^ tag) | (d[2] ^ tag) | (d[3] ^ tag)) & 15) || p == 3) continue;
            xp += 2048;
            lds_st1(ctrl + C_XSEEN, ++p);
        }
        STAMP(9);
        if (!late)
            for (int q = p; q < 3; ++q) bring_x(xi - 2032 + XLEAD * 1024, q, lane);
        while (!late && released(pa) <= NSTEP - 16) late = (unsigned)__builtin_amdgcn_s_memrealtime() - tstart > WATCH_TICKS;
        if (!late) bring_x(xi - 2032 + XLEAD * 1024, 3, lane);
        lds_st1(ctrl + C_XSEEN, p + 8);
        if (!late) {
            asm volatile("s_waitcnt vmcnt(16)" ::: "memory");
            if (lane >= p && lane < 2) lds_st(ctrl + C_XRDY + lane * 4, 1);
            asm volatile("s_waitcnt vmcnt(8)" ::: "memory");
            if (lane >= p && lane == 2) lds_st(ctrl + C_XRDY + lane * 4, 1);
        }
        asm volatile("s_waitcnt vmcnt(0)" ::: "memory");
        if (lane >= p && lane < 4) lds_st(ctrl + C_XRDY + lane * 4, late ? 2 : 1);
        STAMP(10);
        STAMP_OUT(ws);
        return;
    }
    unsigned pre;
    asm volatile("s_load_dword %0, %1, 0\n\ts_load_dword %0, %1, 64\n\ts_load_dword %0, %1, 128\n\t"
                 "s_load_dword %0, %1, 192\n\ts_load_dword %0, %1, 256"
                 : "=&s"(pre) : "s"(__builtin_amdgcn_kernarg_segment_ptr()) : "memory");
#define NOW(t) asm volatile("s_memrealtime %0" : "=s"(t) :: "memory")
    if (wave == 6) {
        __builtin_amdgcn_s_setprio(3);
        NOW(t64);
        u64x2 own = args_ld(0);
        const int pd = ct == 3 ? 0 : ((ct ^ 4) + j) % 7 + 1, pc = j >> 2, pm = (j - (QROT ? (NRANK - pd) >> 1 : 0)) & 3;
        u64x2 to = args_ld(pd * 16);
        unsigned sv[8];
#pragma unroll
        for (int g = 0; g < 8; ++g) {
            sv[g] = (pm * 128 + g * 16 + (lane >> 2)) * KLOC + ((pc + ROT * pm) & 7) * 64 + (((lane & 3) ^ ((lane >> 4) & 3)) << 4) - (g & 3) * 1024;
            asm volatile("" : "+v"(sv[g]));
        }
        asm volatile("s_barrier\n\ts_waitcnt lgkmcnt(0)" : "+s"(t64), "+s"(own), "+s"(to) : "s"(pre) : "memory");
        char* const ws = (char*)own[0];
        char* const pw = (char*)to[0];
        const char* const base = (const char*)to[1];
        unsigned cv;
        asm volatile("s_load_dword %0, %1, %2" : "=s"(cv) : "s"(ws), "i"(WS_TICKET) : "memory");
        if (pd && j < XJIT) {
            dma4(base, lds0 + LDS_PUSH, sv[0], sv[1], sv[2], sv[3]);
            dma4(base, lds0 + LDS_PUSH + 4096, sv[4], sv[5], sv[6], sv[7]);
        }
        AFTER_X();
        int pd2 = pd;
        asm volatile("" : "+s"(pd2) :: "memory");
        const unsigned tstart = (unsigned)t64;
        const unsigned due = tstart + PUSH0 + ((j < 4 * NFAST ? j * PACE0 : j * PACE - 4 * NFAST * (PACE - PACE0)) >> 2);
        unsigned doff[2], poff[2];
#pragma unroll
        for (int m = 0; m < 2; ++m) {
            doff[m] = lane * 16 + WS_DATA + frag_off(NRANK - pd2, pc, pm * 8 + m);
            asm volatile("" : "+v"(doff[m]));
        }
        const unsigned roff = lds0 + LDS_PUSH + (lane & 15) * 64 + (((lane >> 4) ^ ((lane >> 2) & 3)) << 4);
        asm volatile("s_waitcnt lgkmcnt(0)" : "+s"(cv) : "s"(due), "v"(roff) : "memory");
        const unsigned call = (cv >> 8) + 1;
        lds_st1(ctrl + C_CALL, (int)call);
        const unsigned cur = (call % 3) * SLOT_BYTES, old = ((call + 2) % 3) * SLOT_BYTES;
        STAMP(12);
        if (pd) {
            char* dp[4];
#pragma unroll
            for (int w = 0; w < 4; ++w) { dp[w] = pw + cur + w * (2 * NFRAG * 1024); asm volatile("" : "+s"(dp[w])); }
            __builtin_amdgcn_s_setprio(0);
            if (j >= 20) touch_x(ws + WS_X + (b ^ 8) * 4 * XSTRIDE);
            if (j >= XJIT) {
                while ((int)(due - XAHEAD - (unsigned)__builtin_amdgcn_s_memrealtime()) > 0) __builtin_amdgcn_s_sleep(4);
                __builtin_amdgcn_s_setprio(3);
                dma4(base, lds0 + LDS_PUSH, sv[0], sv[1], sv[2], sv[3]);
                dma4(base, lds0 + LDS_PUSH + 4096, sv[4], sv[5], sv[6], sv[7]);
                __builtin_amdgcn_s_setprio(0);
            }
            for (int left; (left = (int)(due - (unsigned)__builtin_amdgcn_s_memrealtime())) > 0;) {
                if (left > 40) __builtin_amdgcn_s_sleep(8);
                else __builtin_amdgcn_s_sleep(1);
            }
            __builtin_amdgcn_s_setprio(3);
            asm volatile("s_waitcnt vmcnt(0)" ::: "memory");
            i32x4 dat[8];
#pragma unroll
            for (int g = 0; g < 8; ++g) dat[g] = *(lds_i32x4_p)(roff + g * 1024);
#pragma unroll
            for (int g = 0; g < 8; ++g)
                asm volatile("global_store_dwordx4 %1, %0, %2 sc0 sc1\n\ts_nop 1" :: "v"(dat[g]), "v"(doff[g & 1]), "s"(dp[g >> 1]) : "memory");
            __builtin_amdgcn_s_setprio(0);
            STAMP(3);
            asm volatile("" : "+s"(pd2) :: "memory");
#pragma unroll
            for (int m = 0; m < 2; ++m) poff[m] = lane * 16 + WS_DATA + frag_off(pd2, pc, pm * 8 + m);
            asm volatile("s_waitcnt vmcnt(0)" :: "v"(poff[0]), "v"(poff[1]) : "memory");
            STAMP(7);
            const i32x4 poison = {(int)POISON, (int)POISON, (int)POISON, (int)POISON};
#pragma unroll
            for (int g = 0; g < 8; ++g)
                asm volatile("global_store_dwordx4 %1, %0, %2 " POISON_BITS "\n\ts_nop 1"
                             :: "v"(poison), "v"(poff[g & 1]), "s"(ws + old + (g >> 1) * (2 * NFRAG * 1024)) : "memory");
        }
        take_ticket(a, call, lane);
        if (j < 20 || !pd) touch_x(ws + WS_X + (b ^ 8) * 4 * XSTRIDE);
        __builtin_amdgcn_s_setprio(0);
        for (int p = 0, seen = 0; seen < 8;) {
            while ((seen = __builtin_amdgcn_readfirstlane(lds_ld(ctrl + C_XSEEN))) <= p) __builtin_amdgcn_s_sleep(3);
            const int p0 = p;
            for (; p < (seen & 7); ++p) bring_x(ws + WS_X + (b ^ 8) * 4 * XSTRIDE, p, lane);
            asm volatile("s_waitcnt vmcnt(0)" ::: "memory");
            if (lane >= p0 && lane < p) lds_st(ctrl + C_XRDY + lane * 4, 1);
        }
        STAMP(4);
        STAMP(5);
        STAMP_OUT(ws);
        return;
    }

    if (wave >= 4) {
        __builtin_amdgcn_s_setprio(LPRIO);
        u64x2 col = args_ld(128 + ct * 16), rk = args_ld(264);
        Ldr l;
        const int s0 = wave & 1;
        unsigned f0[8];
#pragma unroll
        for (int i = 0; i < 4; ++i) {
            l.dv[i] = (lane >> 3) * NN + n0 + (((lane & 7) ^ (((lane >> 4) & 3) | ((i >> 1) << 2))) << 4) + i * (8 * NN - 1024);
            f0[i] = l.dv[i] + (4 * P + (s0 ^ mi)) * 64 * NN;
            f0[i + 4] = f0[i] + 32 * NN;
            asm volatile("" : "+v"(l.dv[i]), "+v"(f0[i]), "+v"(f0[i + 4]));
        }
        l.woff = w_chunk(min(lane, NSTEP - 1), P, mi);
        l.fa = ctrl + C_FULL + (lane ? 64 + lane * 8 : 0) + s0 * 4;
        l.pa = ctrl + C_PROG + (lane & 3) * 260;
        asm volatile("s_barrier\n\ts_waitcnt lgkmcnt(0)" : "+s"(col), "+s"(rk), "+v"(l.woff), "+v"(l.fa), "+v"(l.pa) : "s"(pre) : "memory");
        AFTER_X();
        if (HOLDW) __builtin_amdgcn_s_sleep(HOLDW);
        l.w = (const char*)col[0];
        dma4(l.w, lds0 + s0 * 8192, f0[0], f0[1], f0[2], f0[3]);
        dma4(l.w, lds0 + s0 * 8192 + 4096, f0[4], f0[5], f0[6], f0[7]);
        STAMP(0);
        const int r8 = (int)rk[1] * 8;
        l.woff = (((l.woff + r8) & 63) - r8) * 64 * NN;
        l.freed = 0;
        if (wave & 1) fills<1, 1>(l STAMP_PASS);
        else fills<0, 1>(l STAMP_PASS);
        STAMP(20);
        STAMP_OUT(ws);
        return;
    }

    __builtin_amdgcn_s_setprio(CPRIO);
    NOW(t64);
    u64x2 own = args_ld(0);
    Ctx c;
#pragma unroll
    for (int i = 0; i < 4; ++i) {
        c.sv[i] = (mi * 128 + wave * 32 + i * 8 + (lane >> 3)) * KLOC + P * 256 + (mi >> 1) * 128 +
                  (((lane & 7) ^ ((i * 4 + (lane >> 4)) & 7)) << 4) - i * 1024;
        asm volatile("" : "+v"(c.sv[i]));
    }
    c.ll = lds0 + LDS_LOC + wave * 4096;
    asm volatile("s_barrier\n\ts_waitcnt lgkmcnt(0)" : "+s"(t64), "+s"(own), "+s"(c.ll) : "s"(pre) : "memory");
    AFTER_X();
    if (HOLDX) __builtin_amdgcn_s_sleep(HOLDX);
    dma4((const char*)own[1], c.ll, c.sv[0], c.sv[1], c.sv[2], c.sv[3]);
    char* const ws = (char*)own[0];
    const unsigned tstart = (unsigned)t64;
    STAMP(19);
    c.xl2 = (const char*)own[1] + (128 - (mi >> 1) * 256);
    c.la = c.ll + P * 2048 + (lane & 15) * 128 + ((((mi & 1) * 4 + (lane >> 4)) ^ ((lane & 15) >> 1)) << 4);
    c.l16 = lane * 16;
    c.chk = ctrl + C_FULL + min(lane & 3, 2) * 4;
    c.foff = (lane & 7) * 128;
    c.flags = ws + WS_FLAGS;
    c.t0 = tstart;
    c.bad = 0;
    typedef const __attribute__((address_space(4))) float* cfp;
    float scale = *(cfp)(uintptr_t)a.sx * *(cfp)(uintptr_t)a.sw;
    char* outp = (char*)a.col[ct].out;
    asm volatile("" : "+s"(outp));
    c.prog = ctrl + C_PROG + wave * 256 + lane * 4;
    c.one = 1;
    asm volatile("" : "+v"(c.one), "+v"(c.chk), "+v"(c.prog), "+v"(c.foff));
    {
        const unsigned q = (lane & 15) >> 1, kg = lane >> 4;
        const unsigned r0 = lds0 + (kg * 16 + q) * 128 + (lane & 1) * 8, f = (q >> 1) | ((kg & 1) << 2);
#pragma unroll
        for (int t = 0; t < 8; ++t) {
            c.rb[0][t] = r0 + ((t ^ f) << 4);
            c.rb[1][t] = c.rb[0][t] + 65536;
            asm volatile("" : "+v"(c.rb[0][t]), "+v"(c.rb[1][t]));
        }
    }
    i32x4 acc[2][8];
#pragma unroll
    for (int m = 0; m < 2; ++m)
#pragma unroll
        for (int t = 0; t < 8; ++t) { acc[m][t] = i32x4{0, 0, 0, 0}; asm volatile("" : "+v"(acc[m][t])); }
    i32x2 B[2][16];
    i32x4 AL[2][2][2], A[NFRAG];
    Part X;
    if constexpr (LP0 == 0) {
        asm volatile("s_waitcnt vmcnt(0)" ::: "memory");
        read_local(AL[0], c);
        {
            LGKM(0);
            dma4(c.xl2, c.ll, c.sv[0], c.sv[1], c.sv[2], c.sv[3]);
        }
    }
    STAMP(1);
    unsigned call;
    while ((call = __builtin_amdgcn_readfirstlane(lds_ld(ctrl + C_CALL))) == 0) __builtin_amdgcn_s_sleep(1);
    c.call = call;
    c.sent = call <= 2 ? 0u : POISON;
    c.tag = call & 15;
#pragma unroll
    for (int i = 0; i < 7; ++i) {
        c.ab[i] = ws + WS_DATA + (call % 3) * SLOT_BYTES + ((mi * 4 + wave) * 2 + P) * (NFRAG * 1024) + i * 8192 + 4096;
        asm volatile("" : "+s"(c.ab[i]));
    }
    c.xo = ws + WS_X + (b * 4 + wave) * XSTRIDE + 4096;
    c.xi = ws + WS_X + ((b ^ 8) * 4 + wave) * XSTRIDE + 4096;
    asm volatile("" : "+s"(c.xo), "+s"(c.xi), "+s"(c.sent), "+s"(c.tag), "+s"(c.flags), "+s"(c.bad), "+v"(c.l16));
    c.xf = ctrl + C_XRDY;
    c.xa = lds0 + LDS_PUSH + wave * 2048 + c.l16;
    c.xa3 = lds0 + LDS_X3 + wave * 2048 + c.l16;
    unsigned tw = lds0 + LDS_PUSH + wave * 2048 + (lane & 15) * 128 + (((lane >> 4) ^ (lane & 7)) << 4);
    unsigned tw1 = tw ^ 64;
    unsigned tr = lds0 + LDS_PUSH + wave * 2048 + (lane >> 3) * 128 + (((lane & 7) ^ (lane >> 3)) << 4);
    unsigned oo = (lane >> 3) * NN * 4 + (lane & 7) * 16;
    char* o0 = outp + (size_t)(unsigned)(((mi * 128 + wave * 32 + P * 16) * NN + n0) * 4);
    char* o1 = o0 + 8 * NN * 4;
    asm volatile("" : "+v"(c.xf), "+v"(c.xa), "+v"(c.xa3), "+v"(tw), "+v"(tw1), "+v"(tr), "+v"(oo), "+s"(o0), "+s"(o1), "+v"(scale));
    check<1>(c);
    STAMP(3);
#pragma unroll
    for (int t = 0; t < 8; ++t) { read_b<0, 0>(B[0], t, c); read_b<0, 1>(B[0], t, c); }
    check<2>(c);
    step<0>(c, acc, AL, A, B, X STAMP_PASS);
    asm volatile("s_nop 7\n\ts_nop 1" ::: "memory");
    SB();
    STAMP(20);

    if (c.bad) scale = __builtin_nanf("");
    i32x4 Z[2];
#define STORE_Z(p) asm volatile("global_store_dwordx4 %2, %0, %3 offset:%5 " OUT_BITS "\n\t" \
                                "global_store_dwordx4 %2, %1, %4 offset:%5 " OUT_BITS "\n\ts_nop 1" \
                                :: "v"(Z[0]), "v"(Z[1]), "v"(oo), "s"(o0), "s"(o1), "i"((p) * 128) : "memory")
#pragma unroll
    for (int p = 0; p < 4; ++p) {
        asm volatile(
            "v_cmp_ne_u32 vcc, 0, %0\n\t"
            "s_cbranch_vccz .Lagain%=\n"
            ".Lgo%=:\n\t"
            ".subsection 1\n"
            ".Lagain%=:\n\t"
            "ds_read_b32 %0, %3 offset:%5\n\t"
            "ds_read_b128 %1, %4 offset:%6\n\t"
            "ds_read_b128 %2, %4 offset:%6+1024\n\t"
            "s_waitcnt lgkmcnt(0)\n\t"
            "v_cmp_ne_u32 vcc, 0, %0\n\t"
            "s_cbranch_vccz .Lagain%=\n\t"
            "s_branch .Lgo%=\n\t"
            ".subsection 0"
            : "+v"(X.f), "+v"(X.x[0]), "+v"(X.x[1])
            : "v"(c.xf), "v"(p < 3 ? c.xa : c.xa3), "i"(p * 4), "i"(p < 3 ? p * 8192 : 0) : "memory", "vcc");
        i32x4 x[2] = {X.x[0], X.x[1]};
        if (__builtin_expect(__any(((((unsigned)x[0][0] ^ c.tag) | ((unsigned)x[1][0] ^ c.tag)) & 15) | (X.f & 2)), 0)) {
            bool late;
            do {
                asm volatile("global_load_dwordx4 %0, %2, %3 offset:%4 sc0 sc1\n\tglobal_load_dwordx4 %1, %2, %3 offset:%5 sc0 sc1\n\t"
                             "s_waitcnt vmcnt(0)"
                             : "=&v"(x[0]), "=&v"(x[1]) : "v"(c.l16), "s"(c.xi), "i"(p * 2048 - 4096), "i"(p * 2048 - 3072) : "memory");
                late = (unsigned)__builtin_amdgcn_s_memrealtime() - c.t0 > DEADLINE_TICKS + DEADLINE_TICKS / 2;
            } while (untagged(x[0], x[1], c.tag) && !late);
            if (late) { c.bad = 1; scale = __builtin_nanf(""); }
        }
        if (p == 0) STAMP(21);
        if (p == 3) STAMP(22);
        if (p < 3) { SB(); X = fetch_x(c, p + 1); SB(); }
        f32x4 y[2];
#pragma unroll
        for (int h = 0; h < 2; ++h) {
            x[h][0] >>= 4;
            const i32x4 s = acc[0][2 * p + h] + x[h];
            y[h] = f32x4{(float)s[0] * scale, (float)s[1] * scale, (float)s[2] * scale, (float)s[3] * scale};
        }
        *(lds_vi32x4_p)(tw + (p % 3) * 8192) = __builtin_bit_cast(i32x4, y[0]);
        *(lds_vi32x4_p)(tw1 + (p % 3) * 8192) = __builtin_bit_cast(i32x4, y[1]);
        if (p) STORE_Z(p - 1);
        Z[0] = *(lds_vi32x4_p)(tr + (p % 3) * 8192);
        Z[1] = *(lds_vi32x4_p)(tr + (p % 3) * 8192 + 1024);
    }
    STORE_Z(3);
    if (c.bad && lane == 0) __hip_atomic_store((unsigned*)(a.peer[0].ws + WS_TMO), call, __ATOMIC_RELAXED, __HIP_MEMORY_SCOPE_AGENT);
    STAMP(23);
    STAMP(24);
    STAMP_OUT(ws);
}

extern "C" void kernel_launch(int rank, int nranks, ncclComm_t comm,
                              hipStream_t stream,
                              void* const* d_in, const int* in_sizes, int n_in,
                              void* d_out, int out_size,
                              const int* in_dtypes, int out_dtype,
                              void* const* d_out_all,
                              void* const* d_ws_all, size_t ws_bytes) {
    if (nranks != NRANK || (unsigned)rank >= (unsigned)NRANK || ws_bytes < WS_NEEDED || n_in < 4 ||
        out_size != MLOC * NN || in_sizes[0] != NRANK * MLOC * KLOC || in_sizes[1] != NRANK * KLOC * NN) {
        fprintf(stderr, "unexpected problem shape\n");
        return;
    }
    static bool attr_set = false;
    if (!attr_set) {
        HIP_CHECK(hipFuncSetAttribute((const void*)a2a_gemm_kernel, hipFuncAttributeMaxDynamicSharedMemorySize,
                                      (int)LDS_BYTES));
        attr_set = true;
    }
    Args a;
    for (int i = 0; i < NRANK; ++i) {
        const int p = (rank + i) & (NRANK - 1), c0 = ((i - (int)((uintptr_t)d_in[1] >> 7)) & 7) * 128;
        a.peer[i].ws = (char*)d_ws_all[p];
        a.peer[i].x = (const int8_t*)d_in[0] + (size_t)p * MLOC * KLOC;
        a.col[i].w = (const int8_t*)d_in[1] + (size_t)rank * KLOC * NN + c0;
        a.col[i].out = (float*)d_out + c0;
    }
    a.sx = (const float*)d_in[2];
    a.sw = (const float*)d_in[3];
    a.rank = rank;
    a.pad = 0;
    hipLaunchKernelGGL(a2a_gemm_kernel, dim3(NWG), dim3(NTHREADS), LDS_BYTES, stream, a);
    HIP_CHECK(hipGetLastError());
}
```

```cpp
#include <hip/hip_runtime.h>
#include <rccl/rccl.h>
#include <cstdint>
#include <cstdio>

#define HIP_CHECK(expr) do { hipError_t _e = (expr); if (_e != hipSuccess) { \
    fprintf(stderr, "HIP error %s at %s:%d\n", hipGetErrorString(_e), __FILE__, __LINE__); \
    return; } } while (0)

typedef int i32x2 __attribute__((ext_vector_type(2)));
typedef int i32x4 __attribute__((ext_vector_type(4)));
typedef float f32x4 __attribute__((ext_vector_type(4)));
typedef __attribute__((address_space(3))) i32x2* lds_i32x2_p;
typedef __attribute__((address_space(3))) i32x4* lds_i32x4_p;
typedef __attribute__((address_space(3))) int* lds_int_p;
typedef volatile __attribute__((address_space(3))) int* lds_vint_p;
typedef volatile __attribute__((address_space(3))) i32x4* lds_vi32x4_p;

#define STAMP(i) do {} while (0)
#define CSTAMP(i) do {} while (0)
#define STAMP_DECL do {} while (0)
#define STAMP_OUT(wsp) do {} while (0)
#define STAMP_ARG
#define STAMP_PASS

namespace {

constexpr int NRANK = 8;
constexpr int NN = 4096;
constexpr int KLOC = 512;
constexpr int MLOC = 512;
constexpr int NWG = 256;
constexpr int NTHREADS = 512;

#define NSPLIT 40
#define LP0 0
#define LP1 2
#define PACE 60
#define PUSH0 50
#define TFIRST 250
#define PACE0 57
#define NFAST 2
#define POISON_BITS ""
#define OUT_BITS "sc1"
#define XST_BITS "sc0 sc1"
#define XLD_BITS "sc0 sc1"
#define XRE_BITS "sc0 sc1"
#define XPF_BITS "sc1"
#define XSTRIDE 9216
#define QROT 1
#define ADIST 6
#define DPS 2
#define TICKET_AT 0
#define LEAD 0
#define ADIST2 ADIST
#define AINIT ADIST
#define SPF 0
#define FIRST_AT 2
#define LDW 3
#define LDW_FROM 14
#define LSLEEP 2
#define XLEAD 1
#define XJIT 8
#define XAHEAD 150
#define HOLDW 10
#define HOLDX 4
#define CMAP 2
#define ROT 4
#define LPRIO 0
#define CPRIO 1
#define CS0 10
#define CS1 20
constexpr int NS1 = 4 + NSPLIT / 2, NS2 = 56 - NSPLIT, NSTEP = NS1 + NS2;
constexpr int NFRAG = 56;
static_assert(NSPLIT % 2 == 0 && NSPLIT >= 8 && NSTEP <= 64 && NSTEP % 2 == 0 && LP0 == 0 && LP1 == 2, "");

constexpr unsigned WS_TMO = 0;
constexpr unsigned WS_TICKET = 4096;
constexpr unsigned WS_FLAGS = 8192;
constexpr unsigned WS_DATA = 1u << 20;
constexpr unsigned SLOT_BYTES = 32 * NFRAG * 1024;
constexpr unsigned WS_X = WS_DATA + 3u * SLOT_BYTES;
constexpr size_t WS_NEEDED = (size_t)WS_X + NWG * 4u * XSTRIDE;
constexpr unsigned POISON = 0x80808080u;

constexpr unsigned LDS_PUSH = 16 * 8192;
constexpr unsigned LDS_LOC = LDS_PUSH + 8192;
constexpr unsigned LDS_X3 = (NSTEP & 15) * 8192;
constexpr unsigned LDS_CTRL = LDS_LOC + 16384;
constexpr unsigned C_FULL = 0;
constexpr unsigned C_DUE = 8;
constexpr unsigned C_CNT = 12;
constexpr unsigned C_CALL = 16;
constexpr unsigned C_XSEEN = 24;
constexpr unsigned C_XRDY = 32;
constexpr unsigned C_TICKET = 768;
constexpr unsigned C_PROG = 1024;
constexpr unsigned LDS_BYTES = LDS_CTRL + 2048 + 256;

#define DEADLINE 50000000u
constexpr unsigned WATCH_TICKS = 3000;
constexpr unsigned DEADLINE_TICKS = DEADLINE;

struct Args {
    struct { char* ws; const int8_t* x; } peer[NRANK];
    struct { const int8_t* w; float* out; } col[NRANK];
    const float* sx;
    const float* sw;
    int rank;
    int pad;
};
typedef unsigned long long u64;
typedef u64 u64x2 __attribute__((ext_vector_type(2)));
__device__ __forceinline__ u64x2 args_ld(unsigned off) {
    u64x2 r;
    asm volatile("s_load_dwordx4 %0, %1, %2" : "=s"(r) : "s"(__builtin_amdgcn_kernarg_segment_ptr()), "s"(off) : "memory");
    return r;
}

#define SB() __builtin_amdgcn_sched_barrier(0)
#define LGKM(n) __builtin_amdgcn_s_waitcnt(0xC07F | ((n) << 8))

#define DMA4(BITS) asm volatile( \
        "s_mov_b32 m0, %5\n\t" \
        "s_nop 0\n\t" \
        "global_load_lds_dwordx4 %0, %4 " BITS "\n\t" \
        "global_load_lds_dwordx4 %1, %4 offset:1024 " BITS "\n\t" \
        "global_load_lds_dwordx4 %2, %4 offset:2048 " BITS "\n\t" \
        "global_load_lds_dwordx4 %3, %4 offset:3072 " BITS \
        :: "v"(v0), "v"(v1), "v"(v2), "v"(v3), "s"(gbase), "s"(lds_addr) : "memory")
template <bool SYS = false>
__device__ __forceinline__ void dma4(const char* gbase, unsigned lds_addr,
                                     unsigned v0, unsigned v1, unsigned v2, unsigned v3) {
    if constexpr (SYS) DMA4("sc0 sc1");
    else DMA4("");
}
template <int N> __device__ __forceinline__ void wait_vm(i32x4& a) {
    asm volatile("s_waitcnt vmcnt(%1)" : "+v"(a) : "i"(N) : "memory");
}
template <int N> __device__ __forceinline__ void wait_vm(i32x4& a, i32x4& b) {
    asm volatile("s_waitcnt vmcnt(%2)" : "+v"(a), "+v"(b) : "i"(N) : "memory");
}
__device__ __forceinline__ int lds_ld(unsigned addr) {
    return __hip_atomic_load((lds_int_p)addr, __ATOMIC_RELAXED, __HIP_MEMORY_SCOPE_WORKGROUP);
}
__device__ __forceinline__ void lds_st(unsigned addr, int v) {
    __hip_atomic_store((lds_int_p)addr, v, __ATOMIC_RELAXED, __HIP_MEMORY_SCOPE_WORKGROUP);
}
__device__ __forceinline__ void lds_st1(unsigned addr, int v) {
    asm volatile("s_mov_b64 exec, 1\n\tds_write_b32 %0, %1\n\ts_mov_b64 exec, -1" :: "v"(addr), "v"(v) : "memory");
}
__device__ __forceinline__ void lds_st2(unsigned addr, int v0, int v1) {
    asm volatile("s_mov_b64 exec, 1\n\tds_write_b64 %0, %1\n\ts_mov_b64 exec, -1" :: "v"(addr), "v"(i32x2{v0, v1}) : "memory");
}
__device__ __forceinline__ void lds_wait(unsigned addr, int need) {
    while (__builtin_amdgcn_readfirstlane(
               __hip_atomic_load((lds_int_p)addr, __ATOMIC_ACQUIRE, __HIP_MEMORY_SCOPE_WORKGROUP)) < need)
        __builtin_amdgcn_s_sleep(1);
    asm volatile("" ::: "memory");
}

constexpr int imin(int x, int y) { return x < y ? x : y; }
constexpr int imax(int x, int y) { return x > y ? x : y; }
constexpr int local_of(int s) { return s >= LP0 && s < LP0 + 2 ? s - LP0 : s >= LP1 && s < LP1 + 2 ? 2 + s - LP1 : -1; }
constexpr int frag_lo(int s) {
    return s >= NS1 ? NSPLIT + s - NS1 : 2 * (s - imin(2, imax(0, s - LP0)) - imin(2, imax(0, s - LP1)));
}
__device__ __forceinline__ int pos_of(int rel, int mi) {
    if (!QROT) return rel - 1;
    const int v = (rel + 2 * mi) & 7;
    return v - (v > 2 * mi ? 1 : 0);
}
__device__ __forceinline__ int rel_of(int pos, int mi) {
    if (!QROT) return pos + 1;
    return (pos + (pos >= 2 * mi ? 1 : 0) - 2 * mi) & 7;
}
__device__ __forceinline__ unsigned frag_off(int rel, int cls, int rg) {
    const int i = 7 * cls + pos_of(rel, rg >> 3), m = rg & 1;
    const int P = i < NSPLIT ? i & 1 : m, frag = i < NSPLIT ? (i & ~1) + (m ^ P) : i;
    return (((rg >> 1) * 2 + P) * NFRAG + frag) * 1024;
}

struct Ctx {
    const char* ab[7];
    char* xo;
    const char* xi;
    const char* xl;
    const char* xl2;
    unsigned sv[4];
    unsigned ll;
    unsigned la;
    unsigned sent;
    unsigned tag;
    unsigned xf, xa, xa3;
    unsigned rb[2][8];
    unsigned l16;
    unsigned chk;
    const char* flags;
    unsigned foff;
    unsigned call, t0;
    unsigned bad;
    unsigned prog;
    int one;
    int seen;
};

__device__ __forceinline__ void read_local(i32x4 (&AL)[2][2], const Ctx& c) {
#pragma unroll
    for (int s = 0; s < 2; ++s) {
        AL[s][0] = *(lds_i32x4_p)(c.la ^ (s * 64));
        AL[s][1] = *(lds_i32x4_p)(c.la ^ (s * 64) ^ 2048);
    }
}
template <int S, int H>
__device__ __forceinline__ void read_b(i32x2 (&B)[16], int t, const Ctx& c) {
    B[2 * t + H] = __builtin_amdgcn_ds_read_tr8_b64_v2i32((lds_i32x2_p)(c.rb[(S >> 3) & 1][t] + (S & 7) * 8192 + H * 1024));
}
__device__ __forceinline__ void mma(i32x4& acc, const i32x4& a, const i32x2 (&B)[16], int t) {
    asm volatile("v_mfma_i32_16x16x64_i8 %0, %1, %2, %0"
                 : "+v"(acc) : "v"(__builtin_shufflevector(B[2 * t], B[2 * t + 1], 0, 1, 2, 3)), "v"(a));
}
template <int F>
__device__ __forceinline__ void load_frag(i32x4 (&A)[NFRAG], const Ctx& c) {
    asm volatile("global_load_dwordx4 %0, %1, %2 offset:%3 sc0 sc1"
                 : "=v"(A[F]) : "v"(c.l16), "s"(c.ab[F / 8]), "i"((F % 8) * 1024 - 4096) : "memory");
}
template <int F, int END>
__device__ __forceinline__ void first_frags(i32x4 (&A)[NFRAG], const Ctx& c, int cnt) {
    if constexpr (F < END) {
        asm volatile(
            "v_cmp_gt_i32 vcc, %[need], %[cnt]\n\t"
            "s_cbranch_vccz .Lgo%=\n"
            ".Lagain%=:\n\t"
            "ds_read_b32 %[cnt], %[at]\n\t"
            "s_waitcnt lgkmcnt(0)\n\t"
            "v_cmp_gt_i32 vcc, %[need], %[cnt]\n\t"
            "s_cbranch_vccnz .Lagain%=\n"
            ".Lgo%=:"
            : [cnt] "+v"(cnt) : [at] "v"(LDS_CTRL + C_CNT), [need] "i"(F / 2 + 1) : "memory", "vcc");
        load_frag<F>(A, c);
        load_frag<F + 1>(A, c);
        first_frags<F + 2, END>(A, c, cnt);
    }
}
template <int FULL>
__device__ __forceinline__ void check(const Ctx& c) {
    while (__any(__hip_atomic_load((lds_int_p)c.chk, __ATOMIC_ACQUIRE, __HIP_MEMORY_SCOPE_WORKGROUP) < FULL))
        __builtin_amdgcn_s_sleep(1);
    asm volatile("" ::: "memory");
}

#define CP_WAIT \
        "s_or_b64 vcc, %[low], %[miss]\n\t" \
        "s_cbranch_scc1 .Lslow%=\n" \
        ".Lback%=:\n\t" \
        ".subsection 1\n" \
        ".Lagain%=:\n\t" \
        "s_sleep 1\n" \
        ".Lslow%=:\n\t" \
        "ds_read_b32 %[p0], %[chk]\n\t" \
        "s_waitcnt lgkmcnt(0)\n\t" \
        "v_cmp_gt_i32 vcc, %[full], %[p0]\n\t" \
        "s_cbranch_vccnz .Lagain%=\n\t"
template <int FULL>
__device__ __forceinline__ void checkpoint(Ctx& c, unsigned long long low) {
    unsigned p0;
    asm volatile(CP_WAIT "s_branch .Lback%=\n\t.subsection 0"
                 : [p0] "=&v"(p0) : [low] "s"(low), [miss] "i"(0), [chk] "v"(c.chk), [full] "i"(FULL) : "memory", "vcc", "scc");
}
template <int FULL, int F>
__device__ __forceinline__ void checkpoint(i32x4 (&A)[NFRAG], Ctx& c, unsigned long long low, unsigned long long miss) {
    unsigned p0, p1, fl, prev;
    asm volatile(
        CP_WAIT
        "s_cmp_eq_u64 %[miss], 0\n\t"
        "s_cbranch_scc1 .Lback%=\n\t"
        "s_mov_b32 %[prev], 0\n"
        ".Lretry%=:\n\t"
        "global_load_dword %[p0], %[l16], %[ab] offset:%[o0] sc0 sc1\n\t"
        "global_load_dword %[p1], %[l16], %[ab] offset:%[o1] sc0 sc1\n\t"
        "global_load_dwordx4 %[a0], %[l16], %[ab] offset:%[o0] sc0 sc1\n\t"
        "global_load_dwordx4 %[a1], %[l16], %[ab] offset:%[o1] sc0 sc1\n\t"
        "global_load_dword %[fl], %[foff], %[flags] sc0 sc1\n\t"
        "s_memrealtime vcc\n\t"
        "s_waitcnt vmcnt(0) lgkmcnt(0)\n\t"
        "s_cmp_lg_u32 %[prev], 0\n\t"
        "s_cbranch_scc1 .Lback%=\n\t"
        "s_sub_u32 %[prev], vcc_lo, %[t0]\n\t"
        "v_cmp_eq_u32 vcc, %[sent], %[p0]\n\t"
        "v_cmp_eq_u32 %[miss], %[sent], %[p1]\n\t"
        "s_or_b64 vcc, vcc, %[miss]\n\t"
        "s_cbranch_scc0 .Lback%=\n\t"
        "s_cmp_gt_u32 %[prev], %[dl]\n\t"
        "s_cselect_b32 %[bad], 1, %[bad]\n\t"
        "s_cbranch_scc1 .Lback%=\n\t"
        "v_subrev_u32 %[fl], %[call], %[fl]\n\t"
        "v_cmp_gt_i32 vcc, 0, %[fl]\n\t"
        "s_cmp_eq_u64 vcc, 0\n\t"
        "s_cselect_b32 %[prev], 1, 0\n\t"
        "s_branch .Lretry%=\n\t"
        ".subsection 0"
        : [a0] "+v"(A[F]), [a1] "+v"(A[F + 1]), [p0] "=&v"(p0), [p1] "=&v"(p1), [fl] "=&v"(fl), [prev] "=&s"(prev),
          [bad] "+s"(c.bad), [miss] "+s"(miss)
        : [low] "s"(low), [chk] "v"(c.chk), [full] "i"(FULL), [l16] "v"(c.l16), [ab] "s"(c.ab[F / 8]), [foff] "v"(c.foff), [flags] "s"(c.flags),
          [o0] "i"((F % 8) * 1024 - 4096), [o1] "i"((F % 8) * 1024 - 3072), [sent] "s"(c.sent), [call] "s"(c.call),
          [t0] "s"(c.t0), [dl] "i"(DEADLINE_TICKS)
        : "memory", "vcc", "scc");
}

static_assert(ADIST2 >= ADIST && ADIST % 2 == 0 && AINIT % 2 == 0 && AINIT <= ADIST && FIRST_AT >= 2 && FIRST_AT <= 3 && NS2 % 2 == 0, "");
constexpr int issued_before(int s) {
    return s < FIRST_AT ? 0 : imin(NFRAG, imin(frag_lo(s) + (s > NS1 ? ADIST2 : ADIST), AINIT + 4 * (s - FIRST_AT)));
}
constexpr int span(int s) {
    return s + 1 >= NSTEP ? 0 : s + 1 < NS1 ? 1 : (s + 1 - NS1) % 2 == 0 ? 2 : 0;
}
constexpr int dumps(int s) { return s >= NS1 && (s - NS1) * DPS < 8 ? DPS : 0; }
constexpr int ops_before_wait(int s) {
    int n = issued_before(s + 1 == FIRST_AT ? s : s + 1);
    for (int t = 0; t <= s; ++t) n += t < s ? dumps(t) : 0;
    return n;
}
constexpr int ops_through_frag(int f) {
    int s = FIRST_AT - 1;
    while (issued_before(s + 1) <= f) ++s;
    int n = f + 1;
    for (int t = 0; t < s; ++t) n += dumps(t);
    return n;
}

__device__ __forceinline__ bool untagged(const i32x4& x0, const i32x4& x1, unsigned tag) {
    return __any((((unsigned)x0[0] ^ tag) | ((unsigned)x1[0] ^ tag)) & 15);
}
constexpr unsigned x_lds(int p) { return p == 3 ? LDS_X3 : LDS_PUSH + p * 8192; }
__device__ __forceinline__ void bring_x(const char* xp, int p, int lane) {
    const unsigned v0 = lane * 16, v2 = lane * 16 + XSTRIDE - 2048;
    dma4<true>(xp + p * 2048, x_lds(p), v0, v0, v2, v2);
    dma4<true>(xp + p * 2048 + 2 * XSTRIDE, x_lds(p) + 4096, v0, v0, v2, v2);
}
struct Part { int f; i32x4 x[2]; };
__device__ __forceinline__ Part fetch_x(const Ctx& c, int p) {
    Part r;
    r.f = *(lds_vint_p)(c.xf + p * 4);
    r.x[0] = *(lds_vi32x4_p)(p < 3 ? c.xa + p * 8192 : c.xa3);
    r.x[1] = *(lds_vi32x4_p)((p < 3 ? c.xa + p * 8192 : c.xa3) + 1024);
    return r;
}
__device__ __forceinline__ void touch_x(const char* xi) {
    unsigned w;
#define TOUCH4(O) "s_load_dword %0, %1, " #O "\n\ts_load_dword %0, %1, %2+" #O "\n\ts_load_dword %0, %1, %3+" #O "\n\ts_load_dword %0, %1, %4+" #O "\n\t"
    asm volatile(TOUCH4(0) TOUCH4(2048) TOUCH4(4096) TOUCH4(6144) "s_waitcnt lgkmcnt(0)"
                 : "=&s"(w) : "s"(xi + 2032 - XLEAD * 1024), "i"(XSTRIDE), "i"(2 * XSTRIDE), "i"(3 * XSTRIDE) : "memory");
}
__device__ __forceinline__ unsigned long long or64(unsigned long long x, unsigned long long y) {
    unsigned long long r;
    asm volatile("s_or_b64 %0, %1, %2" : "=s"(r) : "s"(x), "s"(y) : "scc");
    return r;
}

template <int S>
__device__ __forceinline__ void step(Ctx& c, i32x4 (&acc)[2][8], i32x4 (&AL)[2][2][2], i32x4 (&A)[NFRAG],
                                     i32x2 (&B)[2][16], Part& X STAMP_ARG) {
    constexpr bool PH1 = S < NS1, LOC = S < 4, NEXT = S + 1 < NSTEP;
    constexpr int SPAN = span(S), FULL = imin(S + SPAN + 2, NSTEP);
    constexpr bool CP = SPAN > 0, VAL = CP && S + 1 >= 4;
    constexpr bool FREE = (S & 1) == 1 && S + 13 < NSTEP;
    constexpr int F = frag_lo(S), FN = frag_lo(S + 1);
    constexpr int L0 = issued_before(S), L1 = S + 1 == FIRST_AT ? L0 : issued_before(S + 1);
    unsigned long long m0 = 0, m1 = 0, m2 = 0;
    if constexpr ((S & 3) == 0) STAMP(4 + (S >> 2));
    if constexpr (S == 2) STAMP(15);
    if constexpr (S == CS0) { STAMP(28); CSTAMP(30); }
    if constexpr (S == CS1) { STAMP(29); CSTAMP(31); }
    if constexpr (S == FIRST_AT) first_frags<0, AINIT>(A, c, 0);
    const i32x4& a0 = LOC ? AL[LOC ? S / 2 : 0][S & 1][0] : A[F];
    const i32x4& a1 = LOC ? AL[LOC ? S / 2 : 0][S & 1][1] : A[PH1 ? F + 1 : F];
    LGKM(S == 2 ? 6 : 8);
    SB();
#pragma unroll
    for (int g = 0; g < 16; ++g) {
        const int t = g >> 1;
        if (g == 8) { if constexpr (NEXT) LGKM(8 + FREE); else LGKM(0); SB(); }
        if ((g & 1) == 0) { mma(acc[0][t], a0, B[S & 1], t); SB(); }
        else if constexpr (PH1) { mma(acc[1][t], a1, B[S & 1], t); SB(); }
        if (CP && g == 0) { c.seen = lds_ld(c.chk); SB(); }
        if constexpr (NEXT) {
            if (g & 1) read_b<S + 1, 1>(B[(S + 1) & 1], t, c);
            else if (g != 14) read_b<S + 1, 0>(B[(S + 1) & 1], t, c);
            if (g == 13) read_b<S + 1, 0>(B[(S + 1) & 1], 7, c);
        }
        if (!NEXT && g == 8) X = fetch_x(c, 0);
        if (g == 1) { if constexpr (L0 < L1) load_frag<L0>(A, c); }
        if (FREE && g == 2) __hip_atomic_fetch_add((lds_int_p)c.prog, c.one, __ATOMIC_RELAXED, __HIP_MEMORY_SCOPE_WORKGROUP);
        if (g == 3) { if constexpr (L0 + 2 < L1) load_frag<L0 + 1>(A, c); }
        if (g == 4) { if constexpr (L0 + 3 < L1) load_frag<L0 + 2>(A, c); }
        if (g == 5) { if constexpr (L0 + 1 < L1) load_frag<L1 - 1>(A, c); }
        if (CP && g == 8) m2 = __builtin_amdgcn_sicmp(c.seen, FULL, 40);
        if constexpr (VAL) {
            if (g == 9) wait_vm<ops_before_wait(S) - ops_through_frag(FN + 1)>(A[FN], A[FN + 1]);
            if (g == 10) m0 = __builtin_amdgcn_uicmp(A[FN][0], c.sent, 32);
            if (g == 11) m1 = __builtin_amdgcn_uicmp(A[FN + 1][0], c.sent, 32);
            if (g == 12) m0 = or64(m0, m1);
            if (g == 14) checkpoint<FULL, FN>(A, c, m2, m0);
        } else if constexpr (CP) {
            if (g == 14) checkpoint<FULL>(c, m2);
        }
        if (S == 1 && g == 9) {
            asm volatile("s_waitcnt vmcnt(0)" ::: "memory");
            read_local(AL[1], c);
        }
        if constexpr (dumps(S) > 0) {
            if (g == 15 || (DPS == 2 && g == 11)) {
                const int q = (S - NS1) * DPS + (g == 15 ? DPS - 1 : 0);
                acc[1][q][0] = (acc[1][q][0] << 4) | c.tag;
                asm volatile("global_store_dwordx4 %1, %0, %2 offset:%3 " XST_BITS "\n\ts_nop 1"
                             :: "v"(acc[1][q]), "v"(c.l16), "s"(c.xo), "i"(q * 1024 - 4096) : "memory");
            }
        }
        SB();
    }
    if constexpr (NEXT) step<S + 1>(c, acc, AL, A, B, X STAMP_PASS);
}

template <int S>
__device__ __forceinline__ void allowed(int& due, int n) {
    if constexpr (span(S) > 0) {
        constexpr int F = issued_before(S + span(S) + 1), NEED = F <= NSPLIT ? (F + 1) / 2 : F - NSPLIT / 2;
        if (n < NEED) due = imin(S + span(S) + 2, NSTEP) - 1;
    }
    if constexpr (S > FIRST_AT) allowed<S - 1>(due, n);
}
#define LORD 0
constexpr int LSEQ[1][2][3] = {{{0, 2, 4}, {1, 3, 5}}};
constexpr int fill_step(int h, int k) { return k < 3 ? LSEQ[LORD][h][k] : imin(NSTEP, 2 * k + h); }
constexpr int owed(int h, int k) {
    int m = fill_step(h, k);
    for (int i = k + 1; i < 3; ++i) m = imin(m, fill_step(h, i));
    return m;
}
__device__ __forceinline__ int w_chunk(int s, int P, int mi) {
    const int i = max(s < NS1 ? 2 * s - 8 + P : NSPLIT + s - NS1, 0), cls = (i * 37) >> 8;
    const int rem = rel_of(i - 7 * cls, mi) * 8 + ((cls + ROT * mi) & 7);
    return s < 4 ? 4 * P + (s ^ mi) : rem;
}
struct Ldr {
    const char* w;
    unsigned dv[4];
    int woff;
    unsigned fa, pa;
    int freed;
};
__device__ __forceinline__ void fill(const Ldr& l, int off, unsigned slot) {
    const char* p = l.w + off;
    dma4(p, slot, l.dv[0], l.dv[1], l.dv[2], l.dv[3]);
    dma4(p + 32 * NN, slot + 4096, l.dv[0], l.dv[1], l.dv[2], l.dv[3]);
}
__device__ __forceinline__ int released(unsigned pa) {
    int v = lds_ld(pa);
    v = min(v, __builtin_amdgcn_update_dpp(v, v, 0xB1, 0xF, 0xF, false));
    v = min(v, __builtin_amdgcn_update_dpp(v, v, 0x4E, 0xF, 0xF, false));
    return 2 * __builtin_amdgcn_readfirstlane(v);
}
template <int H, int K>
__device__ __forceinline__ void fills(Ldr& l STAMP_ARG) {
    constexpr int S = fill_step(H, K);
    if constexpr (S < NSTEP) {
        if constexpr (S >= 16) {
            if (__builtin_expect(l.freed <= S - 16, 0)) {
                while ((l.freed = released(l.pa)) <= S - 16) __builtin_amdgcn_s_sleep(LSLEEP);
            }
        }
        if constexpr (K > 1) fill(l, __builtin_amdgcn_readlane(l.woff, S), (S & 15) * 8192);
        constexpr int W = imin(LDW, imax(1, K - LDW_FROM + 2));
        asm volatile("s_waitcnt vmcnt(%2)\n\tds_max_i32 %0, %1" :: "v"(l.fa), "v"(owed(H, K - W + 1)), "i"(8 * W) : "memory");
        if constexpr (K < 6 || (K & 1) == 1) STAMP(K < 6 ? K : 3 + K / 2);
        fills<H, K + 1>(l STAMP_PASS);
    } else {
#pragma unroll
        for (int k = LDW - 1; k >= 0; --k)
            asm volatile("s_waitcnt vmcnt(%2)\n\tds_max_i32 %0, %1" :: "v"(l.fa), "v"(k ? fill_step(H, K - k) : NSTEP), "i"(8 * k) : "memory");
    }
}

__device__ __forceinline__ void take_ticket(const Args& a, unsigned call, int lane) {
    unsigned tk = 0;
    if (lane == 0) tk = __hip_atomic_fetch_add((unsigned*)(a.peer[0].ws + WS_TICKET), 1u, __ATOMIC_RELAXED, __HIP_MEMORY_SCOPE_AGENT);
    if ((__builtin_amdgcn_readfirstlane(tk) & 255) == 255) {
        if (lane < NRANK)
            __hip_atomic_store((unsigned*)(a.peer[lane].ws + WS_FLAGS + ((NRANK - lane) & (NRANK - 1)) * 128), call,
                               __ATOMIC_RELAXED, __HIP_MEMORY_SCOPE_SYSTEM);
    }
}

}

__global__ __launch_bounds__(NTHREADS, 2) __attribute__((aligned(4096))) void a2a_gemm_kernel(Args a) {
    extern __shared__ __attribute__((aligned(16))) char smem[];
    const int lane = threadIdx.x & 63;
    int wave = __builtin_amdgcn_readfirstlane(threadIdx.x >> 6);
    const int b = blockIdx.x;
    const int xcd = b & 7, j = b >> 3;
    const unsigned lds0 = (unsigned)(uintptr_t)(__attribute__((address_space(3))) char*)smem;
    const unsigned ctrl = lds0 + LDS_CTRL;
    const int P = j & 1;
    const int mi = (j >> 1) & 3;
    const int ct = (xcd + CMAP * (j >> 3)) & 7;
    const int n0 = (j >> 3) * 1024;
    STAMP_DECL;
#define NEXT_ROLE() asm volatile("" : "+s"(wave) :: "memory")
    u64 t64;

    unsigned pre = 0;
    if (wave == 7) {
        {
            u64 pc;
            unsigned len;
            asm volatile("s_getpc_b64 %0\n.Lpc%=:\n\ts_mov_b32 %1, .Lend%=-.Lpc%=-4\n\t.subsection 2\n.Lend%=:\n\t.subsection 0" : "=s"(pc), "=s"(len));
            asm volatile("s_mov_b32 m0, %2\n\ts_mov_b64 exec, 0xffff\n\tglobal_load_lds_dword %0, %1\n\ts_mov_b64 exec, -1"
                         :: "v"(min((unsigned)(j * 2048 + lane * 128), len)), "s"(pc), "i"(LDS_CTRL + 2048) : "memory");
        }
        lds_st(ctrl + lane * 4, lane == 1 ? owed(1, 0) : lane == C_DUE / 4 ? FIRST_AT + 2 : 0);
        *(lds_i32x4_p)(ctrl + C_PROG + lane * 16) = i32x4{0, 0, 0, 0};
        asm volatile("s_waitcnt lgkmcnt(0)" ::: "memory");
    } else {
        asm volatile("s_load_dword %0, %1, 0\n\ts_load_dword %0, %1, 64\n\ts_load_dword %0, %1, 128\n\t"
                     "s_load_dword %0, %1, 192\n\ts_load_dword %0, %1, 256"
                     : "=&s"(pre) : "s"(__builtin_amdgcn_kernarg_segment_ptr()) : "memory");
    }
    asm volatile("s_barrier" : "+s"(wave) :: "memory");
#define STAMP0()
#define STAMP0B()
#define NOW(t) asm volatile("s_memrealtime %0" : "=s"(t) :: "memory")
    Ctx c;
    if (wave < 4) {
    __builtin_amdgcn_s_setprio(CPRIO);
    STAMP0();
    u64x2 own = args_ld(0);
#pragma unroll
    for (int i = 0; i < 4; ++i) {
        c.sv[i] = (mi * 128 + wave * 32 + i * 8 + (lane >> 3)) * KLOC + P * 256 + (mi >> 1) * 128 +
                  (((lane & 7) ^ ((i * 4 + (lane >> 4)) & 7)) << 4) - i * 1024;
        asm volatile("" : "+v"(c.sv[i]));
    }
    unsigned ll = lds0 + LDS_LOC + wave * 4096;
    asm volatile("s_waitcnt lgkmcnt(0)" : "+s"(own), "+s"(ll) : "s"(pre) : "memory");
    STAMP0B();
    if (HOLDX) __builtin_amdgcn_s_sleep(HOLDX);
    dma4((const char*)own[1], ll, c.sv[0], c.sv[1], c.sv[2], c.sv[3]);
    }
    NEXT_ROLE();
    Ldr l;
    if ((wave & 6) == 4) {
        __builtin_amdgcn_s_setprio(LPRIO);
        u64x2 col = args_ld(128 + ct * 16);
        unsigned f0 = (4 * P + ((wave & 1) ^ mi)) * 64 * NN, slot = lds0 + (wave & 1) * 8192;
#pragma unroll
        for (int i = 0; i < 4; ++i) {
            l.dv[i] = (lane >> 3) * NN + n0 + (((lane & 7) ^ (((lane >> 4) & 3) | ((i >> 1) << 2))) << 4) + i * (8 * NN - 1024);
            asm volatile("" : "+v"(l.dv[i]));
        }
        asm volatile("s_waitcnt lgkmcnt(0)" : "+s"(col), "+s"(f0), "+s"(slot) : "s"(pre) : "memory");
        if (HOLDW) __builtin_amdgcn_s_sleep(HOLDW);
        l.w = (const char*)col[0];
        fill(l, f0, slot);
        fill(l, f0 ^ (2 * 64 * NN), slot + 16384);
        STAMP(0);
    }
    NEXT_ROLE();
    if (wave < 4) {
    char* const ws = a.peer[0].ws;
    const unsigned tstart = (unsigned)__builtin_amdgcn_s_memrealtime();
    STAMP(19);
    c.ll = lds0 + LDS_LOC + wave * 4096;
    c.xl2 = (const char*)a.peer[0].x + (128 - (mi >> 1) * 256);
    c.la = c.ll + P * 2048 + (lane & 15) * 128 + ((((mi & 1) * 4 + (lane >> 4)) ^ ((lane & 15) >> 1)) << 4);
    c.l16 = lane * 16;
    c.chk = ctrl + C_FULL + min(lane & 3, 2) * 4;
    c.foff = (lane & 7) * 128;
    c.flags = ws + WS_FLAGS;
    c.t0 = tstart;
    c.bad = 0;
    typedef const __attribute__((address_space(4))) float* cfp;
    float scale = *(cfp)(uintptr_t)a.sx * *(cfp)(uintptr_t)a.sw;
    char* outp = (char*)a.col[ct].out;
    asm volatile("" : "+s"(outp));
    c.prog = ctrl + C_PROG + wave * 256 + lane * 4;
    c.one = 1;
    asm volatile("" : "+v"(c.one), "+v"(c.chk), "+v"(c.prog), "+v"(c.foff));
    {
        const unsigned q = (lane & 15) >> 1, kg = lane >> 4;
        const unsigned r0 = lds0 + (kg * 16 + q) * 128 + (lane & 1) * 8, f = (q >> 1) | ((kg & 1) << 2);
#pragma unroll
        for (int t = 0; t < 8; ++t) {
            c.rb[0][t] = r0 + ((t ^ f) << 4);
            c.rb[1][t] = c.rb[0][t] + 65536;
            asm volatile("" : "+v"(c.rb[0][t]), "+v"(c.rb[1][t]));
        }
    }
    i32x4 acc[2][8];
#pragma unroll
    for (int m = 0; m < 2; ++m)
#pragma unroll
        for (int t = 0; t < 8; ++t) { acc[m][t] = i32x4{0, 0, 0, 0}; asm volatile("" : "+v"(acc[m][t])); }
    i32x2 B[2][16];
    i32x4 AL[2][2][2], A[NFRAG];
    Part X;
    if constexpr (LP0 == 0) {
        asm volatile("s_waitcnt vmcnt(0)" ::: "memory");
        read_local(AL[0], c);
        {
            LGKM(0);
            dma4(c.xl2, c.ll, c.sv[0], c.sv[1], c.sv[2], c.sv[3]);
        }
    }
    STAMP(1);
    unsigned call;
    while ((call = __builtin_amdgcn_readfirstlane(lds_ld(ctrl + C_CALL))) == 0) __builtin_amdgcn_s_sleep(1);
    c.call = call;
    c.sent = call <= 2 ? 0u : POISON;
    c.tag = call & 15;
#pragma unroll
    for (int i = 0; i < 7; ++i) {
        c.ab[i] = ws + WS_DATA + (call % 3) * SLOT_BYTES + ((mi * 4 + wave) * 2 + P) * (NFRAG * 1024) + i * 8192 + 4096;
        asm volatile("" : "+s"(c.ab[i]));
    }
    c.xo = ws + WS_X + (b * 4 + wave) * XSTRIDE + 4096;
    c.xi = ws + WS_X + ((b ^ 8) * 4 + wave) * XSTRIDE + 4096;
    asm volatile("" : "+s"(c.xo), "+s"(c.xi), "+s"(c.sent), "+s"(c.tag), "+s"(c.flags), "+s"(c.bad), "+v"(c.l16));
    c.xf = ctrl + C_XRDY;
    c.xa = lds0 + LDS_PUSH + wave * 2048 + c.l16;
    c.xa3 = lds0 + LDS_X3 + wave * 2048 + c.l16;
    unsigned tw = lds0 + LDS_PUSH + wave * 2048 + (lane & 15) * 128 + (((lane >> 4) ^ (lane & 7)) << 4);
    unsigned tw1 = tw ^ 64;
    unsigned tr = lds0 + LDS_PUSH + wave * 2048 + (lane >> 3) * 128 + (((lane & 7) ^ (lane >> 3)) << 4);
    unsigned oo = (lane >> 3) * NN * 4 + (lane & 7) * 16;
    char* o0 = outp + (size_t)(unsigned)(((mi * 128 + wave * 32 + P * 16) * NN + n0) * 4);
    char* o1 = o0 + 8 * NN * 4;
    asm volatile("" : "+v"(c.xf), "+v"(c.xa), "+v"(c.xa3), "+v"(tw), "+v"(tw1), "+v"(tr), "+v"(oo), "+s"(o0), "+s"(o1), "+v"(scale));
    check<1>(c);
    STAMP(3);
#pragma unroll
    for (int t = 0; t < 8; ++t) { read_b<0, 0>(B[0], t, c); read_b<0, 1>(B[0], t, c); }
    check<2>(c);
    step<0>(c, acc, AL, A, B, X STAMP_PASS);
    asm volatile("s_nop 7\n\ts_nop 1" ::: "memory");
    SB();
    STAMP(20);

    if (c.bad) scale = __builtin_nanf("");
    i32x4 Z[2];
#define STORE_Z(p) asm volatile("global_store_dwordx4 %2, %0, %3 offset:%5 " OUT_BITS "\n\t" \
                                "global_store_dwordx4 %2, %1, %4 offset:%5 " OUT_BITS "\n\ts_nop 1" \
                                :: "v"(Z[0]), "v"(Z[1]), "v"(oo), "s"(o0), "s"(o1), "i"((p) * 128) : "memory")
#pragma unroll
    for (int p = 0; p < 4; ++p) {
        asm volatile(
            "v_cmp_ne_u32 vcc, 0, %0\n\t"
            "s_cbranch_vccz .Lagain%=\n"
            ".Lgo%=:\n\t"
            ".subsection 1\n"
            ".Lagain%=:\n\t"
            "ds_read_b32 %0, %3 offset:%5\n\t"
            "ds_read_b128 %1, %4 offset:%6\n\t"
            "ds_read_b128 %2, %4 offset:%6+1024\n\t"
            "s_waitcnt lgkmcnt(0)\n\t"
            "v_cmp_ne_u32 vcc, 0, %0\n\t"
            "s_cbranch_vccz .Lagain%=\n\t"
            "s_branch .Lgo%=\n\t"
            ".subsection 0"
            : "+v"(X.f), "+v"(X.x[0]), "+v"(X.x[1])
            : "v"(c.xf), "v"(p < 3 ? c.xa : c.xa3), "i"(p * 4), "i"(p < 3 ? p * 8192 : 0) : "memory", "vcc");
        i32x4 x[2] = {X.x[0], X.x[1]};
        if (__builtin_expect(__any(((((unsigned)x[0][0] ^ c.tag) | ((unsigned)x[1][0] ^ c.tag)) & 15) | (X.f & 2)), 0)) {
            bool late;
            do {
                asm volatile("global_load_dwordx4 %0, %2, %3 offset:%4 sc0 sc1\n\tglobal_load_dwordx4 %1, %2, %3 offset:%5 sc0 sc1\n\t"
                             "s_waitcnt vmcnt(0)"
                             : "=&v"(x[0]), "=&v"(x[1]) : "v"(c.l16), "s"(c.xi), "i"(p * 2048 - 4096), "i"(p * 2048 - 3072) : "memory");
                late = (unsigned)__builtin_amdgcn_s_memrealtime() - c.t0 > DEADLINE_TICKS + DEADLINE_TICKS / 2;
            } while (untagged(x[0], x[1], c.tag) && !late);
            if (late) { c.bad = 1; scale = __builtin_nanf(""); }
        }
        if (p == 0) STAMP(21);
        if (p == 3) STAMP(22);
        if (p < 3) { SB(); X = fetch_x(c, p + 1); SB(); }
        f32x4 y[2];
#pragma unroll
        for (int h = 0; h < 2; ++h) {
            x[h][0] >>= 4;
            const i32x4 s = acc[0][2 * p + h] + x[h];
            y[h] = f32x4{(float)s[0] * scale, (float)s[1] * scale, (float)s[2] * scale, (float)s[3] * scale};
        }
        *(lds_vi32x4_p)(tw + (p % 3) * 8192) = __builtin_bit_cast(i32x4, y[0]);
        *(lds_vi32x4_p)(tw1 + (p % 3) * 8192) = __builtin_bit_cast(i32x4, y[1]);
        if (p) STORE_Z(p - 1);
        Z[0] = *(lds_vi32x4_p)(tr + (p % 3) * 8192);
        Z[1] = *(lds_vi32x4_p)(tr + (p % 3) * 8192 + 1024);
    }
    STORE_Z(3);
    if (c.bad && lane == 0) __hip_atomic_store((unsigned*)(a.peer[0].ws + WS_TMO), call, __ATOMIC_RELAXED, __HIP_MEMORY_SCOPE_AGENT);
    STAMP(23);
    STAMP(24);
    STAMP_OUT(ws);
    __builtin_amdgcn_endpgm();
    }
    NEXT_ROLE();
    if (wave < 6) {
        l.w = (const char*)a.col[ct].w;
        l.fa = ctrl + C_FULL + (lane ? 64 + lane * 8 : 0) + (wave & 1) * 4;
        l.pa = ctrl + C_PROG + (lane & 3) * 260;
        const int r8 = a.rank * 8;
        l.woff = (((w_chunk(min(lane, NSTEP - 1), P, mi) + r8) & 63) - r8) * 64 * NN;
        l.freed = 0;
        if (wave & 1) fills<1, 1>(l STAMP_PASS);
        else fills<0, 1>(l STAMP_PASS);
        STAMP(20);
        STAMP_OUT(ws);
        __builtin_amdgcn_endpgm();
    }

    NEXT_ROLE();
    if (wave == 7) {
        const unsigned tstart = (unsigned)__builtin_amdgcn_s_memrealtime();
        char* const ws = a.peer[0].ws;
        STAMP(0);
        unsigned d[8];
        unsigned long long now;
        const char* const xi = ws + WS_X + (b ^ 8) * 4 * XSTRIDE + 2032 - XLEAD * 1024;
        const char *xp = xi, *xq = xi + 6144;
#define LOOK_X(BITS) asm volatile( \
            "s_load_dword %0, %9, 0 " BITS "\n\ts_load_dword %1, %9, %11 " BITS "\n\ts_load_dword %2, %9, %12 " BITS "\n\t" \
            "s_load_dword %3, %9, %13 " BITS "\n\ts_load_dword %4, %10, 0 " BITS "\n\ts_load_dword %5, %10, %11 " BITS "\n\t" \
            "s_load_dword %6, %10, %12 " BITS "\n\ts_load_dword %7, %10, %13 " BITS "\n\ts_memrealtime %8\n\ts_waitcnt lgkmcnt(0)" \
            : "=&s"(d[0]), "=&s"(d[1]), "=&s"(d[2]), "=&s"(d[3]), "=&s"(d[4]), "=&s"(d[5]), "=&s"(d[6]), "=&s"(d[7]), "=&s"(now) \
            : "s"(xp), "s"(xq), "i"(XSTRIDE), "i"(2 * XSTRIDE), "i"(3 * XSTRIDE) : "memory")
        int due = 1000;
        allowed<NSTEP - 1>(due, lane);
        constexpr int NI = NSPLIT / 2 + NS2;
        const int q = min(lane, NI - 1);
        const int offs = (q < NSPLIT / 2 ? 2 * q + (j >> 4) : q + NSPLIT / 2) * 1024;
        unsigned call;
        while ((call = __builtin_amdgcn_readfirstlane(lds_ld(ctrl + C_CALL))) == 0) __builtin_amdgcn_s_sleep(1);
        const unsigned sent = call <= 2 ? 0u : POISON;
        const char* af = ws + WS_DATA + (call % 3) * SLOT_BYTES + ((mi * 4 + 2 + ((j >> 3) & 1)) * 2 + P) * (NFRAG * 1024) + xcd * 128 + 112;
        int n = 0;
        STAMP(1);
        while (n < NSPLIT / 2) {
            unsigned o[4];
#pragma unroll
            for (int i = 0; i < 4; ++i) o[i] = __builtin_amdgcn_readlane(offs, n + i);
            do {
                asm volatile("s_load_dword %0, %5, %6 glc\n\ts_load_dword %1, %5, %7 glc\n\t"
                             "s_load_dword %2, %5, %8 glc\n\ts_load_dword %3, %5, %9 glc\n\t"
                             "s_memrealtime %4\n\ts_waitcnt lgkmcnt(0)"
                             : "=&s"(d[0]), "=&s"(d[1]), "=&s"(d[2]), "=&s"(d[3]), "=&s"(now)
                             : "s"(af), "s"(o[0]), "s"(o[1]), "s"(o[2]), "s"(o[3]) : "memory");
            } while (d[0] == sent && (unsigned)now - tstart < WATCH_TICKS);
            n += d[1] == sent ? 1 : d[2] == sent ? 2 : d[3] == sent ? 3 : 4;
            lds_st2(ctrl + C_DUE, __builtin_amdgcn_readlane(due, n + (n >= 4 ? LEAD : 0)), n);
        }
        while (n + 2 * LEAD < NI) {
            const char* p = af + (n + NSPLIT / 2) * 1024;
            do {
                asm volatile("s_load_dword %0, %9, 0 glc\n\ts_load_dword %1, %9, 0x400 glc\n\ts_load_dword %2, %9, 0x800 glc\n\t"
                             "s_load_dword %3, %9, 0xc00 glc\n\ts_load_dword %4, %9, 0x1000 glc\n\ts_load_dword %5, %9, 0x1400 glc\n\t"
                             "s_load_dword %6, %9, 0x1800 glc\n\ts_load_dword %7, %9, 0x1c00 glc\n\t"
                             "s_memrealtime %8\n\ts_waitcnt lgkmcnt(0)"
                             : "=&s"(d[0]), "=&s"(d[1]), "=&s"(d[2]), "=&s"(d[3]), "=&s"(d[4]), "=&s"(d[5]), "=&s"(d[6]), "=&s"(d[7]), "=&s"(now)
                             : "s"(p) : "memory");
            } while (d[0] == sent && (unsigned)now - tstart < WATCH_TICKS);
            n += d[1] == sent ? 1 : d[2] == sent ? 2 : d[3] == sent ? 3 : d[4] == sent ? 4 : d[5] == sent ? 5 : d[6] == sent ? 6 : d[7] == sent ? 7 : 8;
            lds_st1(ctrl + C_DUE, __builtin_amdgcn_readlane(due, n + 2 * LEAD));
        }
        lds_st2(ctrl + C_DUE, 1000, 1000);
        STAMP(7);
        const unsigned tag = call & 15, pa = ctrl + C_PROG + (lane & 3) * 260;
        bool late = false;
        while (released(pa) < imin(NS1, NSTEP - 14) && !(late = (unsigned)__builtin_amdgcn_s_memrealtime() - tstart > WATCH_TICKS))
            __builtin_amdgcn_s_sleep(6);
        STAMP(8);
        int p = 0;
        while (!late) {
            LOOK_X("glc");
            if (!(((d[4] ^ tag) | (d[5] ^ tag) | (d[6] ^ tag) | (d[7] ^ tag)) & 15)) break;
            late = (unsigned)now - tstart > WATCH_TICKS;
            if ((((d[0] ^ tag) | (d[1] ^ tag) | (d[2] ^ tag) | (d[3] ^ tag)) & 15) || p == 3) continue;
            xp += 2048;
            lds_st1(ctrl + C_XSEEN, ++p);
        }
        STAMP(9);
        if (!late)
            for (int q = p; q < 3; ++q) bring_x(xi - 2032 + XLEAD * 1024, q, lane);
        while (!late && released(pa) <= NSTEP - 16) late = (unsigned)__builtin_amdgcn_s_memrealtime() - tstart > WATCH_TICKS;
        if (!late) bring_x(xi - 2032 + XLEAD * 1024, 3, lane);
        lds_st1(ctrl + C_XSEEN, p + 8);
        if (!late) {
            asm volatile("s_waitcnt vmcnt(16)" ::: "memory");
            if (lane >= p && lane < 2) lds_st(ctrl + C_XRDY + lane * 4, 1);
            asm volatile("s_waitcnt vmcnt(8)" ::: "memory");
            if (lane >= p && lane == 2) lds_st(ctrl + C_XRDY + lane * 4, 1);
        }
        asm volatile("s_waitcnt vmcnt(0)" ::: "memory");
        if (lane >= p && lane < 4) lds_st(ctrl + C_XRDY + lane * 4, late ? 2 : 1);
        STAMP(10);
        STAMP_OUT(ws);
        __builtin_amdgcn_endpgm();
    }
    {
        __builtin_amdgcn_s_setprio(3);
        u64x2 own = args_ld(0);
        const int pd = ct == 3 ? 0 : ((ct ^ 4) + j) % 7 + 1, pc = j >> 2, pm = (j - (QROT ? (NRANK - pd) >> 1 : 0)) & 3;
        u64x2 to = args_ld(pd * 16);
        unsigned sv[8];
#pragma unroll
        for (int g = 0; g < 8; ++g) {
            sv[g] = (pm * 128 + g * 16 + (lane >> 2)) * KLOC + ((pc + ROT * pm) & 7) * 64 + (((lane & 3) ^ ((lane >> 4) & 3)) << 4) - (g & 3) * 1024;
            asm volatile("" : "+v"(sv[g]));
        }
        asm volatile("s_waitcnt lgkmcnt(0)" : "+s"(own), "+s"(to) : "s"(pre) : "memory");
        char* const ws = (char*)own[0];
        char* const pw = (char*)to[0];
        const char* const base = (const char*)to[1];
        unsigned cv;
        asm volatile("s_load_dword %0, %2, %3\n\ts_memrealtime %1" : "=&s"(cv), "=&s"(t64) : "s"(ws), "i"(WS_TICKET) : "memory");
        if (pd && j < XJIT) {
            dma4(base, lds0 + LDS_PUSH, sv[0], sv[1], sv[2], sv[3]);
            dma4(base, lds0 + LDS_PUSH + 4096, sv[4], sv[5], sv[6], sv[7]);
        }
        int pd2 = pd;
        asm volatile("" : "+s"(pd2) :: "memory");
        const unsigned slot = PUSH0 + ((j < 4 * NFAST ? j * PACE0 : j * PACE - 4 * NFAST * (PACE - PACE0)) >> 2);
        unsigned doff[2], poff[2];
#pragma unroll
        for (int m = 0; m < 2; ++m) {
            doff[m] = lane * 16 + WS_DATA + frag_off(NRANK - pd2, pc, pm * 8 + m);
            asm volatile("" : "+v"(doff[m]));
        }
        const unsigned roff = lds0 + LDS_PUSH + (lane & 15) * 64 + (((lane >> 4) ^ ((lane >> 2) & 3)) << 4);
        asm volatile("s_waitcnt lgkmcnt(0)" : "+s"(cv), "+s"(t64) : "s"(slot), "v"(roff) : "memory");
        const unsigned due = (unsigned)t64 + slot;
        const unsigned call = (cv >> 8) + 1;
        lds_st1(ctrl + C_CALL, (int)call);
        const unsigned cur = (call % 3) * SLOT_BYTES, old = ((call + 2) % 3) * SLOT_BYTES;
        STAMP(12);
        if (pd) {
            char* dp[4];
#pragma unroll
            for (int w = 0; w < 4; ++w) { dp[w] = pw + cur + w * (2 * NFRAG * 1024); asm volatile("" : "+s"(dp[w])); }
            __builtin_amdgcn_s_setprio(0);
            if (j >= 20) touch_x(ws + WS_X + (b ^ 8) * 4 * XSTRIDE);
            if (j >= XJIT) {
                while ((int)(due - XAHEAD - (unsigned)__builtin_amdgcn_s_memrealtime()) > 0) __builtin_amdgcn_s_sleep(4);
                __builtin_amdgcn_s_setprio(3);
                dma4(base, lds0 + LDS_PUSH, sv[0], sv[1], sv[2], sv[3]);
                dma4(base, lds0 + LDS_PUSH + 4096, sv[4], sv[5], sv[6], sv[7]);
                __builtin_amdgcn_s_setprio(0);
            }
            for (int left; (left = (int)(due - (unsigned)__builtin_amdgcn_s_memrealtime())) > 0;) {
                if (left > 40) __builtin_amdgcn_s_sleep(8);
                else __builtin_amdgcn_s_sleep(1);
            }
            __builtin_amdgcn_s_setprio(3);
            asm volatile("s_waitcnt vmcnt(0)" ::: "memory");
            i32x4 dat[8];
#pragma unroll
            for (int g = 0; g < 8; ++g) dat[g] = *(lds_i32x4_p)(roff + g * 1024);
#pragma unroll
            for (int g = 0; g < 8; ++g)
                asm volatile("global_store_dwordx4 %1, %0, %2 sc0 sc1\n\ts_nop 1" :: "v"(dat[g]), "v"(doff[g & 1]), "s"(dp[g >> 1]) : "memory");
            __builtin_amdgcn_s_setprio(0);
            STAMP(3);
            asm volatile("" : "+s"(pd2) :: "memory");
#pragma unroll
            for (int m = 0; m < 2; ++m) poff[m] = lane * 16 + WS_DATA + frag_off(pd2, pc, pm * 8 + m);
            asm volatile("s_waitcnt vmcnt(0)" :: "v"(poff[0]), "v"(poff[1]) : "memory");
            STAMP(7);
            const i32x4 poison = {(int)POISON, (int)POISON, (int)POISON, (int)POISON};
#pragma unroll
            for (int g = 0; g < 8; ++g)
                asm volatile("global_store_dwordx4 %1, %0, %2 " POISON_BITS "\n\ts_nop 1"
                             :: "v"(poison), "v"(poff[g & 1]), "s"(ws + old + (g >> 1) * (2 * NFRAG * 1024)) : "memory");
        }
        take_ticket(a, call, lane);
        if (j < 20 || !pd) touch_x(ws + WS_X + (b ^ 8) * 4 * XSTRIDE);
        __builtin_amdgcn_s_setprio(0);
        for (int p = 0, seen = 0; seen < 8;) {
            while ((seen = __builtin_amdgcn_readfirstlane(lds_ld(ctrl + C_XSEEN))) <= p) __builtin_amdgcn_s_sleep(3);
            const int p0 = p;
            for (; p < (seen & 7); ++p) bring_x(ws + WS_X + (b ^ 8) * 4 * XSTRIDE, p, lane);
            asm volatile("s_waitcnt vmcnt(0)" ::: "memory");
            if (lane >= p0 && lane < p) lds_st(ctrl + C_XRDY + lane * 4, 1);
        }
        STAMP(4);
        STAMP(5);
        STAMP_OUT(ws);
    }

}

extern "C" void kernel_launch(int rank, int nranks, ncclComm_t comm,
                              hipStream_t stream,
                              void* const* d_in, const int* in_sizes, int n_in,
                              void* d_out, int out_size,
                              const int* in_dtypes, int out_dtype,
                              void* const* d_out_all,
                              void* const* d_ws_all, size_t ws_bytes) {
    if (nranks != NRANK || (unsigned)rank >= (unsigned)NRANK || ws_bytes < WS_NEEDED || n_in < 4 ||
        out_size != MLOC * NN || in_sizes[0] != NRANK * MLOC * KLOC || in_sizes[1] != NRANK * KLOC * NN) {
        fprintf(stderr, "unexpected problem shape\n");
        return;
    }
    static bool attr_set = false;
    if (!attr_set) {
        HIP_CHECK(hipFuncSetAttribute((const void*)a2a_gemm_kernel, hipFuncAttributeMaxDynamicSharedMemorySize,
                                      (int)LDS_BYTES));
        attr_set = true;
    }
    Args a;
    for (int i = 0; i < NRANK; ++i) {
        const int p = (rank + i) & (NRANK - 1), c0 = ((i - (int)((uintptr_t)d_in[1] >> 7)) & 7) * 128;
        a.peer[i].ws = (char*)d_ws_all[p];
        a.peer[i].x = (const int8_t*)d_in[0] + (size_t)p * MLOC * KLOC;
        a.col[i].w = (const int8_t*)d_in[1] + (size_t)rank * KLOC * NN + c0;
        a.col[i].out = (float*)d_out + c0;
    }
    a.sx = (const float*)d_in[2];
    a.sw = (const float*)d_in[3];
    a.rank = rank;
    a.pad = 0;
    hipLaunchKernelGGL(a2a_gemm_kernel, dim3(NWG), dim3(NTHREADS), LDS_BYTES, stream, a);
    HIP_CHECK(hipGetLastError());
}
```

```cpp
#include <hip/hip_runtime.h>
#include <rccl/rccl.h>
#include <cstdint>
#include <cstdio>

#define HIP_CHECK(expr) do { hipError_t _e = (expr); if (_e != hipSuccess) { \
    fprintf(stderr, "HIP error %s at %s:%d\n", hipGetErrorString(_e), __FILE__, __LINE__); \
    return; } } while (0)

typedef int i32x2 __attribute__((ext_vector_type(2)));
typedef int i32x4 __attribute__((ext_vector_type(4)));
typedef float f32x4 __attribute__((ext_vector_type(4)));
typedef __attribute__((address_space(3))) i32x2* lds_i32x2_p;
typedef __attribute__((address_space(3))) i32x4* lds_i32x4_p;
typedef __attribute__((address_space(3))) int* lds_int_p;
typedef volatile __attribute__((address_space(3))) int* lds_vint_p;
typedef volatile __attribute__((address_space(3))) i32x4* lds_vi32x4_p;

#define STAMP(i) do {} while (0)
#define CSTAMP(i) do {} while (0)
#define STAMP_DECL do {} while (0)
#define STAMP_OUT(wsp) do {} while (0)
#define STAMP_ARG
#define STAMP_PASS

namespace {

constexpr int NRANK = 8;
constexpr int NN = 4096;
constexpr int KLOC = 512;
constexpr int MLOC = 512;
constexpr int NWG = 256;
constexpr int NTHREADS = 512;

#define NSPLIT 40
#define LP0 0
#define LP1 2
#define PACE 62
#define PUSH0 50
#define TFIRST 250
#define PACE0 57
#define NFAST 2
#define POISON_BITS ""
#define OUT_BITS "sc1"
#define XST_BITS "sc0 sc1"
#define XLD_BITS "sc0 sc1"
#define XRE_BITS "sc0 sc1"
#define XPF_BITS "sc1"
#define XSTRIDE 9216
#define QROT 1
#define ADIST 6
#define TICKET_AT 0
#define LEAD 0
#define ADIST2 ADIST
#define AINIT ADIST
#define SPF 0
#define FIRST_AT 2
#define LDW 3
#define LDW_FROM 14
#define LSLEEP 2
#define XLEAD 1
#define XJIT 8
#define XAHEAD 150
#define HOLDW 10
#define HOLDX 4
#define CMAP 2
#define PAUSE 0
#define PDFIX 1
#define XFIT (2 * XBIAS)
#define ROT 4
#define LPRIO 0
#define CPRIO 1
#define CS0 10
#define CS1 20
constexpr int NS1 = 4 + NSPLIT / 2, NS2 = 56 - NSPLIT, NSTEP = NS1 + NS2;
constexpr int NFRAG = 56;
static_assert(NSPLIT % 2 == 0 && NSPLIT >= 8 && NSTEP <= 64 && NSTEP % 2 == 0 && LP0 == 0 && LP1 == 2, "");

constexpr unsigned WS_TMO = 0;
constexpr unsigned WS_TICKET = 4096;
constexpr unsigned WS_FLAGS = 8192;
constexpr unsigned WS_DATA = 1u << 20;
constexpr unsigned SLOT_BYTES = 32 * NFRAG * 1024;
constexpr unsigned WS_X = WS_DATA + 3u * SLOT_BYTES;
constexpr unsigned WS_RAW = WS_X + NWG * 4u * XSTRIDE;
constexpr size_t WS_NEEDED = (size_t)WS_RAW + NWG * 4u * 8192;
constexpr int XBIAS = 1 << 21;
constexpr unsigned POISON = 0x80808080u;

constexpr unsigned LDS_PUSH = 16 * 8192;
constexpr unsigned LDS_LOC = LDS_PUSH + 8192;
constexpr unsigned LDS_CTRL = LDS_LOC + 16384;
constexpr unsigned C_FULL = 0;
constexpr unsigned C_DUE = 8;
constexpr unsigned C_CNT = 12;
constexpr unsigned C_CALL = 16;
constexpr unsigned C_PAUSE = 20;
constexpr unsigned C_XSEEN = 24;
constexpr unsigned C_XRDY = 32;
constexpr unsigned C_TICKET = 768;
constexpr unsigned C_PROG = 1024;
constexpr unsigned LDS_BYTES = LDS_CTRL + 2048 + 256;

#define DEADLINE 50000000u
constexpr unsigned WATCH_TICKS = 3000;
constexpr unsigned DEADLINE_TICKS = DEADLINE;

struct Args {
    struct { char* ws; const int8_t* x; } peer[NRANK];
    struct { const int8_t* w; float* out; } col[NRANK];
    const float* sx;
    const float* sw;
    int rank;
    int pad;
};
typedef unsigned long long u64;
typedef u64 u64x2 __attribute__((ext_vector_type(2)));
__device__ __forceinline__ u64x2 args_ld(unsigned off) {
    u64x2 r;
    asm volatile("s_load_dwordx4 %0, %1, %2" : "=s"(r) : "s"(__builtin_amdgcn_kernarg_segment_ptr()), "s"(off) : "memory");
    return r;
}

#define SB() __builtin_amdgcn_sched_barrier(0)
#define LGKM(n) __builtin_amdgcn_s_waitcnt(0xC07F | ((n) << 8))

#define DMA4(BITS) asm volatile( \
        "s_mov_b32 m0, %5\n\t" \
        "s_nop 0\n\t" \
        "global_load_lds_dwordx4 %0, %4 " BITS "\n\t" \
        "global_load_lds_dwordx4 %1, %4 offset:1024 " BITS "\n\t" \
        "global_load_lds_dwordx4 %2, %4 offset:2048 " BITS "\n\t" \
        "global_load_lds_dwordx4 %3, %4 offset:3072 " BITS \
        :: "v"(v0), "v"(v1), "v"(v2), "v"(v3), "s"(gbase), "s"(lds_addr) : "memory")
template <bool SYS = false>
__device__ __forceinline__ void dma4(const char* gbase, unsigned lds_addr,
                                     unsigned v0, unsigned v1, unsigned v2, unsigned v3) {
    if constexpr (SYS) DMA4("sc0 sc1");
    else DMA4("");
}
template <int N> __device__ __forceinline__ void wait_vm(i32x4& a) {
    asm volatile("s_waitcnt vmcnt(%1)" : "+v"(a) : "i"(N) : "memory");
}
template <int N> __device__ __forceinline__ void wait_vm(i32x4& a, i32x4& b) {
    asm volatile("s_waitcnt vmcnt(%2)" : "+v"(a), "+v"(b) : "i"(N) : "memory");
}
__device__ __forceinline__ int lds_ld(unsigned addr) {
    return __hip_atomic_load((lds_int_p)addr, __ATOMIC_RELAXED, __HIP_MEMORY_SCOPE_WORKGROUP);
}
__device__ __forceinline__ void lds_st(unsigned addr, int v) {
    __hip_atomic_store((lds_int_p)addr, v, __ATOMIC_RELAXED, __HIP_MEMORY_SCOPE_WORKGROUP);
}
__device__ __forceinline__ void lds_st1(unsigned addr, int v) {
    asm volatile("s_mov_b64 exec, 1\n\tds_write_b32 %0, %1\n\ts_mov_b64 exec, -1" :: "v"(addr), "v"(v) : "memory");
}
__device__ __forceinline__ void lds_st2(unsigned addr, int v0, int v1) {
    asm volatile("s_mov_b64 exec, 1\n\tds_write_b64 %0, %1\n\ts_mov_b64 exec, -1" :: "v"(addr), "v"(i32x2{v0, v1}) : "memory");
}
__device__ __forceinline__ void lds_wait(unsigned addr, int need) {
    while (__builtin_amdgcn_readfirstlane(
               __hip_atomic_load((lds_int_p)addr, __ATOMIC_ACQUIRE, __HIP_MEMORY_SCOPE_WORKGROUP)) < need)
        __builtin_amdgcn_s_sleep(1);
    asm volatile("" ::: "memory");
}

constexpr int imin(int x, int y) { return x < y ? x : y; }
constexpr int imax(int x, int y) { return x > y ? x : y; }
constexpr int local_of(int s) { return s >= LP0 && s < LP0 + 2 ? s - LP0 : s >= LP1 && s < LP1 + 2 ? 2 + s - LP1 : -1; }
constexpr int frag_lo(int s) {
    return s >= NS1 ? NSPLIT + s - NS1 : 2 * (s - imin(2, imax(0, s - LP0)) - imin(2, imax(0, s - LP1)));
}
__device__ __forceinline__ int pos_of(int rel, int mi) {
    if (!QROT) return rel - 1;
    const int v = (rel + 2 * mi) & 7;
    return v - (v > 2 * mi ? 1 : 0);
}
__device__ __forceinline__ int rel_of(int pos, int mi) {
    if (!QROT) return pos + 1;
    return (pos + (pos >= 2 * mi ? 1 : 0) - 2 * mi) & 7;
}
__device__ __forceinline__ unsigned frag_off(int rel, int cls, int rg) {
    const int i = 7 * cls + pos_of(rel, rg >> 3), m = rg & 1;
    const int P = i < NSPLIT ? i & 1 : m, frag = i < NSPLIT ? (i & ~1) + (m ^ P) : i;
    return (((rg >> 1) * 2 + P) * NFRAG + frag) * 1024;
}

struct Ctx {
    const char* ab[7];
    char* xo;
    const char* xi;
    const char* xl;
    const char* xl2;
    unsigned sv[4];
    unsigned ll;
    unsigned la;
    unsigned sent;
    unsigned tag;
    char* ro;
    unsigned sel[3];
    unsigned any, raw[2], b23;
    float scale;
    unsigned xf, xa;
    unsigned rb[2][8];
    unsigned l16;
    unsigned chk;
    const char* flags;
    unsigned foff;
    unsigned call, t0;
    unsigned bad;
    unsigned prog;
    int one;
    int seen;
};

__device__ __forceinline__ void read_local(i32x4 (&AL)[2][2], const Ctx& c) {
#pragma unroll
    for (int s = 0; s < 2; ++s) {
        AL[s][0] = *(lds_i32x4_p)(c.la ^ (s * 64));
        AL[s][1] = *(lds_i32x4_p)(c.la ^ (s * 64) ^ 2048);
    }
}
template <int S, int H>
__device__ __forceinline__ void read_b(i32x2 (&B)[16], int t, const Ctx& c) {
    B[2 * t + H] = __builtin_amdgcn_ds_read_tr8_b64_v2i32((lds_i32x2_p)(c.rb[(S >> 3) & 1][t] + (S & 7) * 8192 + H * 1024));
}
__device__ __forceinline__ void mma(i32x4& acc, const i32x4& a, const i32x2 (&B)[16], int t) {
    asm volatile("v_mfma_i32_16x16x64_i8 %0, %1, %2, %0"
                 : "+v"(acc) : "v"(__builtin_shufflevector(B[2 * t], B[2 * t + 1], 0, 1, 2, 3)), "v"(a));
}
template <int F>
__device__ __forceinline__ void load_frag(i32x4 (&A)[NFRAG], const Ctx& c) {
    asm volatile("global_load_dwordx4 %0, %1, %2 offset:%3 sc0 sc1"
                 : "=v"(A[F]) : "v"(c.l16), "s"(c.ab[F / 8]), "i"((F % 8) * 1024 - 4096) : "memory");
}
template <int F, int END>
__device__ __forceinline__ void first_frags(i32x4 (&A)[NFRAG], const Ctx& c, int cnt) {
    if constexpr (F < END) {
        asm volatile(
            "v_cmp_gt_i32 vcc, %[need], %[cnt]\n\t"
            "s_cbranch_vccz .Lgo%=\n"
            ".Lagain%=:\n\t"
            "ds_read_b32 %[cnt], %[at]\n\t"
            "s_waitcnt lgkmcnt(0)\n\t"
            "v_cmp_gt_i32 vcc, %[need], %[cnt]\n\t"
            "s_cbranch_vccnz .Lagain%=\n"
            ".Lgo%=:"
            : [cnt] "+v"(cnt) : [at] "v"(LDS_CTRL + C_CNT), [need] "i"(F / 2 + 1) : "memory", "vcc");
        load_frag<F>(A, c);
        load_frag<F + 1>(A, c);
        first_frags<F + 2, END>(A, c, cnt);
    }
}
template <int FULL>
__device__ __forceinline__ void check(const Ctx& c) {
    while (__any(__hip_atomic_load((lds_int_p)c.chk, __ATOMIC_ACQUIRE, __HIP_MEMORY_SCOPE_WORKGROUP) < FULL))
        __builtin_amdgcn_s_sleep(1);
    asm volatile("" ::: "memory");
}

#define CP_WAIT \
        "s_or_b64 vcc, %[low], %[miss]\n\t" \
        "s_cbranch_scc1 .Lslow%=\n" \
        ".Lback%=:\n\t" \
        ".subsection 1\n" \
        ".Lagain%=:\n\t" \
        "s_sleep 1\n" \
        ".Lslow%=:\n\t" \
        "ds_read_b32 %[p0], %[chk]\n\t" \
        "s_waitcnt lgkmcnt(0)\n\t" \
        "v_cmp_gt_i32 vcc, %[full], %[p0]\n\t" \
        "s_cbranch_vccnz .Lagain%=\n\t"
template <int FULL>
__device__ __forceinline__ void checkpoint(Ctx& c, unsigned long long low) {
    unsigned p0;
    asm volatile(CP_WAIT "s_branch .Lback%=\n\t.subsection 0"
                 : [p0] "=&v"(p0) : [low] "s"(low), [miss] "i"(0), [chk] "v"(c.chk), [full] "i"(FULL) : "memory", "vcc", "scc");
}
template <int FULL, int F>
__device__ __forceinline__ void checkpoint(i32x4 (&A)[NFRAG], Ctx& c, unsigned long long low, unsigned long long miss) {
    unsigned p0, p1, fl, prev;
    asm volatile(
        CP_WAIT
        "s_cmp_eq_u64 %[miss], 0\n\t"
        "s_cbranch_scc1 .Lback%=\n\t"
        "s_mov_b32 %[prev], 0\n"
        ".Lretry%=:\n\t"
        "global_load_dword %[p0], %[l16], %[ab] offset:%[o0] sc0 sc1\n\t"
        "global_load_dword %[p1], %[l16], %[ab] offset:%[o1] sc0 sc1\n\t"
        "global_load_dwordx4 %[a0], %[l16], %[ab] offset:%[o0] sc0 sc1\n\t"
        "global_load_dwordx4 %[a1], %[l16], %[ab] offset:%[o1] sc0 sc1\n\t"
        "global_load_dword %[fl], %[foff], %[flags] sc0 sc1\n\t"
        "s_memrealtime vcc\n\t"
        "s_waitcnt vmcnt(0) lgkmcnt(0)\n\t"
        "s_cmp_lg_u32 %[prev], 0\n\t"
        "s_cbranch_scc1 .Lback%=\n\t"
        "s_sub_u32 %[prev], vcc_lo, %[t0]\n\t"
        "v_cmp_eq_u32 vcc, %[sent], %[p0]\n\t"
        "v_cmp_eq_u32 %[miss], %[sent], %[p1]\n\t"
        "s_or_b64 vcc, vcc, %[miss]\n\t"
        "s_cbranch_scc0 .Lback%=\n\t"
        "s_cmp_gt_u32 %[prev], %[dl]\n\t"
        "s_cselect_b32 %[bad], 1, %[bad]\n\t"
        "s_cbranch_scc1 .Lback%=\n\t"
        "v_subrev_u32 %[fl], %[call], %[fl]\n\t"
        "v_cmp_gt_i32 vcc, 0, %[fl]\n\t"
        "s_cmp_eq_u64 vcc, 0\n\t"
        "s_cselect_b32 %[prev], 1, 0\n\t"
        "s_branch .Lretry%=\n\t"
        ".subsection 0"
        : [a0] "+v"(A[F]), [a1] "+v"(A[F + 1]), [p0] "=&v"(p0), [p1] "=&v"(p1), [fl] "=&v"(fl), [prev] "=&s"(prev),
          [bad] "+s"(c.bad), [miss] "+s"(miss)
        : [low] "s"(low), [chk] "v"(c.chk), [full] "i"(FULL), [l16] "v"(c.l16), [ab] "s"(c.ab[F / 8]), [foff] "v"(c.foff), [flags] "s"(c.flags),
          [o0] "i"((F % 8) * 1024 - 4096), [o1] "i"((F % 8) * 1024 - 3072), [sent] "s"(c.sent), [call] "s"(c.call),
          [t0] "s"(c.t0), [dl] "i"(DEADLINE_TICKS)
        : "memory", "vcc", "scc");
}

static_assert(ADIST2 >= ADIST && ADIST % 2 == 0 && AINIT % 2 == 0 && AINIT <= ADIST && FIRST_AT >= 2 && FIRST_AT <= 3 && NS2 % 2 == 0, "");
constexpr int issued_before(int s) {
    return s < FIRST_AT ? 0 : imin(NFRAG, imin(frag_lo(s) + (s > NS1 ? ADIST2 : ADIST), AINIT + 4 * (s - FIRST_AT)));
}
constexpr int span(int s) {
    return s + 1 >= NSTEP ? 0 : s + 1 < NS1 ? 1 : (s + 1 - NS1) % 2 == 0 ? 2 : 0;
}
constexpr int dumps(int s) { return s >= NS1 && s < NS1 + 3 ? 2 : 0; }
constexpr int ops_before_wait(int s) {
    int n = issued_before(s + 1 == FIRST_AT ? s : s + 1);
    for (int t = 0; t <= s; ++t) n += t < s ? dumps(t) : 0;
    return n;
}
constexpr int ops_through_frag(int f) {
    int s = FIRST_AT - 1;
    while (issued_before(s + 1) <= f) ++s;
    int n = f + 1;
    for (int t = 0; t < s; ++t) n += dumps(t);
    return n;
}

__device__ __forceinline__ bool untagged(const i32x4& x0, const i32x4& x1, unsigned tag) {
    return __any((((unsigned)x0[0] ^ tag) | ((unsigned)x1[0] ^ tag)) & 3);
}
constexpr unsigned x_lds(int p) { return LDS_PUSH + p * 8192; }
__device__ __forceinline__ void bring_x(const char* xp, int p, int lane) {
    const unsigned v0 = lane * 16, v2 = lane * 16 + XSTRIDE - 2048;
    dma4<true>(xp + p * 2048, x_lds(p), v0, v0, v2, v2);
    dma4<true>(xp + p * 2048 + 2 * XSTRIDE, x_lds(p) + 4096, v0, v0, v2, v2);
}
struct Part { int f; i32x4 x[2]; unsigned long long odd; };
__device__ __forceinline__ Part fetch_x(const Ctx& c, int p) {
    Part r;
    r.f = *(lds_vint_p)(c.xf + p * 4);
    r.x[0] = *(lds_vi32x4_p)(c.xa + p * 8192);
    r.x[1] = *(lds_vi32x4_p)(c.xa + p * 8192 + 1024);
    return r;
}
__device__ __forceinline__ unsigned long long odd_x(const Ctx& c, const Part& X) {
    const unsigned odd = ((((unsigned)X.x[0][0] ^ c.tag) | ((unsigned)X.x[1][0] ^ c.tag)) & 3) |
                         (((unsigned)X.x[0][1] | (unsigned)X.x[1][1]) & c.b23);
    return __builtin_amdgcn_uicmp(odd + 1, (unsigned)X.f, 33);
}
__device__ __forceinline__ void touch_x(const char* xi) {
    unsigned w;
#define TOUCH4(O) "s_load_dword %0, %1, " #O "\n\ts_load_dword %0, %1, %2+" #O "\n\ts_load_dword %0, %1, %3+" #O "\n\ts_load_dword %0, %1, %4+" #O "\n\t"
    asm volatile(TOUCH4(0) TOUCH4(2048) TOUCH4(4096) "s_waitcnt lgkmcnt(0)"
                 : "=&s"(w) : "s"(xi + 2032 - XLEAD * 1024), "i"(XSTRIDE), "i"(2 * XSTRIDE), "i"(3 * XSTRIDE) : "memory");
}
__device__ __forceinline__ unsigned long long or64(unsigned long long x, unsigned long long y) {
    unsigned long long r;
    asm volatile("s_or_b64 %0, %1, %2" : "=s"(r) : "s"(x), "s"(y) : "scc");
    return r;
}

template <int Q>
__device__ __forceinline__ void piece(Ctx& c, i32x4 (&T)[8], int part) {
    constexpr int U = Q / 3, KK = Q % 3;
    i32x4& t = T[4 * U + KK];
    if (part == 0) {
        t[0] = (t[0] << 2) | c.tag;
        t[1] |= c.raw[U];
        t[0] = __builtin_amdgcn_perm(T[4 * U + 3][0], t[0], c.sel[KK]);
    } else if (part == 1) {
#pragma unroll
        for (int e = 1; e < 4; ++e) t[e] = __builtin_amdgcn_perm(T[4 * U + 3][e], t[e], c.sel[KK]);
    } else {
        asm volatile("global_store_dwordx4 %1, %0, %2 offset:%3 sc0 sc1\n\ts_nop 1" :: "v"(t), "v"(c.l16), "s"(c.xo), "i"(Q * 1024 - 4096) : "memory");
    }
}
template <int U>
__device__ __forceinline__ void fits(Ctx& c, i32x4 (&T)[8], int part) {
    if (part < 4) {
        const i32x4& t = T[4 * U + part];
        c.any = (part ? c.any : 0u) | t[0] | t[1] | t[2] | t[3];
        asm volatile("" : "+v"(c.any));
    } else {
        asm volatile(
            "s_mov_b32 %0, 0\n\t"
            "v_cmp_le_u32 vcc, %8, %1\n\t"
            "s_cbranch_vccnz .Lraw%=\n"
            ".Lfit%=:\n\t"
            ".subsection 1\n"
            ".Lraw%=:\n\t"
            "global_store_dwordx4 %6, %2, %7 offset:%9 sc0 sc1\n\t"
            "global_store_dwordx4 %6, %3, %7 offset:%9+1024 sc0 sc1\n\t"
            "global_store_dwordx4 %6, %4, %7 offset:%9+2048 sc0 sc1\n\t"
            "global_store_dwordx4 %6, %5, %7 offset:%9+3072 sc0 sc1\n\t"
            "s_mov_b32 %0, 0x800000\n\t"
            "s_waitcnt vmcnt(0)\n\t"
            "s_branch .Lfit%=\n\t"
            ".subsection 0"
            : "=&s"(c.raw[U]) : "v"(c.any), "v"(T[4 * U]), "v"(T[4 * U + 1]), "v"(T[4 * U + 2]), "v"(T[4 * U + 3]),
              "v"(c.l16), "s"(c.ro), "i"(XFIT), "i"(U * 4096 - 4096) : "memory", "vcc");
    }
}
template <int D>
__device__ __forceinline__ void pack_x(Ctx& c, i32x4 (&T)[8], int g) {
    if constexpr (D < 2) { if (g >= 1 && g <= 5) fits<D>(c, T, g - 1); }
    if (g == 7) piece<2 * D>(c, T, 0);
    if (g == 9) piece<2 * D>(c, T, 1);
    if (g == 10) piece<2 * D>(c, T, 2);
    if (g == 11) piece<2 * D + 1>(c, T, 0);
    if (g == 13) piece<2 * D + 1>(c, T, 1);
    if (g == 15) piece<2 * D + 1>(c, T, 2);
}

template <int S>
__device__ __forceinline__ void step(Ctx& c, i32x4 (&acc)[2][8], i32x4 (&AL)[2][2][2], i32x4 (&A)[NFRAG],
                                     i32x2 (&B)[2][16], Part& X STAMP_ARG) {
    constexpr bool PH1 = S < NS1, LOC = S < 4, NEXT = S + 1 < NSTEP;
    constexpr int SPAN = span(S), FULL = imin(S + SPAN + 2, NSTEP);
    constexpr bool CP = SPAN > 0, VAL = CP && S + 1 >= 4;
    constexpr bool FREE = (S & 1) == 1 && S + 15 < NSTEP;
    constexpr int F = frag_lo(S), FN = frag_lo(S + 1);
    constexpr int L0 = issued_before(S), L1 = S + 1 == FIRST_AT ? L0 : issued_before(S + 1);
    unsigned long long m0 = 0, m1 = 0, m2 = 0;
    if constexpr ((S & 3) == 0) STAMP(4 + (S >> 2));
    if constexpr (S == 2) STAMP(15);
    if constexpr (S == CS0) { STAMP(28); CSTAMP(30); }
    if constexpr (S == CS1) { STAMP(29); CSTAMP(31); }
    if constexpr (S == FIRST_AT) first_frags<0, AINIT>(A, c, 0);
    const i32x4& a0 = LOC ? AL[LOC ? S / 2 : 0][S & 1][0] : A[F];
    const i32x4& a1 = LOC ? AL[LOC ? S / 2 : 0][S & 1][1] : A[PH1 ? F + 1 : F];
    LGKM(S == 2 ? 6 : 8);
    SB();
#pragma unroll
    for (int g = 0; g < 16; ++g) {
        const int t = g >> 1;
        if (g == 8) { if constexpr (NEXT) LGKM(8 + FREE); else LGKM(0); SB(); }
        if ((g & 1) == 0) { mma(acc[0][t], a0, B[S & 1], t); SB(); }
        else if constexpr (PH1) { mma(acc[1][t], a1, B[S & 1], t); SB(); }
        if (CP && g == 0) { c.seen = lds_ld(c.chk); SB(); }
        if constexpr (NEXT) {
            if (g & 1) read_b<S + 1, 1>(B[(S + 1) & 1], t, c);
            else if (g != 14) read_b<S + 1, 0>(B[(S + 1) & 1], t, c);
            if (g == 13) read_b<S + 1, 0>(B[(S + 1) & 1], 7, c);
        }
        if (!NEXT && g == 0) X = fetch_x(c, 0);
        if (!NEXT && g == 2) { if (c.bad & 1) c.scale = __builtin_nanf(""); asm volatile("" : "+v"(c.scale)); }
        if (!NEXT && g == 10) { X.odd = odd_x(c, X); asm volatile("" : "+s"(X.odd)); }
        if (g == 1) { if constexpr (L0 < L1) load_frag<L0>(A, c); }
        if (FREE && g == 2) __hip_atomic_fetch_add((lds_int_p)c.prog, c.one, __ATOMIC_RELAXED, __HIP_MEMORY_SCOPE_WORKGROUP);
        if (g == 3) { if constexpr (L0 + 2 < L1) load_frag<L0 + 1>(A, c); }
        if (g == 4) { if constexpr (L0 + 3 < L1) load_frag<L0 + 2>(A, c); }
        if (g == 5) { if constexpr (L0 + 1 < L1) load_frag<L1 - 1>(A, c); }
        if (CP && g == 8) m2 = __builtin_amdgcn_sicmp(c.seen, FULL, 40);
        if constexpr (VAL) {
            if (g == 9) wait_vm<ops_before_wait(S) - ops_through_frag(FN + 1)>(A[FN], A[FN + 1]);
            if (g == 10) m0 = __builtin_amdgcn_uicmp(A[FN][0], c.sent, 32);
            if (g == 11) m1 = __builtin_amdgcn_uicmp(A[FN + 1][0], c.sent, 32);
            if (g == 12) m0 = or64(m0, m1);
            if (g == 14) checkpoint<FULL, FN>(A, c, m2, m0);
        } else if constexpr (CP) {
            if (g == 14) checkpoint<FULL>(c, m2);
        }
        if (S == 1 && g == 9) {
            asm volatile("s_waitcnt vmcnt(0)" ::: "memory");
            read_local(AL[1], c);
        }
        if constexpr (dumps(S) > 0) pack_x<S - NS1>(c, acc[1], g);
        SB();
    }
    if constexpr (NEXT) step<S + 1>(c, acc, AL, A, B, X STAMP_PASS);
}

template <int S>
__device__ __forceinline__ void allowed(int& due, int n) {
    if constexpr (span(S) > 0) {
        constexpr int F = issued_before(S + span(S) + 1), NEED = F <= NSPLIT ? (F + 1) / 2 : F - NSPLIT / 2;
        if (n < NEED) due = imin(S + span(S) + 2, NSTEP) - 1;
    }
    if constexpr (S > FIRST_AT) allowed<S - 1>(due, n);
}
#define LORD 0
constexpr int LSEQ[1][2][3] = {{{0, 2, 4}, {1, 3, 5}}};
constexpr int fill_step(int h, int k) { return k < 3 ? LSEQ[LORD][h][k] : imin(NSTEP, 2 * k + h); }
constexpr int owed(int h, int k) {
    int m = fill_step(h, k);
    for (int i = k + 1; i < 3; ++i) m = imin(m, fill_step(h, i));
    return m;
}
__device__ __forceinline__ int w_chunk(int s, int P, int mi) {
    const int i = max(s < NS1 ? 2 * s - 8 + P : NSPLIT + s - NS1, 0), cls = (i * 37) >> 8;
    const int rem = rel_of(i - 7 * cls, mi) * 8 + ((cls + ROT * mi) & 7);
    return s < 4 ? 4 * P + (s ^ mi) : rem;
}
struct Ldr {
    const char* w;
    unsigned dv[4];
    int woff;
    unsigned fa, pa;
    int freed;
    int pz;
};
__device__ __forceinline__ void fill(const Ldr& l, int off, unsigned slot) {
    const char* p = l.w + off;
    dma4(p, slot, l.dv[0], l.dv[1], l.dv[2], l.dv[3]);
    dma4(p + 32 * NN, slot + 4096, l.dv[0], l.dv[1], l.dv[2], l.dv[3]);
}
__device__ __forceinline__ int released(unsigned pa) {
    int v = lds_ld(pa);
    v = min(v, __builtin_amdgcn_update_dpp(v, v, 0xB1, 0xF, 0xF, false));
    v = min(v, __builtin_amdgcn_update_dpp(v, v, 0x4E, 0xF, 0xF, false));
    return 2 * __builtin_amdgcn_readfirstlane(v);
}
template <int H, int K>
__device__ __forceinline__ void fills(Ldr& l STAMP_ARG) {
    constexpr int S = fill_step(H, K);
    if constexpr (S < NSTEP) {
        if constexpr (S >= 16) {
            if (__builtin_expect(l.freed <= S - 16, 0)) {
                while ((l.freed = released(l.pa)) <= S - 16) __builtin_amdgcn_s_sleep(LSLEEP);
            }
        }
        if constexpr (PAUSE && K > 1 && K < 12) {
            if (l.pz != 2) while ((l.pz = __builtin_amdgcn_readfirstlane(lds_ld(LDS_CTRL + C_PAUSE))) == 1) __builtin_amdgcn_s_sleep(1);
        }
        if constexpr (K > 1) fill(l, __builtin_amdgcn_readlane(l.woff, S), (S & 15) * 8192);
        constexpr int W = imin(LDW, imax(1, K - LDW_FROM + 2));
        asm volatile("s_waitcnt vmcnt(%2)\n\tds_max_i32 %0, %1" :: "v"(l.fa), "v"(owed(H, K - W + 1)), "i"(8 * W) : "memory");
        if constexpr (K < 6 || (K & 1) == 1) STAMP(K < 6 ? K : 3 + K / 2);
        fills<H, K + 1>(l STAMP_PASS);
    } else {
#pragma unroll
        for (int k = LDW - 1; k >= 0; --k)
            asm volatile("s_waitcnt vmcnt(%2)\n\tds_max_i32 %0, %1" :: "v"(l.fa), "v"(k ? fill_step(H, K - k) : NSTEP), "i"(8 * k) : "memory");
    }
}

__device__ __forceinline__ void take_ticket(const Args& a, unsigned call, int lane) {
    unsigned tk = 0;
    if (lane == 0) tk = __hip_atomic_fetch_add((unsigned*)(a.peer[0].ws + WS_TICKET), 1u, __ATOMIC_RELAXED, __HIP_MEMORY_SCOPE_AGENT);
    if ((__builtin_amdgcn_readfirstlane(tk) & 255) == 255) {
        if (lane < NRANK)
            __hip_atomic_store((unsigned*)(a.peer[lane].ws + WS_FLAGS + ((NRANK - lane) & (NRANK - 1)) * 128), call,
                               __ATOMIC_RELAXED, __HIP_MEMORY_SCOPE_SYSTEM);
    }
}

}

__global__ __launch_bounds__(NTHREADS, 2) __attribute__((aligned(4096))) void a2a_gemm_kernel(Args a) {
    extern __shared__ __attribute__((aligned(16))) char smem[];
    const int lane = threadIdx.x & 63;
    int wave = __builtin_amdgcn_readfirstlane(threadIdx.x >> 6);
    const int b = blockIdx.x;
    const int xcd = b & 7, j = b >> 3;
    const unsigned lds0 = (unsigned)(uintptr_t)(__attribute__((address_space(3))) char*)smem;
    const unsigned ctrl = lds0 + LDS_CTRL;
    const int P = j & 1;
    const int mi = (j >> 1) & 3;
    const int ct = (xcd + CMAP * (j >> 3)) & 7;
    const int n0 = (j >> 3) * 1024;
    STAMP_DECL;
#define NEXT_ROLE() asm volatile("" : "+s"(wave) :: "memory")
    u64 t64;

    unsigned pre = 0;
    if (wave == 7) {
        {
            u64 pc;
            unsigned len;
            asm volatile("s_getpc_b64 %0\n.Lpc%=:\n\ts_mov_b32 %1, .Lend%=-.Lpc%=-4\n\t.subsection 2\n.Lend%=:\n\t.subsection 0" : "=s"(pc), "=s"(len));
            asm volatile("s_mov_b32 m0, %2\n\ts_mov_b64 exec, 0xffff\n\tglobal_load_lds_dword %0, %1\n\ts_mov_b64 exec, -1"
                         :: "v"(min((unsigned)(j * 2048 + lane * 128), len)), "s"(pc), "i"(LDS_CTRL + 2048) : "memory");
        }
        lds_st(ctrl + lane * 4, lane == 1 ? owed(1, 0) : lane == C_DUE / 4 ? FIRST_AT + 2 : 0);
        *(lds_i32x4_p)(ctrl + C_PROG + lane * 16) = i32x4{0, 0, 0, 0};
        asm volatile("s_waitcnt lgkmcnt(0)" ::: "memory");
    } else {
        asm volatile("s_load_dword %0, %1, 0\n\ts_load_dword %0, %1, 64\n\ts_load_dword %0, %1, 128\n\t"
                     "s_load_dword %0, %1, 192\n\ts_load_dword %0, %1, 256"
                     : "=&s"(pre) : "s"(__builtin_amdgcn_kernarg_segment_ptr()) : "memory");
    }
    asm volatile("s_barrier" : "+s"(wave) :: "memory");
#define STAMP0()
#define STAMP0B()
#define NOW(t) asm volatile("s_memrealtime %0" : "=s"(t) :: "memory")
    Ctx c;
    if (wave < 4) {
    __builtin_amdgcn_s_setprio(CPRIO);
    STAMP0();
    u64x2 own = args_ld(0);
#pragma unroll
    for (int i = 0; i < 4; ++i) {
        c.sv[i] = (mi * 128 + wave * 32 + i * 8 + (lane >> 3)) * KLOC + P * 256 + (mi >> 1) * 128 +
                  (((lane & 7) ^ ((i * 4 + (lane >> 4)) & 7)) << 4) - i * 1024;
        asm volatile("" : "+v"(c.sv[i]));
    }
    unsigned ll = lds0 + LDS_LOC + wave * 4096;
    asm volatile("s_waitcnt lgkmcnt(0)" : "+s"(own), "+s"(ll) : "s"(pre) : "memory");
    STAMP0B();
    if (HOLDX) __builtin_amdgcn_s_sleep(HOLDX);
    dma4((const char*)own[1], ll, c.sv[0], c.sv[1], c.sv[2], c.sv[3]);
    }
    NEXT_ROLE();
    Ldr l;
    if ((wave & 6) == 4) {
        __builtin_amdgcn_s_setprio(LPRIO);
        u64x2 col = args_ld(128 + ct * 16);
        unsigned f0 = (4 * P + ((wave & 1) ^ mi)) * 64 * NN, slot = lds0 + (wave & 1) * 8192;
#pragma unroll
        for (int i = 0; i < 4; ++i) {
            l.dv[i] = (lane >> 3) * NN + n0 + (((lane & 7) ^ (((lane >> 4) & 3) | ((i >> 1) << 2))) << 4) + i * (8 * NN - 1024);
            asm volatile("" : "+v"(l.dv[i]));
        }
        asm volatile("s_waitcnt lgkmcnt(0)" : "+s"(col), "+s"(f0), "+s"(slot) : "s"(pre) : "memory");
        if (HOLDW) __builtin_amdgcn_s_sleep(HOLDW);
        l.w = (const char*)col[0];
        fill(l, f0, slot);
        fill(l, f0 ^ (2 * 64 * NN), slot + 16384);
        STAMP(0);
    }
    NEXT_ROLE();
    if (wave < 4) {
    char* const ws = a.peer[0].ws;
    const unsigned tstart = (unsigned)__builtin_amdgcn_s_memrealtime();
    STAMP(19);
    c.ll = lds0 + LDS_LOC + wave * 4096;
    c.xl2 = (const char*)a.peer[0].x + (128 - (mi >> 1) * 256);
    c.la = c.ll + P * 2048 + (lane & 15) * 128 + ((((mi & 1) * 4 + (lane >> 4)) ^ ((lane & 15) >> 1)) << 4);
    c.l16 = lane * 16;
    c.chk = ctrl + C_FULL + min(lane & 3, 2) * 4;
    c.foff = (lane & 7) * 128;
    c.flags = ws + WS_FLAGS;
    c.t0 = tstart;
    c.bad = 0;
    typedef const __attribute__((address_space(4))) float* cfp;
    c.scale = *(cfp)(uintptr_t)a.sx * *(cfp)(uintptr_t)a.sw;
    char* outp = (char*)a.col[ct].out;
    asm volatile("" : "+s"(outp));
    c.prog = ctrl + C_PROG + wave * 256 + lane * 4;
    c.one = 1;
    asm volatile("" : "+v"(c.one), "+v"(c.chk), "+v"(c.prog), "+v"(c.foff));
    {
        const unsigned q = (lane & 15) >> 1, kg = lane >> 4;
        const unsigned r0 = lds0 + (kg * 16 + q) * 128 + (lane & 1) * 8, f = (q >> 1) | ((kg & 1) << 2);
#pragma unroll
        for (int t = 0; t < 8; ++t) {
            c.rb[0][t] = r0 + ((t ^ f) << 4);
            c.rb[1][t] = c.rb[0][t] + 65536;
            asm volatile("" : "+v"(c.rb[0][t]), "+v"(c.rb[1][t]));
        }
    }
    i32x4 acc[2][8];
#pragma unroll
    for (int m = 0; m < 2; ++m)
#pragma unroll
        for (int t = 0; t < 8; ++t) { acc[m][t] = (m ? XBIAS : -XBIAS) + i32x4{0, 0, 0, 0}; asm volatile("" : "+v"(acc[m][t])); }
    i32x2 B[2][16];
    i32x4 AL[2][2][2], A[NFRAG];
    Part X;
    if constexpr (LP0 == 0) {
        asm volatile("s_waitcnt vmcnt(0)" ::: "memory");
        read_local(AL[0], c);
        {
            LGKM(0);
            dma4(c.xl2, c.ll, c.sv[0], c.sv[1], c.sv[2], c.sv[3]);
        }
    }
    STAMP(1);
    unsigned call;
    while ((call = __builtin_amdgcn_readfirstlane(lds_ld(ctrl + C_CALL))) == 0) __builtin_amdgcn_s_sleep(1);
    c.call = call;
    c.sent = call <= 2 ? 0u : POISON;
    c.tag = call % 3 + 1;
    c.ro = ws + WS_RAW + (b * 4 + wave) * 8192 + 4096;
    c.sel[0] = 0x04020100; c.sel[1] = 0x05020100; c.sel[2] = 0x06020100;
    asm volatile("" : "+s"(c.ro), "+s"(c.sel[0]), "+s"(c.sel[1]), "+s"(c.sel[2]));
#pragma unroll
    for (int i = 0; i < 7; ++i) {
        c.ab[i] = ws + WS_DATA + (call % 3) * SLOT_BYTES + ((mi * 4 + wave) * 2 + P) * (NFRAG * 1024) + i * 8192 + 4096;
        asm volatile("" : "+s"(c.ab[i]));
    }
    c.xo = ws + WS_X + (b * 4 + wave) * XSTRIDE + 4096;
    c.xi = ws + WS_X + ((b ^ 8) * 4 + wave) * XSTRIDE + 4096;
    asm volatile("" : "+s"(c.xo), "+s"(c.xi), "+s"(c.sent), "+s"(c.tag), "+s"(c.flags), "+s"(c.bad), "+v"(c.l16));
    c.xf = ctrl + C_XRDY;
    c.xa = lds0 + LDS_PUSH + wave * 2048 + c.l16;
    unsigned tw = lds0 + LDS_PUSH + wave * 2048 + (lane & 15) * 128 + (((lane >> 4) ^ (lane & 7)) << 4);
    unsigned tw1 = tw ^ 64;
    unsigned tr = lds0 + LDS_PUSH + wave * 2048 + (lane >> 3) * 128 + (((lane & 7) ^ (lane >> 3)) << 4);
    unsigned oo = (lane >> 3) * NN * 4 + (lane & 7) * 16;
    char* o0 = outp + (size_t)(unsigned)(((mi * 128 + wave * 32 + P * 16) * NN + n0) * 4);
    char* o1 = o0 + 8 * NN * 4;
    unsigned sel0 = 0x0c0c0703, sel1 = 0x0c070100;
    c.b23 = 0x800000;
    const char* ri = ws + WS_RAW + ((b ^ 8) * 4 + wave) * 8192 + 4096;
    asm volatile("" : "+s"(sel0), "+s"(sel1), "+s"(c.b23), "+s"(ri));
    asm volatile("" : "+v"(c.xf), "+v"(c.xa), "+v"(tw), "+v"(tw1), "+v"(tr), "+v"(oo), "+s"(o0), "+s"(o1), "+v"(c.scale));
    check<1>(c);
    STAMP(3);
#pragma unroll
    for (int t = 0; t < 8; ++t) { read_b<0, 0>(B[0], t, c); read_b<0, 1>(B[0], t, c); }
    check<2>(c);
    step<0>(c, acc, AL, A, B, X STAMP_PASS);
    SB();
    STAMP(20);

    i32x4 Z[2];
#define STORE_Z(p) asm volatile("global_store_dwordx4 %2, %0, %3 offset:%5 " OUT_BITS "\n\t" \
                                "global_store_dwordx4 %2, %1, %4 offset:%5 " OUT_BITS "\n\ts_nop 1" \
                                :: "v"(Z[0]), "v"(Z[1]), "v"(oo), "s"(o0), "s"(o1), "i"((p) * 128) : "memory")
    i32x4 Q[6];
#pragma unroll
    for (int p = 0; p < 4; ++p) {
        if (p < 3) {
            i32x4 &x0 = Q[2 * p], &x1 = Q[2 * p + 1];
            if (p) X.odd = odd_x(c, X);
            if (__builtin_expect(X.odd != 0, 0)) {
                asm volatile("s_nop 7\n\ts_nop 1" ::: "memory");
                while (__builtin_amdgcn_readfirstlane(X.f) == 0) X = fetch_x(c, p);
                if (__builtin_expect(untagged(X.x[0], X.x[1], c.tag) | (__builtin_amdgcn_readfirstlane(X.f) != 1), 0)) {
                    bool late;
                    do {
                        asm volatile("global_load_dwordx4 %0, %2, %3 offset:%4 sc0 sc1\n\tglobal_load_dwordx4 %1, %2, %3 offset:%5 sc0 sc1\n\t"
                                     "s_waitcnt vmcnt(0)"
                                     : "=&v"(X.x[0]), "=&v"(X.x[1]) : "v"(c.l16), "s"(c.xi), "i"(p * 2048 - 4096), "i"(p * 2048 - 3072) : "memory");
                        late = (unsigned)__builtin_amdgcn_s_memrealtime() - c.t0 > DEADLINE_TICKS + DEADLINE_TICKS / 2;
                    } while (untagged(X.x[0], X.x[1], c.tag) && !late);
                    if (late) { c.bad = 1; c.scale = __builtin_nanf(""); }
                }
#pragma unroll
                for (int h = 0; h < 2; ++h) {
                    if (__builtin_expect(__builtin_amdgcn_readfirstlane(X.x[h][1]) & 0x800000, 0)) {
                        X.x[h] = i32x4{0, 0, 0, 0};
                        if ((2 * p + h) % 3 == 0) {
#pragma unroll
                            for (int t = 0; t < 4; ++t) {
                                i32x4 r;
                                asm volatile("global_load_dwordx4 %0, %1, %2 offset:%3 sc0 sc1\n\ts_waitcnt vmcnt(0)"
                                             : "=&v"(r) : "v"(c.l16), "s"(ri), "i"(t * 1024 + (p & 1) * 4096 - 4096) : "memory");
                                acc[0][4 * (p & 1) + t] += r;
                            }
                        }
                    }
                }
            }
            x0 = X.x[0];
            x1 = X.x[1];
            if (p < 2) { SB(); X = fetch_x(c, p + 1); SB(); }
        }
        if (p == 0) STAMP(21);
        if (p == 2) STAMP(22);
        const int u3 = p / 2 * 3;
        i32x4 s[2];
#pragma unroll
        for (int h = 0; h < 2; ++h) {
            const int k = (p & 1) * 2 + h;
            const i32x4& own = acc[0][2 * p + h];
#pragma unroll
            for (int e = 0; e < 4; ++e) {
                if (k == 3) s[h][e] = own[e] + __builtin_amdgcn_perm(Q[u3 + 2][e], __builtin_amdgcn_perm(Q[u3 + 1][e], Q[u3][e], sel0), sel1);
                else if (e == 0) s[h][e] = own[e] + __builtin_amdgcn_ubfe(Q[u3 + k][e], 2, 22);
                else asm("v_mad_u32_u24 %0, %1, 1, %2" : "=v"(s[h][e]) : "v"(Q[u3 + k][e]), "v"(own[e]));
            }
        }
        f32x4 y[2];
#pragma unroll
        for (int h = 0; h < 2; ++h)
            y[h] = f32x4{(float)s[h][0] * c.scale, (float)s[h][1] * c.scale, (float)s[h][2] * c.scale, (float)s[h][3] * c.scale};
        *(lds_vi32x4_p)(tw + (p % 3) * 8192) = __builtin_bit_cast(i32x4, y[0]);
        *(lds_vi32x4_p)(tw1 + (p % 3) * 8192) = __builtin_bit_cast(i32x4, y[1]);
        if (p) STORE_Z(p - 1);
        Z[0] = *(lds_vi32x4_p)(tr + (p % 3) * 8192);
        Z[1] = *(lds_vi32x4_p)(tr + (p % 3) * 8192 + 1024);
    }
    STORE_Z(3);
    if (c.bad && lane == 0) __hip_atomic_store((unsigned*)(a.peer[0].ws + WS_TMO), call, __ATOMIC_RELAXED, __HIP_MEMORY_SCOPE_AGENT);
    STAMP(23);
    STAMP(24);
    STAMP_OUT(ws);
    __builtin_amdgcn_endpgm();
    }
    NEXT_ROLE();
    if (wave < 6) {
        l.w = (const char*)a.col[ct].w;
        l.fa = ctrl + C_FULL + (lane ? 64 + lane * 8 : 0) + (wave & 1) * 4;
        l.pa = ctrl + C_PROG + (lane & 3) * 260;
        const int r8 = a.rank * 8;
        l.woff = (((w_chunk(min(lane, NSTEP - 1), P, mi) + r8) & 63) - r8) * 64 * NN;
        l.freed = 0;
        l.pz = 0;
        if (wave & 1) fills<1, 1>(l STAMP_PASS);
        else fills<0, 1>(l STAMP_PASS);
        STAMP(20);
        STAMP_OUT(ws);
        __builtin_amdgcn_endpgm();
    }

    NEXT_ROLE();
    if (wave == 7) {
        const unsigned tstart = (unsigned)__builtin_amdgcn_s_memrealtime();
        char* const ws = a.peer[0].ws;
        STAMP(0);
        unsigned d[8];
        unsigned long long now;
        const char* const xi = ws + WS_X + (b ^ 8) * 4 * XSTRIDE + 2032 - XLEAD * 1024;
        const char *xp = xi, *xq = xi + 4096;
#define LOOK_X(BITS) asm volatile( \
            "s_load_dword %0, %9, 0 " BITS "\n\ts_load_dword %1, %9, %11 " BITS "\n\ts_load_dword %2, %9, %12 " BITS "\n\t" \
            "s_load_dword %3, %9, %13 " BITS "\n\ts_load_dword %4, %10, 0 " BITS "\n\ts_load_dword %5, %10, %11 " BITS "\n\t" \
            "s_load_dword %6, %10, %12 " BITS "\n\ts_load_dword %7, %10, %13 " BITS "\n\ts_memrealtime %8\n\ts_waitcnt lgkmcnt(0)" \
            : "=&s"(d[0]), "=&s"(d[1]), "=&s"(d[2]), "=&s"(d[3]), "=&s"(d[4]), "=&s"(d[5]), "=&s"(d[6]), "=&s"(d[7]), "=&s"(now) \
            : "s"(xp), "s"(xq), "i"(XSTRIDE), "i"(2 * XSTRIDE), "i"(3 * XSTRIDE) : "memory")
        int due = 1000;
        allowed<NSTEP - 1>(due, lane);
        constexpr int NI = NSPLIT / 2 + NS2;
        const int q = min(lane, NI - 1);
        const int offs = (q < NSPLIT / 2 ? 2 * q + (j >> 4) : q + NSPLIT / 2) * 1024;
        unsigned call;
        while ((call = __builtin_amdgcn_readfirstlane(lds_ld(ctrl + C_CALL))) == 0) __builtin_amdgcn_s_sleep(1);
        const unsigned sent = call <= 2 ? 0u : POISON;
        const char* af = ws + WS_DATA + (call % 3) * SLOT_BYTES + ((mi * 4 + 2 + ((j >> 3) & 1)) * 2 + P) * (NFRAG * 1024) + xcd * 128 + 112;
        int n = 0;
        STAMP(1);
        while (n < NSPLIT / 2) {
            unsigned o[4];
#pragma unroll
            for (int i = 0; i < 4; ++i) o[i] = __builtin_amdgcn_readlane(offs, n + i);
            do {
                asm volatile("s_load_dword %0, %5, %6 glc\n\ts_load_dword %1, %5, %7 glc\n\t"
                             "s_load_dword %2, %5, %8 glc\n\ts_load_dword %3, %5, %9 glc\n\t"
                             "s_memrealtime %4\n\ts_waitcnt lgkmcnt(0)"
                             : "=&s"(d[0]), "=&s"(d[1]), "=&s"(d[2]), "=&s"(d[3]), "=&s"(now)
                             : "s"(af), "s"(o[0]), "s"(o[1]), "s"(o[2]), "s"(o[3]) : "memory");
            } while (d[0] == sent && (unsigned)now - tstart < WATCH_TICKS);
            n += d[1] == sent ? 1 : d[2] == sent ? 2 : d[3] == sent ? 3 : 4;
            lds_st2(ctrl + C_DUE, __builtin_amdgcn_readlane(due, n + (n >= 4 ? LEAD : 0)), n);
        }
        while (n + 2 * LEAD < NI) {
            const char* p = af + (n + NSPLIT / 2) * 1024;
            do {
                asm volatile("s_load_dword %0, %9, 0 glc\n\ts_load_dword %1, %9, 0x400 glc\n\ts_load_dword %2, %9, 0x800 glc\n\t"
                             "s_load_dword %3, %9, 0xc00 glc\n\ts_load_dword %4, %9, 0x1000 glc\n\ts_load_dword %5, %9, 0x1400 glc\n\t"
                             "s_load_dword %6, %9, 0x1800 glc\n\ts_load_dword %7, %9, 0x1c00 glc\n\t"
                             "s_memrealtime %8\n\ts_waitcnt lgkmcnt(0)"
                             : "=&s"(d[0]), "=&s"(d[1]), "=&s"(d[2]), "=&s"(d[3]), "=&s"(d[4]), "=&s"(d[5]), "=&s"(d[6]), "=&s"(d[7]), "=&s"(now)
                             : "s"(p) : "memory");
            } while (d[0] == sent && (unsigned)now - tstart < WATCH_TICKS);
            n += d[1] == sent ? 1 : d[2] == sent ? 2 : d[3] == sent ? 3 : d[4] == sent ? 4 : d[5] == sent ? 5 : d[6] == sent ? 6 : d[7] == sent ? 7 : 8;
            lds_st1(ctrl + C_DUE, __builtin_amdgcn_readlane(due, n + 2 * LEAD));
        }
        lds_st2(ctrl + C_DUE, 1000, 1000);
        STAMP(7);
        const unsigned tag = call % 3 + 1, pa = ctrl + C_PROG + (lane & 3) * 260;
        bool late = false;
        while (released(pa) < imin(NS1, NSTEP - 16) && !(late = (unsigned)__builtin_amdgcn_s_memrealtime() - tstart > WATCH_TICKS))
            __builtin_amdgcn_s_sleep(6);
        lds_wait(ctrl + C_PAUSE, 2);
        STAMP(8);
        int p = 0;
        while (!late) {
            LOOK_X("glc");
            if (!(((d[4] ^ tag) | (d[5] ^ tag) | (d[6] ^ tag) | (d[7] ^ tag)) & 3)) break;
            late = (unsigned)now - tstart > WATCH_TICKS;
            if ((((d[0] ^ tag) | (d[1] ^ tag) | (d[2] ^ tag) | (d[3] ^ tag)) & 3) || p == 2) continue;
            xp += 2048;
            lds_st1(ctrl + C_XSEEN, ++p);
        }
        STAMP(9);
        if (!late)
            for (int q = p; q < 3; ++q) bring_x(xi - 2032 + XLEAD * 1024, q, lane);
        lds_st1(ctrl + C_XSEEN, p + 8);
        if (!late) {
            asm volatile("s_waitcnt vmcnt(16)" ::: "memory");
            if (lane >= p && lane == 0) lds_st(ctrl + C_XRDY + lane * 4, 1);
            asm volatile("s_waitcnt vmcnt(8)" ::: "memory");
            if (lane >= p && lane == 1) lds_st(ctrl + C_XRDY + lane * 4, 1);
        }
        asm volatile("s_waitcnt vmcnt(0)" ::: "memory");
        if (lane >= p && lane < 3) lds_st(ctrl + C_XRDY + lane * 4, late ? 16 : 1);
        STAMP(10);
        STAMP_OUT(ws);
        __builtin_amdgcn_endpgm();
    }
    {
        __builtin_amdgcn_s_setprio(3);
        u64x2 own = args_ld(0);
        const int pd = PDFIX == 1 ? ((ct ^ 4) + 1) & 7 : PDFIX == 2 ? (ct == 3 ? 0 : xcd ? xcd : (3 - CMAP * (j >> 3)) & 7) : ct == 3 ? 0 : ((ct ^ 4) + j) % 7 + 1, pc = j >> 2, pm = (j - (QROT ? (NRANK - pd) >> 1 : 0)) & 3;
        u64x2 to = args_ld(pd * 16);
        unsigned sv[8];
#pragma unroll
        for (int g = 0; g < 8; ++g) {
            sv[g] = (pm * 128 + g * 16 + (lane >> 2)) * KLOC + ((pc + ROT * pm) & 7) * 64 + (((lane & 3) ^ ((lane >> 4) & 3)) << 4) - (g & 3) * 1024;
            asm volatile("" : "+v"(sv[g]));
        }
        asm volatile("s_waitcnt lgkmcnt(0)" : "+s"(own), "+s"(to) : "s"(pre) : "memory");
        char* const ws = (char*)own[0];
        char* const pw = (char*)to[0];
        const char* const base = (const char*)to[1];
        unsigned cv;
        asm volatile("s_load_dword %0, %2, %3\n\ts_memrealtime %1" : "=&s"(cv), "=&s"(t64) : "s"(ws), "i"(WS_TICKET) : "memory");
        if (pd && j < XJIT) {
            dma4(base, lds0 + LDS_PUSH, sv[0], sv[1], sv[2], sv[3]);
            dma4(base, lds0 + LDS_PUSH + 4096, sv[4], sv[5], sv[6], sv[7]);
        }
        int pd2 = pd;
        asm volatile("" : "+s"(pd2) :: "memory");
        const unsigned slot = PUSH0 + ((j < 4 * NFAST ? j * PACE0 : j * PACE - 4 * NFAST * (PACE - PACE0)) >> 2);
        unsigned doff[2], poff[2];
#pragma unroll
        for (int m = 0; m < 2; ++m) {
            doff[m] = lane * 16 + WS_DATA + frag_off(NRANK - pd2, pc, pm * 8 + m);
            asm volatile("" : "+v"(doff[m]));
        }
        const unsigned roff = lds0 + LDS_PUSH + (lane & 15) * 64 + (((lane >> 4) ^ ((lane >> 2) & 3)) << 4);
        asm volatile("s_waitcnt lgkmcnt(0)" : "+s"(cv), "+s"(t64) : "s"(slot), "v"(roff) : "memory");
        const unsigned due = (unsigned)t64 + slot;
        const unsigned call = (cv >> 8) + 1;
        lds_st1(ctrl + C_CALL, (int)call);
        const unsigned cur = (call % 3) * SLOT_BYTES, old = ((call + 2) % 3) * SLOT_BYTES;
        STAMP(12);
        if (pd) {
            char* dp[4];
#pragma unroll
            for (int w = 0; w < 4; ++w) { dp[w] = pw + cur + w * (2 * NFRAG * 1024); asm volatile("" : "+s"(dp[w])); }
            __builtin_amdgcn_s_setprio(0);
            if (j >= 20) touch_x(ws + WS_X + (b ^ 8) * 4 * XSTRIDE);
            if (j >= XJIT) {
                while ((int)(due - XAHEAD - (unsigned)__builtin_amdgcn_s_memrealtime()) > 0) __builtin_amdgcn_s_sleep(4);
                __builtin_amdgcn_s_setprio(3);
                dma4(base, lds0 + LDS_PUSH, sv[0], sv[1], sv[2], sv[3]);
                dma4(base, lds0 + LDS_PUSH + 4096, sv[4], sv[5], sv[6], sv[7]);
                __builtin_amdgcn_s_setprio(0);
            }
            if (PAUSE) {
                for (int left; (left = (int)(due - PAUSE - (unsigned)__builtin_amdgcn_s_memrealtime())) > 0;) {
                    if (left > 40) __builtin_amdgcn_s_sleep(8);
                    else __builtin_amdgcn_s_sleep(1);
                }
                lds_st1(ctrl + C_PAUSE, 1);
            }
            for (int left; (left = (int)(due - (unsigned)__builtin_amdgcn_s_memrealtime())) > 0;) {
                if (left > 40) __builtin_amdgcn_s_sleep(8);
                else __builtin_amdgcn_s_sleep(1);
            }
            __builtin_amdgcn_s_setprio(3);
            STAMP(2);
            asm volatile("s_waitcnt vmcnt(0)" ::: "memory");
            STAMP(11);
            i32x4 dat[8];
#pragma unroll
            for (int g = 0; g < 8; ++g) dat[g] = *(lds_i32x4_p)(roff + g * 1024);
#pragma unroll
            for (int g = 0; g < 8; ++g)
                asm volatile("global_store_dwordx4 %1, %0, %2 sc0 sc1\n\ts_nop 1" :: "v"(dat[g]), "v"(doff[g & 1]), "s"(dp[g >> 1]) : "memory");
            lds_st1(ctrl + C_PAUSE, 2);
            __builtin_amdgcn_s_setprio(0);
            STAMP(3);
            asm volatile("" : "+s"(pd2) :: "memory");
#pragma unroll
            for (int m = 0; m < 2; ++m) poff[m] = lane * 16 + WS_DATA + frag_off(pd2, pc, pm * 8 + m);
            asm volatile("s_waitcnt vmcnt(0)" :: "v"(poff[0]), "v"(poff[1]) : "memory");
            STAMP(7);
            const i32x4 poison = {(int)POISON, (int)POISON, (int)POISON, (int)POISON};
#pragma unroll
            for (int g = 0; g < 8; ++g)
                asm volatile("global_store_dwordx4 %1, %0, %2 " POISON_BITS "\n\ts_nop 1"
                             :: "v"(poison), "v"(poff[g & 1]), "s"(ws + old + (g >> 1) * (2 * NFRAG * 1024)) : "memory");
        }
        lds_st1(ctrl + C_PAUSE, 2);
        take_ticket(a, call, lane);
        if (j < 20 || !pd) touch_x(ws + WS_X + (b ^ 8) * 4 * XSTRIDE);
        __builtin_amdgcn_s_setprio(0);
        for (int p = 0, seen = 0; seen < 8;) {
            while ((seen = __builtin_amdgcn_readfirstlane(lds_ld(ctrl + C_XSEEN))) <= p) __builtin_amdgcn_s_sleep(3);
            const int p0 = p;
            for (; p < (seen & 7); ++p) bring_x(ws + WS_X + (b ^ 8) * 4 * XSTRIDE, p, lane);
            asm volatile("s_waitcnt vmcnt(0)" ::: "memory");
            if (lane >= p0 && lane < p) lds_st(ctrl + C_XRDY + lane * 4, 1);
        }
        STAMP(4);
        STAMP(5);
        STAMP_OUT(ws);
    }

}

extern "C" void kernel_launch(int rank, int nranks, ncclComm_t comm,
                              hipStream_t stream,
                              void* const* d_in, const int* in_sizes, int n_in,
                              void* d_out, int out_size,
                              const int* in_dtypes, int out_dtype,
                              void* const* d_out_all,
                              void* const* d_ws_all, size_t ws_bytes) {
    if (nranks != NRANK || (unsigned)rank >= (unsigned)NRANK || ws_bytes < WS_NEEDED || n_in < 4 ||
        out_size != MLOC * NN || in_sizes[0] != NRANK * MLOC * KLOC || in_sizes[1] != NRANK * KLOC * NN) {
        fprintf(stderr, "unexpected problem shape\n");
        return;
    }
    static bool attr_set = false;
    if (!attr_set) {
        HIP_CHECK(hipFuncSetAttribute((const void*)a2a_gemm_kernel, hipFuncAttributeMaxDynamicSharedMemorySize,
                                      (int)LDS_BYTES));
        attr_set = true;
    }
    Args a;
    for (int i = 0; i < NRANK; ++i) {
        const int p = (rank + i) & (NRANK - 1), c0 = ((i - (int)((uintptr_t)d_in[1] >> 7)) & 7) * 128;
        a.peer[i].ws = (char*)d_ws_all[p];
        a.peer[i].x = (const int8_t*)d_in[0] + (size_t)p * MLOC * KLOC;
        a.col[i].w = (const int8_t*)d_in[1] + (size_t)rank * KLOC * NN + c0;
        a.col[i].out = (float*)d_out + c0;
    }
    a.sx = (const float*)d_in[2];
    a.sw = (const float*)d_in[3];
    a.rank = rank;
    a.pad = 0;
    hipLaunchKernelGGL(a2a_gemm_kernel, dim3(NWG), dim3(NTHREADS), LDS_BYTES, stream, a);
    HIP_CHECK(hipGetLastError());
}
```

```cpp
#include <hip/hip_runtime.h>
#include <rccl/rccl.h>
#include <cstdint>
#include <cstdio>

#define HIP_CHECK(expr) do { hipError_t _e = (expr); if (_e != hipSuccess) { \
    fprintf(stderr, "HIP error %s at %s:%d\n", hipGetErrorString(_e), __FILE__, __LINE__); \
    return; } } while (0)

typedef int i32x2 __attribute__((ext_vector_type(2)));
typedef int i32x4 __attribute__((ext_vector_type(4)));
typedef float f32x4 __attribute__((ext_vector_type(4)));
typedef __attribute__((address_space(3))) i32x2* lds_i32x2_p;
typedef __attribute__((address_space(3))) i32x4* lds_i32x4_p;
typedef __attribute__((address_space(3))) int* lds_int_p;
typedef volatile __attribute__((address_space(3))) int* lds_vint_p;
typedef volatile __attribute__((address_space(3))) i32x4* lds_vi32x4_p;

#define STAMP(i) do {} while (0)
#define CSTAMP(i) do {} while (0)
#define STAMP_DECL do {} while (0)
#define STAMP_OUT(wsp) do {} while (0)
#define STAMP_ARG
#define STAMP_PASS

namespace {

constexpr int NRANK = 8;
constexpr int NN = 4096;
constexpr int KLOC = 512;
constexpr int MLOC = 512;
constexpr int NWG = 256;
constexpr int NTHREADS = 512;

#define NSPLIT 44
#define LP0 0
#define LP1 2
#define PACE 64
#define PUSH0 50
#define TFIRST 250
#define PACE0 58
#define NFAST 2
#define POISON_BITS ""
#define OUT_BITS "sc1"
#define XST_BITS "sc0 sc1"
#define XLD_BITS "sc0 sc1"
#define XRE_BITS "sc0 sc1"
#define XPF_BITS "sc1"
#define XSTRIDE 7168
#define QROT 1
#define ADIST 6
#define TICKET_AT 0
#define LEAD 0
#define ADIST2 ADIST
#define AINIT ADIST
#define SPF 0
#define FIRST_AT 2
#define LDW 3
#define LDW_FROM 14
#define LSLEEP 2
#define XLEAD 0
#define XJIT 8
#define XAHEAD 150
#define HOLDW 10
#define HOLDX 4
#define CMAP 2
#define PAUSE 45
#define PDFIX 0
#define XFIT (2 * XBIAS)
#define WREST 0
#define DUMPNOW 1
#define T2AT 2
#define T2PER 4
#define AUP 99
#define AUP2 99
#define LATEW 0
#define TLBW 0
#define ROT 4
#define LPRIO 0
#define CPRIO 1
#define CS0 10
#define CS1 20
constexpr int NS1 = 4 + NSPLIT / 2, NS2 = 56 - NSPLIT, NSTEP = NS1 + NS2;
constexpr int NFRAG = 56;
static_assert(NSPLIT % 2 == 0 && NSPLIT >= 8 && NSTEP <= 64 && NSTEP % 2 == 0 && LP0 == 0 && LP1 == 2, "");

constexpr unsigned WS_TMO = 0;
constexpr unsigned WS_TICKET = 4096;
constexpr unsigned WS_FLAGS = 8192;
constexpr unsigned WS_DATA = 2u << 20;
constexpr unsigned SLOT_BYTES = 2u << 20;
constexpr unsigned WS_X = WS_DATA + 3u * SLOT_BYTES;
constexpr unsigned WS_RAW = WS_X + NWG * 4u * XSTRIDE;
constexpr size_t WS_NEEDED = (size_t)WS_RAW + NWG * 4u * 8192;
constexpr int XBIAS = 1 << 21;
constexpr unsigned POISON = 0x80808080u;

constexpr unsigned LDS_PUSH = 16 * 8192;
constexpr unsigned LDS_LOC = LDS_PUSH + 8192;
constexpr unsigned LDS_H1 = (NSTEP & 15) * 8192;
constexpr unsigned LDS_CTRL = LDS_LOC + 16384;
constexpr unsigned C_FULL = 0;
constexpr unsigned C_DUE = 8;
constexpr unsigned C_CNT = 12;
constexpr unsigned C_CALL = 16;
constexpr unsigned C_PAUSE = 20;
constexpr unsigned C_H0 = 32;
constexpr unsigned C_H1 = 48;
constexpr unsigned C_TICKET = 768;
constexpr unsigned C_PROG = 1024;
constexpr unsigned LDS_BYTES = LDS_CTRL + 2048 + 256;

#define DEADLINE 50000000u
constexpr unsigned WATCH_TICKS = 3000;
constexpr unsigned DEADLINE_TICKS = DEADLINE;

struct Args {
    struct { char* ws; const int8_t* x; } peer[NRANK];
    struct { const int8_t* w; float* out; } col[NRANK];
    const float* sx;
    const float* sw;
    int rank;
    int pad;
};
typedef unsigned long long u64;
typedef u64 u64x2 __attribute__((ext_vector_type(2)));
__device__ __forceinline__ u64x2 args_ld(unsigned off) {
    u64x2 r;
    asm volatile("s_load_dwordx4 %0, %1, %2" : "=s"(r) : "s"(__builtin_amdgcn_kernarg_segment_ptr()), "s"(off) : "memory");
    return r;
}

#define SB() __builtin_amdgcn_sched_barrier(0)
#define LGKM(n) __builtin_amdgcn_s_waitcnt(0xC07F | ((n) << 8))

#define DMA4(BITS) asm volatile( \
        "s_mov_b32 m0, %5\n\t" \
        "s_nop 0\n\t" \
        "global_load_lds_dwordx4 %0, %4 " BITS "\n\t" \
        "global_load_lds_dwordx4 %1, %4 offset:1024 " BITS "\n\t" \
        "global_load_lds_dwordx4 %2, %4 offset:2048 " BITS "\n\t" \
        "global_load_lds_dwordx4 %3, %4 offset:3072 " BITS \
        :: "v"(v0), "v"(v1), "v"(v2), "v"(v3), "s"(gbase), "s"(lds_addr) : "memory")
template <bool SYS = false>
__device__ __forceinline__ void dma4(const char* gbase, unsigned lds_addr,
                                     unsigned v0, unsigned v1, unsigned v2, unsigned v3) {
    if constexpr (SYS) DMA4("sc0 sc1");
    else DMA4("");
}
template <int N> __device__ __forceinline__ void wait_vm(i32x4& a) {
    asm volatile("s_waitcnt vmcnt(%1)" : "+v"(a) : "i"(N) : "memory");
}
template <int N> __device__ __forceinline__ void wait_vm(i32x4& a, i32x4& b) {
    asm volatile("s_waitcnt vmcnt(%2)" : "+v"(a), "+v"(b) : "i"(N) : "memory");
}
__device__ __forceinline__ int lds_ld(unsigned addr) {
    return __hip_atomic_load((lds_int_p)addr, __ATOMIC_RELAXED, __HIP_MEMORY_SCOPE_WORKGROUP);
}
__device__ __forceinline__ void lds_st(unsigned addr, int v) {
    __hip_atomic_store((lds_int_p)addr, v, __ATOMIC_RELAXED, __HIP_MEMORY_SCOPE_WORKGROUP);
}
__device__ __forceinline__ void lds_st1(unsigned addr, int v) {
    asm volatile("s_mov_b64 exec, 1\n\tds_write_b32 %0, %1\n\ts_mov_b64 exec, -1" :: "v"(addr), "v"(v) : "memory");
}
__device__ __forceinline__ void lds_st2(unsigned addr, int v0, int v1) {
    asm volatile("s_mov_b64 exec, 1\n\tds_write_b64 %0, %1\n\ts_mov_b64 exec, -1" :: "v"(addr), "v"(i32x2{v0, v1}) : "memory");
}
__device__ __forceinline__ void lds_wait(unsigned addr, int need) {
    while (__builtin_amdgcn_readfirstlane(
               __hip_atomic_load((lds_int_p)addr, __ATOMIC_ACQUIRE, __HIP_MEMORY_SCOPE_WORKGROUP)) < need)
        __builtin_amdgcn_s_sleep(1);
    asm volatile("" ::: "memory");
}

constexpr int imin(int x, int y) { return x < y ? x : y; }
constexpr int imax(int x, int y) { return x > y ? x : y; }
constexpr int local_of(int s) { return s >= LP0 && s < LP0 + 2 ? s - LP0 : s >= LP1 && s < LP1 + 2 ? 2 + s - LP1 : -1; }
constexpr int frag_lo(int s) {
    return s >= NS1 ? NSPLIT + s - NS1 : 2 * (s - imin(2, imax(0, s - LP0)) - imin(2, imax(0, s - LP1)));
}
__device__ __forceinline__ int pos_of(int rel, int mi) {
    if (!QROT) return rel - 1;
    const int v = (rel + 2 * mi) & 7;
    return v - (v > 2 * mi ? 1 : 0);
}
__device__ __forceinline__ int rel_of(int pos, int mi) {
    if (!QROT) return pos + 1;
    return (pos + (pos >= 2 * mi ? 1 : 0) - 2 * mi) & 7;
}
__device__ __forceinline__ unsigned frag_off(int rel, int cls, int rg) {
    const int i = 7 * cls + pos_of(rel, rg >> 3), m = rg & 1;
    const int P = i < NSPLIT ? i & 1 : m, frag = i < NSPLIT ? (i & ~1) + (m ^ P) : i;
    return (((rg >> 1) * 2 + P) * NFRAG + frag) * 1024;
}

struct Ctx {
    const char* ab[7];
    char* xo;
    const char* xl;
    const char* xl2;
    unsigned sv[4];
    unsigned ll;
    unsigned la;
    unsigned sent;
    unsigned tag;
    char* ro;
    unsigned sel[3];
    unsigned any, raw[2];
    float scale;
    unsigned hf, h0, h1;
    unsigned rb[2][8];
    unsigned l16;
    unsigned chk;
    const char* flags;
    unsigned foff;
    unsigned call, t0;
    unsigned bad;
    unsigned prog;
    int one;
    int seen;
};

__device__ __forceinline__ void read_local(i32x4 (&AL)[2][2], const Ctx& c) {
#pragma unroll
    for (int s = 0; s < 2; ++s) {
        AL[s][0] = *(lds_i32x4_p)(c.la ^ (s * 64));
        AL[s][1] = *(lds_i32x4_p)(c.la ^ (s * 64) ^ 2048);
    }
}
template <int S, int H>
__device__ __forceinline__ void read_b(i32x2 (&B)[16], int t, const Ctx& c) {
    B[2 * t + H] = __builtin_amdgcn_ds_read_tr8_b64_v2i32((lds_i32x2_p)(c.rb[(S >> 3) & 1][t] + (S & 7) * 8192 + H * 1024));
}
__device__ __forceinline__ void mma(i32x4& acc, const i32x4& a, const i32x2 (&B)[16], int t) {
    asm volatile("v_mfma_i32_16x16x64_i8 %0, %1, %2, %0"
                 : "+v"(acc) : "v"(__builtin_shufflevector(B[2 * t], B[2 * t + 1], 0, 1, 2, 3)), "v"(a));
}
template <int F>
__device__ __forceinline__ void load_frag(i32x4 (&A)[NFRAG], const Ctx& c) {
    asm volatile("global_load_dwordx4 %0, %1, %2 offset:%3 sc0 sc1"
                 : "=v"(A[F]) : "v"(c.l16), "s"(c.ab[F / 8]), "i"((F % 8) * 1024 - 4096) : "memory");
}
template <int F, int END>
__device__ __forceinline__ void first_frags(i32x4 (&A)[NFRAG], const Ctx& c, int cnt) {
    if constexpr (F < END) {
        asm volatile(
            "v_cmp_gt_i32 vcc, %[need], %[cnt]\n\t"
            "s_cbranch_vccz .Lgo%=\n"
            ".Lagain%=:\n\t"
            "ds_read_b32 %[cnt], %[at]\n\t"
            "s_waitcnt lgkmcnt(0)\n\t"
            "v_cmp_gt_i32 vcc, %[need], %[cnt]\n\t"
            "s_cbranch_vccnz .Lagain%=\n"
            ".Lgo%=:"
            : [cnt] "+v"(cnt) : [at] "v"(LDS_CTRL + C_CNT), [need] "i"(F / 2 + 1) : "memory", "vcc");
        load_frag<F>(A, c);
        load_frag<F + 1>(A, c);
        first_frags<F + 2, END>(A, c, cnt);
    }
}
template <int FULL>
__device__ __forceinline__ void check(const Ctx& c) {
    while (__any(__hip_atomic_load((lds_int_p)c.chk, __ATOMIC_ACQUIRE, __HIP_MEMORY_SCOPE_WORKGROUP) < FULL))
        __builtin_amdgcn_s_sleep(1);
    asm volatile("" ::: "memory");
}

#define CP_WAIT \
        "s_or_b64 vcc, %[low], %[miss]\n\t" \
        "s_cbranch_scc1 .Lslow%=\n" \
        ".Lback%=:\n\t" \
        ".subsection 1\n" \
        ".Lagain%=:\n\t" \
        "s_sleep 1\n" \
        ".Lslow%=:\n\t" \
        "ds_read_b32 %[p0], %[chk]\n\t" \
        "s_waitcnt lgkmcnt(0)\n\t" \
        "v_cmp_gt_i32 vcc, %[full], %[p0]\n\t" \
        "s_cbranch_vccnz .Lagain%=\n\t"
template <int FULL>
__device__ __forceinline__ void checkpoint(Ctx& c, unsigned long long low) {
    unsigned p0;
    asm volatile(CP_WAIT "s_branch .Lback%=\n\t.subsection 0"
                 : [p0] "=&v"(p0) : [low] "s"(low), [miss] "i"(0), [chk] "v"(c.chk), [full] "i"(FULL) : "memory", "vcc", "scc");
}
template <int FULL, int F>
__device__ __forceinline__ void checkpoint(i32x4 (&A)[NFRAG], Ctx& c, unsigned long long low, unsigned long long miss) {
    unsigned p0, p1, fl, prev;
    asm volatile(
        CP_WAIT
        "s_cmp_eq_u64 %[miss], 0\n\t"
        "s_cbranch_scc1 .Lback%=\n\t"
        "s_mov_b32 %[prev], 0\n"
        ".Lretry%=:\n\t"
        "global_load_dword %[p0], %[l16], %[ab] offset:%[o0] sc0 sc1\n\t"
        "global_load_dword %[p1], %[l16], %[ab] offset:%[o1] sc0 sc1\n\t"
        "global_load_dwordx4 %[a0], %[l16], %[ab] offset:%[o0] sc0 sc1\n\t"
        "global_load_dwordx4 %[a1], %[l16], %[ab] offset:%[o1] sc0 sc1\n\t"
        "global_load_dword %[fl], %[foff], %[flags] sc0 sc1\n\t"
        "s_memrealtime vcc\n\t"
        "s_waitcnt vmcnt(0) lgkmcnt(0)\n\t"
        "s_cmp_lg_u32 %[prev], 0\n\t"
        "s_cbranch_scc1 .Lback%=\n\t"
        "s_sub_u32 %[prev], vcc_lo, %[t0]\n\t"
        "v_cmp_eq_u32 vcc, %[sent], %[p0]\n\t"
        "v_cmp_eq_u32 %[miss], %[sent], %[p1]\n\t"
        "s_or_b64 vcc, vcc, %[miss]\n\t"
        "s_cbranch_scc0 .Lback%=\n\t"
        "s_cmp_gt_u32 %[prev], %[dl]\n\t"
        "s_cselect_b32 %[bad], 1, %[bad]\n\t"
        "s_cbranch_scc1 .Lback%=\n\t"
        "v_subrev_u32 %[fl], %[call], %[fl]\n\t"
        "v_cmp_gt_i32 vcc, 0, %[fl]\n\t"
        "s_cmp_eq_u64 vcc, 0\n\t"
        "s_cselect_b32 %[prev], 1, 0\n\t"
        "s_branch .Lretry%=\n\t"
        ".subsection 0"
        : [a0] "+v"(A[F]), [a1] "+v"(A[F + 1]), [p0] "=&v"(p0), [p1] "=&v"(p1), [fl] "=&v"(fl), [prev] "=&s"(prev),
          [bad] "+s"(c.bad), [miss] "+s"(miss)
        : [low] "s"(low), [chk] "v"(c.chk), [full] "i"(FULL), [l16] "v"(c.l16), [ab] "s"(c.ab[F / 8]), [foff] "v"(c.foff), [flags] "s"(c.flags),
          [o0] "i"((F % 8) * 1024 - 4096), [o1] "i"((F % 8) * 1024 - 3072), [sent] "s"(c.sent), [call] "s"(c.call),
          [t0] "s"(c.t0), [dl] "i"(DEADLINE_TICKS)
        : "memory", "vcc", "scc");
}

static_assert(2 * T2AT <= ADIST && T2PER <= 4 && ADIST % 2 == 0 && AINIT % 2 == 0 && AINIT <= ADIST && FIRST_AT >= 2 && FIRST_AT <= 3 && NS2 % 2 == 0, "");
constexpr int issued_before(int s) {
    if (s <= NS1 - T2AT) return s < FIRST_AT ? 0 : imin(NSPLIT, imin(frag_lo(s) + ADIST + (s > AUP ? 2 : 0) + (s > AUP2 ? 2 : 0), AINIT + 4 * (s - FIRST_AT)));
    const int ramp = NSPLIT + T2PER * (s - NS1 + T2AT);
    return imin(NFRAG, s <= NS1 ? ramp : imin(ramp, imax(NSPLIT + T2PER * T2AT, frag_lo(s) + ADIST2)));
}
constexpr int span(int s) {
    return s + 1 >= NSTEP ? 0 : s + 1 < NS1 ? 1 : (s + 1 - NS1) % 2 == 0 ? 2 : 0;
}
constexpr int dumps(int s) { return DUMPNOW ? (s == NS1 - 1 ? 6 : 0) : s >= NS1 && s < NS1 + 3 ? 2 : 0; }
constexpr int ops_before_wait(int s) {
    int n = issued_before(s + 1 == FIRST_AT ? s : s + 1);
    for (int t = 0; t <= s; ++t) n += t < s || DUMPNOW ? dumps(t) : 0;
    return n;
}
constexpr int ops_through_frag(int f) {
    int s = FIRST_AT - 1;
    while (issued_before(s + 1) <= f) ++s;
    int n = f + 1;
    for (int t = 0; t < s; ++t) n += dumps(t);
    return n;
}

struct Four { int f; i32x4 x[4]; };
__device__ __forceinline__ Four fetch4(unsigned flag, unsigned at) {
    Four r;
    r.f = *(lds_vint_p)flag;
#pragma unroll
    for (int t = 0; t < 4; ++t) r.x[t] = *(lds_vi32x4_p)(at + t * 1024);
    return r;
}
__device__ __forceinline__ void await4(Four& r, unsigned flag, unsigned at) {
    asm volatile(
        "v_cmp_ne_u32 vcc, 0, %0\n\t"
        "s_cbranch_vccz .Lagain%=\n"
        ".Lgo%=:\n\t"
        ".subsection 1\n"
        ".Lagain%=:\n\t"
        "ds_read_b32 %0, %5\n\t"
        "s_waitcnt lgkmcnt(0)\n\t"
        "v_cmp_ne_u32 vcc, 0, %0\n\t"
        "s_cbranch_vccz .Lagain%=\n\t"
        "ds_read_b128 %1, %6\n\t"
        "ds_read_b128 %2, %6 offset:1024\n\t"
        "ds_read_b128 %3, %6 offset:2048\n\t"
        "ds_read_b128 %4, %6 offset:3072\n\t"
        "s_waitcnt lgkmcnt(0)\n\t"
        "s_branch .Lgo%=\n\t"
        ".subsection 0"
        : "+v"(r.f), "+v"(r.x[0]), "+v"(r.x[1]), "+v"(r.x[2]), "+v"(r.x[3]) : "v"(flag), "v"(at) : "memory", "vcc");
}
struct Out { unsigned tw, tw1, tr, oo; char *o0, *o1; };
__device__ __forceinline__ Out out_of(char* rows, unsigned lds, int lane) {
    Out o;
    o.tw = lds + (lane & 15) * 128 + (((lane >> 4) ^ (lane & 7)) << 4);
    o.tw1 = o.tw ^ 64;
    o.tr = lds + (lane >> 3) * 128 + (((lane & 7) ^ (lane >> 3)) << 4);
    o.oo = (lane >> 3) * NN * 4 + (lane & 7) * 16;
    o.o0 = rows;
    o.o1 = rows + 8 * NN * 4;
    asm volatile("" : "+v"(o.tw), "+v"(o.tw1), "+v"(o.tr), "+v"(o.oo), "+s"(o.o0), "+s"(o.o1));
    return o;
}
__device__ __forceinline__ void finish(const i32x4* u, const i32x4* v, float scale, const Out& o) {
    i32x4 Z[4];
#pragma unroll
    for (int p = 0; p < 2; ++p) {
#pragma unroll
        for (int h = 0; h < 2; ++h) {
            const i32x4 s = u[2 * p + h] + v[2 * p + h];
            const f32x4 y = {(float)s[0] * scale, (float)s[1] * scale, (float)s[2] * scale, (float)s[3] * scale};
            *(lds_vi32x4_p)((h ? o.tw1 : o.tw) + p * 2048) = __builtin_bit_cast(i32x4, y);
        }
        Z[2 * p] = *(lds_vi32x4_p)(o.tr + p * 2048);
        Z[2 * p + 1] = *(lds_vi32x4_p)(o.tr + p * 2048 + 1024);
        SB();
    }
#pragma unroll
    for (int p = 0; p < 2; ++p)
        asm volatile("global_store_dwordx4 %2, %0, %3 offset:%5 " OUT_BITS "\n\t"
                     "global_store_dwordx4 %2, %1, %4 offset:%5 " OUT_BITS "\n\ts_nop 1"
                     :: "v"(Z[2 * p]), "v"(Z[2 * p + 1]), "v"(o.oo), "s"(o.o0), "s"(o.o1), "i"(p * 128) : "memory");
}
constexpr int XLOOK = 6144 - 16 - XLEAD * 1024;
__device__ __forceinline__ void touch_x(const char* xi) {
    unsigned w;
    asm volatile("s_load_dword %0, %1, 0\n\ts_load_dword %0, %1, %2\n\ts_load_dword %0, %1, %3\n\ts_load_dword %0, %1, %4\n\ts_waitcnt lgkmcnt(0)"
                 : "=&s"(w) : "s"(xi + XLOOK), "i"(XSTRIDE), "i"(2 * XSTRIDE), "i"(3 * XSTRIDE) : "memory");
}
__device__ __forceinline__ unsigned long long or64(unsigned long long x, unsigned long long y) {
    unsigned long long r;
    asm volatile("s_or_b64 %0, %1, %2" : "=s"(r) : "s"(x), "s"(y) : "scc");
    return r;
}

template <int Q>
__device__ __forceinline__ void piece(Ctx& c, i32x4 (&T)[8], int part) {
    constexpr int U = Q / 3, KK = Q % 3;
    i32x4& t = T[4 * U + KK];
    if (part == 0) {
        t[0] = (t[0] << 2) | c.tag;
        if (KK == 0) t[1] |= c.raw[U];
        t[0] = __builtin_amdgcn_perm(T[4 * U + 3][0], t[0], c.sel[KK]);
    } else if (part == 1) {
#pragma unroll
        for (int e = 1; e < 4; ++e) t[e] = __builtin_amdgcn_perm(T[4 * U + 3][e], t[e], c.sel[KK]);
    } else {
        asm volatile("global_store_dwordx4 %1, %0, %2 offset:%3 sc0 sc1\n\ts_nop 1" :: "v"(t), "v"(c.l16), "s"(c.xo), "i"(Q * 1024 - 4096) : "memory");
    }
}
template <int U>
__device__ __forceinline__ void fits(Ctx& c, i32x4 (&T)[8], int part) {
    if (part < 4) {
        const i32x4& t = T[4 * U + part];
        c.any = (part ? c.any : 0u) | t[0] | t[1] | t[2] | t[3];
        asm volatile("" : "+v"(c.any));
    } else {
        asm volatile(
            "s_mov_b32 %0, 0\n\t"
            "v_cmp_le_u32 vcc, %8, %1\n\t"
            "s_cbranch_vccnz .Lraw%=\n"
            ".Lfit%=:\n\t"
            ".subsection 1\n"
            ".Lraw%=:\n\t"
            "global_store_dwordx4 %6, %2, %7 offset:%9 sc0 sc1\n\t"
            "global_store_dwordx4 %6, %3, %7 offset:%9+1024 sc0 sc1\n\t"
            "global_store_dwordx4 %6, %4, %7 offset:%9+2048 sc0 sc1\n\t"
            "global_store_dwordx4 %6, %5, %7 offset:%9+3072 sc0 sc1\n\t"
            "s_mov_b32 %0, 0x800000\n\t"
            "s_waitcnt vmcnt(0)\n\t"
            "s_branch .Lfit%=\n\t"
            ".subsection 0"
            : "=&s"(c.raw[U]) : "v"(c.any), "v"(T[4 * U]), "v"(T[4 * U + 1]), "v"(T[4 * U + 2]), "v"(T[4 * U + 3]),
              "v"(c.l16), "s"(c.ro), "i"(XFIT), "i"(U * 4096 - 4096) : "memory", "vcc");
    }
}
template <int D>
__device__ __forceinline__ void pack_x(Ctx& c, i32x4 (&T)[8], int g) {
    if constexpr (D < 2) { if (g >= 1 && g <= 5) fits<D>(c, T, g - 1); }
    if (g == 7) piece<2 * D>(c, T, 0);
    if (g == 9) piece<2 * D>(c, T, 1);
    if (g == 10) piece<2 * D>(c, T, 2);
    if (g == 11) piece<2 * D + 1>(c, T, 0);
    if (g == 13) piece<2 * D + 1>(c, T, 1);
    if (g == 15) piece<2 * D + 1>(c, T, 2);
}

template <int S>
__device__ __forceinline__ void step(Ctx& c, i32x4 (&acc)[2][8], i32x4 (&AL)[2][2][2], i32x4 (&A)[NFRAG],
                                     i32x2 (&B)[2][16], Four& X STAMP_ARG) {
    constexpr bool PH1 = S < NS1, LOC = S < 4, NEXT = S + 1 < NSTEP;
    constexpr int SPAN = span(S), FULL = imin(S + SPAN + 2, NSTEP);
    constexpr bool HAND = DUMPNOW && S == NS1 - 1;
    constexpr bool CP = SPAN > 0 && !HAND, VAL = CP && S + 1 >= 4;
    constexpr bool FREE = (S & 1) == 1 && S + 13 < NSTEP;
    constexpr int F = frag_lo(S), FN = frag_lo(S + 1);
    constexpr int L0 = issued_before(S), L1 = S + 1 == FIRST_AT ? L0 : issued_before(S + 1);
    unsigned long long m0 = 0, m1 = 0, m2 = 0;
    if constexpr ((S & 3) == 0) STAMP(4 + (S >> 2));
    if constexpr (S == 2) STAMP(15);
    if constexpr (S == CS0) { STAMP(28); CSTAMP(30); }
    if constexpr (S == CS1) { STAMP(29); CSTAMP(31); }
    if constexpr (S == FIRST_AT) first_frags<0, AINIT>(A, c, 0);
    const i32x4& a0 = LOC ? AL[LOC ? S / 2 : 0][S & 1][0] : A[F];
    const i32x4& a1 = LOC ? AL[LOC ? S / 2 : 0][S & 1][1] : A[PH1 ? F + 1 : F];
    LGKM(S == 2 ? 6 : 8);
    SB();
#pragma unroll
    for (int g = 0; g < 16; ++g) {
        const int t = g >> 1;
        if (g == 8) { if constexpr (NEXT) LGKM(8 + FREE); else LGKM(6); SB(); }
        if constexpr (HAND) {
            const int u = (g >> 3) * 4 + (g & 3);
            if (g & 4) mma(acc[0][u], a0, B[S & 1], u);
            else mma(acc[1][u], a1, B[S & 1], u);
            SB();
        } else if ((g & 1) == 0) { mma(acc[0][t], a0, B[S & 1], t); SB(); }
        else if constexpr (PH1) { mma(acc[1][t], a1, B[S & 1], t); SB(); }
        if (CP && g == 0) { c.seen = lds_ld(c.chk); SB(); }
        if constexpr (NEXT) {
            if (g & 1) read_b<S + 1, 1>(B[(S + 1) & 1], t, c);
            else if (g != 14) read_b<S + 1, 0>(B[(S + 1) & 1], t, c);
            if (g == 13) read_b<S + 1, 0>(B[(S + 1) & 1], 7, c);
        }
        if constexpr (!NEXT) {
            if (g == 0) X = fetch4(c.hf + C_H1 - C_H0, c.h1);
            if (g == 2) { if (c.bad & 1) c.scale = __builtin_nanf(""); asm volatile("" : "+v"(c.scale)); }
            if (g >= 6 && g <= 12 && (g & 1) == 0) *(lds_vi32x4_p)(c.h0 + (g - 6) * 512) = acc[0][(g - 6) / 2];
            if (g == 12) lds_st1(c.hf, c.one);
        }
        if (g == 1) { if constexpr (L0 < L1) load_frag<L0>(A, c); }
        if (FREE && g == 2) __hip_atomic_fetch_add((lds_int_p)c.prog, c.one, __ATOMIC_RELAXED, __HIP_MEMORY_SCOPE_WORKGROUP);
        if (g == 3) { if constexpr (L0 + 1 < L1) load_frag<L0 + 1>(A, c); }
        if (g == 4) { if constexpr (L0 + 2 < L1) load_frag<L0 + 2>(A, c); }
        if (g == 5) { if constexpr (L0 + 3 < L1) load_frag<L0 + 3>(A, c); }
        if (CP && g == 8) m2 = __builtin_amdgcn_sicmp(c.seen, FULL, 40);
        if constexpr (VAL) {
            if (g == 9 + LATEW) wait_vm<ops_before_wait(S) - ops_through_frag(FN + 1)>(A[FN], A[FN + 1]);
            if (g == 10 + LATEW) m0 = __builtin_amdgcn_uicmp(A[FN][0], c.sent, 32);
            if (g == 11 + LATEW) m1 = __builtin_amdgcn_uicmp(A[FN + 1][0], c.sent, 32);
            if (g == 12 + 2 * LATEW) m0 = or64(m0, m1);
            if (g == 14) checkpoint<FULL, FN>(A, c, m2, m0);
        } else if constexpr (CP) {
            if (g == 14) checkpoint<FULL>(c, m2);
        }
        if (S == 1 && g == 9) {
            asm volatile("s_waitcnt vmcnt(0)" ::: "memory");
            read_local(AL[1], c);
        }
        if constexpr (HAND) {
            if (g >= 3 && g <= 7) fits<0>(c, acc[1], g - 3);
            if (g >= 8 && g <= 10) piece<0>(c, acc[1], g - 8);
            if (g >= 10 && g <= 12) piece<1>(c, acc[1], g - 10);
            if (g >= 12 && g <= 14) piece<2>(c, acc[1], g - 12);
            if (g == 14) { fits<1>(c, acc[1], 0); fits<1>(c, acc[1], 1); }
            if (g == 15) { fits<1>(c, acc[1], 2); fits<1>(c, acc[1], 3); fits<1>(c, acc[1], 4); }
        } else if constexpr (dumps(S) > 0) pack_x<S - NS1>(c, acc[1], g);
        SB();
    }
    if constexpr (HAND) {
#pragma unroll
        for (int i = 0; i < 3; ++i) piece<3>(c, acc[1], i);
        c.seen = lds_ld(c.chk);
#pragma unroll
        for (int i = 0; i < 3; ++i) piece<4>(c, acc[1], i);
#pragma unroll
        for (int i = 0; i < 3; ++i) piece<5>(c, acc[1], i);
        m2 = __builtin_amdgcn_sicmp(c.seen, FULL, 40);
        wait_vm<ops_before_wait(S) - ops_through_frag(FN + 1)>(A[FN], A[FN + 1]);
        m0 = or64(__builtin_amdgcn_uicmp(A[FN][0], c.sent, 32), __builtin_amdgcn_uicmp(A[FN + 1][0], c.sent, 32));
        checkpoint<FULL, FN>(A, c, m2, m0);
        SB();
    }
    if constexpr (NEXT) step<S + 1>(c, acc, AL, A, B, X STAMP_PASS);
}

template <int S>
__device__ __forceinline__ void allowed(int& due, int n) {
    if constexpr (span(S) > 0) {
        constexpr int F = issued_before(S + span(S) + 1), NEED = F <= NSPLIT ? (F + 1) / 2 : F - NSPLIT / 2;
        if (n < NEED) due = imin(S + span(S) + 2, NSTEP) - 1;
    }
    if constexpr (S > FIRST_AT) allowed<S - 1>(due, n);
}
#define LORD 0
constexpr int LSEQ[1][2][3] = {{{0, 2, 4}, {1, 3, 5}}};
constexpr int fill_step(int h, int k) { return k < 3 ? LSEQ[LORD][h][k] : imin(NSTEP, 2 * k + h); }
constexpr int owed(int h, int k) {
    int m = fill_step(h, k);
    for (int i = k + 1; i < 3; ++i) m = imin(m, fill_step(h, i));
    return m;
}
__device__ __forceinline__ int w_chunk(int s, int P, int mi) {
    const int i = max(s < NS1 ? 2 * s - 8 + P : NSPLIT + s - NS1, 0), cls = (i * 37) >> 8;
    const int rem = rel_of(i - 7 * cls, mi) * 8 + ((cls + ROT * mi) & 7);
    return s < 4 ? 4 * P + (s ^ mi) : rem;
}
struct Ldr {
    const char* w;
    unsigned dv[4];
    int woff;
    unsigned fa, pa;
    int freed;
    int pz;
};
__device__ __forceinline__ void fill(const Ldr& l, int off, unsigned slot) {
    const char* p = l.w + off;
    dma4(p, slot, l.dv[0], l.dv[1], l.dv[2], l.dv[3]);
    dma4(p + 32 * NN, slot + 4096, l.dv[0], l.dv[1], l.dv[2], l.dv[3]);
}
__device__ __forceinline__ int released(unsigned pa) {
    int v = lds_ld(pa);
    v = min(v, __builtin_amdgcn_update_dpp(v, v, 0xB1, 0xF, 0xF, false));
    v = min(v, __builtin_amdgcn_update_dpp(v, v, 0x4E, 0xF, 0xF, false));
    return 2 * __builtin_amdgcn_readfirstlane(v);
}
template <int H, int K>
__device__ __forceinline__ void fills(Ldr& l STAMP_ARG) {
    constexpr int S = fill_step(H, K);
    if constexpr (S < NSTEP) {
        if constexpr (S >= 16) {
            if (__builtin_expect(l.freed <= S - 16, 0)) {
                while ((l.freed = released(l.pa)) <= S - 16) __builtin_amdgcn_s_sleep(LSLEEP);
            }
        }
        if constexpr (PAUSE && K > 1 && K < 12) {
            if (l.pz != 2) while ((l.pz = __builtin_amdgcn_readfirstlane(lds_ld(LDS_CTRL + C_PAUSE))) == 1) __builtin_amdgcn_s_sleep(1);
        }
        if constexpr (K > 1) fill(l, __builtin_amdgcn_readlane(l.woff, S), (S & 15) * 8192);
        constexpr int W = imin(LDW, imax(1, K - LDW_FROM + 2));
        asm volatile("s_waitcnt vmcnt(%2)\n\tds_max_i32 %0, %1" :: "v"(l.fa), "v"(owed(H, K - W + 1)), "i"(8 * W) : "memory");
        if constexpr (K < 6 || (K & 1) == 1) STAMP(K < 6 ? K : 3 + K / 2);
        fills<H, K + 1>(l STAMP_PASS);
    } else {
#pragma unroll
        for (int k = LDW - 1; k >= 0; --k)
            asm volatile("s_waitcnt vmcnt(%2)\n\tds_max_i32 %0, %1" :: "v"(l.fa), "v"(k ? fill_step(H, K - k) : NSTEP), "i"(8 * k) : "memory");
    }
}

__device__ __forceinline__ void take_ticket(const Args& a, unsigned call, int lane) {
    unsigned tk = 0;
    if (lane == 0) tk = __hip_atomic_fetch_add((unsigned*)(a.peer[0].ws + WS_TICKET), 1u, __ATOMIC_RELAXED, __HIP_MEMORY_SCOPE_AGENT);
    if ((__builtin_amdgcn_readfirstlane(tk) & 255) == 255) {
        if (lane < NRANK)
            __hip_atomic_store((unsigned*)(a.peer[lane].ws + WS_FLAGS + ((NRANK - lane) & (NRANK - 1)) * 128), call,
                               __ATOMIC_RELAXED, __HIP_MEMORY_SCOPE_SYSTEM);
    }
}

__device__ __forceinline__ void help(const Args& a, int w, int b, int lane STAMP_ARG) {
    __builtin_amdgcn_s_setprio(0);
    const unsigned t0 = (unsigned)__builtin_amdgcn_s_memrealtime();
    const int j = b >> 3, ct = ((b & 7) + CMAP * (j >> 3)) & 7;
    const unsigned call = __builtin_amdgcn_readfirstlane(lds_ld(LDS_CTRL + C_CALL)), tag = call % 3 + 1;
    const char* xi = a.peer[0].ws + WS_X + ((b ^ 8) * 4 + w) * XSTRIDE;
    typedef const __attribute__((address_space(4))) float* cfp;
    float scale = *(cfp)(uintptr_t)a.sx * *(cfp)(uintptr_t)a.sw;
    const Out o = out_of((char*)a.col[ct].out + (size_t)(unsigned)(((((j >> 1) & 3) * 128 + w * 32 + (j & 1) * 16) * NN + (j >> 3) * 1024) * 4),
                         LDS_LOC + w * 4096, lane);
    const unsigned l16 = lane * 16, pa = LDS_CTRL + C_PROG + (lane & 3) * 260;
    unsigned sel0 = 0x0c0c0703, sel1 = 0x0c070100;
    unsigned hf = LDS_CTRL + C_H0 + w * 4, h0 = LDS_LOC + w * 4096 + l16, h1 = LDS_H1 + w * 4096 + l16;
    asm volatile("" : "+s"(sel0), "+s"(sel1), "+v"(scale), "+v"(hf), "+v"(h0), "+v"(h1));
    while (released(pa) < NSTEP - 14) __builtin_amdgcn_s_sleep(4);
    STAMP(13);
    unsigned long long now;
    unsigned d;
    do {
        asm volatile("s_load_dword %0, %2, %3 glc\n\ts_memrealtime %1\n\ts_waitcnt lgkmcnt(0)" : "=&s"(d), "=&s"(now) : "s"(xi), "i"(XLOOK) : "memory");
    } while ((d & 3) != tag && (unsigned)now - t0 < WATCH_TICKS);
    STAMP(14);
    __builtin_amdgcn_s_setprio(2);
    i32x4 Q[6];
    bool late;
    do {
        asm volatile("global_load_dwordx4 %3, %6, %7 offset:-1024 sc0 sc1\n\tglobal_load_dwordx4 %4, %6, %7 sc0 sc1\n\t"
                     "global_load_dwordx4 %5, %6, %7 offset:1024 sc0 sc1\n\tglobal_load_dwordx4 %0, %6, %7 offset:-4096 sc0 sc1\n\t"
                     "global_load_dwordx4 %1, %6, %7 offset:-3072 sc0 sc1\n\tglobal_load_dwordx4 %2, %6, %7 offset:-2048 sc0 sc1\n\t"
                     "s_waitcnt vmcnt(0)"
                     : "=&v"(Q[0]), "=&v"(Q[1]), "=&v"(Q[2]), "=&v"(Q[3]), "=&v"(Q[4]), "=&v"(Q[5]) : "v"(l16), "s"(xi + 4096) : "memory");
        unsigned diff = 0;
#pragma unroll
        for (int q = 0; q < 6; ++q) diff |= (unsigned)Q[q][0] ^ tag;
        if (__builtin_expect(!__any(diff & 3), 1)) { late = false; break; }
        late = (unsigned)__builtin_amdgcn_s_memrealtime() - t0 > DEADLINE_TICKS + DEADLINE_TICKS / 2;
    } while (!late);
    STAMP(15);
    i32x4 T[2][4];
    Four H;
#pragma unroll
    for (int u = 1; u >= 0; --u) {
#pragma unroll
        for (int e = 0; e < 4; ++e) {
#pragma unroll
            for (int k = 0; k < 3; ++k) T[u][k][e] = e ? Q[3 * u + k][e] & 0xffffff : __builtin_amdgcn_ubfe(Q[3 * u + k][e], 2, 22);
            T[u][3][e] = __builtin_amdgcn_perm(Q[3 * u + 2][e], __builtin_amdgcn_perm(Q[3 * u + 1][e], Q[3 * u][e], sel0), sel1);
        }
        if (__builtin_expect(__builtin_amdgcn_readfirstlane(Q[3 * u][1]) & 0x800000, 0))
            asm volatile("global_load_dwordx4 %0, %4, %5 sc0 sc1\n\tglobal_load_dwordx4 %1, %4, %5 offset:1024 sc0 sc1\n\t"
                         "global_load_dwordx4 %2, %4, %5 offset:2048 sc0 sc1\n\tglobal_load_dwordx4 %3, %4, %5 offset:3072 sc0 sc1\n\t"
                         "s_waitcnt vmcnt(0)"
                         : "=&v"(T[u][0]), "=&v"(T[u][1]), "=&v"(T[u][2]), "=&v"(T[u][3])
                         : "v"(l16), "s"(a.peer[0].ws + WS_RAW + ((b ^ 8) * 4 + w) * 8192 + u * 4096) : "memory");
        if (u) {
#pragma unroll
            for (int t = 0; t < 4; ++t) *(lds_vi32x4_p)(h1 + t * 1024) = T[1][t];
            lds_st1(LDS_CTRL + C_H1 + w * 4, 1);
            H = fetch4(hf, h0);
            SB();
        }
    }
    if (late) {
        scale = __builtin_nanf("");
        if (lane == 0) __hip_atomic_store((unsigned*)(a.peer[0].ws + WS_TMO), call, __ATOMIC_RELAXED, __HIP_MEMORY_SCOPE_AGENT);
    }
    __builtin_amdgcn_s_setprio(0);
    await4(H, hf, h0);
    __builtin_amdgcn_s_setprio(1);
    STAMP(16);
    finish(H.x, T[0], scale, o);
    STAMP(17);
}

}

__global__ __launch_bounds__(NTHREADS, 2) __attribute__((aligned(4096))) void a2a_gemm_kernel(Args a) {
    extern __shared__ __attribute__((aligned(16))) char smem[];
    const int lane = threadIdx.x & 63;
    int wave = __builtin_amdgcn_readfirstlane(threadIdx.x >> 6);
    const int b = blockIdx.x;
    const int xcd = b & 7, j = b >> 3;
    const unsigned lds0 = (unsigned)(uintptr_t)(__attribute__((address_space(3))) char*)smem;
    const unsigned ctrl = lds0 + LDS_CTRL;
    const int P = j & 1;
    const int mi = (j >> 1) & 3;
    const int ct = (xcd + CMAP * (j >> 3)) & 7;
    const int n0 = (j >> 3) * 1024;
    STAMP_DECL;
#define NEXT_ROLE() asm volatile("" : "+s"(wave) :: "memory")
    u64 t64;

    unsigned pre = 0;
    if (wave == 7) {
        {
            u64 pc;
            unsigned len;
            asm volatile("s_getpc_b64 %0\n.Lpc%=:\n\ts_mov_b32 %1, .Lend%=-.Lpc%=-4\n\t.subsection 2\n.Lend%=:\n\t.subsection 0" : "=s"(pc), "=s"(len));
            asm volatile("s_mov_b32 m0, %2\n\ts_mov_b64 exec, 0xffff\n\tglobal_load_lds_dword %0, %1\n\ts_mov_b64 exec, -1"
                         :: "v"(min((unsigned)(j * 2048 + lane * 128), len)), "s"(pc), "i"(LDS_CTRL + 2048) : "memory");
        }
        lds_st(ctrl + lane * 4, lane == 1 ? owed(1, 0) : lane == C_DUE / 4 ? FIRST_AT + 2 : 0);
        *(lds_i32x4_p)(ctrl + C_PROG + lane * 16) = i32x4{0, 0, 0, 0};
        asm volatile("s_waitcnt lgkmcnt(0)" ::: "memory");
    } else {
        asm volatile("s_load_dword %0, %1, 0\n\ts_load_dword %0, %1, 64\n\ts_load_dword %0, %1, 128\n\t"
                     "s_load_dword %0, %1, 192\n\ts_load_dword %0, %1, 256"
                     : "=&s"(pre) : "s"(__builtin_amdgcn_kernarg_segment_ptr()) : "memory");
    }
    asm volatile("s_barrier" : "+s"(wave) :: "memory");
#define STAMP0()
#define STAMP0B()
#define NOW(t) asm volatile("s_memrealtime %0" : "=s"(t) :: "memory")
    Ctx c;
    if (wave < 4) {
    __builtin_amdgcn_s_setprio(CPRIO);
    STAMP0();
    u64x2 own = args_ld(0);
#pragma unroll
    for (int i = 0; i < 4; ++i) {
        c.sv[i] = (mi * 128 + wave * 32 + i * 8 + (lane >> 3)) * KLOC + P * 256 + (mi >> 1) * 128 +
                  (((lane & 7) ^ ((i * 4 + (lane >> 4)) & 7)) << 4) - i * 1024;
        asm volatile("" : "+v"(c.sv[i]));
    }
    unsigned ll = lds0 + LDS_LOC + wave * 4096;
    asm volatile("s_waitcnt lgkmcnt(0)" : "+s"(own), "+s"(ll) : "s"(pre) : "memory");
    STAMP0B();
    if (HOLDX) __builtin_amdgcn_s_sleep(HOLDX);
    dma4((const char*)own[1], ll, c.sv[0], c.sv[1], c.sv[2], c.sv[3]);
    }
    NEXT_ROLE();
    Ldr l;
    if ((wave & 6) == 4) {
        __builtin_amdgcn_s_setprio(LPRIO);
        u64x2 col = args_ld(128 + ct * 16);
        unsigned f0 = (4 * P + ((wave & 1) ^ mi)) * 64 * NN, slot = lds0 + (wave & 1) * 8192;
#pragma unroll
        for (int i = 0; i < 4; ++i) {
            l.dv[i] = (lane >> 3) * NN + n0 + (((lane & 7) ^ (((lane >> 4) & 3) | ((i >> 1) << 2))) << 4) + i * (8 * NN - 1024);
            asm volatile("" : "+v"(l.dv[i]));
        }
        asm volatile("s_waitcnt lgkmcnt(0)" : "+s"(col), "+s"(f0), "+s"(slot) : "s"(pre) : "memory");
        if (HOLDW) __builtin_amdgcn_s_sleep(HOLDW);
        l.w = (const char*)col[0];
        fill(l, f0, slot);
        fill(l, f0 ^ (2 * 64 * NN), slot + 16384);
        STAMP(0);
    }
    NEXT_ROLE();
    if (wave < 4) {
    char* const ws = a.peer[0].ws;
    const unsigned tstart = (unsigned)__builtin_amdgcn_s_memrealtime();
    STAMP(19);
    c.ll = lds0 + LDS_LOC + wave * 4096;
    c.xl2 = (const char*)a.peer[0].x + (128 - (mi >> 1) * 256);
    c.la = c.ll + P * 2048 + (lane & 15) * 128 + ((((mi & 1) * 4 + (lane >> 4)) ^ ((lane & 15) >> 1)) << 4);
    c.l16 = lane * 16;
    c.chk = ctrl + C_FULL + min(lane & 3, 2) * 4;
    c.foff = (lane & 7) * 128;
    c.flags = ws + WS_FLAGS;
    c.t0 = tstart;
    c.bad = 0;
    typedef const __attribute__((address_space(4))) float* cfp;
    c.scale = *(cfp)(uintptr_t)a.sx * *(cfp)(uintptr_t)a.sw;
    char* outp = (char*)a.col[ct].out;
    asm volatile("" : "+s"(outp));
    c.prog = ctrl + C_PROG + wave * 256 + lane * 4;
    c.one = 1;
    asm volatile("" : "+v"(c.one), "+v"(c.chk), "+v"(c.prog), "+v"(c.foff));
    {
        const unsigned q = (lane & 15) >> 1, kg = lane >> 4;
        const unsigned r0 = lds0 + (kg * 16 + q) * 128 + (lane & 1) * 8, f = (q >> 1) | ((kg & 1) << 2);
#pragma unroll
        for (int t = 0; t < 8; ++t) {
            c.rb[0][t] = r0 + ((t ^ f) << 4);
            c.rb[1][t] = c.rb[0][t] + 65536;
            asm volatile("" : "+v"(c.rb[0][t]), "+v"(c.rb[1][t]));
        }
    }
    i32x4 acc[2][8];
#pragma unroll
    for (int m = 0; m < 2; ++m)
#pragma unroll
        for (int t = 0; t < 8; ++t) { acc[m][t] = (m ? XBIAS : -XBIAS) + i32x4{0, 0, 0, 0}; asm volatile("" : "+v"(acc[m][t])); }
    i32x2 B[2][16];
    i32x4 AL[2][2][2], A[NFRAG];
    Four X;
    if constexpr (LP0 == 0) {
        asm volatile("s_waitcnt vmcnt(0)" ::: "memory");
        read_local(AL[0], c);
        {
            LGKM(0);
            dma4(c.xl2, c.ll, c.sv[0], c.sv[1], c.sv[2], c.sv[3]);
        }
    }
    STAMP(1);
    unsigned call;
    while ((call = __builtin_amdgcn_readfirstlane(lds_ld(ctrl + C_CALL))) == 0) __builtin_amdgcn_s_sleep(1);
    c.call = call;
    c.sent = call <= 2 ? 0u : POISON;
    c.tag = call % 3 + 1;
    c.ro = ws + WS_RAW + (b * 4 + wave) * 8192 + 4096;
    c.sel[0] = 0x04020100; c.sel[1] = 0x05020100; c.sel[2] = 0x06020100;
    asm volatile("" : "+s"(c.ro), "+s"(c.sel[0]), "+s"(c.sel[1]), "+s"(c.sel[2]));
#pragma unroll
    for (int i = 0; i < 7; ++i) {
        c.ab[i] = ws + WS_DATA + (call % 3) * SLOT_BYTES + ((mi * 4 + wave) * 2 + P) * (NFRAG * 1024) + i * 8192 + 4096;
        asm volatile("" : "+s"(c.ab[i]));
    }
    c.xo = ws + WS_X + (b * 4 + wave) * XSTRIDE + 4096;
    asm volatile("" : "+s"(c.xo), "+s"(c.sent), "+s"(c.tag), "+s"(c.flags), "+s"(c.bad), "+v"(c.l16));
    c.hf = ctrl + C_H0 + wave * 4;
    c.h0 = c.ll + c.l16;
    c.h1 = lds0 + LDS_H1 + wave * 4096 + c.l16;
    const Out o = out_of(outp + (size_t)(unsigned)(((mi * 128 + wave * 32 + P * 16) * NN + n0 + 64) * 4), lds0 + LDS_H1 + wave * 4096, lane);
    asm volatile("" : "+v"(c.hf), "+v"(c.h0), "+v"(c.h1), "+v"(c.scale));
    check<1>(c);
    STAMP(3);
#pragma unroll
    for (int t = 0; t < 8; ++t) { read_b<0, 0>(B[0], t, c); read_b<0, 1>(B[0], t, c); }
    check<2>(c);
    step<0>(c, acc, AL, A, B, X STAMP_PASS);
    SB();
    STAMP(20);

    await4(X, c.hf + C_H1 - C_H0, c.h1);
    STAMP(21);
    finish(&acc[0][4], X.x, c.scale, o);
    if (c.bad && lane == 0) __hip_atomic_store((unsigned*)(a.peer[0].ws + WS_TMO), call, __ATOMIC_RELAXED, __HIP_MEMORY_SCOPE_AGENT);
    STAMP(23);
    STAMP(24);
    STAMP_OUT(ws);
    __builtin_amdgcn_endpgm();
    }
    NEXT_ROLE();
    if (wave == 6) {
        __builtin_amdgcn_s_setprio(3);
        u64x2 own = args_ld(0);
        const int pd = PDFIX == 3 ? (xcd - j) & 7 : PDFIX == 4 ? (xcd - (j & 3) - 4 * (j >> 3)) & 7 : PDFIX == 1 ? ((ct ^ 4) + 1) & 7 : PDFIX == 2 ? (ct == 3 ? 0 : xcd ? xcd : (3 - CMAP * (j >> 3)) & 7) : ct == 3 ? 0 : ((ct ^ 4) + j) % 7 + 1, pc = j >> 2, pm = (j - (QROT ? (NRANK - pd) >> 1 : 0)) & 3;
        u64x2 to = args_ld(pd * 16);
        unsigned sv[8];
#pragma unroll
        for (int g = 0; g < 8; ++g) {
            sv[g] = (pm * 128 + g * 16 + (lane >> 2)) * KLOC + ((pc + ROT * pm) & 7) * 64 + (((lane & 3) ^ ((lane >> 4) & 3)) << 4) - (g & 3) * 1024;
            asm volatile("" : "+v"(sv[g]));
        }
        asm volatile("s_waitcnt lgkmcnt(0)" : "+s"(own), "+s"(to) : "s"(pre) : "memory");
        char* const ws = (char*)own[0];
        char* const pw = (char*)to[0];
        const char* const base = (const char*)to[1];
        unsigned cv;
        asm volatile("s_load_dword %0, %2, %3\n\ts_memrealtime %1" : "=&s"(cv), "=&s"(t64) : "s"(ws), "i"(WS_TICKET) : "memory");
        if (pd && j < XJIT) {
            dma4(base, lds0 + LDS_PUSH, sv[0], sv[1], sv[2], sv[3]);
            dma4(base, lds0 + LDS_PUSH + 4096, sv[4], sv[5], sv[6], sv[7]);
        }
        int pd2 = pd;
        asm volatile("" : "+s"(pd2) :: "memory");
        const unsigned slot = PUSH0 + ((j < 4 * NFAST ? j * PACE0 : j * PACE - 4 * NFAST * (PACE - PACE0)) >> 2);
        unsigned doff[2], poff[2];
#pragma unroll
        for (int m = 0; m < 2; ++m) {
            doff[m] = lane * 16 + WS_DATA + frag_off(NRANK - pd2, pc, pm * 8 + m);
            asm volatile("" : "+v"(doff[m]));
        }
        const unsigned roff = lds0 + LDS_PUSH + (lane & 15) * 64 + (((lane >> 4) ^ ((lane >> 2) & 3)) << 4);
        asm volatile("s_waitcnt lgkmcnt(0)" : "+s"(cv), "+s"(t64) : "s"(slot), "v"(roff) : "memory");
        const unsigned due = (unsigned)t64 + slot;
        const unsigned call = (cv >> 8) + 1;
        lds_st1(ctrl + C_CALL, (int)call);
        const unsigned cur = (call % 3) * SLOT_BYTES, old = ((call + 2) % 3) * SLOT_BYTES;
        STAMP(12);
        if (pd) {
            char* dp[4];
#pragma unroll
            for (int w = 0; w < 4; ++w) { dp[w] = pw + cur + w * (2 * NFRAG * 1024); asm volatile("" : "+s"(dp[w])); }
            __builtin_amdgcn_s_setprio(0);
            if (j >= 20) touch_x(ws + WS_X + (b ^ 8) * 4 * XSTRIDE);
            if (j >= XJIT) {
                while ((int)(due - XAHEAD - (unsigned)__builtin_amdgcn_s_memrealtime()) > 0) __builtin_amdgcn_s_sleep(4);
                __builtin_amdgcn_s_setprio(3);
                dma4(base, lds0 + LDS_PUSH, sv[0], sv[1], sv[2], sv[3]);
                dma4(base, lds0 + LDS_PUSH + 4096, sv[4], sv[5], sv[6], sv[7]);
                __builtin_amdgcn_s_setprio(0);
            }
            if (PAUSE) {
                for (int left; (left = (int)(due - PAUSE - (unsigned)__builtin_amdgcn_s_memrealtime())) > 0;) {
                    if (left > 40) __builtin_amdgcn_s_sleep(8);
                    else __builtin_amdgcn_s_sleep(1);
                }
                lds_st1(ctrl + C_PAUSE, 1);
            }
            for (int left; (left = (int)(due - (unsigned)__builtin_amdgcn_s_memrealtime())) > 0;) {
                if (left > 40) __builtin_amdgcn_s_sleep(8);
                else __builtin_amdgcn_s_sleep(1);
            }
            __builtin_amdgcn_s_setprio(3);
            STAMP(2);
            asm volatile("s_waitcnt vmcnt(0)" ::: "memory");
            STAMP(11);
            i32x4 dat[8];
#pragma unroll
            for (int g = 0; g < 8; ++g) dat[g] = *(lds_i32x4_p)(roff + g * 1024);
#pragma unroll
            for (int g = 0; g < 8; ++g)
                asm volatile("global_store_dwordx4 %1, %0, %2 sc0 sc1\n\ts_nop 1" :: "v"(dat[g]), "v"(doff[g & 1]), "s"(dp[g >> 1]) : "memory");
            lds_st1(ctrl + C_PAUSE, 2);
            __builtin_amdgcn_s_setprio(0);
            STAMP(3);
            asm volatile("" : "+s"(pd2) :: "memory");
#pragma unroll
            for (int m = 0; m < 2; ++m) poff[m] = lane * 16 + WS_DATA + frag_off(pd2, pc, pm * 8 + m);
            asm volatile("s_waitcnt vmcnt(0)" :: "v"(poff[0]), "v"(poff[1]) : "memory");
            STAMP(7);
            const i32x4 poison = {(int)POISON, (int)POISON, (int)POISON, (int)POISON};
#pragma unroll
            for (int g = 0; g < 8; ++g)
                asm volatile("global_store_dwordx4 %1, %0, %2 " POISON_BITS "\n\ts_nop 1"
                             :: "v"(poison), "v"(poff[g & 1]), "s"(ws + old + (g >> 1) * (2 * NFRAG * 1024)) : "memory");
        }
        lds_st1(ctrl + C_PAUSE, 2);
        take_ticket(a, call, lane);
        if (j < 20 || !pd) touch_x(ws + WS_X + (b ^ 8) * 4 * XSTRIDE);
    }

    NEXT_ROLE();
    if (wave < 6) {
        l.w = (const char*)a.col[ct].w;
        l.fa = ctrl + C_FULL + (lane ? 64 + lane * 8 : 0) + (wave & 1) * 4;
        l.pa = ctrl + C_PROG + (lane & 3) * 260;
        const int r8 = a.rank * 8;
        l.woff = (((w_chunk(min(lane, NSTEP - 1), P, mi) + r8) & 63) - r8) * 64 * NN;
        l.freed = 0;
        l.pz = 0;
        if (wave & 1) fills<1, 1>(l STAMP_PASS);
        else fills<0, 1>(l STAMP_PASS);
        STAMP(20);
    }

    NEXT_ROLE();
    if (wave == 7) {
        const unsigned tstart = (unsigned)__builtin_amdgcn_s_memrealtime();
        char* const ws = a.peer[0].ws;
        STAMP(0);
        unsigned d[8];
        unsigned long long now;
        int due = 1000;
        allowed<NSTEP - 1>(due, lane);
        constexpr int NI = NSPLIT / 2 + NS2;
        const int q = min(lane, NI - 1);
        const int offs = (q < NSPLIT / 2 ? 2 * q + (j >> 4) : q + NSPLIT / 2) * 1024;
        unsigned call;
        while ((call = __builtin_amdgcn_readfirstlane(lds_ld(ctrl + C_CALL))) == 0) __builtin_amdgcn_s_sleep(1);
        const unsigned sent = call <= 2 ? 0u : POISON;
        const char* af = ws + WS_DATA + (call % 3) * SLOT_BYTES + ((mi * 4 + 2 + ((j >> 3) & 1)) * 2 + P) * (NFRAG * 1024) + xcd * 128 + 112;
        int n = 0;
        STAMP(1);
        const unsigned pa = ctrl + C_PROG + (lane & 3) * 260;
        while (n < NSPLIT / 2) {
            if (WREST) while (n > released(pa) + WREST && (unsigned)__builtin_amdgcn_s_memrealtime() - tstart < WATCH_TICKS) __builtin_amdgcn_s_sleep(4);
            unsigned o[4];
#pragma unroll
            for (int i = 0; i < 4; ++i) o[i] = __builtin_amdgcn_readlane(offs, n + i);
            do {
                asm volatile("s_load_dword %0, %5, %6 glc\n\ts_load_dword %1, %5, %7 glc\n\t"
                             "s_load_dword %2, %5, %8 glc\n\ts_load_dword %3, %5, %9 glc\n\t"
                             "s_memrealtime %4\n\ts_waitcnt lgkmcnt(0)"
                             : "=&s"(d[0]), "=&s"(d[1]), "=&s"(d[2]), "=&s"(d[3]), "=&s"(now)
                             : "s"(af), "s"(o[0]), "s"(o[1]), "s"(o[2]), "s"(o[3]) : "memory");
            } while (d[0] == sent && (unsigned)now - tstart < WATCH_TICKS);
            n += d[1] == sent ? 1 : d[2] == sent ? 2 : d[3] == sent ? 3 : 4;
            lds_st2(ctrl + C_DUE, __builtin_amdgcn_readlane(due, n + (n >= 4 ? LEAD : 0)), n);
        }
        while (n + 2 * LEAD < NI) {
            if (WREST) while (n > 2 * released(pa) - NSPLIT / 2 + 2 * WREST && (unsigned)__builtin_amdgcn_s_memrealtime() - tstart < WATCH_TICKS) __builtin_amdgcn_s_sleep(4);
            const char* p = af + (n + NSPLIT / 2) * 1024;
            do {
                asm volatile("s_load_dword %0, %9, 0 glc\n\ts_load_dword %1, %9, 0x400 glc\n\ts_load_dword %2, %9, 0x800 glc\n\t"
                             "s_load_dword %3, %9, 0xc00 glc\n\ts_load_dword %4, %9, 0x1000 glc\n\ts_load_dword %5, %9, 0x1400 glc\n\t"
                             "s_load_dword %6, %9, 0x1800 glc\n\ts_load_dword %7, %9, 0x1c00 glc\n\t"
                             "s_memrealtime %8\n\ts_waitcnt lgkmcnt(0)"
                             : "=&s"(d[0]), "=&s"(d[1]), "=&s"(d[2]), "=&s"(d[3]), "=&s"(d[4]), "=&s"(d[5]), "=&s"(d[6]), "=&s"(d[7]), "=&s"(now)
                             : "s"(p) : "memory");
            } while (d[0] == sent && (unsigned)now - tstart < WATCH_TICKS);
            n += d[1] == sent ? 1 : d[2] == sent ? 2 : d[3] == sent ? 3 : d[4] == sent ? 4 : d[5] == sent ? 5 : d[6] == sent ? 6 : d[7] == sent ? 7 : 8;
            lds_st1(ctrl + C_DUE, __builtin_amdgcn_readlane(due, n + 2 * LEAD));
        }
        lds_st2(ctrl + C_DUE, 1000, 1000);
        STAMP(7);
    }
    NEXT_ROLE();
    help(a, wave & 3, b, lane STAMP_PASS);
    STAMP_OUT(a.peer[0].ws);
}

extern "C" void kernel_launch(int rank, int nranks, ncclComm_t comm,
                              hipStream_t stream,
                              void* const* d_in, const int* in_sizes, int n_in,
                              void* d_out, int out_size,
                              const int* in_dtypes, int out_dtype,
                              void* const* d_out_all,
                              void* const* d_ws_all, size_t ws_bytes) {
    if (nranks != NRANK || (unsigned)rank >= (unsigned)NRANK || ws_bytes < WS_NEEDED || n_in < 4 ||
        out_size != MLOC * NN || in_sizes[0] != NRANK * MLOC * KLOC || in_sizes[1] != NRANK * KLOC * NN) {
        fprintf(stderr, "unexpected problem shape\n");
        return;
    }
    static bool attr_set = false;
    if (!attr_set) {
        HIP_CHECK(hipFuncSetAttribute((const void*)a2a_gemm_kernel, hipFuncAttributeMaxDynamicSharedMemorySize,
                                      (int)LDS_BYTES));
        attr_set = true;
    }
    Args a;
    for (int i = 0; i < NRANK; ++i) {
        const int p = (rank + i) & (NRANK - 1), c0 = ((i - (int)((uintptr_t)d_in[1] >> 7)) & 7) * 128;
        a.peer[i].ws = (char*)d_ws_all[p];
        a.peer[i].x = (const int8_t*)d_in[0] + (size_t)p * MLOC * KLOC;
        a.col[i].w = (const int8_t*)d_in[1] + (size_t)rank * KLOC * NN + c0;
        a.col[i].out = (float*)d_out + c0;
    }
    a.sx = (const float*)d_in[2];
    a.sw = (const float*)d_in[3];
    a.rank = rank;
    a.pad = 0;
    hipLaunchKernelGGL(a2a_gemm_kernel, dim3(NWG), dim3(NTHREADS), LDS_BYTES, stream, a);
    HIP_CHECK(hipGetLastError());
}
```

```cpp
#include <hip/hip_runtime.h>
#include <rccl/rccl.h>
#include <cstdint>
#include <cstdio>

#define HIP_CHECK(expr) do { hipError_t _e = (expr); if (_e != hipSuccess) { \
    fprintf(stderr, "HIP error %s at %s:%d\n", hipGetErrorString(_e), __FILE__, __LINE__); \
    return; } } while (0)

typedef int i32x2 __attribute__((ext_vector_type(2)));
typedef int i32x4 __attribute__((ext_vector_type(4)));
typedef float f32x4 __attribute__((ext_vector_type(4)));
typedef __attribute__((address_space(3))) i32x2* lds_i32x2_p;
typedef __attribute__((address_space(3))) i32x4* lds_i32x4_p;
typedef __attribute__((address_space(3))) int* lds_int_p;
typedef volatile __attribute__((address_space(3))) int* lds_vint_p;
typedef volatile __attribute__((address_space(3))) i32x4* lds_vi32x4_p;

#define STAMP(i) do {} while (0)
#define CSTAMP(i) do {} while (0)
#define STAMP_DECL do {} while (0)
#define STAMP_OUT(wsp) do {} while (0)
#define STAMP_ARG
#define STAMP_PASS

namespace {

constexpr int NRANK = 8;
constexpr int NN = 4096;
constexpr int KLOC = 512;
constexpr int MLOC = 512;
constexpr int NWG = 256;
constexpr int NTHREADS = 512;

#define NSPLIT 44
#define LP0 0
#define LP1 2
#define PACE 64
#define PUSH0 50
#define TFIRST 250
#define PACE0 58
#define NFAST 2
#define POISON_BITS "sc1"
#define OUT_BITS "sc1"
#define XST_BITS "sc0 sc1"
#define XLD_BITS "sc0 sc1"
#define XRE_BITS "sc0 sc1"
#define XPF_BITS "sc1"
#define XSTRIDE 7168
#define QROT 1
#define ADIST 6
#define TICKET_AT 0
#define LEAD 0
#define ADIST2 ADIST
#define AINIT ADIST
#define SPF 0
#define FIRST_AT 3
#define LDW 3
#define LDW_FROM 14
#define LSLEEP 2
#define XLEAD 0
#define XJIT 8
#define XAHEAD 150
#define HOLDW 10
#define HOLDX 4
#define CMAP 2
#define PAUSE 45
#define PDFIX 0
#define XFIT (2 * XBIAS)
#define WREST 0
#define DUMPNOW 1
#define T2AT 2
#define T2PER 4
#define AUP 99
#define AUP2 99
#define LATEW 0
#define LUP 14
#define PEARLY 0
#define OUT2_BITS OUT_BITS
#define ROT 4
#define LPRIO 0
#define CPRIO 1
#define CS0 10
#define CS1 20
constexpr int NS1 = 4 + NSPLIT / 2, NS2 = 56 - NSPLIT, NSTEP = NS1 + NS2;
constexpr int NFRAG = 56;
static_assert(NSPLIT % 2 == 0 && NSPLIT >= 8 && NSTEP <= 64 && NSTEP % 2 == 0 && LP0 == 0 && LP1 == 2, "");

constexpr unsigned WS_TMO = 0;
constexpr unsigned WS_TICKET = 4096;
constexpr unsigned WS_FLAGS = 8192;
constexpr unsigned WS_DATA = 2u << 20;
constexpr unsigned SLOT_BYTES = 2u << 20;
constexpr unsigned WS_X = WS_DATA + 3u * SLOT_BYTES;
constexpr unsigned WS_RAW = WS_X + NWG * 4u * XSTRIDE;
constexpr size_t WS_NEEDED = (size_t)WS_RAW + NWG * 4u * 8192;
constexpr int XBIAS = 1 << 21;
constexpr unsigned POISON = 0x80808080u;

constexpr unsigned LDS_PUSH = 16 * 8192;
constexpr unsigned LDS_LOC = LDS_PUSH + 8192;
constexpr unsigned LDS_H1 = (NSTEP & 15) * 8192;
constexpr unsigned LDS_CTRL = LDS_LOC + 16384;
constexpr unsigned C_FULL = 0;
constexpr unsigned C_DUE = 8;
constexpr unsigned C_CNT = 12;
constexpr unsigned C_CALL = 16;
constexpr unsigned C_PAUSE = 20;
constexpr unsigned C_H0 = 32;
constexpr unsigned C_H1 = 48;
constexpr unsigned C_TICKET = 768;
constexpr unsigned C_PROG = 1024;
constexpr unsigned LDS_BYTES = LDS_CTRL + 2048 + 256;

#define DEADLINE 50000000u
constexpr unsigned WATCH_TICKS = 3000;
constexpr unsigned DEADLINE_TICKS = DEADLINE;

struct Args {
    struct { char* ws; const int8_t* x; } peer[NRANK];
    struct { const int8_t* w; float* out; } col[NRANK];
    const float* sx;
    const float* sw;
    int rank;
    int pad;
};
typedef unsigned long long u64;
typedef u64 u64x2 __attribute__((ext_vector_type(2)));
__device__ __forceinline__ u64x2 args_ld(unsigned off) {
    u64x2 r;
    asm volatile("s_load_dwordx4 %0, %1, %2" : "=s"(r) : "s"(__builtin_amdgcn_kernarg_segment_ptr()), "s"(off) : "memory");
    return r;
}

#define SB() __builtin_amdgcn_sched_barrier(0)
#define LGKM(n) __builtin_amdgcn_s_waitcnt(0xC07F | ((n) << 8))

#define DMA4(BITS) asm volatile( \
        "s_mov_b32 m0, %5\n\t" \
        "s_nop 0\n\t" \
        "global_load_lds_dwordx4 %0, %4 " BITS "\n\t" \
        "global_load_lds_dwordx4 %1, %4 offset:1024 " BITS "\n\t" \
        "global_load_lds_dwordx4 %2, %4 offset:2048 " BITS "\n\t" \
        "global_load_lds_dwordx4 %3, %4 offset:3072 " BITS \
        :: "v"(v0), "v"(v1), "v"(v2), "v"(v3), "s"(gbase), "s"(lds_addr) : "memory")
template <bool SYS = false>
__device__ __forceinline__ void dma4(const char* gbase, unsigned lds_addr,
                                     unsigned v0, unsigned v1, unsigned v2, unsigned v3) {
    if constexpr (SYS) DMA4("sc0 sc1");
    else DMA4("");
}
template <int N> __device__ __forceinline__ void wait_vm(i32x4& a) {
    asm volatile("s_waitcnt vmcnt(%1)" : "+v"(a) : "i"(N) : "memory");
}
template <int N> __device__ __forceinline__ void wait_vm(i32x4& a, i32x4& b) {
    asm volatile("s_waitcnt vmcnt(%2)" : "+v"(a), "+v"(b) : "i"(N) : "memory");
}
__device__ __forceinline__ int lds_ld(unsigned addr) {
    return __hip_atomic_load((lds_int_p)addr, __ATOMIC_RELAXED, __HIP_MEMORY_SCOPE_WORKGROUP);
}
__device__ __forceinline__ void lds_st(unsigned addr, int v) {
    __hip_atomic_store((lds_int_p)addr, v, __ATOMIC_RELAXED, __HIP_MEMORY_SCOPE_WORKGROUP);
}
__device__ __forceinline__ void lds_st1(unsigned addr, int v) {
    asm volatile("s_mov_b64 exec, 1\n\tds_write_b32 %0, %1\n\ts_mov_b64 exec, -1" :: "v"(addr), "v"(v) : "memory");
}
__device__ __forceinline__ void lds_st2(unsigned addr, int v0, int v1) {
    asm volatile("s_mov_b64 exec, 1\n\tds_write_b64 %0, %1\n\ts_mov_b64 exec, -1" :: "v"(addr), "v"(i32x2{v0, v1}) : "memory");
}
__device__ __forceinline__ void lds_wait(unsigned addr, int need) {
    while (__builtin_amdgcn_readfirstlane(
               __hip_atomic_load((lds_int_p)addr, __ATOMIC_ACQUIRE, __HIP_MEMORY_SCOPE_WORKGROUP)) < need)
        __builtin_amdgcn_s_sleep(1);
    asm volatile("" ::: "memory");
}

constexpr int imin(int x, int y) { return x < y ? x : y; }
constexpr int imax(int x, int y) { return x > y ? x : y; }
constexpr int local_of(int s) { return s >= LP0 && s < LP0 + 2 ? s - LP0 : s >= LP1 && s < LP1 + 2 ? 2 + s - LP1 : -1; }
constexpr int frag_lo(int s) {
    return s >= NS1 ? NSPLIT + s - NS1 : 2 * (s - imin(2, imax(0, s - LP0)) - imin(2, imax(0, s - LP1)));
}
__device__ __forceinline__ int pos_of(int rel, int mi) {
    if (!QROT) return rel - 1;
    const int v = (rel + 2 * mi) & 7;
    return v - (v > 2 * mi ? 1 : 0);
}
__device__ __forceinline__ int rel_of(int pos, int mi) {
    if (!QROT) return pos + 1;
    return (pos + (pos >= 2 * mi ? 1 : 0) - 2 * mi) & 7;
}
__device__ __forceinline__ unsigned frag_off(int rel, int cls, int rg) {
    const int i = 7 * cls + pos_of(rel, rg >> 3), m = rg & 1;
    const int P = i < NSPLIT ? i & 1 : m, frag = i < NSPLIT ? (i & ~1) + (m ^ P) : i;
    return (((rg >> 1) * 2 + P) * NFRAG + frag) * 1024;
}

struct Ctx {
    const char* ab[7];
    char* xo;
    const char* xl;
    const char* xl2;
    unsigned sv[4];
    unsigned ll;
    unsigned la;
    unsigned sent;
    unsigned tag;
    char* ro;
    unsigned sel[3];
    unsigned any, raw[2];
    float scale;
    unsigned hf, h0, h1;
    unsigned rb[2][8];
    unsigned l16;
    unsigned chk;
    const char* flags;
    unsigned foff;
    unsigned call, t0;
    unsigned bad;
    unsigned prog;
    int one;
    int seen;
};

__device__ __forceinline__ void read_local(i32x4 (&AL)[2][2], const Ctx& c) {
#pragma unroll
    for (int s = 0; s < 2; ++s) {
        AL[s][0] = *(lds_i32x4_p)(c.la ^ (s * 64));
        AL[s][1] = *(lds_i32x4_p)(c.la ^ (s * 64) ^ 2048);
    }
}
template <int S, int H>
__device__ __forceinline__ void read_b(i32x2 (&B)[16], int t, const Ctx& c) {
    B[2 * t + H] = __builtin_amdgcn_ds_read_tr8_b64_v2i32((lds_i32x2_p)(c.rb[(S >> 3) & 1][t] + (S & 7) * 8192 + H * 1024));
}
__device__ __forceinline__ void mma(i32x4& acc, const i32x4& a, const i32x2 (&B)[16], int t) {
    asm volatile("v_mfma_i32_16x16x64_i8 %0, %1, %2, %0"
                 : "+v"(acc) : "v"(__builtin_shufflevector(B[2 * t], B[2 * t + 1], 0, 1, 2, 3)), "v"(a));
}
template <int F>
__device__ __forceinline__ void load_frag(i32x4 (&A)[NFRAG], const Ctx& c) {
    asm volatile("global_load_dwordx4 %0, %1, %2 offset:%3 sc0 sc1"
                 : "=v"(A[F]) : "v"(c.l16), "s"(c.ab[F / 8]), "i"((F % 8) * 1024 - 4096) : "memory");
}
template <int F, int END>
__device__ __forceinline__ void first_frags(i32x4 (&A)[NFRAG], const Ctx& c, int cnt) {
    if constexpr (F < END) {
        asm volatile(
            "v_cmp_gt_i32 vcc, %[need], %[cnt]\n\t"
            "s_cbranch_vccz .Lgo%=\n"
            ".Lagain%=:\n\t"
            "ds_read_b32 %[cnt], %[at]\n\t"
            "s_waitcnt lgkmcnt(0)\n\t"
            "v_cmp_gt_i32 vcc, %[need], %[cnt]\n\t"
            "s_cbranch_vccnz .Lagain%=\n"
            ".Lgo%=:"
            : [cnt] "+v"(cnt) : [at] "v"(LDS_CTRL + C_CNT), [need] "i"(F / 2 + 1) : "memory", "vcc");
        load_frag<F>(A, c);
        load_frag<F + 1>(A, c);
        first_frags<F + 2, END>(A, c, cnt);
    }
}
template <int FULL>
__device__ __forceinline__ void check(const Ctx& c) {
    while (__any(__hip_atomic_load((lds_int_p)c.chk, __ATOMIC_ACQUIRE, __HIP_MEMORY_SCOPE_WORKGROUP) < FULL))
        __builtin_amdgcn_s_sleep(1);
    asm volatile("" ::: "memory");
}

#define CP_WAIT \
        "s_or_b64 vcc, %[low], %[miss]\n\t" \
        "s_cbranch_scc1 .Lslow%=\n" \
        ".Lback%=:\n\t" \
        ".subsection 1\n" \
        ".Lagain%=:\n\t" \
        "s_sleep 1\n" \
        ".Lslow%=:\n\t" \
        "ds_read_b32 %[p0], %[chk]\n\t" \
        "s_waitcnt lgkmcnt(0)\n\t" \
        "v_cmp_gt_i32 vcc, %[full], %[p0]\n\t" \
        "s_cbranch_vccnz .Lagain%=\n\t"
template <int FULL>
__device__ __forceinline__ void checkpoint(Ctx& c, unsigned long long low) {
    unsigned p0;
    asm volatile(CP_WAIT "s_branch .Lback%=\n\t.subsection 0"
                 : [p0] "=&v"(p0) : [low] "s"(low), [miss] "i"(0), [chk] "v"(c.chk), [full] "i"(FULL) : "memory", "vcc", "scc");
}
template <int FULL, int F>
__device__ __forceinline__ void checkpoint(i32x4 (&A)[NFRAG], Ctx& c, unsigned long long low, unsigned long long miss) {
    unsigned p0, p1, fl, prev;
    asm volatile(
        CP_WAIT
        "s_cmp_eq_u64 %[miss], 0\n\t"
        "s_cbranch_scc1 .Lback%=\n\t"
        "s_mov_b32 %[prev], 0\n"
        ".Lretry%=:\n\t"
        "global_load_dword %[p0], %[l16], %[ab] offset:%[o0] sc0 sc1\n\t"
        "global_load_dword %[p1], %[l16], %[ab] offset:%[o1] sc0 sc1\n\t"
        "global_load_dwordx4 %[a0], %[l16], %[ab] offset:%[o0] sc0 sc1\n\t"
        "global_load_dwordx4 %[a1], %[l16], %[ab] offset:%[o1] sc0 sc1\n\t"
        "global_load_dword %[fl], %[foff], %[flags] sc0 sc1\n\t"
        "s_memrealtime vcc\n\t"
        "s_waitcnt vmcnt(0) lgkmcnt(0)\n\t"
        "s_cmp_lg_u32 %[prev], 0\n\t"
        "s_cbranch_scc1 .Lback%=\n\t"
        "s_sub_u32 %[prev], vcc_lo, %[t0]\n\t"
        "v_cmp_eq_u32 vcc, %[sent], %[p0]\n\t"
        "v_cmp_eq_u32 %[miss], %[sent], %[p1]\n\t"
        "s_or_b64 vcc, vcc, %[miss]\n\t"
        "s_cbranch_scc0 .Lback%=\n\t"
        "s_cmp_gt_u32 %[prev], %[dl]\n\t"
        "s_cselect_b32 %[bad], 1, %[bad]\n\t"
        "s_cbranch_scc1 .Lback%=\n\t"
        "v_subrev_u32 %[fl], %[call], %[fl]\n\t"
        "v_cmp_gt_i32 vcc, 0, %[fl]\n\t"
        "s_cmp_eq_u64 vcc, 0\n\t"
        "s_cselect_b32 %[prev], 1, 0\n\t"
        "s_branch .Lretry%=\n\t"
        ".subsection 0"
        : [a0] "+v"(A[F]), [a1] "+v"(A[F + 1]), [p0] "=&v"(p0), [p1] "=&v"(p1), [fl] "=&v"(fl), [prev] "=&s"(prev),
          [bad] "+s"(c.bad), [miss] "+s"(miss)
        : [low] "s"(low), [chk] "v"(c.chk), [full] "i"(FULL), [l16] "v"(c.l16), [ab] "s"(c.ab[F / 8]), [foff] "v"(c.foff), [flags] "s"(c.flags),
          [o0] "i"((F % 8) * 1024 - 4096), [o1] "i"((F % 8) * 1024 - 3072), [sent] "s"(c.sent), [call] "s"(c.call),
          [t0] "s"(c.t0), [dl] "i"(DEADLINE_TICKS)
        : "memory", "vcc", "scc");
}

static_assert(2 * T2AT <= ADIST && T2PER <= 4 && ADIST % 2 == 0 && AINIT % 2 == 0 && AINIT <= ADIST && FIRST_AT >= 2 && FIRST_AT <= 3 && NS2 % 2 == 0, "");
constexpr int issued_before(int s) {
    if (s <= NS1 - T2AT) return s < FIRST_AT ? 0 : imin(NSPLIT, imin(frag_lo(s) + ADIST + (s > AUP ? 2 : 0) + (s > AUP2 ? 2 : 0), AINIT + 4 * (s - FIRST_AT)));
    const int ramp = NSPLIT + T2PER * (s - NS1 + T2AT);
    return imin(NFRAG, s <= NS1 ? ramp : imin(ramp, imax(NSPLIT + T2PER * T2AT, frag_lo(s) + ADIST2)));
}
constexpr int span(int s) {
    return s + 1 >= NSTEP ? 0 : s + 1 < NS1 ? 1 : (s + 1 - NS1) % 2 == 0 ? 2 : 0;
}
constexpr int dumps(int s) { return DUMPNOW ? (s == NS1 - 1 ? 6 : 0) : s >= NS1 && s < NS1 + 3 ? 2 : 0; }
constexpr int ops_before_wait(int s) {
    int n = issued_before(s + 1 == FIRST_AT ? s : s + 1);
    for (int t = 0; t <= s; ++t) n += t < s || DUMPNOW ? dumps(t) : 0;
    return n;
}
constexpr int ops_through_frag(int f) {
    int s = FIRST_AT - 1;
    while (issued_before(s + 1) <= f) ++s;
    int n = f + 1;
    for (int t = 0; t < s; ++t) n += dumps(t);
    return n;
}

struct Four { int f; i32x4 x[4]; };
__device__ __forceinline__ Four fetch4(unsigned flag, unsigned at) {
    Four r;
    r.f = *(lds_vint_p)flag;
#pragma unroll
    for (int t = 0; t < 4; ++t) r.x[t] = *(lds_vi32x4_p)(at + t * 1024);
    return r;
}
__device__ __forceinline__ void await4(Four& r, unsigned flag, unsigned at) {
    asm volatile(
        "v_cmp_ne_u32 vcc, 0, %0\n\t"
        "s_cbranch_vccz .Lagain%=\n"
        ".Lgo%=:\n\t"
        ".subsection 1\n"
        ".Lagain%=:\n\t"
        "ds_read_b32 %0, %5\n\t"
        "s_waitcnt lgkmcnt(0)\n\t"
        "v_cmp_ne_u32 vcc, 0, %0\n\t"
        "s_cbranch_vccz .Lagain%=\n\t"
        "ds_read_b128 %1, %6\n\t"
        "ds_read_b128 %2, %6 offset:1024\n\t"
        "ds_read_b128 %3, %6 offset:2048\n\t"
        "ds_read_b128 %4, %6 offset:3072\n\t"
        "s_waitcnt lgkmcnt(0)\n\t"
        "s_branch .Lgo%=\n\t"
        ".subsection 0"
        : "+v"(r.f), "+v"(r.x[0]), "+v"(r.x[1]), "+v"(r.x[2]), "+v"(r.x[3]) : "v"(flag), "v"(at) : "memory", "vcc");
}
struct Out { unsigned tw, tw1, tr, oo; char *o0, *o1; };
__device__ __forceinline__ Out out_of(char* rows, unsigned lds, int lane) {
    Out o;
    o.tw = lds + (lane & 15) * 128 + (((lane >> 4) ^ (lane & 7)) << 4);
    o.tw1 = o.tw ^ 64;
    o.tr = lds + (lane >> 3) * 128 + (((lane & 7) ^ (lane >> 3)) << 4);
    o.oo = (lane >> 3) * NN * 4 + (lane & 7) * 16;
    o.o0 = rows;
    o.o1 = rows + 8 * NN * 4;
    asm volatile("" : "+v"(o.tw), "+v"(o.tw1), "+v"(o.tr), "+v"(o.oo), "+s"(o.o0), "+s"(o.o1));
    return o;
}
__device__ __forceinline__ void finish(const i32x4* u, const i32x4* v, float scale, const Out& o) {
    i32x4 Z[4];
#pragma unroll
    for (int p = 0; p < 2; ++p) {
#pragma unroll
        for (int h = 0; h < 2; ++h) {
            const i32x4 s = u[2 * p + h] + v[2 * p + h];
            const f32x4 y = {(float)s[0] * scale, (float)s[1] * scale, (float)s[2] * scale, (float)s[3] * scale};
            *(lds_vi32x4_p)((h ? o.tw1 : o.tw) + p * 2048) = __builtin_bit_cast(i32x4, y);
        }
        Z[2 * p] = *(lds_vi32x4_p)(o.tr + p * 2048);
        Z[2 * p + 1] = *(lds_vi32x4_p)(o.tr + p * 2048 + 1024);
        SB();
    }
#pragma unroll
    for (int p = 0; p < 2; ++p)
        asm volatile("global_store_dwordx4 %2, %0, %3 offset:%5 " OUT_BITS "\n\t"
                     "global_store_dwordx4 %2, %1, %4 offset:%5 " OUT2_BITS "\n\ts_nop 1"
                     :: "v"(Z[2 * p]), "v"(Z[2 * p + 1]), "v"(o.oo), "s"(o.o0), "s"(o.o1), "i"(p * 128) : "memory");
}
constexpr int XLOOK = 6144 - 16 - XLEAD * 1024;
__device__ __forceinline__ void touch_x(const char* xi) {
    unsigned w;
    asm volatile("s_load_dword %0, %1, 0\n\ts_load_dword %0, %1, %2\n\ts_load_dword %0, %1, %3\n\ts_load_dword %0, %1, %4\n\ts_waitcnt lgkmcnt(0)"
                 : "=&s"(w) : "s"(xi + XLOOK), "i"(XSTRIDE), "i"(2 * XSTRIDE), "i"(3 * XSTRIDE) : "memory");
}
__device__ __forceinline__ unsigned long long or64(unsigned long long x, unsigned long long y) {
    unsigned long long r;
    asm volatile("s_or_b64 %0, %1, %2" : "=s"(r) : "s"(x), "s"(y) : "scc");
    return r;
}

template <int Q>
__device__ __forceinline__ void piece(Ctx& c, i32x4 (&T)[8], int part) {
    constexpr int U = Q / 3, KK = Q % 3;
    i32x4& t = T[4 * U + KK];
    if (part == 0) {
        t[0] = (t[0] << 2) | c.tag;
        if (KK == 0) t[1] |= c.raw[U];
        t[0] = __builtin_amdgcn_perm(T[4 * U + 3][0], t[0], c.sel[KK]);
    } else if (part == 1) {
#pragma unroll
        for (int e = 1; e < 4; ++e) t[e] = __builtin_amdgcn_perm(T[4 * U + 3][e], t[e], c.sel[KK]);
    } else {
        asm volatile("global_store_dwordx4 %1, %0, %2 offset:%3 sc0 sc1\n\ts_nop 1" :: "v"(t), "v"(c.l16), "s"(c.xo), "i"(Q * 1024 - 4096) : "memory");
    }
}
template <int U>
__device__ __forceinline__ void fits(Ctx& c, i32x4 (&T)[8], int part) {
    if (part < 4) {
        const i32x4& t = T[4 * U + part];
        c.any = (part ? c.any : 0u) | t[0] | t[1] | t[2] | t[3];
        asm volatile("" : "+v"(c.any));
    } else {
        asm volatile(
            "s_mov_b32 %0, 0\n\t"
            "v_cmp_le_u32 vcc, %8, %1\n\t"
            "s_cbranch_vccnz .Lraw%=\n"
            ".Lfit%=:\n\t"
            ".subsection 1\n"
            ".Lraw%=:\n\t"
            "global_store_dwordx4 %6, %2, %7 offset:%9 sc0 sc1\n\t"
            "global_store_dwordx4 %6, %3, %7 offset:%9+1024 sc0 sc1\n\t"
            "global_store_dwordx4 %6, %4, %7 offset:%9+2048 sc0 sc1\n\t"
            "global_store_dwordx4 %6, %5, %7 offset:%9+3072 sc0 sc1\n\t"
            "s_mov_b32 %0, 0x800000\n\t"
            "s_waitcnt vmcnt(0)\n\t"
            "s_branch .Lfit%=\n\t"
            ".subsection 0"
            : "=&s"(c.raw[U]) : "v"(c.any), "v"(T[4 * U]), "v"(T[4 * U + 1]), "v"(T[4 * U + 2]), "v"(T[4 * U + 3]),
              "v"(c.l16), "s"(c.ro), "i"(XFIT), "i"(U * 4096 - 4096) : "memory", "vcc");
    }
}
template <int D>
__device__ __forceinline__ void pack_x(Ctx& c, i32x4 (&T)[8], int g) {
    if constexpr (D < 2) { if (g >= 1 && g <= 5) fits<D>(c, T, g - 1); }
    if (g == 7) piece<2 * D>(c, T, 0);
    if (g == 9) piece<2 * D>(c, T, 1);
    if (g == 10) piece<2 * D>(c, T, 2);
    if (g == 11) piece<2 * D + 1>(c, T, 0);
    if (g == 13) piece<2 * D + 1>(c, T, 1);
    if (g == 15) piece<2 * D + 1>(c, T, 2);
}

template <int S>
__device__ __forceinline__ void step(Ctx& c, i32x4 (&acc)[2][8], i32x4 (&AL)[2][2][2], i32x4 (&A)[NFRAG],
                                     i32x2 (&B)[2][16], Four& X STAMP_ARG) {
    constexpr bool PH1 = S < NS1, LOC = S < 4, NEXT = S + 1 < NSTEP;
    constexpr int SPAN = span(S), FULL = imin(S + SPAN + 2, NSTEP);
    constexpr bool HAND = DUMPNOW && S == NS1 - 1;
    constexpr bool CP = SPAN > 0 && !HAND, VAL = CP && S + 1 >= 4;
    constexpr bool FREE = (S & 1) == 1 && S + 13 < NSTEP;
    constexpr int F = frag_lo(S), FN = frag_lo(S + 1);
    constexpr int L0 = issued_before(S), L1 = S + 1 == FIRST_AT ? L0 : issued_before(S + 1);
    unsigned long long m0 = 0, m1 = 0, m2 = 0;
    if constexpr ((S & 3) == 0) STAMP(4 + (S >> 2));
    if constexpr (S == 2) STAMP(15);
    if constexpr (S == CS0) { STAMP(28); CSTAMP(30); }
    if constexpr (S == CS1) { STAMP(29); CSTAMP(31); }
    if constexpr (S == FIRST_AT) first_frags<0, AINIT>(A, c, 0);
    const i32x4& a0 = LOC ? AL[LOC ? S / 2 : 0][S & 1][0] : A[F];
    const i32x4& a1 = LOC ? AL[LOC ? S / 2 : 0][S & 1][1] : A[PH1 ? F + 1 : F];
    LGKM(S == 2 ? 6 : 8);
    SB();
#pragma unroll
    for (int g = 0; g < 16; ++g) {
        const int t = g >> 1;
        if (g == 8) { if constexpr (NEXT) LGKM(8 + FREE); else LGKM(6); SB(); }
        if constexpr (HAND) {
            const int u = (g >> 3) * 4 + (g & 3);
            if (g & 4) mma(acc[0][u], a0, B[S & 1], u);
            else mma(acc[1][u], a1, B[S & 1], u);
            SB();
        } else if ((g & 1) == 0) { mma(acc[0][t], a0, B[S & 1], t); SB(); }
        else if constexpr (PH1) { mma(acc[1][t], a1, B[S & 1], t); SB(); }
        if (CP && g == 0) { c.seen = lds_ld(c.chk); SB(); }
        if constexpr (NEXT) {
            if (g & 1) read_b<S + 1, 1>(B[(S + 1) & 1], t, c);
            else if (g != 14) read_b<S + 1, 0>(B[(S + 1) & 1], t, c);
            if (g == 13) read_b<S + 1, 0>(B[(S + 1) & 1], 7, c);
        }
        if constexpr (!NEXT) {
            if (g == 0) X = fetch4(c.hf + C_H1 - C_H0, c.h1);
            if (g == 2) { if (c.bad & 1) c.scale = __builtin_nanf(""); asm volatile("" : "+v"(c.scale)); }
            if (g >= 6 && g <= 12 && (g & 1) == 0) *(lds_vi32x4_p)(c.h0 + (g - 6) * 512) = acc[0][(g - 6) / 2];
            if (g == 12) lds_st1(c.hf, c.one);
        }
        if (g == 1) { if constexpr (L0 < L1) load_frag<L0>(A, c); }
        if (FREE && g == 2) __hip_atomic_fetch_add((lds_int_p)c.prog, c.one, __ATOMIC_RELAXED, __HIP_MEMORY_SCOPE_WORKGROUP);
        if (g == 3) { if constexpr (L0 + 1 < L1) load_frag<L0 + 1>(A, c); }
        if (g == 4) { if constexpr (L0 + 2 < L1) load_frag<L0 + 2>(A, c); }
        if (g == 5) { if constexpr (L0 + 3 < L1) load_frag<L0 + 3>(A, c); }
        if (CP && g == 8) m2 = __builtin_amdgcn_sicmp(c.seen, FULL, 40);
        if constexpr (VAL) {
            if (g == 9 + LATEW) wait_vm<ops_before_wait(S) - ops_through_frag(FN + 1)>(A[FN], A[FN + 1]);
            if (g == 10 + LATEW) m0 = __builtin_amdgcn_uicmp(A[FN][0], c.sent, 32);
            if (g == 11 + LATEW) m1 = __builtin_amdgcn_uicmp(A[FN + 1][0], c.sent, 32);
            if (g == 12 + 2 * LATEW) m0 = or64(m0, m1);
            if (g == 14) checkpoint<FULL, FN>(A, c, m2, m0);
        } else if constexpr (CP) {
            if (g == 14) checkpoint<FULL>(c, m2);
        }
        if (S == 1 && g == 9) {
            asm volatile("s_waitcnt vmcnt(0)" ::: "memory");
            read_local(AL[1], c);
        }
        if constexpr (HAND) {
            if (g >= 3 && g <= 7) fits<0>(c, acc[1], g - 3);
            if (g >= 8 && g <= 10) piece<0>(c, acc[1], g - 8);
            if (g >= 10 && g <= 12) piece<1>(c, acc[1], g - 10);
            if (g >= 12 && g <= 14) piece<2>(c, acc[1], g - 12);
            if (g == 14) { fits<1>(c, acc[1], 0); fits<1>(c, acc[1], 1); }
            if (g == 15) { fits<1>(c, acc[1], 2); fits<1>(c, acc[1], 3); fits<1>(c, acc[1], 4); }
        } else if constexpr (dumps(S) > 0) pack_x<S - NS1>(c, acc[1], g);
        SB();
    }
    if constexpr (HAND) {
#pragma unroll
        for (int i = 0; i < 3; ++i) piece<3>(c, acc[1], i);
        c.seen = lds_ld(c.chk);
#pragma unroll
        for (int i = 0; i < 3; ++i) piece<4>(c, acc[1], i);
#pragma unroll
        for (int i = 0; i < 3; ++i) piece<5>(c, acc[1], i);
        m2 = __builtin_amdgcn_sicmp(c.seen, FULL, 40);
        wait_vm<ops_before_wait(S) - ops_through_frag(FN + 1)>(A[FN], A[FN + 1]);
        m0 = or64(__builtin_amdgcn_uicmp(A[FN][0], c.sent, 32), __builtin_amdgcn_uicmp(A[FN + 1][0], c.sent, 32));
        checkpoint<FULL, FN>(A, c, m2, m0);
        SB();
    }
    if constexpr (NEXT) step<S + 1>(c, acc, AL, A, B, X STAMP_PASS);
}

template <int S>
__device__ __forceinline__ void allowed(int& due, int n) {
    if constexpr (span(S) > 0) {
        constexpr int F = issued_before(S + span(S) + 1), NEED = F <= NSPLIT ? (F + 1) / 2 : F - NSPLIT / 2;
        if (n < NEED) due = imin(S + span(S) + 2, NSTEP) - 1;
    }
    if constexpr (S > FIRST_AT) allowed<S - 1>(due, n);
}
#define LORD 0
constexpr int LSEQ[1][2][3] = {{{0, 2, 4}, {1, 3, 5}}};
constexpr int fill_step(int h, int k) { return k < 3 ? LSEQ[LORD][h][k] : imin(NSTEP, 2 * k + h); }
constexpr int owed(int h, int k) {
    int m = fill_step(h, k);
    for (int i = k + 1; i < 3; ++i) m = imin(m, fill_step(h, i));
    return m;
}
__device__ __forceinline__ int w_chunk(int s, int P, int mi) {
    const int i = max(s < NS1 ? 2 * s - 8 + P : NSPLIT + s - NS1, 0), cls = (i * 37) >> 8;
    const int rem = rel_of(i - 7 * cls, mi) * 8 + ((cls + ROT * mi) & 7);
    return s < 4 ? 4 * P + (s ^ mi) : rem;
}
struct Ldr {
    const char* w;
    unsigned dv[4];
    int woff;
    unsigned fa, pa;
    int freed;
    int pz;
};
__device__ __forceinline__ void fill(const Ldr& l, int off, unsigned slot) {
    const char* p = l.w + off;
    dma4(p, slot, l.dv[0], l.dv[1], l.dv[2], l.dv[3]);
    dma4(p + 32 * NN, slot + 4096, l.dv[0], l.dv[1], l.dv[2], l.dv[3]);
}
__device__ __forceinline__ int released(unsigned pa) {
    int v = lds_ld(pa);
    v = min(v, __builtin_amdgcn_update_dpp(v, v, 0xB1, 0xF, 0xF, false));
    v = min(v, __builtin_amdgcn_update_dpp(v, v, 0x4E, 0xF, 0xF, false));
    return 2 * __builtin_amdgcn_readfirstlane(v);
}
template <int H, int K>
__device__ __forceinline__ void fills(Ldr& l STAMP_ARG) {
    constexpr int S = fill_step(H, K);
    if constexpr (S < NSTEP) {
        if constexpr (S >= 16) {
            if (__builtin_expect(l.freed <= S - 16, 0)) {
                while ((l.freed = released(l.pa)) <= S - 16) __builtin_amdgcn_s_sleep(LSLEEP);
            }
        }
        if constexpr (PAUSE && K > 1 && K < 12) {
            if (l.pz != 2) while ((l.pz = __builtin_amdgcn_readfirstlane(lds_ld(LDS_CTRL + C_PAUSE))) == 1) __builtin_amdgcn_s_sleep(1);
        }
        if constexpr (K == LUP) __builtin_amdgcn_s_setprio(2);
        if constexpr (K > 1) fill(l, __builtin_amdgcn_readlane(l.woff, S), (S & 15) * 8192);
        constexpr int W = imin(LDW, imax(1, K - LDW_FROM + 2));
        asm volatile("s_waitcnt vmcnt(%2)\n\tds_max_i32 %0, %1" :: "v"(l.fa), "v"(owed(H, K - W + 1)), "i"(8 * W) : "memory");
        if constexpr (K < 6 || (K & 1) == 1) STAMP(K < 6 ? K : 3 + K / 2);
        fills<H, K + 1>(l STAMP_PASS);
    } else {
#pragma unroll
        for (int k = LDW - 1; k >= 0; --k)
            asm volatile("s_waitcnt vmcnt(%2)\n\tds_max_i32 %0, %1" :: "v"(l.fa), "v"(k ? fill_step(H, K - k) : NSTEP), "i"(8 * k) : "memory");
    }
}

__device__ __forceinline__ void take_ticket(const Args& a, unsigned call, int lane) {
    unsigned tk = 0;
    if (lane == 0) tk = __hip_atomic_fetch_add((unsigned*)(a.peer[0].ws + WS_TICKET), 1u, __ATOMIC_RELAXED, __HIP_MEMORY_SCOPE_AGENT);
    if ((__builtin_amdgcn_readfirstlane(tk) & 255) == 255) {
        if (lane < NRANK)
            __hip_atomic_store((unsigned*)(a.peer[lane].ws + WS_FLAGS + ((NRANK - lane) & (NRANK - 1)) * 128), call,
                               __ATOMIC_RELAXED, __HIP_MEMORY_SCOPE_SYSTEM);
    }
}

__device__ __forceinline__ void help(const Args& a, int w, int b, int lane STAMP_ARG) {
    __builtin_amdgcn_s_setprio(0);
    const unsigned t0 = (unsigned)__builtin_amdgcn_s_memrealtime();
    const int j = b >> 3, ct = ((b & 7) + CMAP * (j >> 3)) & 7;
    const unsigned call = __builtin_amdgcn_readfirstlane(lds_ld(LDS_CTRL + C_CALL)), tag = call % 3 + 1;
    const char* xi = a.peer[0].ws + WS_X + ((b ^ 8) * 4 + w) * XSTRIDE;
    typedef const __attribute__((address_space(4))) float* cfp;
    float scale = *(cfp)(uintptr_t)a.sx * *(cfp)(uintptr_t)a.sw;
    const Out o = out_of((char*)a.col[ct].out + (size_t)(unsigned)(((((j >> 1) & 3) * 128 + w * 32 + (j & 1) * 16) * NN + (j >> 3) * 1024) * 4),
                         LDS_LOC + w * 4096, lane);
    const unsigned l16 = lane * 16, pa = LDS_CTRL + C_PROG + (lane & 3) * 260;
    unsigned sel0 = 0x0c0c0703, sel1 = 0x0c070100;
    unsigned hf = LDS_CTRL + C_H0 + w * 4, h0 = LDS_LOC + w * 4096 + l16, h1 = LDS_H1 + w * 4096 + l16;
    asm volatile("" : "+s"(sel0), "+s"(sel1), "+v"(scale), "+v"(hf), "+v"(h0), "+v"(h1));
    while (released(pa) < NSTEP - 14) __builtin_amdgcn_s_sleep(4);
    STAMP(13);
    unsigned long long now;
    unsigned d;
    do {
        asm volatile("s_load_dword %0, %2, %3 glc\n\ts_memrealtime %1\n\ts_waitcnt lgkmcnt(0)" : "=&s"(d), "=&s"(now) : "s"(xi), "i"(XLOOK) : "memory");
    } while ((d & 3) != tag && (unsigned)now - t0 < WATCH_TICKS);
    STAMP(14);
    __builtin_amdgcn_s_setprio(2);
    i32x4 Q[6];
    bool late;
    do {
        asm volatile("global_load_dwordx4 %3, %6, %7 offset:-1024 sc0 sc1\n\tglobal_load_dwordx4 %4, %6, %7 sc0 sc1\n\t"
                     "global_load_dwordx4 %5, %6, %7 offset:1024 sc0 sc1\n\tglobal_load_dwordx4 %0, %6, %7 offset:-4096 sc0 sc1\n\t"
                     "global_load_dwordx4 %1, %6, %7 offset:-3072 sc0 sc1\n\tglobal_load_dwordx4 %2, %6, %7 offset:-2048 sc0 sc1\n\t"
                     "s_waitcnt vmcnt(0)"
                     : "=&v"(Q[0]), "=&v"(Q[1]), "=&v"(Q[2]), "=&v"(Q[3]), "=&v"(Q[4]), "=&v"(Q[5]) : "v"(l16), "s"(xi + 4096) : "memory");
        unsigned diff = 0;
#pragma unroll
        for (int q = 0; q < 6; ++q) diff |= (unsigned)Q[q][0] ^ tag;
        if (__builtin_expect(!__any(diff & 3), 1)) { late = false; break; }
        late = (unsigned)__builtin_amdgcn_s_memrealtime() - t0 > DEADLINE_TICKS + DEADLINE_TICKS / 2;
    } while (!late);
    STAMP(15);
    i32x4 T[2][4];
    Four H;
#pragma unroll
    for (int u = 1; u >= 0; --u) {
#pragma unroll
        for (int e = 0; e < 4; ++e) {
#pragma unroll
            for (int k = 0; k < 3; ++k) T[u][k][e] = e ? Q[3 * u + k][e] & 0xffffff : __builtin_amdgcn_ubfe(Q[3 * u + k][e], 2, 22);
            T[u][3][e] = __builtin_amdgcn_perm(Q[3 * u + 2][e], __builtin_amdgcn_perm(Q[3 * u + 1][e], Q[3 * u][e], sel0), sel1);
        }
        if (__builtin_expect(__builtin_amdgcn_readfirstlane(Q[3 * u][1]) & 0x800000, 0))
            asm volatile("global_load_dwordx4 %0, %4, %5 sc0 sc1\n\tglobal_load_dwordx4 %1, %4, %5 offset:1024 sc0 sc1\n\t"
                         "global_load_dwordx4 %2, %4, %5 offset:2048 sc0 sc1\n\tglobal_load_dwordx4 %3, %4, %5 offset:3072 sc0 sc1\n\t"
                         "s_waitcnt vmcnt(0)"
                         : "=&v"(T[u][0]), "=&v"(T[u][1]), "=&v"(T[u][2]), "=&v"(T[u][3])
                         : "v"(l16), "s"(a.peer[0].ws + WS_RAW + ((b ^ 8) * 4 + w) * 8192 + u * 4096) : "memory");
        if (u) {
#pragma unroll
            for (int t = 0; t < 4; ++t) *(lds_vi32x4_p)(h1 + t * 1024) = T[1][t];
            lds_st1(LDS_CTRL + C_H1 + w * 4, 1);
            H = fetch4(hf, h0);
            SB();
        }
    }
    if (late) {
        scale = __builtin_nanf("");
        if (lane == 0) __hip_atomic_store((unsigned*)(a.peer[0].ws + WS_TMO), call, __ATOMIC_RELAXED, __HIP_MEMORY_SCOPE_AGENT);
    }
    __builtin_amdgcn_s_setprio(0);
    await4(H, hf, h0);
    __builtin_amdgcn_s_setprio(1);
    STAMP(16);
    finish(H.x, T[0], scale, o);
    STAMP(17);
}

}

__global__ __launch_bounds__(NTHREADS, 2) __attribute__((aligned(4096))) void a2a_gemm_kernel(Args a) {
    extern __shared__ __attribute__((aligned(16))) char smem[];
    const int lane = threadIdx.x & 63;
    int wave = __builtin_amdgcn_readfirstlane(threadIdx.x >> 6);
    const int b = blockIdx.x;
    const int xcd = b & 7, j = b >> 3;
    const unsigned lds0 = (unsigned)(uintptr_t)(__attribute__((address_space(3))) char*)smem;
    const unsigned ctrl = lds0 + LDS_CTRL;
    const int P = j & 1;
    const int mi = (j >> 1) & 3;
    const int ct = (xcd + CMAP * (j >> 3)) & 7;
    const int n0 = (j >> 3) * 1024;
    STAMP_DECL;
#define NEXT_ROLE() asm volatile("" : "+s"(wave) :: "memory")
    u64 t64;

    unsigned pre = 0;
    if (wave == 7) {
        {
            u64 pc;
            unsigned len;
            asm volatile("s_getpc_b64 %0\n.Lpc%=:\n\ts_mov_b32 %1, .Lend%=-.Lpc%=-4\n\t.subsection 2\n.Lend%=:\n\t.subsection 0" : "=s"(pc), "=s"(len));
            asm volatile("s_mov_b32 m0, %2\n\ts_mov_b64 exec, 0xffff\n\tglobal_load_lds_dword %0, %1\n\ts_mov_b64 exec, -1"
                         :: "v"(min((unsigned)(j * 2048 + lane * 128), len)), "s"(pc), "i"(LDS_CTRL + 2048) : "memory");
        }
        lds_st(ctrl + lane * 4, lane == 1 ? owed(1, 0) : lane == C_DUE / 4 ? FIRST_AT + 2 : 0);
        *(lds_i32x4_p)(ctrl + C_PROG + lane * 16) = i32x4{0, 0, 0, 0};
        asm volatile("s_waitcnt lgkmcnt(0)" ::: "memory");
    } else {
        asm volatile("s_load_dword %0, %1, 0\n\ts_load_dword %0, %1, 64\n\ts_load_dword %0, %1, 128\n\t"
                     "s_load_dword %0, %1, 192\n\ts_load_dword %0, %1, 256"
                     : "=&s"(pre) : "s"(__builtin_amdgcn_kernarg_segment_ptr()) : "memory");
    }
    asm volatile("s_barrier" : "+s"(wave) :: "memory");
#define STAMP0()
#define STAMP0B()
#define NOW(t) asm volatile("s_memrealtime %0" : "=s"(t) :: "memory")
    Ctx c;
    if (wave < 4) {
    __builtin_amdgcn_s_setprio(CPRIO);
    STAMP0();
    u64x2 own = args_ld(0);
#pragma unroll
    for (int i = 0; i < 4; ++i) {
        c.sv[i] = (mi * 128 + wave * 32 + i * 8 + (lane >> 3)) * KLOC + P * 256 + (mi >> 1) * 128 +
                  (((lane & 7) ^ ((i * 4 + (lane >> 4)) & 7)) << 4) - i * 1024;
        asm volatile("" : "+v"(c.sv[i]));
    }
    unsigned ll = lds0 + LDS_LOC + wave * 4096;
    asm volatile("s_waitcnt lgkmcnt(0)" : "+s"(own), "+s"(ll) : "s"(pre) : "memory");
    STAMP0B();
    if (HOLDX) __builtin_amdgcn_s_sleep(HOLDX);
    dma4((const char*)own[1], ll, c.sv[0], c.sv[1], c.sv[2], c.sv[3]);
    }
    NEXT_ROLE();
    Ldr l;
    if ((wave & 6) == 4) {
        __builtin_amdgcn_s_setprio(LPRIO);
        u64x2 col = args_ld(128 + ct * 16);
        unsigned f0 = (4 * P + ((wave & 1) ^ mi)) * 64 * NN, slot = lds0 + (wave & 1) * 8192;
#pragma unroll
        for (int i = 0; i < 4; ++i) {
            l.dv[i] = (lane >> 3) * NN + n0 + (((lane & 7) ^ (((lane >> 4) & 3) | ((i >> 1) << 2))) << 4) + i * (8 * NN - 1024);
            asm volatile("" : "+v"(l.dv[i]));
        }
        asm volatile("s_waitcnt lgkmcnt(0)" : "+s"(col), "+s"(f0), "+s"(slot) : "s"(pre) : "memory");
        if (HOLDW) __builtin_amdgcn_s_sleep(HOLDW);
        l.w = (const char*)col[0];
        fill(l, f0, slot);
        fill(l, f0 ^ (2 * 64 * NN), slot + 16384);
        STAMP(0);
    }
    NEXT_ROLE();
    if (wave < 4) {
    char* const ws = a.peer[0].ws;
    const unsigned tstart = (unsigned)__builtin_amdgcn_s_memrealtime();
    STAMP(19);
    c.ll = lds0 + LDS_LOC + wave * 4096;
    c.xl2 = (const char*)a.peer[0].x + (128 - (mi >> 1) * 256);
    c.la = c.ll + P * 2048 + (lane & 15) * 128 + ((((mi & 1) * 4 + (lane >> 4)) ^ ((lane & 15) >> 1)) << 4);
    c.l16 = lane * 16;
    c.chk = ctrl + C_FULL + min(lane & 3, 2) * 4;
    c.foff = (lane & 7) * 128;
    c.flags = ws + WS_FLAGS;
    c.t0 = tstart;
    c.bad = 0;
    typedef const __attribute__((address_space(4))) float* cfp;
    c.scale = *(cfp)(uintptr_t)a.sx * *(cfp)(uintptr_t)a.sw;
    char* outp = (char*)a.col[ct].out;
    asm volatile("" : "+s"(outp));
    c.prog = ctrl + C_PROG + wave * 256 + lane * 4;
    c.one = 1;
    asm volatile("" : "+v"(c.one), "+v"(c.chk), "+v"(c.prog), "+v"(c.foff));
    {
        const unsigned q = (lane & 15) >> 1, kg = lane >> 4;
        const unsigned r0 = lds0 + (kg * 16 + q) * 128 + (lane & 1) * 8, f = (q >> 1) | ((kg & 1) << 2);
#pragma unroll
        for (int t = 0; t < 8; ++t) {
            c.rb[0][t] = r0 + ((t ^ f) << 4);
            c.rb[1][t] = c.rb[0][t] + 65536;
            asm volatile("" : "+v"(c.rb[0][t]), "+v"(c.rb[1][t]));
        }
    }
    i32x4 acc[2][8];
#pragma unroll
    for (int m = 0; m < 2; ++m)
#pragma unroll
        for (int t = 0; t < 8; ++t) { acc[m][t] = (m ? XBIAS : -XBIAS) + i32x4{0, 0, 0, 0}; asm volatile("" : "+v"(acc[m][t])); }
    i32x2 B[2][16];
    i32x4 AL[2][2][2], A[NFRAG];
    Four X;
    if constexpr (LP0 == 0) {
        asm volatile("s_waitcnt vmcnt(0)" ::: "memory");
        read_local(AL[0], c);
        {
            LGKM(0);
            dma4(c.xl2, c.ll, c.sv[0], c.sv[1], c.sv[2], c.sv[3]);
        }
    }
    STAMP(1);
    unsigned call;
    while ((call = __builtin_amdgcn_readfirstlane(lds_ld(ctrl + C_CALL))) == 0) __builtin_amdgcn_s_sleep(1);
    c.call = call;
    c.sent = call <= 2 ? 0u : POISON;
    c.tag = call % 3 + 1;
    c.ro = ws + WS_RAW + (b * 4 + wave) * 8192 + 4096;
    c.sel[0] = 0x04020100; c.sel[1] = 0x05020100; c.sel[2] = 0x06020100;
    asm volatile("" : "+s"(c.ro), "+s"(c.sel[0]), "+s"(c.sel[1]), "+s"(c.sel[2]));
#pragma unroll
    for (int i = 0; i < 7; ++i) {
        c.ab[i] = ws + WS_DATA + (call % 3) * SLOT_BYTES + ((mi * 4 + wave) * 2 + P) * (NFRAG * 1024) + i * 8192 + 4096;
        asm volatile("" : "+s"(c.ab[i]));
    }
    c.xo = ws + WS_X + (b * 4 + wave) * XSTRIDE + 4096;
    asm volatile("" : "+s"(c.xo), "+s"(c.sent), "+s"(c.tag), "+s"(c.flags), "+s"(c.bad), "+v"(c.l16));
    c.hf = ctrl + C_H0 + wave * 4;
    c.h0 = c.ll + c.l16;
    c.h1 = lds0 + LDS_H1 + wave * 4096 + c.l16;
    const Out o = out_of(outp + (size_t)(unsigned)(((mi * 128 + wave * 32 + P * 16) * NN + n0 + 64) * 4), lds0 + LDS_H1 + wave * 4096, lane);
    asm volatile("" : "+v"(c.hf), "+v"(c.h0), "+v"(c.h1), "+v"(c.scale));
    check<1>(c);
    STAMP(3);
#pragma unroll
    for (int t = 0; t < 8; ++t) { read_b<0, 0>(B[0], t, c); read_b<0, 1>(B[0], t, c); }
    check<2>(c);
    step<0>(c, acc, AL, A, B, X STAMP_PASS);
    SB();
    STAMP(20);

    await4(X, c.hf + C_H1 - C_H0, c.h1);
    STAMP(21);
    finish(&acc[0][4], X.x, c.scale, o);
    if (c.bad && lane == 0) __hip_atomic_store((unsigned*)(a.peer[0].ws + WS_TMO), call, __ATOMIC_RELAXED, __HIP_MEMORY_SCOPE_AGENT);
    STAMP(23);
    STAMP(24);
    STAMP_OUT(ws);
    __builtin_amdgcn_endpgm();
    }
    NEXT_ROLE();
    if (wave == 6) {
        __builtin_amdgcn_s_setprio(3);
        u64x2 own = args_ld(0);
        const int pd = PDFIX == 3 ? (xcd - j) & 7 : PDFIX == 4 ? (xcd - (j & 3) - 4 * (j >> 3)) & 7 : PDFIX == 1 ? ((ct ^ 4) + 1) & 7 : PDFIX == 2 ? (ct == 3 ? 0 : xcd ? xcd : (3 - CMAP * (j >> 3)) & 7) : ct == 3 ? 0 : ((ct ^ 4) + j) % 7 + 1, pc = j >> 2, pm = (j - (QROT ? (NRANK - pd) >> 1 : 0)) & 3;
        u64x2 to = args_ld(pd * 16);
        unsigned sv[8];
#pragma unroll
        for (int g = 0; g < 8; ++g) {
            sv[g] = (pm * 128 + g * 16 + (lane >> 2)) * KLOC + ((pc + ROT * pm) & 7) * 64 + (((lane & 3) ^ ((lane >> 4) & 3)) << 4) - (g & 3) * 1024;
            asm volatile("" : "+v"(sv[g]));
        }
        asm volatile("s_waitcnt lgkmcnt(0)" : "+s"(own), "+s"(to) : "s"(pre) : "memory");
        char* const ws = (char*)own[0];
        char* const pw = (char*)to[0];
        const char* const base = (const char*)to[1];
        unsigned cv;
        asm volatile("s_load_dword %0, %2, %3\n\ts_memrealtime %1" : "=&s"(cv), "=&s"(t64) : "s"(ws), "i"(WS_TICKET) : "memory");
        if (pd && j < XJIT) {
            dma4(base, lds0 + LDS_PUSH, sv[0], sv[1], sv[2], sv[3]);
            dma4(base, lds0 + LDS_PUSH + 4096, sv[4], sv[5], sv[6], sv[7]);
        }
        int pd2 = pd;
        asm volatile("" : "+s"(pd2) :: "memory");
        const unsigned slot = PUSH0 + ((j < 4 * NFAST ? j * PACE0 : j * PACE - 4 * NFAST * (PACE - PACE0)) >> 2);
        unsigned doff[2], poff[2];
#pragma unroll
        for (int m = 0; m < 2; ++m) {
            doff[m] = lane * 16 + WS_DATA + frag_off(NRANK - pd2, pc, pm * 8 + m);
            asm volatile("" : "+v"(doff[m]));
        }
        const unsigned roff = lds0 + LDS_PUSH + (lane & 15) * 64 + (((lane >> 4) ^ ((lane >> 2) & 3)) << 4);
        asm volatile("s_waitcnt lgkmcnt(0)" : "+s"(cv), "+s"(t64) : "s"(slot), "v"(roff) : "memory");
        const unsigned due = (unsigned)t64 + slot;
        const unsigned call = (cv >> 8) + 1;
        lds_st1(ctrl + C_CALL, (int)call);
        const unsigned cur = (call % 3) * SLOT_BYTES, old = ((call + 2) % 3) * SLOT_BYTES;
        STAMP(12);
        if (pd) {
            char* dp[4];
#pragma unroll
            for (int w = 0; w < 4; ++w) { dp[w] = pw + cur + w * (2 * NFRAG * 1024); asm volatile("" : "+s"(dp[w])); }
            __builtin_amdgcn_s_setprio(0);
            if (j >= 20) touch_x(ws + WS_X + (b ^ 8) * 4 * XSTRIDE);
            if (j >= XJIT) {
                while ((int)(due - XAHEAD - (unsigned)__builtin_amdgcn_s_memrealtime()) > 0) __builtin_amdgcn_s_sleep(4);
                __builtin_amdgcn_s_setprio(3);
                dma4(base, lds0 + LDS_PUSH, sv[0], sv[1], sv[2], sv[3]);
                dma4(base, lds0 + LDS_PUSH + 4096, sv[4], sv[5], sv[6], sv[7]);
                __builtin_amdgcn_s_setprio(0);
            }
            if (PAUSE) {
                for (int left; (left = (int)(due - PAUSE - (unsigned)__builtin_amdgcn_s_memrealtime())) > 0;) {
                    if (left > 40) __builtin_amdgcn_s_sleep(8);
                    else __builtin_amdgcn_s_sleep(1);
                }
                lds_st1(ctrl + C_PAUSE, 1);
            }
            for (int left; (left = (int)(due - (unsigned)__builtin_amdgcn_s_memrealtime())) > 0;) {
                if (left > 40) __builtin_amdgcn_s_sleep(8);
                else __builtin_amdgcn_s_sleep(1);
            }
            __builtin_amdgcn_s_setprio(3);
            STAMP(2);
            asm volatile("s_waitcnt vmcnt(0)" ::: "memory");
            STAMP(11);
            i32x4 dat[8];
#pragma unroll
            for (int g = 0; g < 8; ++g) dat[g] = *(lds_i32x4_p)(roff + g * 1024);
#pragma unroll
            for (int g = 0; g < 8; ++g)
                asm volatile("global_store_dwordx4 %1, %0, %2 sc0 sc1\n\ts_nop 1" :: "v"(dat[g]), "v"(doff[g & 1]), "s"(dp[g >> 1]) : "memory");
            lds_st1(ctrl + C_PAUSE, 2);
            __builtin_amdgcn_s_setprio(0);
            STAMP(3);
            asm volatile("" : "+s"(pd2) :: "memory");
#pragma unroll
            for (int m = 0; m < 2; ++m) poff[m] = lane * 16 + WS_DATA + frag_off(pd2, pc, pm * 8 + m);
            if (!PEARLY) asm volatile("s_waitcnt vmcnt(0)" :: "v"(poff[0]), "v"(poff[1]) : "memory");
            STAMP(7);
            const i32x4 poison = {(int)POISON, (int)POISON, (int)POISON, (int)POISON};
#pragma unroll
            for (int g = 0; g < 8; ++g)
                asm volatile("global_store_dwordx4 %1, %0, %2 " POISON_BITS "\n\ts_nop 1"
                             :: "v"(poison), "v"(poff[g & 1]), "s"(ws + old + (g >> 1) * (2 * NFRAG * 1024)) : "memory");
            if (PEARLY) asm volatile("s_waitcnt vmcnt(0)" ::: "memory");
        }
        lds_st1(ctrl + C_PAUSE, 2);
        take_ticket(a, call, lane);
        if (j < 20 || !pd) touch_x(ws + WS_X + (b ^ 8) * 4 * XSTRIDE);
    }

    NEXT_ROLE();
    if (wave < 6) {
        l.w = (const char*)a.col[ct].w;
        l.fa = ctrl + C_FULL + (lane ? 64 + lane * 8 : 0) + (wave & 1) * 4;
        l.pa = ctrl + C_PROG + (lane & 3) * 260;
        const int r8 = a.rank * 8;
        l.woff = (((w_chunk(min(lane, NSTEP - 1), P, mi) + r8) & 63) - r8) * 64 * NN;
        l.freed = 0;
        l.pz = 0;
        if (wave & 1) fills<1, 1>(l STAMP_PASS);
        else fills<0, 1>(l STAMP_PASS);
        STAMP(20);
    }

    NEXT_ROLE();
    if (wave == 7) {
        const unsigned tstart = (unsigned)__builtin_amdgcn_s_memrealtime();
        char* const ws = a.peer[0].ws;
        STAMP(0);
        unsigned d[8];
        unsigned long long now;
        int due = 1000;
        allowed<NSTEP - 1>(due, lane);
        constexpr int NI = NSPLIT / 2 + NS2;
        const int q = min(lane, NI - 1);
        const int offs = (q < NSPLIT / 2 ? 2 * q + (j >> 4) : q + NSPLIT / 2) * 1024;
        unsigned call;
        while ((call = __builtin_amdgcn_readfirstlane(lds_ld(ctrl + C_CALL))) == 0) __builtin_amdgcn_s_sleep(1);
        const unsigned sent = call <= 2 ? 0u : POISON;
        const char* af = ws + WS_DATA + (call % 3) * SLOT_BYTES + ((mi * 4 + 2 + ((j >> 3) & 1)) * 2 + P) * (NFRAG * 1024) + xcd * 128 + 112;
        int n = 0;
        STAMP(1);
        const unsigned pa = ctrl + C_PROG + (lane & 3) * 260;
        while (n < NSPLIT / 2) {
            if (WREST) while (n > released(pa) + WREST && (unsigned)__builtin_amdgcn_s_memrealtime() - tstart < WATCH_TICKS) __builtin_amdgcn_s_sleep(4);
            unsigned o[4];
#pragma unroll
            for (int i = 0; i < 4; ++i) o[i] = __builtin_amdgcn_readlane(offs, n + i);
            do {
                asm volatile("s_load_dword %0, %5, %6 glc\n\ts_load_dword %1, %5, %7 glc\n\t"
                             "s_load_dword %2, %5, %8 glc\n\ts_load_dword %3, %5, %9 glc\n\t"
                             "s_memrealtime %4\n\ts_waitcnt lgkmcnt(0)"
                             : "=&s"(d[0]), "=&s"(d[1]), "=&s"(d[2]), "=&s"(d[3]), "=&s"(now)
                             : "s"(af), "s"(o[0]), "s"(o[1]), "s"(o[2]), "s"(o[3]) : "memory");
            } while (d[0] == sent && (unsigned)now - tstart < WATCH_TICKS);
            n += d[1] == sent ? 1 : d[2] == sent ? 2 : d[3] == sent ? 3 : 4;
            lds_st2(ctrl + C_DUE, __builtin_amdgcn_readlane(due, n + (n >= 4 ? LEAD : 0)), n);
        }
        while (n + 2 * LEAD < NI) {
            if (WREST) while (n > 2 * released(pa) - NSPLIT / 2 + 2 * WREST && (unsigned)__builtin_amdgcn_s_memrealtime() - tstart < WATCH_TICKS) __builtin_amdgcn_s_sleep(4);
            const char* p = af + (n + NSPLIT / 2) * 1024;
            do {
                asm volatile("s_load_dword %0, %9, 0 glc\n\ts_load_dword %1, %9, 0x400 glc\n\ts_load_dword %2, %9, 0x800 glc\n\t"
                             "s_load_dword %3, %9, 0xc00 glc\n\ts_load_dword %4, %9, 0x1000 glc\n\ts_load_dword %5, %9, 0x1400 glc\n\t"
                             "s_load_dword %6, %9, 0x1800 glc\n\ts_load_dword %7, %9, 0x1c00 glc\n\t"
                             "s_memrealtime %8\n\ts_waitcnt lgkmcnt(0)"
                             : "=&s"(d[0]), "=&s"(d[1]), "=&s"(d[2]), "=&s"(d[3]), "=&s"(d[4]), "=&s"(d[5]), "=&s"(d[6]), "=&s"(d[7]), "=&s"(now)
                             : "s"(p) : "memory");
            } while (d[0] == sent && (unsigned)now - tstart < WATCH_TICKS);
            n += d[1] == sent ? 1 : d[2] == sent ? 2 : d[3] == sent ? 3 : d[4] == sent ? 4 : d[5] == sent ? 5 : d[6] == sent ? 6 : d[7] == sent ? 7 : 8;
            lds_st1(ctrl + C_DUE, __builtin_amdgcn_readlane(due, n + 2 * LEAD));
        }
        lds_st2(ctrl + C_DUE, 1000, 1000);
        STAMP(7);
    }
    NEXT_ROLE();
    help(a, wave & 3, b, lane STAMP_PASS);
    STAMP_OUT(a.peer[0].ws);
}

extern "C" void kernel_launch(int rank, int nranks, ncclComm_t comm,
                              hipStream_t stream,
                              void* const* d_in, const int* in_sizes, int n_in,
                              void* d_out, int out_size,
                              const int* in_dtypes, int out_dtype,
                              void* const* d_out_all,
                              void* const* d_ws_all, size_t ws_bytes) {
    if (nranks != NRANK || (unsigned)rank >= (unsigned)NRANK || ws_bytes < WS_NEEDED || n_in < 4 ||
        out_size != MLOC * NN || in_sizes[0] != NRANK * MLOC * KLOC || in_sizes[1] != NRANK * KLOC * NN) {
        fprintf(stderr, "unexpected problem shape\n");
        return;
    }
    static bool attr_set = false;
    if (!attr_set) {
        HIP_CHECK(hipFuncSetAttribute((const void*)a2a_gemm_kernel, hipFuncAttributeMaxDynamicSharedMemorySize,
                                      (int)LDS_BYTES));
        attr_set = true;
    }
    Args a;
    for (int i = 0; i < NRANK; ++i) {
        const int p = (rank + i) & (NRANK - 1), c0 = ((i - (int)((uintptr_t)d_in[1] >> 7)) & 7) * 128;
        a.peer[i].ws = (char*)d_ws_all[p];
        a.peer[i].x = (const int8_t*)d_in[0] + (size_t)p * MLOC * KLOC;
        a.col[i].w = (const int8_t*)d_in[1] + (size_t)rank * KLOC * NN + c0;
        a.col[i].out = (float*)d_out + c0;
    }
    a.sx = (const float*)d_in[2];
    a.sw = (const float*)d_in[3];
    a.rank = rank;
    a.pad = 0;
    hipLaunchKernelGGL(a2a_gemm_kernel, dim3(NWG), dim3(NTHREADS), LDS_BYTES, stream, a);
    HIP_CHECK(hipGetLastError());
}
```

```cpp
#include <hip/hip_runtime.h>
#include <rccl/rccl.h>
#include <cstdint>
#include <cstdio>

#define HIP_CHECK(expr) do { hipError_t _e = (expr); if (_e != hipSuccess) { \
    fprintf(stderr, "HIP error %s at %s:%d\n", hipGetErrorString(_e), __FILE__, __LINE__); \
    return; } } while (0)

typedef int i32x2 __attribute__((ext_vector_type(2)));
typedef int i32x4 __attribute__((ext_vector_type(4)));
typedef float f32x4 __attribute__((ext_vector_type(4)));
typedef __attribute__((address_space(3))) i32x2* lds_i32x2_p;
typedef __attribute__((address_space(3))) i32x4* lds_i32x4_p;
typedef __attribute__((address_space(3))) int* lds_int_p;
typedef __attribute__((address_space(1))) unsigned* gu32_p;
typedef volatile __attribute__((address_space(3))) int* lds_vint_p;
typedef volatile __attribute__((address_space(3))) i32x4* lds_vi32x4_p;

namespace {

constexpr int NRANK = 8;
constexpr int NN = 4096;
constexpr int KLOC = 512;
constexpr int MLOC = 512;
constexpr int NWG = 256;
constexpr int NTHREADS = 512;

constexpr int NSPLIT = 44;
constexpr int PUSH0 = 50;
constexpr int PACE0 = 58;
constexpr int PACE = 64;
constexpr int PAUSE = 45;
constexpr int XJIT = 8;
constexpr int XAHEAD = 150;
constexpr int ADIST = 6;
constexpr int FIRST_AT = 3;
constexpr int T2AT = 2, T2PER = 4;
constexpr int LDW = 3;
constexpr int LDW_FROM = 14;
constexpr int XSTRIDE = 7168;
constexpr int NS1 = 4 + NSPLIT / 2, NS2 = 56 - NSPLIT, NSTEP = NS1 + NS2;
constexpr int NFRAG = 56;
static_assert(NSPLIT % 4 == 0 && NSTEP <= 64 && (NSTEP & 15) != 15, "");

constexpr unsigned WS_TMO = 0;
constexpr unsigned WS_TICKET = 4096;
constexpr unsigned WS_FLAGS = 8192;
constexpr unsigned WS_DATA = 2u << 20;
constexpr unsigned SLOT_BYTES = 2u << 20;
constexpr unsigned WS_X = WS_DATA + 3u * SLOT_BYTES;
constexpr unsigned WS_RAW = WS_X + NWG * 4u * XSTRIDE;
constexpr size_t WS_NEEDED = (size_t)WS_RAW + NWG * 4u * 8192;
constexpr int XBIAS = 1 << 21;
constexpr unsigned POISON = 0x80808080u;

constexpr unsigned LDS_PUSH = 16 * 8192;
constexpr unsigned LDS_LOC = LDS_PUSH + 8192;
constexpr unsigned LDS_H1 = (NSTEP & 15) * 8192;
constexpr unsigned LDS_CTRL = LDS_LOC + 16384;
constexpr unsigned C_FULL = 0;
constexpr unsigned C_DUE = 8;
constexpr unsigned C_CNT = 12;
constexpr unsigned C_CALL = 16;
constexpr unsigned C_PAUSE = 20;
constexpr unsigned C_H0 = 32;
constexpr unsigned C_H1 = 48;
constexpr unsigned C_PROG = 1024;
constexpr unsigned LDS_BYTES = LDS_CTRL + 2048 + 256;

constexpr unsigned WATCH_TICKS = 3000;
constexpr unsigned DEADLINE_TICKS = 50000000u;

struct Args {
    struct { char* ws; const int8_t* x; } peer[NRANK];
    struct { const int8_t* w; float* out; } col[NRANK];
    const float* sx;
    const float* sw;
    int rank;
    int pad;
};
typedef unsigned long long u64;
typedef u64 u64x2 __attribute__((ext_vector_type(2)));
__device__ __forceinline__ u64x2 args_ld(unsigned off) {
    u64x2 r;
    asm volatile("s_load_dwordx4 %0, %1, %2" : "=s"(r) : "s"(__builtin_amdgcn_kernarg_segment_ptr()), "s"(off) : "memory");
    return r;
}

#define SB() __builtin_amdgcn_sched_barrier(0)
#define LGKM(n) __builtin_amdgcn_s_waitcnt(0xC07F | ((n) << 8))

__device__ __forceinline__ void dma4(const char* gbase, unsigned lds_addr,
                                     unsigned v0, unsigned v1, unsigned v2, unsigned v3) {
    asm volatile(
        "s_mov_b32 m0, %5\n\t"
        "s_nop 0\n\t"
        "global_load_lds_dwordx4 %0, %4\n\t"
        "global_load_lds_dwordx4 %1, %4 offset:1024\n\t"
        "global_load_lds_dwordx4 %2, %4 offset:2048\n\t"
        "global_load_lds_dwordx4 %3, %4 offset:3072"
        :: "v"(v0), "v"(v1), "v"(v2), "v"(v3), "s"(gbase), "s"(lds_addr) : "memory");
}
template <int N> __device__ __forceinline__ void wait_vm(i32x4& a, i32x4& b) {
    asm volatile("s_waitcnt vmcnt(%2)" : "+v"(a), "+v"(b) : "i"(N) : "memory");
}
__device__ __forceinline__ int lds_ld(unsigned addr) {
    return __hip_atomic_load((lds_int_p)addr, __ATOMIC_RELAXED, __HIP_MEMORY_SCOPE_WORKGROUP);
}
__device__ __forceinline__ void lds_st(unsigned addr, int v) {
    __hip_atomic_store((lds_int_p)addr, v, __ATOMIC_RELAXED, __HIP_MEMORY_SCOPE_WORKGROUP);
}
__device__ __forceinline__ void lds_st1(unsigned addr, int v) {
    asm volatile("s_mov_b64 exec, 1\n\tds_write_b32 %0, %1\n\ts_mov_b64 exec, -1" :: "v"(addr), "v"(v) : "memory");
}
__device__ __forceinline__ void lds_st2(unsigned addr, int v0, int v1) {
    asm volatile("s_mov_b64 exec, 1\n\tds_write_b64 %0, %1\n\ts_mov_b64 exec, -1" :: "v"(addr), "v"(i32x2{v0, v1}) : "memory");
}

constexpr int imin(int x, int y) { return x < y ? x : y; }
constexpr int imax(int x, int y) { return x > y ? x : y; }
constexpr int frag_lo(int s) {
    return s >= NS1 ? NSPLIT + s - NS1 : 2 * imax(0, s - 4);
}
__device__ __forceinline__ int pos_of(int rel, int mi) {
    const int v = (rel + 2 * mi) & 7;
    return v - (v > 2 * mi ? 1 : 0);
}
__device__ __forceinline__ int rel_of(int pos, int mi) {
    return (pos + (pos >= 2 * mi ? 1 : 0) - 2 * mi) & 7;
}
__device__ __forceinline__ unsigned frag_off(int rel, int cls, int rg) {
    const int i = 7 * cls + pos_of(rel, rg >> 3), m = rg & 1;
    const int P = i < NSPLIT ? i & 1 : m, frag = i < NSPLIT ? (i & ~1) + (m ^ P) : i;
    return (((rg >> 1) * 2 + P) * NFRAG + frag) * 1024;
}

struct Ctx {
    const char* ab[7];
    char* xo;
    const char* xl2;
    unsigned sv[4];
    unsigned ll;
    unsigned la;
    unsigned sent;
    unsigned tag;
    char* ro;
    unsigned sel[3];
    unsigned any, raw[2];
    float scale;
    unsigned hf, h0, h1;
    unsigned rb[2][8];
    unsigned l16;
    unsigned chk;
    const char* flags;
    unsigned foff;
    unsigned call, t0;
    unsigned bad;
    unsigned prog;
    int one;
    int seen;
};

__device__ __forceinline__ void read_local(i32x4 (&AL)[2][2], const Ctx& c) {
#pragma unroll
    for (int s = 0; s < 2; ++s) {
        AL[s][0] = *(lds_i32x4_p)(c.la ^ (s * 64));
        AL[s][1] = *(lds_i32x4_p)(c.la ^ (s * 64) ^ 2048);
    }
}
template <int S, int H>
__device__ __forceinline__ void read_b(i32x2 (&B)[16], int t, const Ctx& c) {
    B[2 * t + H] = __builtin_amdgcn_ds_read_tr8_b64_v2i32((lds_i32x2_p)(c.rb[(S >> 3) & 1][t] + (S & 7) * 8192 + H * 1024));
}
__device__ __forceinline__ void mma(i32x4& acc, const i32x4& a, const i32x2 (&B)[16], int t) {
    asm volatile("v_mfma_i32_16x16x64_i8 %0, %1, %2, %0"
                 : "+v"(acc) : "v"(__builtin_shufflevector(B[2 * t], B[2 * t + 1], 0, 1, 2, 3)), "v"(a));
}
template <int F>
__device__ __forceinline__ void load_frag(i32x4 (&A)[NFRAG], const Ctx& c) {
    asm volatile("global_load_dwordx4 %0, %1, %2 offset:%3 sc0 sc1"
                 : "=v"(A[F]) : "v"(c.l16), "s"(c.ab[F / 8]), "i"((F % 8) * 1024 - 4096) : "memory");
}
template <int F, int END>
__device__ __forceinline__ void first_frags(i32x4 (&A)[NFRAG], const Ctx& c, int cnt) {
    if constexpr (F < END) {
        asm volatile(
            "v_cmp_gt_i32 vcc, %[need], %[cnt]\n\t"
            "s_cbranch_vccz .Lgo%=\n"
            ".Lagain%=:\n\t"
            "ds_read_b32 %[cnt], %[at]\n\t"
            "s_waitcnt lgkmcnt(0)\n\t"
            "v_cmp_gt_i32 vcc, %[need], %[cnt]\n\t"
            "s_cbranch_vccnz .Lagain%=\n"
            ".Lgo%=:"
            : [cnt] "+v"(cnt) : [at] "v"(LDS_CTRL + C_CNT), [need] "i"(F / 2 + 1) : "memory", "vcc");
        load_frag<F>(A, c);
        load_frag<F + 1>(A, c);
        first_frags<F + 2, END>(A, c, cnt);
    }
}
template <int FULL>
__device__ __forceinline__ void check(const Ctx& c) {
    while (__any(__hip_atomic_load((lds_int_p)c.chk, __ATOMIC_ACQUIRE, __HIP_MEMORY_SCOPE_WORKGROUP) < FULL))
        __builtin_amdgcn_s_sleep(1);
    asm volatile("" ::: "memory");
}

#define CP_WAIT \
        "s_or_b64 vcc, %[low], %[miss]\n\t" \
        "s_cbranch_scc1 .Lslow%=\n" \
        ".Lback%=:\n\t" \
        ".subsection 1\n" \
        ".Lagain%=:\n\t" \
        "s_sleep 1\n" \
        ".Lslow%=:\n\t" \
        "ds_read_b32 %[p0], %[chk]\n\t" \
        "s_waitcnt lgkmcnt(0)\n\t" \
        "v_cmp_gt_i32 vcc, %[full], %[p0]\n\t" \
        "s_cbranch_vccnz .Lagain%=\n\t"
template <int FULL>
__device__ __forceinline__ void checkpoint(Ctx& c, unsigned long long low) {
    unsigned p0;
    asm volatile(CP_WAIT "s_branch .Lback%=\n\t.subsection 0"
                 : [p0] "=&v"(p0) : [low] "s"(low), [miss] "i"(0), [chk] "v"(c.chk), [full] "i"(FULL) : "memory", "vcc", "scc");
}
template <int FULL, int F>
__device__ __forceinline__ void checkpoint(i32x4 (&A)[NFRAG], Ctx& c, unsigned long long low, unsigned long long miss) {
    unsigned p0, p1, fl, prev;
    asm volatile(
        CP_WAIT
        "s_cmp_eq_u64 %[miss], 0\n\t"
        "s_cbranch_scc1 .Lback%=\n\t"
        "s_mov_b32 %[prev], 0\n"
        ".Lretry%=:\n\t"
        "global_load_dword %[p0], %[l16], %[ab] offset:%[o0] sc0 sc1\n\t"
        "global_load_dword %[p1], %[l16], %[ab] offset:%[o1] sc0 sc1\n\t"
        "global_load_dwordx4 %[a0], %[l16], %[ab] offset:%[o0] sc0 sc1\n\t"
        "global_load_dwordx4 %[a1], %[l16], %[ab] offset:%[o1] sc0 sc1\n\t"
        "global_load_dword %[fl], %[foff], %[flags] sc0 sc1\n\t"
        "s_memrealtime vcc\n\t"
        "s_waitcnt vmcnt(0) lgkmcnt(0)\n\t"
        "s_cmp_lg_u32 %[prev], 0\n\t"
        "s_cbranch_scc1 .Lback%=\n\t"
        "s_sub_u32 %[prev], vcc_lo, %[t0]\n\t"
        "v_cmp_eq_u32 vcc, %[sent], %[p0]\n\t"
        "v_cmp_eq_u32 %[miss], %[sent], %[p1]\n\t"
        "s_or_b64 vcc, vcc, %[miss]\n\t"
        "s_cbranch_scc0 .Lback%=\n\t"
        "s_cmp_gt_u32 %[prev], %[dl]\n\t"
        "s_cselect_b32 %[bad], 1, %[bad]\n\t"
        "s_cbranch_scc1 .Lback%=\n\t"
        "v_subrev_u32 %[fl], %[call], %[fl]\n\t"
        "v_cmp_gt_i32 vcc, 0, %[fl]\n\t"
        "s_cmp_eq_u64 vcc, 0\n\t"
        "s_cselect_b32 %[prev], 1, 0\n\t"
        "s_branch .Lretry%=\n\t"
        ".subsection 0"
        : [a0] "+v"(A[F]), [a1] "+v"(A[F + 1]), [p0] "=&v"(p0), [p1] "=&v"(p1), [fl] "=&v"(fl), [prev] "=&s"(prev),
          [bad] "+s"(c.bad), [miss] "+s"(miss)
        : [low] "s"(low), [chk] "v"(c.chk), [full] "i"(FULL), [l16] "v"(c.l16), [ab] "s"(c.ab[F / 8]), [foff] "v"(c.foff), [flags] "s"(c.flags),
          [o0] "i"((F % 8) * 1024 - 4096), [o1] "i"((F % 8) * 1024 - 3072), [sent] "s"(c.sent), [call] "s"(c.call),
          [t0] "s"(c.t0), [dl] "i"(DEADLINE_TICKS)
        : "memory", "vcc", "scc");
}

static_assert(2 * T2AT <= ADIST && T2PER <= 4 && ADIST % 2 == 0 && FIRST_AT >= 2 && FIRST_AT <= 3, "");
constexpr int issued_before(int s) {
    if (s <= NS1 - T2AT) return s < FIRST_AT ? 0 : imin(NSPLIT, frag_lo(s) + ADIST);
    const int ramp = NSPLIT + T2PER * (s - NS1 + T2AT);
    return imin(NFRAG, s <= NS1 ? ramp : imin(ramp, imax(NSPLIT + T2PER * T2AT, frag_lo(s) + ADIST)));
}
constexpr int span(int s) {
    return s + 1 >= NSTEP ? 0 : s + 1 < NS1 ? 1 : (s + 1 - NS1) % 2 == 0 ? 2 : 0;
}
constexpr int ops_before_wait(int s) {
    return issued_before(s + 1) + (s >= NS1 - 1 ? 6 : 0);
}
constexpr int ops_through_frag(int f) {
    return f + 1 + (f >= issued_before(NS1) ? 6 : 0);
}

struct Four { int f; i32x4 x[4]; };
__device__ __forceinline__ Four fetch4(unsigned flag, unsigned at) {
    Four r;
    r.f = *(lds_vint_p)flag;
#pragma unroll
    for (int t = 0; t < 4; ++t) r.x[t] = *(lds_vi32x4_p)(at + t * 1024);
    return r;
}
__device__ __forceinline__ void await4(Four& r, unsigned flag, unsigned at) {
    asm volatile(
        "v_cmp_ne_u32 vcc, 0, %0\n\t"
        "s_cbranch_vccz .Lagain%=\n"
        ".Lgo%=:\n\t"
        ".subsection 1\n"
        ".Lagain%=:\n\t"
        "ds_read_b32 %0, %5\n\t"
        "s_waitcnt lgkmcnt(0)\n\t"
        "v_cmp_ne_u32 vcc, 0, %0\n\t"
        "s_cbranch_vccz .Lagain%=\n\t"
        "ds_read_b128 %1, %6\n\t"
        "ds_read_b128 %2, %6 offset:1024\n\t"
        "ds_read_b128 %3, %6 offset:2048\n\t"
        "ds_read_b128 %4, %6 offset:3072\n\t"
        "s_waitcnt lgkmcnt(0)\n\t"
        "s_branch .Lgo%=\n\t"
        ".subsection 0"
        : "+v"(r.f), "+v"(r.x[0]), "+v"(r.x[1]), "+v"(r.x[2]), "+v"(r.x[3]) : "v"(flag), "v"(at) : "memory", "vcc");
}
struct Out { unsigned tw, tw1, tr, oo; char *o0, *o1; };
__device__ __forceinline__ Out out_of(char* rows, unsigned lds, int lane) {
    Out o;
    o.tw = lds + (lane & 15) * 128 + (((lane >> 4) ^ (lane & 7)) << 4);
    o.tw1 = o.tw ^ 64;
    o.tr = lds + (lane >> 3) * 128 + (((lane & 7) ^ (lane >> 3)) << 4);
    o.oo = (lane >> 3) * NN * 4 + (lane & 7) * 16;
    o.o0 = rows;
    o.o1 = rows + 8 * NN * 4;
    asm volatile("" : "+v"(o.tw), "+v"(o.tw1), "+v"(o.tr), "+v"(o.oo), "+s"(o.o0), "+s"(o.o1));
    return o;
}
__device__ __forceinline__ void finish(const i32x4* u, const i32x4* v, float scale, const Out& o) {
    i32x4 Z[4];
#pragma unroll
    for (int p = 0; p < 2; ++p) {
#pragma unroll
        for (int h = 0; h < 2; ++h) {
            const i32x4 s = u[2 * p + h] + v[2 * p + h];
            const f32x4 y = {(float)s[0] * scale, (float)s[1] * scale, (float)s[2] * scale, (float)s[3] * scale};
            *(lds_vi32x4_p)((h ? o.tw1 : o.tw) + p * 2048) = __builtin_bit_cast(i32x4, y);
        }
        Z[2 * p] = *(lds_vi32x4_p)(o.tr + p * 2048);
        Z[2 * p + 1] = *(lds_vi32x4_p)(o.tr + p * 2048 + 1024);
        SB();
    }
#pragma unroll
    for (int p = 0; p < 2; ++p)
        asm volatile("global_store_dwordx4 %2, %0, %3 offset:%5 sc1\n\t"
                     "global_store_dwordx4 %2, %1, %4 offset:%5 sc1\n\ts_nop 1"
                     :: "v"(Z[2 * p]), "v"(Z[2 * p + 1]), "v"(o.oo), "s"(o.o0), "s"(o.o1), "i"(p * 128) : "memory");
}
constexpr int XLOOK = 6144 - 16;
__device__ __forceinline__ void touch_x(const char* xi) {
    unsigned w;
    asm volatile("s_load_dword %0, %1, 0\n\ts_load_dword %0, %1, %2\n\ts_load_dword %0, %1, %3\n\ts_load_dword %0, %1, %4\n\ts_waitcnt lgkmcnt(0)"
                 : "=&s"(w) : "s"(xi + XLOOK), "i"(XSTRIDE), "i"(2 * XSTRIDE), "i"(3 * XSTRIDE) : "memory");
}
__device__ __forceinline__ unsigned long long or64(unsigned long long x, unsigned long long y) {
    unsigned long long r;
    asm volatile("s_or_b64 %0, %1, %2" : "=s"(r) : "s"(x), "s"(y) : "scc");
    return r;
}

template <int Q>
__device__ __forceinline__ void piece(Ctx& c, i32x4 (&T)[8], int part) {
    constexpr int U = Q / 3, KK = Q % 3;
    i32x4& t = T[4 * U + KK];
    if (part == 0) {
        t[0] = (t[0] << 2) | c.tag;
        if (KK == 0) t[1] |= c.raw[U];
        t[0] = __builtin_amdgcn_perm(T[4 * U + 3][0], t[0], c.sel[KK]);
    } else if (part == 1) {
#pragma unroll
        for (int e = 1; e < 4; ++e) t[e] = __builtin_amdgcn_perm(T[4 * U + 3][e], t[e], c.sel[KK]);
    } else {
        asm volatile("global_store_dwordx4 %1, %0, %2 offset:%3 sc0 sc1\n\ts_nop 1" :: "v"(t), "v"(c.l16), "s"(c.xo), "i"(Q * 1024 - 4096) : "memory");
    }
}
template <int U>
__device__ __forceinline__ void fits(Ctx& c, i32x4 (&T)[8], int part) {
    if (part < 4) {
        const i32x4& t = T[4 * U + part];
        c.any = (part ? c.any : 0u) | t[0] | t[1] | t[2] | t[3];
        asm volatile("" : "+v"(c.any));
    } else {
        asm volatile(
            "s_mov_b32 %0, 0\n\t"
            "v_cmp_le_u32 vcc, %8, %1\n\t"
            "s_cbranch_vccnz .Lraw%=\n"
            ".Lfit%=:\n\t"
            ".subsection 1\n"
            ".Lraw%=:\n\t"
            "global_store_dwordx4 %6, %2, %7 offset:%9 sc0 sc1\n\t"
            "global_store_dwordx4 %6, %3, %7 offset:%9+1024 sc0 sc1\n\t"
            "global_store_dwordx4 %6, %4, %7 offset:%9+2048 sc0 sc1\n\t"
            "global_store_dwordx4 %6, %5, %7 offset:%9+3072 sc0 sc1\n\t"
            "s_mov_b32 %0, 0x800000\n\t"
            "s_waitcnt vmcnt(0)\n\t"
            "s_branch .Lfit%=\n\t"
            ".subsection 0"
            : "=&s"(c.raw[U]) : "v"(c.any), "v"(T[4 * U]), "v"(T[4 * U + 1]), "v"(T[4 * U + 2]), "v"(T[4 * U + 3]),
              "v"(c.l16), "s"(c.ro), "i"(2 * XBIAS), "i"(U * 4096 - 4096) : "memory", "vcc");
    }
}

template <int S>
__device__ __forceinline__ void step(Ctx& c, i32x4 (&acc)[2][8], i32x4 (&AL)[2][2][2], i32x4 (&A)[NFRAG],
                                     i32x2 (&B)[2][16], Four& X) {
    constexpr bool PH1 = S < NS1, LOC = S < 4, NEXT = S + 1 < NSTEP;
    constexpr int SPAN = span(S), FULL = imin(S + SPAN + 2, NSTEP);
    constexpr bool HAND = S == NS1 - 1;
    constexpr bool CP = SPAN > 0 && !HAND, VAL = CP && S + 1 >= 4;
    constexpr bool FREE = (S & 1) == 1 && S + 13 < NSTEP;
    constexpr int F = frag_lo(S), FN = frag_lo(S + 1);
    constexpr int L0 = issued_before(S), L1 = S + 1 == FIRST_AT ? L0 : issued_before(S + 1);
    unsigned long long m0 = 0, m1 = 0, m2 = 0;
    if constexpr (S == FIRST_AT) first_frags<0, ADIST>(A, c, 0);
    const i32x4& a0 = LOC ? AL[LOC ? S / 2 : 0][S & 1][0] : A[F];
    const i32x4& a1 = LOC ? AL[LOC ? S / 2 : 0][S & 1][1] : A[PH1 ? F + 1 : F];
    LGKM(S == 2 ? 6 : 8);
    SB();
#pragma unroll
    for (int g = 0; g < 16; ++g) {
        const int t = g >> 1;
        if (g == 8) { if constexpr (NEXT) LGKM(8 + FREE); else LGKM(6); SB(); }
        if constexpr (HAND) {
            const int u = (g >> 3) * 4 + (g & 3);
            if (g & 4) mma(acc[0][u], a0, B[S & 1], u);
            else mma(acc[1][u], a1, B[S & 1], u);
            SB();
        } else if ((g & 1) == 0) { mma(acc[0][t], a0, B[S & 1], t); SB(); }
        else if constexpr (PH1) { mma(acc[1][t], a1, B[S & 1], t); SB(); }
        if (CP && g == 0) { c.seen = lds_ld(c.chk); SB(); }
        if constexpr (NEXT) {
            if (g & 1) read_b<S + 1, 1>(B[(S + 1) & 1], t, c);
            else if (g != 14) read_b<S + 1, 0>(B[(S + 1) & 1], t, c);
            if (g == 13) read_b<S + 1, 0>(B[(S + 1) & 1], 7, c);
        }
        if constexpr (!NEXT) {
            if (g == 0) X = fetch4(c.hf + C_H1 - C_H0, c.h1);
            if (g == 2) { if (c.bad) c.scale = __builtin_nanf(""); asm volatile("" : "+v"(c.scale)); }
            if (g >= 6 && g <= 12 && (g & 1) == 0) *(lds_vi32x4_p)(c.h0 + (g - 6) * 512) = acc[0][(g - 6) / 2];
            if (g == 12) lds_st1(c.hf, c.one);
        }
        if (g == 1) { if constexpr (L0 < L1) load_frag<L0>(A, c); }
        if (FREE && g == 2) __hip_atomic_fetch_add((lds_int_p)c.prog, c.one, __ATOMIC_RELAXED, __HIP_MEMORY_SCOPE_WORKGROUP);
        if (g == 3) { if constexpr (L0 + 1 < L1) load_frag<L0 + 1>(A, c); }
        if (g == 4) { if constexpr (L0 + 2 < L1) load_frag<L0 + 2>(A, c); }
        if (g == 5) { if constexpr (L0 + 3 < L1) load_frag<L0 + 3>(A, c); }
        if (CP && g == 8) m2 = __builtin_amdgcn_sicmp(c.seen, FULL, 40);
        if constexpr (VAL) {
            if (g == 9) wait_vm<ops_before_wait(S) - ops_through_frag(FN + 1)>(A[FN], A[FN + 1]);
            if (g == 10) m0 = __builtin_amdgcn_uicmp(A[FN][0], c.sent, 32);
            if (g == 11) m1 = __builtin_amdgcn_uicmp(A[FN + 1][0], c.sent, 32);
            if (g == 12) m0 = or64(m0, m1);
            if (g == 14) checkpoint<FULL, FN>(A, c, m2, m0);
        } else if constexpr (CP) {
            if (g == 14) checkpoint<FULL>(c, m2);
        }
        if (S == 1 && g == 9) {
            asm volatile("s_waitcnt vmcnt(0)" ::: "memory");
            read_local(AL[1], c);
        }
        if constexpr (HAND) {
            if (g >= 3 && g <= 7) fits<0>(c, acc[1], g - 3);
            if (g >= 8 && g <= 10) piece<0>(c, acc[1], g - 8);
            if (g >= 10 && g <= 12) piece<1>(c, acc[1], g - 10);
            if (g >= 12 && g <= 14) piece<2>(c, acc[1], g - 12);
            if (g == 14) { fits<1>(c, acc[1], 0); fits<1>(c, acc[1], 1); }
            if (g == 15) { fits<1>(c, acc[1], 2); fits<1>(c, acc[1], 3); fits<1>(c, acc[1], 4); }
        }
        SB();
    }
    if constexpr (HAND) {
#pragma unroll
        for (int i = 0; i < 3; ++i) piece<3>(c, acc[1], i);
        c.seen = lds_ld(c.chk);
#pragma unroll
        for (int i = 0; i < 3; ++i) piece<4>(c, acc[1], i);
#pragma unroll
        for (int i = 0; i < 3; ++i) piece<5>(c, acc[1], i);
        m2 = __builtin_amdgcn_sicmp(c.seen, FULL, 40);
        wait_vm<ops_before_wait(S) - ops_through_frag(FN + 1)>(A[FN], A[FN + 1]);
        m0 = or64(__builtin_amdgcn_uicmp(A[FN][0], c.sent, 32), __builtin_amdgcn_uicmp(A[FN + 1][0], c.sent, 32));
        checkpoint<FULL, FN>(A, c, m2, m0);
        SB();
    }
    if constexpr (NEXT) step<S + 1>(c, acc, AL, A, B, X);
}

template <int S>
__device__ __forceinline__ void allowed(int& due, int n) {
    if constexpr (span(S) > 0) {
        constexpr int F = issued_before(S + span(S) + 1), NEED = F <= NSPLIT ? (F + 1) / 2 : F - NSPLIT / 2;
        if (n < NEED) due = imin(S + span(S) + 2, NSTEP) - 1;
    }
    if constexpr (S > FIRST_AT) allowed<S - 1>(due, n);
}

constexpr int fill_step(int h, int k) { return imin(NSTEP, 2 * k + h); }
__device__ __forceinline__ int w_chunk(int s, int P, int mi) {
    const int i = max(s < NS1 ? 2 * s - 8 + P : NSPLIT + s - NS1, 0), cls = (i * 37) >> 8;
    const int rem = rel_of(i - 7 * cls, mi) * 8 + ((cls + 4 * mi) & 7);
    return s < 4 ? 4 * P + (s ^ mi) : rem;
}
struct Ldr {
    const char* w;
    unsigned dv[4];
    int woff;
    unsigned fa, pa;
    int freed;
    int pz;
};
__device__ __forceinline__ void fill(const Ldr& l, int off, unsigned slot) {
    const char* p = l.w + off;
    dma4(p, slot, l.dv[0], l.dv[1], l.dv[2], l.dv[3]);
    dma4(p + 32 * NN, slot + 4096, l.dv[0], l.dv[1], l.dv[2], l.dv[3]);
}
__device__ __forceinline__ int released(unsigned pa) {
    int v = lds_ld(pa);
    v = min(v, __builtin_amdgcn_update_dpp(v, v, 0xB1, 0xF, 0xF, false));
    v = min(v, __builtin_amdgcn_update_dpp(v, v, 0x4E, 0xF, 0xF, false));
    return 2 * __builtin_amdgcn_readfirstlane(v);
}
template <int H, int K>
__device__ __forceinline__ void fills(Ldr& l) {
    constexpr int S = fill_step(H, K);
    if constexpr (S < NSTEP) {
        if constexpr (S >= 16) {
            if (__builtin_expect(l.freed <= S - 16, 0)) {
                while ((l.freed = released(l.pa)) <= S - 16) __builtin_amdgcn_s_sleep(2);
            }
        }
        if constexpr (K > 1 && K < 12) {
            if (l.pz != 2) while ((l.pz = __builtin_amdgcn_readfirstlane(lds_ld(LDS_CTRL + C_PAUSE))) == 1) __builtin_amdgcn_s_sleep(1);
        }
        if constexpr (K == LDW_FROM) __builtin_amdgcn_s_setprio(2);
        if constexpr (K > 1) fill(l, __builtin_amdgcn_readlane(l.woff, S), (S & 15) * 8192);
        constexpr int W = imin(LDW, imax(1, K - LDW_FROM + 2));
        asm volatile("s_waitcnt vmcnt(%2)\n\tds_max_i32 %0, %1" :: "v"(l.fa), "v"(fill_step(H, K - W + 1)), "i"(8 * W) : "memory");
        fills<H, K + 1>(l);
    } else {
#pragma unroll
        for (int k = LDW - 1; k >= 0; --k)
            asm volatile("s_waitcnt vmcnt(%2)\n\tds_max_i32 %0, %1" :: "v"(l.fa), "v"(k ? fill_step(H, K - k) : NSTEP), "i"(8 * k) : "memory");
    }
}

__device__ __forceinline__ void take_ticket(const Args& a, unsigned call, int lane) {
    unsigned tk = 0;
    if (lane == 0) tk = __hip_atomic_fetch_add((unsigned*)(a.peer[0].ws + WS_TICKET), 1u, __ATOMIC_RELAXED, __HIP_MEMORY_SCOPE_AGENT);
    if ((__builtin_amdgcn_readfirstlane(tk) & 255) == 255) {
        if (lane < NRANK)
            __hip_atomic_store((gu32_p)(uintptr_t)(a.peer[lane].ws + WS_FLAGS + ((NRANK - lane) & (NRANK - 1)) * 128), call,
                               __ATOMIC_RELAXED, __HIP_MEMORY_SCOPE_SYSTEM);
    }
}

__device__ __forceinline__ void help(const Args& a, int w, int b, int lane) {
    __builtin_amdgcn_s_setprio(0);
    const unsigned t0 = (unsigned)__builtin_amdgcn_s_memrealtime();
    const int j = b >> 3, ct = ((b & 7) + 2 * (j >> 3)) & 7;
    const unsigned call = __builtin_amdgcn_readfirstlane(lds_ld(LDS_CTRL + C_CALL)), tag = call % 3 + 1;
    const char* xi = a.peer[0].ws + WS_X + ((b ^ 8) * 4 + w) * XSTRIDE;
    typedef const __attribute__((address_space(4))) float* cfp;
    float scale = *(cfp)(uintptr_t)a.sx * *(cfp)(uintptr_t)a.sw;
    const Out o = out_of((char*)a.col[ct].out + (size_t)(unsigned)(((((j >> 1) & 3) * 128 + w * 32 + (j & 1) * 16) * NN + (j >> 3) * 1024) * 4),
                         LDS_LOC + w * 4096, lane);
    const unsigned l16 = lane * 16, pa = LDS_CTRL + C_PROG + (lane & 3) * 260;
    unsigned sel0 = 0x0c0c0703, sel1 = 0x0c070100;
    unsigned hf = LDS_CTRL + C_H0 + w * 4, h0 = LDS_LOC + w * 4096 + l16, h1 = LDS_H1 + w * 4096 + l16;
    asm volatile("" : "+s"(sel0), "+s"(sel1), "+v"(scale), "+v"(hf), "+v"(h0), "+v"(h1));
    while (released(pa) < NSTEP - 14) __builtin_amdgcn_s_sleep(4);
    unsigned long long now;
    unsigned d;
    do {
        asm volatile("s_load_dword %0, %2, %3 glc\n\ts_memrealtime %1\n\ts_waitcnt lgkmcnt(0)" : "=&s"(d), "=&s"(now) : "s"(xi), "i"(XLOOK) : "memory");
    } while ((d & 3) != tag && (unsigned)now - t0 < WATCH_TICKS);
    __builtin_amdgcn_s_setprio(2);
    i32x4 Q[6];
    bool late;
    do {
        asm volatile("global_load_dwordx4 %3, %6, %7 offset:-1024 sc0 sc1\n\tglobal_load_dwordx4 %4, %6, %7 sc0 sc1\n\t"
                     "global_load_dwordx4 %5, %6, %7 offset:1024 sc0 sc1\n\tglobal_load_dwordx4 %0, %6, %7 offset:-4096 sc0 sc1\n\t"
                     "global_load_dwordx4 %1, %6, %7 offset:-3072 sc0 sc1\n\tglobal_load_dwordx4 %2, %6, %7 offset:-2048 sc0 sc1\n\t"
                     "s_waitcnt vmcnt(0)"
                     : "=&v"(Q[0]), "=&v"(Q[1]), "=&v"(Q[2]), "=&v"(Q[3]), "=&v"(Q[4]), "=&v"(Q[5]) : "v"(l16), "s"(xi + 4096) : "memory");
        unsigned diff = 0;
#pragma unroll
        for (int q = 0; q < 6; ++q) diff |= (unsigned)Q[q][0] ^ tag;
        if (__builtin_expect(!__any(diff & 3), 1)) { late = false; break; }
        late = (unsigned)__builtin_amdgcn_s_memrealtime() - t0 > DEADLINE_TICKS + DEADLINE_TICKS / 2;
    } while (!late);
    i32x4 T[2][4];
    Four H;
#pragma unroll
    for (int u = 1; u >= 0; --u) {
#pragma unroll
        for (int e = 0; e < 4; ++e) {
#pragma unroll
            for (int k = 0; k < 3; ++k) T[u][k][e] = e ? Q[3 * u + k][e] & 0xffffff : __builtin_amdgcn_ubfe(Q[3 * u + k][e], 2, 22);
            T[u][3][e] = __builtin_amdgcn_perm(Q[3 * u + 2][e], __builtin_amdgcn_perm(Q[3 * u + 1][e], Q[3 * u][e], sel0), sel1);
        }
        if (__builtin_expect(__builtin_amdgcn_readfirstlane(Q[3 * u][1]) & 0x800000, 0))
            asm volatile("global_load_dwordx4 %0, %4, %5 sc0 sc1\n\tglobal_load_dwordx4 %1, %4, %5 offset:1024 sc0 sc1\n\t"
                         "global_load_dwordx4 %2, %4, %5 offset:2048 sc0 sc1\n\tglobal_load_dwordx4 %3, %4, %5 offset:3072 sc0 sc1\n\t"
                         "s_waitcnt vmcnt(0)"
                         : "=&v"(T[u][0]), "=&v"(T[u][1]), "=&v"(T[u][2]), "=&v"(T[u][3])
                         : "v"(l16), "s"(a.peer[0].ws + WS_RAW + ((b ^ 8) * 4 + w) * 8192 + u * 4096) : "memory");
        if (u) {
#pragma unroll
            for (int t = 0; t < 4; ++t) *(lds_vi32x4_p)(h1 + t * 1024) = T[1][t];
            lds_st1(LDS_CTRL + C_H1 + w * 4, 1);
            H = fetch4(hf, h0);
            SB();
        }
    }
    if (late) {
        scale = __builtin_nanf("");
        if (lane == 0) __hip_atomic_store((unsigned*)(a.peer[0].ws + WS_TMO), call, __ATOMIC_RELAXED, __HIP_MEMORY_SCOPE_AGENT);
    }
    __builtin_amdgcn_s_setprio(0);
    await4(H, hf, h0);
    __builtin_amdgcn_s_setprio(1);
    finish(H.x, T[0], scale, o);
}

}

__global__ __launch_bounds__(NTHREADS, 2) __attribute__((aligned(4096))) void a2a_gemm_kernel(Args a) {
    extern __shared__ __attribute__((aligned(16))) char smem[];
    const int lane = threadIdx.x & 63;
    int wave = __builtin_amdgcn_readfirstlane(threadIdx.x >> 6);
    const int b = blockIdx.x;
    const int xcd = b & 7, j = b >> 3;
    const unsigned lds0 = (unsigned)(uintptr_t)(__attribute__((address_space(3))) char*)smem;
    const unsigned ctrl = lds0 + LDS_CTRL;
    const int P = j & 1;
    const int mi = (j >> 1) & 3;
    const int ct = (xcd + 2 * (j >> 3)) & 7;
    const int n0 = (j >> 3) * 1024;
#define NEXT_ROLE() asm volatile("" : "+s"(wave) :: "memory")
    u64 t64;

    unsigned pre = 0;
    if (wave == 7) {
        {
            u64 pc;
            unsigned len;
            asm volatile("s_getpc_b64 %0\n.Lpc%=:\n\ts_mov_b32 %1, .Lend%=-.Lpc%=-4\n\t.subsection 2\n.Lend%=:\n\t.subsection 0" : "=s"(pc), "=s"(len));
            asm volatile("s_mov_b32 m0, %2\n\ts_mov_b64 exec, 0xffff\n\tglobal_load_lds_dword %0, %1\n\ts_mov_b64 exec, -1"
                         :: "v"(min((unsigned)(j * 2048 + lane * 128), len)), "s"(pc), "i"(LDS_CTRL + 2048) : "memory");
        }
        lds_st(ctrl + lane * 4, lane == 1 ? 1 : lane == C_DUE / 4 ? FIRST_AT + 2 : 0);
        *(lds_i32x4_p)(ctrl + C_PROG + lane * 16) = i32x4{0, 0, 0, 0};
        asm volatile("s_waitcnt lgkmcnt(0)" ::: "memory");
    } else {
        asm volatile("s_load_dword %0, %1, 0\n\ts_load_dword %0, %1, 64\n\ts_load_dword %0, %1, 128\n\t"
                     "s_load_dword %0, %1, 192\n\ts_load_dword %0, %1, 256"
                     : "=&s"(pre) : "s"(__builtin_amdgcn_kernarg_segment_ptr()) : "memory");
    }
    asm volatile("s_barrier" : "+s"(wave) :: "memory");
    Ctx c;
    if (wave < 4) {
        __builtin_amdgcn_s_setprio(1);
        u64x2 own = args_ld(0);
#pragma unroll
        for (int i = 0; i < 4; ++i) {
            c.sv[i] = (mi * 128 + wave * 32 + i * 8 + (lane >> 3)) * KLOC + P * 256 + (mi >> 1) * 128 +
                      (((lane & 7) ^ ((i * 4 + (lane >> 4)) & 7)) << 4) - i * 1024;
            asm volatile("" : "+v"(c.sv[i]));
        }
        unsigned ll = lds0 + LDS_LOC + wave * 4096;
        asm volatile("s_waitcnt lgkmcnt(0)" : "+s"(own), "+s"(ll) : "s"(pre) : "memory");
        __builtin_amdgcn_s_sleep(4);
        dma4((const char*)own[1], ll, c.sv[0], c.sv[1], c.sv[2], c.sv[3]);
    }
    NEXT_ROLE();
    Ldr l;
    if ((wave & 6) == 4) {
        __builtin_amdgcn_s_setprio(0);
        u64x2 col = args_ld(128 + ct * 16);
        unsigned f0 = (4 * P + ((wave & 1) ^ mi)) * 64 * NN, slot = lds0 + (wave & 1) * 8192;
#pragma unroll
        for (int i = 0; i < 4; ++i) {
            l.dv[i] = (lane >> 3) * NN + n0 + (((lane & 7) ^ (((lane >> 4) & 3) | ((i >> 1) << 2))) << 4) + i * (8 * NN - 1024);
            asm volatile("" : "+v"(l.dv[i]));
        }
        asm volatile("s_waitcnt lgkmcnt(0)" : "+s"(col), "+s"(f0), "+s"(slot) : "s"(pre) : "memory");
        __builtin_amdgcn_s_sleep(10);
        l.w = (const char*)col[0];
        fill(l, f0, slot);
        fill(l, f0 ^ (2 * 64 * NN), slot + 16384);
    }
    NEXT_ROLE();
    if (wave < 4) {
        char* const ws = a.peer[0].ws;
        const unsigned tstart = (unsigned)__builtin_amdgcn_s_memrealtime();
        c.ll = lds0 + LDS_LOC + wave * 4096;
        c.xl2 = (const char*)a.peer[0].x + (128 - (mi >> 1) * 256);
        c.la = c.ll + P * 2048 + (lane & 15) * 128 + ((((mi & 1) * 4 + (lane >> 4)) ^ ((lane & 15) >> 1)) << 4);
        c.l16 = lane * 16;
        c.chk = ctrl + C_FULL + min(lane & 3, 2) * 4;
        c.foff = (lane & 7) * 128;
        c.flags = ws + WS_FLAGS;
        c.t0 = tstart;
        c.bad = 0;
        typedef const __attribute__((address_space(4))) float* cfp;
        c.scale = *(cfp)(uintptr_t)a.sx * *(cfp)(uintptr_t)a.sw;
        char* outp = (char*)a.col[ct].out;
        asm volatile("" : "+s"(outp));
        c.prog = ctrl + C_PROG + wave * 256 + lane * 4;
        c.one = 1;
        asm volatile("" : "+v"(c.one), "+v"(c.chk), "+v"(c.prog), "+v"(c.foff));
        {
            const unsigned q = (lane & 15) >> 1, kg = lane >> 4;
            const unsigned r0 = lds0 + (kg * 16 + q) * 128 + (lane & 1) * 8, f = (q >> 1) | ((kg & 1) << 2);
#pragma unroll
            for (int t = 0; t < 8; ++t) {
                c.rb[0][t] = r0 + ((t ^ f) << 4);
                c.rb[1][t] = c.rb[0][t] + 65536;
                asm volatile("" : "+v"(c.rb[0][t]), "+v"(c.rb[1][t]));
            }
        }
        i32x4 acc[2][8];
#pragma unroll
        for (int m = 0; m < 2; ++m)
#pragma unroll
            for (int t = 0; t < 8; ++t) { acc[m][t] = (m ? XBIAS : -XBIAS) + i32x4{0, 0, 0, 0}; asm volatile("" : "+v"(acc[m][t])); }
        i32x2 B[2][16];
        i32x4 AL[2][2][2], A[NFRAG];
        Four X;
        asm volatile("s_waitcnt vmcnt(0)" ::: "memory");
        read_local(AL[0], c);
        LGKM(0);
        dma4(c.xl2, c.ll, c.sv[0], c.sv[1], c.sv[2], c.sv[3]);
        unsigned call;
        while ((call = __builtin_amdgcn_readfirstlane(lds_ld(ctrl + C_CALL))) == 0) __builtin_amdgcn_s_sleep(1);
        c.call = call;
        c.sent = call <= 2 ? 0u : POISON;
        c.tag = call % 3 + 1;
        c.ro = ws + WS_RAW + (b * 4 + wave) * 8192 + 4096;
        c.sel[0] = 0x04020100; c.sel[1] = 0x05020100; c.sel[2] = 0x06020100;
        asm volatile("" : "+s"(c.ro), "+s"(c.sel[0]), "+s"(c.sel[1]), "+s"(c.sel[2]));
#pragma unroll
        for (int i = 0; i < 7; ++i) {
            c.ab[i] = ws + WS_DATA + (call % 3) * SLOT_BYTES + ((mi * 4 + wave) * 2 + P) * (NFRAG * 1024) + i * 8192 + 4096;
            asm volatile("" : "+s"(c.ab[i]));
        }
        c.xo = ws + WS_X + (b * 4 + wave) * XSTRIDE + 4096;
        asm volatile("" : "+s"(c.xo), "+s"(c.sent), "+s"(c.tag), "+s"(c.flags), "+s"(c.bad), "+v"(c.l16));
        c.hf = ctrl + C_H0 + wave * 4;
        c.h0 = c.ll + c.l16;
        c.h1 = lds0 + LDS_H1 + wave * 4096 + c.l16;
        const Out o = out_of(outp + (size_t)(unsigned)(((mi * 128 + wave * 32 + P * 16) * NN + n0 + 64) * 4), lds0 + LDS_H1 + wave * 4096, lane);
        asm volatile("" : "+v"(c.hf), "+v"(c.h0), "+v"(c.h1), "+v"(c.scale));
        check<1>(c);
#pragma unroll
        for (int t = 0; t < 8; ++t) { read_b<0, 0>(B[0], t, c); read_b<0, 1>(B[0], t, c); }
        check<2>(c);
        step<0>(c, acc, AL, A, B, X);
        SB();

        await4(X, c.hf + C_H1 - C_H0, c.h1);
        finish(&acc[0][4], X.x, c.scale, o);
        if (c.bad && lane == 0) __hip_atomic_store((unsigned*)(a.peer[0].ws + WS_TMO), call, __ATOMIC_RELAXED, __HIP_MEMORY_SCOPE_AGENT);
        __builtin_amdgcn_endpgm();
    }
    NEXT_ROLE();
    if (wave == 6) {
        __builtin_amdgcn_s_setprio(3);
        u64x2 own = args_ld(0);
        const int pd = ct == 3 ? 0 : ((ct ^ 4) + j) % 7 + 1, pc = j >> 2, pm = (j - ((NRANK - pd) >> 1)) & 3;
        u64x2 to = args_ld(pd * 16);
        unsigned sv[8];
#pragma unroll
        for (int g = 0; g < 8; ++g) {
            sv[g] = (pm * 128 + g * 16 + (lane >> 2)) * KLOC + ((pc + 4 * pm) & 7) * 64 + (((lane & 3) ^ ((lane >> 4) & 3)) << 4) - (g & 3) * 1024;
            asm volatile("" : "+v"(sv[g]));
        }
        asm volatile("s_waitcnt lgkmcnt(0)" : "+s"(own), "+s"(to) : "s"(pre) : "memory");
        char* const ws = (char*)own[0];
        char* const pw = (char*)to[0];
        const char* const base = (const char*)to[1];
        unsigned cv;
        asm volatile("s_load_dword %0, %2, %3\n\ts_memrealtime %1" : "=&s"(cv), "=&s"(t64) : "s"(ws), "i"(WS_TICKET) : "memory");
        if (pd && j < XJIT) {
            dma4(base, lds0 + LDS_PUSH, sv[0], sv[1], sv[2], sv[3]);
            dma4(base, lds0 + LDS_PUSH + 4096, sv[4], sv[5], sv[6], sv[7]);
        }
        int pd2 = pd;
        asm volatile("" : "+s"(pd2) :: "memory");
        const unsigned slot = PUSH0 + ((j < 8 ? j * PACE0 : j * PACE - 8 * (PACE - PACE0)) >> 2);
        unsigned doff[2], poff[2];
#pragma unroll
        for (int m = 0; m < 2; ++m) {
            doff[m] = lane * 16 + WS_DATA + frag_off(NRANK - pd2, pc, pm * 8 + m);
            asm volatile("" : "+v"(doff[m]));
        }
        const unsigned roff = lds0 + LDS_PUSH + (lane & 15) * 64 + (((lane >> 4) ^ ((lane >> 2) & 3)) << 4);
        asm volatile("s_waitcnt lgkmcnt(0)" : "+s"(cv), "+s"(t64) : "s"(slot), "v"(roff) : "memory");
        const unsigned due = (unsigned)t64 + slot;
        const unsigned call = (cv >> 8) + 1;
        lds_st1(ctrl + C_CALL, (int)call);
        const unsigned cur = (call % 3) * SLOT_BYTES, old = ((call + 2) % 3) * SLOT_BYTES;
        if (pd) {
            char* dp[4];
#pragma unroll
            for (int w = 0; w < 4; ++w) { dp[w] = pw + cur + w * (2 * NFRAG * 1024); asm volatile("" : "+s"(dp[w])); }
            __builtin_amdgcn_s_setprio(0);
            if (j >= 20) touch_x(ws + WS_X + (b ^ 8) * 4 * XSTRIDE);
            if (j >= XJIT) {
                while ((int)(due - XAHEAD - (unsigned)__builtin_amdgcn_s_memrealtime()) > 0) __builtin_amdgcn_s_sleep(4);
                __builtin_amdgcn_s_setprio(3);
                dma4(base, lds0 + LDS_PUSH, sv[0], sv[1], sv[2], sv[3]);
                dma4(base, lds0 + LDS_PUSH + 4096, sv[4], sv[5], sv[6], sv[7]);
                __builtin_amdgcn_s_setprio(0);
            }
            for (int left; (left = (int)(due - PAUSE - (unsigned)__builtin_amdgcn_s_memrealtime())) > 0;) {
                if (left > 40) __builtin_amdgcn_s_sleep(8);
                else __builtin_amdgcn_s_sleep(1);
            }
            lds_st1(ctrl + C_PAUSE, 1);
            for (int left; (left = (int)(due - (unsigned)__builtin_amdgcn_s_memrealtime())) > 0;) {
                if (left > 40) __builtin_amdgcn_s_sleep(8);
                else __builtin_amdgcn_s_sleep(1);
            }
            __builtin_amdgcn_s_setprio(3);
            asm volatile("s_waitcnt vmcnt(0)" ::: "memory");
            i32x4 dat[8];
#pragma unroll
            for (int g = 0; g < 8; ++g) dat[g] = *(lds_i32x4_p)(roff + g * 1024);
#pragma unroll
            for (int g = 0; g < 8; ++g)
                asm volatile("global_store_dwordx4 %1, %0, %2 sc0 sc1\n\ts_nop 1" :: "v"(dat[g]), "v"(doff[g & 1]), "s"(dp[g >> 1]) : "memory");
            lds_st1(ctrl + C_PAUSE, 2);
            __builtin_amdgcn_s_setprio(0);
            asm volatile("" : "+s"(pd2) :: "memory");
#pragma unroll
            for (int m = 0; m < 2; ++m) poff[m] = lane * 16 + WS_DATA + frag_off(pd2, pc, pm * 8 + m);
            asm volatile("s_waitcnt vmcnt(0)" :: "v"(poff[0]), "v"(poff[1]) : "memory");
            const i32x4 poison = {(int)POISON, (int)POISON, (int)POISON, (int)POISON};
#pragma unroll
            for (int g = 0; g < 8; ++g)
                asm volatile("global_store_dwordx4 %1, %0, %2 sc1\n\ts_nop 1"
                             :: "v"(poison), "v"(poff[g & 1]), "s"(ws + old + (g >> 1) * (2 * NFRAG * 1024)) : "memory");
        }
        lds_st1(ctrl + C_PAUSE, 2);
        take_ticket(a, call, lane);
        if (j < 20 || !pd) touch_x(ws + WS_X + (b ^ 8) * 4 * XSTRIDE);
    }

    NEXT_ROLE();
    if (wave < 6) {
        l.w = (const char*)a.col[ct].w;
        l.fa = ctrl + C_FULL + (lane ? 64 + lane * 8 : 0) + (wave & 1) * 4;
        l.pa = ctrl + C_PROG + (lane & 3) * 260;
        const int r8 = a.rank * 8;
        l.woff = (((w_chunk(min(lane, NSTEP - 1), P, mi) + r8) & 63) - r8) * 64 * NN;
        l.freed = 0;
        l.pz = 0;
        if (wave & 1) fills<1, 1>(l);
        else fills<0, 1>(l);
    }

    NEXT_ROLE();
    if (wave == 7) {
        const unsigned tstart = (unsigned)__builtin_amdgcn_s_memrealtime();
        char* const ws = a.peer[0].ws;
        unsigned d[8];
        unsigned long long now;
        int due = 1000;
        allowed<NSTEP - 1>(due, lane);
        constexpr int NI = NSPLIT / 2 + NS2;
        const int q = min(lane, NI - 1);
        const int offs = (q < NSPLIT / 2 ? 2 * q + (j >> 4) : q + NSPLIT / 2) * 1024;
        unsigned call;
        while ((call = __builtin_amdgcn_readfirstlane(lds_ld(ctrl + C_CALL))) == 0) __builtin_amdgcn_s_sleep(1);
        const unsigned sent = call <= 2 ? 0u : POISON;
        const char* af = ws + WS_DATA + (call % 3) * SLOT_BYTES + ((mi * 4 + 2 + ((j >> 3) & 1)) * 2 + P) * (NFRAG * 1024) + xcd * 128 + 112;
        int n = 0;
        while (n < NSPLIT / 2) {
            unsigned o[4];
#pragma unroll
            for (int i = 0; i < 4; ++i) o[i] = __builtin_amdgcn_readlane(offs, n + i);
            do {
                asm volatile("s_load_dword %0, %5, %6 glc\n\ts_load_dword %1, %5, %7 glc\n\t"
                             "s_load_dword %2, %5, %8 glc\n\ts_load_dword %3, %5, %9 glc\n\t"
                             "s_memrealtime %4\n\ts_waitcnt lgkmcnt(0)"
                             : "=&s"(d[0]), "=&s"(d[1]), "=&s"(d[2]), "=&s"(d[3]), "=&s"(now)
                             : "s"(af), "s"(o[0]), "s"(o[1]), "s"(o[2]), "s"(o[3]) : "memory");
            } while (d[0] == sent && (unsigned)now - tstart < WATCH_TICKS);
            n += d[1] == sent ? 1 : d[2] == sent ? 2 : d[3] == sent ? 3 : 4;
            lds_st2(ctrl + C_DUE, __builtin_amdgcn_readlane(due, n), n);
        }
        while (n < NI) {
            const char* p = af + (n + NSPLIT / 2) * 1024;
            do {
                asm volatile("s_load_dword %0, %9, 0 glc\n\ts_load_dword %1, %9, 0x400 glc\n\ts_load_dword %2, %9, 0x800 glc\n\t"
                             "s_load_dword %3, %9, 0xc00 glc\n\ts_load_dword %4, %9, 0x1000 glc\n\ts_load_dword %5, %9, 0x1400 glc\n\t"
                             "s_load_dword %6, %9, 0x1800 glc\n\ts_load_dword %7, %9, 0x1c00 glc\n\t"
                             "s_memrealtime %8\n\ts_waitcnt lgkmcnt(0)"
                             : "=&s"(d[0]), "=&s"(d[1]), "=&s"(d[2]), "=&s"(d[3]), "=&s"(d[4]), "=&s"(d[5]), "=&s"(d[6]), "=&s"(d[7]), "=&s"(now)
                             : "s"(p) : "memory");
            } while (d[0] == sent && (unsigned)now - tstart < WATCH_TICKS);
            n += d[1] == sent ? 1 : d[2] == sent ? 2 : d[3] == sent ? 3 : d[4] == sent ? 4 : d[5] == sent ? 5 : d[6] == sent ? 6 : d[7] == sent ? 7 : 8;
            lds_st1(ctrl + C_DUE, __builtin_amdgcn_readlane(due, n));
        }
        lds_st2(ctrl + C_DUE, 1000, 1000);
    }
    NEXT_ROLE();
    help(a, wave & 3, b, lane);
}

extern "C" void kernel_launch(int rank, int nranks, ncclComm_t comm,
                              hipStream_t stream,
                              void* const* d_in, const int* in_sizes, int n_in,
                              void* d_out, int out_size,
                              const int* in_dtypes, int out_dtype,
                              void* const* d_out_all,
                              void* const* d_ws_all, size_t ws_bytes) {
    if (nranks != NRANK || (unsigned)rank >= (unsigned)NRANK || ws_bytes < WS_NEEDED || n_in < 4 ||
        out_size != MLOC * NN || in_sizes[0] != NRANK * MLOC * KLOC || in_sizes[1] != NRANK * KLOC * NN) {
        fprintf(stderr, "unexpected problem shape\n");
        return;
    }
    static bool attr_set = false;
    if (!attr_set) {
        HIP_CHECK(hipFuncSetAttribute((const void*)a2a_gemm_kernel, hipFuncAttributeMaxDynamicSharedMemorySize,
                                      (int)LDS_BYTES));
        attr_set = true;
    }
    Args a;
    for (int i = 0; i < NRANK; ++i) {
        const int p = (rank + i) & (NRANK - 1), c0 = ((i - (int)((uintptr_t)d_in[1] >> 7)) & 7) * 128;
        a.peer[i].ws = (char*)d_ws_all[p];
        a.peer[i].x = (const int8_t*)d_in[0] + (size_t)p * MLOC * KLOC;
        a.col[i].w = (const int8_t*)d_in[1] + (size_t)rank * KLOC * NN + c0;
        a.col[i].out = (float*)d_out + c0;
    }
    a.sx = (const float*)d_in[2];
    a.sw = (const float*)d_in[3];
    a.rank = rank;
    a.pad = 0;
    hipLaunchKernelGGL(a2a_gemm_kernel, dim3(NWG), dim3(NTHREADS), LDS_BYTES, stream, a);
    HIP_CHECK(hipGetLastError());
}
```
